```python
import numpy as np

kernel_name = "gpu_a2a_gemm_m8192_k8192_n8192_bf16_none"


ABSMAX_RTOL = 0.008

COMPUTE_DTYPE = "bfloat16"
EPILOGUE = "none"
IN_QUANT = None
OUT_QUANT = None
IN_DTYPES = {"x": "bfloat16", "w_mat": "bfloat16"}
WORLD_SIZE = 8

def _to_bf16(x):
    u = np.ascontiguousarray(x, dtype=np.float32).view(np.uint32)
    return ((u + 0x7FFF + ((u >> 16) & 1)) & 0xFFFF0000).view(np.float32)

def setup_inputs():

    x = _to_bf16(np.random.randn(8192, 8192).astype(np.float32))
    w_mat = _to_bf16((np.random.randn(8192, 8192) / np.sqrt(8192)).astype(np.float32))
    return {"x": x, "w_mat": w_mat}

def _epilogue(y):
    return y

def reference(x, w_mat):
    y = (x @ w_mat).astype(np.float32)
    return _epilogue(y)
```

```cpp
#include <hip/hip_runtime.h>
#include <rccl/rccl.h>
#include <cstdio>
#include <cstddef>

#define GEMM_EARLY \
  "s_waitcnt lgkmcnt(0)\n\t" \
  "s_cmp_eq_u32 s17, 1\n\t" \
  "s_cselect_b32 s40, s12, s14\n\t" \
  "s_cselect_b32 s41, s13, s15\n\t" \
  "s_cselect_b32 s10, 0x1ff800, 0\n\t" \
  "s_mul_i32 s10, s10, s16\n\t" \
  "s_add_u32 s19, s75, s10\n\t" \
  "s_lshl4_add_u32 s18, s16, s73\n\t" \
  "s_sub_u32 s40, s40, 0x1000\n\t" \
  "s_subb_u32 s41, s41, 0\n\t" \
  "s_and_b32 s41, s41, 0xffff\n\t" \
  "s_lshl_b32 s84, s18, s74\n\t" \
  "s_add_u32 s84, s84, s19\n\t" \
  "s_add_u32 m0, s24, 0\n\t" \
  "s_nop 0\n\t" \
  "buffer_load_dwordx4 v0, s[40:43], s84 offen offset:0 sc0 sc1 lds\n\t" \
  "buffer_load_dwordx4 v1, s[40:43], s84 offen offset:1024 sc0 sc1 lds\n\t" \
  "buffer_load_dwordx4 v2, s[40:43], s84 offen offset:2048 sc0 sc1 lds\n\t" \
  "buffer_load_dwordx4 v3, s[40:43], s84 offen offset:3072 sc0 sc1 lds\n\t" \
  "s_add_u32 m0, s25, 0\n\t" \
  "s_nop 0\n\t" \
  "buffer_load_dwordx4 v4, s[40:43], s84 offen offset:0 sc0 sc1 lds\n\t" \
  "buffer_load_dwordx4 v5, s[40:43], s84 offen offset:1024 sc0 sc1 lds\n\t" \
  "buffer_load_dwordx4 v6, s[40:43], s84 offen offset:2048 sc0 sc1 lds\n\t" \
  "buffer_load_dwordx4 v7, s[40:43], s84 offen offset:3072 sc0 sc1 lds\n\t" \
  ""
#define GEMM_EARLYB \
  "v_mov_b32 v86, 1\n\t" \
  "s_nop 4\n\t" \
  "buffer_atomic_add v86, %2, %3, 0 offen sc0\n\t" \
  "buffer_load_dwordx4 v[56:59], %4, %5, %6 offen\n\t" \
  ""
#define GEMM_ASM \
  "s_lshl_b32 s84, s73, s74\n\t" \
  "s_add_u32 s84, s84, s75\n\t" \
  "s_add_u32 m0, s26, 0\n\t" \
  "s_nop 0\n\t" \
  "buffer_load_dwordx4 v8, s[40:43], s84 offen offset:0 sc0 sc1 lds\n\t" \
  "buffer_load_dwordx4 v9, s[40:43], s84 offen offset:1024 sc0 sc1 lds\n\t" \
  "buffer_load_dwordx4 v10, s[40:43], s84 offen offset:2048 sc0 sc1 lds\n\t" \
  "buffer_load_dwordx4 v11, s[40:43], s84 offen offset:3072 sc0 sc1 lds\n\t" \
  "s_add_u32 m0, s27, 0\n\t" \
  "s_nop 0\n\t" \
  "buffer_load_dwordx4 v12, s[40:43], s84 offen offset:0 sc0 sc1 lds\n\t" \
  "buffer_load_dwordx4 v13, s[40:43], s84 offen offset:1024 sc0 sc1 lds\n\t" \
  "buffer_load_dwordx4 v14, s[40:43], s84 offen offset:2048 sc0 sc1 lds\n\t" \
  "buffer_load_dwordx4 v15, s[40:43], s84 offen offset:3072 sc0 sc1 lds\n\t" \
  "s_bfe_i32 s10, s8, 0x80000\n\t" \
  "s_add_u32 s73, s73, s10\n\t" \
  "s_lshl_b32 s84, s73, s74\n\t" \
  "s_add_u32 s84, s84, s75\n\t" \
  "v_mfma_f32_32x32x16_bf16 a[0:15], v[50:53], v[50:53], 0\n\t" \
  "v_mfma_f32_32x32x16_bf16 a[16:31], v[50:53], v[50:53], 0\n\t" \
  "v_mfma_f32_32x32x16_bf16 a[32:47], v[50:53], v[50:53], 0\n\t" \
  "v_mfma_f32_32x32x16_bf16 a[48:63], v[50:53], v[50:53], 0\n\t" \
  "v_mfma_f32_32x32x16_bf16 a[64:79], v[50:53], v[50:53], 0\n\t" \
  "v_mfma_f32_32x32x16_bf16 a[80:95], v[50:53], v[50:53], 0\n\t" \
  "v_mfma_f32_32x32x16_bf16 a[96:111], v[50:53], v[50:53], 0\n\t" \
  "v_mfma_f32_32x32x16_bf16 a[112:127], v[50:53], v[50:53], 0\n\t" \
  "v_mfma_f32_32x32x16_bf16 v[128:143], v[50:53], v[50:53], 0\n\t" \
  "v_mfma_f32_32x32x16_bf16 v[144:159], v[50:53], v[50:53], 0\n\t" \
  "v_mfma_f32_32x32x16_bf16 v[160:175], v[50:53], v[50:53], 0\n\t" \
  "v_mfma_f32_32x32x16_bf16 v[176:191], v[50:53], v[50:53], 0\n\t" \
  "v_mfma_f32_32x32x16_bf16 v[192:207], v[50:53], v[50:53], 0\n\t" \
  "v_mfma_f32_32x32x16_bf16 v[208:223], v[50:53], v[50:53], 0\n\t" \
  "v_mfma_f32_32x32x16_bf16 v[224:239], v[50:53], v[50:53], 0\n\t" \
  "v_mfma_f32_32x32x16_bf16 v[240:255], v[50:53], v[50:53], 0\n\t" \
  "s_add_u32 m0, s24, 32768\n\t" \
  "s_nop 0\n\t" \
  "buffer_load_dwordx4 v0, s[40:43], s84 offen offset:0 sc0 sc1 lds\n\t" \
  "buffer_load_dwordx4 v1, s[40:43], s84 offen offset:1024 sc0 sc1 lds\n\t" \
  "buffer_load_dwordx4 v2, s[40:43], s84 offen offset:2048 sc0 sc1 lds\n\t" \
  "buffer_load_dwordx4 v3, s[40:43], s84 offen offset:3072 sc0 sc1 lds\n\t" \
  "s_add_u32 m0, s25, 32768\n\t" \
  "s_nop 0\n\t" \
  "buffer_load_dwordx4 v4, s[40:43], s84 offen offset:0 sc0 sc1 lds\n\t" \
  "buffer_load_dwordx4 v5, s[40:43], s84 offen offset:1024 sc0 sc1 lds\n\t" \
  "buffer_load_dwordx4 v6, s[40:43], s84 offen offset:2048 sc0 sc1 lds\n\t" \
  "buffer_load_dwordx4 v7, s[40:43], s84 offen offset:3072 sc0 sc1 lds\n\t" \
  "s_mov_b32 s87, 24\n\t" \
  "s_mov_b32 s91, 0\n\t" \
  "s_mov_b32 s89, 0\n\t" \
  "s_bfe_i32 s10, s8, 0x80008\n\t" \
  "s_add_u32 s86, s73, s10\n\t" \
  "s_bfe_i32 s10, s8, 0x80010\n\t" \
  "s_add_u32 s73, s86, s10\n\t" \
  "s_waitcnt vmcnt(16)\n\t" \
  "v_readfirstlane_b32 s92, v86\n\t" \
  "s_nop 0\n\t" \
  "s_add_u32 s92, s92, 1\n\t" \
  "s_getreg_b32 s93, hwreg(HW_REG_XCC_ID)\n\t" \
  "s_and_b32 s93, s93, 15\n\t" \
  "s_lshl4_add_u32 s93, s92, s93\n\t" \
  "v_mov_b32 v46, s93\n\t" \
  "s_and_b32 s11, s92, 1\n\t" \
  "s_lshl_b32 s11, s11, 24\n\t" \
  "s_add_u32 s44, s44, s11\n\t" \
  "s_addc_u32 s45, s45, 0\n\t" \
  "v_add_u32 v30, s11, v30\n\t" \
  "v_mov_b32 v43, s92\n\t" \
  "v_mbcnt_lo_u32_b32 v84, -1, 0\n\t" \
  "v_mbcnt_hi_u32_b32 v84, -1, v84\n\t" \
  "v_bfrev_b32 v85, 1\n\t" \
  "v_cmp_gt_u32 vcc, 24, v84\n\t" \
  "s_nop 1\n\t" \
  "v_cndmask_b32 v72, v85, v30, vcc\n\t" \
  "v_cndmask_b32 v73, v30, v85, vcc\n\t" \
  "v_cmp_gt_u32 vcc, 48, v84\n\t" \
  "s_nop 1\n\t" \
  "v_cndmask_b32 v73, v85, v73, vcc\n\t" \
  "v_cndmask_b32 v74, v30, v85, vcc\n\t" \
  "s_lshr_b32 s20, s21, 8\n\t" \
  "s_lshl_b32 s21, s80, 10\n\t" \
  "s_add_u32 s11, s80, s82\n\t" \
  "s_mov_b32 s80, s81\n\t" \
  "s_mov_b32 s81, s11\n\t" \
  "s_sub_u32 s82, 0, s82\n\t" \
  "buffer_store_dwordx4 v[56:59], v72, s[52:55], s21 offen sc0 sc1\n\t" \
  "buffer_load_dwordx4 v[60:63], v29, s[56:59], s80 offen\n\t" \
  "s_waitcnt vmcnt(10)\n\t" \
  "s_barrier\n\t" \
  "ds_read_b64_tr_b16 a[192:193], v16 offset:0\n\t" \
  "ds_read_b64_tr_b16 a[194:195], v16 offset:2048\n\t" \
  "ds_read_b64_tr_b16 a[196:197], v16 offset:16384\n\t" \
  "ds_read_b64_tr_b16 a[198:199], v16 offset:18432\n\t" \
  "ds_read_b128 a[128:131], v24 offset:0\n\t" \
  "ds_read_b128 a[132:135], v25 offset:0\n\t" \
  "ds_read_b64_tr_b16 a[200:201], v17 offset:0\n\t" \
  "ds_read_b64_tr_b16 a[202:203], v17 offset:2048\n\t" \
  "ds_read_b64_tr_b16 a[204:205], v17 offset:16384\n\t" \
  "ds_read_b64_tr_b16 a[206:207], v17 offset:18432\n\t" \
  "ds_read_b64_tr_b16 a[208:209], v18 offset:0\n\t" \
  "ds_read_b64_tr_b16 a[210:211], v18 offset:2048\n\t" \
  "ds_read_b64_tr_b16 a[212:213], v18 offset:16384\n\t" \
  "ds_read_b64_tr_b16 a[214:215], v18 offset:18432\n\t" \
  "ds_read_b64_tr_b16 a[216:217], v19 offset:0\n\t" \
  "ds_read_b64_tr_b16 a[218:219], v19 offset:2048\n\t" \
  "ds_read_b64_tr_b16 a[220:221], v19 offset:16384\n\t" \
  "ds_read_b64_tr_b16 a[222:223], v19 offset:18432\n\t" \
  "s_add_u32 m0, s26, 32768\n\t" \
  "s_waitcnt lgkmcnt(13)\n\t" \
  "v_mfma_f32_16x16x32_bf16 a[0:3], a[192:195], a[128:131], a[0:3]\n\t" \
  "buffer_load_dwordx4 v8, s[40:43], s84 offen offset:0 sc0 sc1 lds\n\t" \
  "s_waitcnt lgkmcnt(12)\n\t" \
  "v_mfma_f32_16x16x32_bf16 a[0:3], a[196:199], a[132:135], a[0:3]\n\t" \
  "ds_read_b128 a[136:139], v24 offset:2048\n\t" \
  "s_waitcnt lgkmcnt(11)\n\t" \
  "v_mfma_f32_16x16x32_bf16 a[4:7], a[200:203], a[128:131], a[4:7]\n\t" \
  "ds_read_b128 a[140:143], v25 offset:2048\n\t" \
  "s_waitcnt lgkmcnt(10)\n\t" \
  "v_mfma_f32_16x16x32_bf16 a[4:7], a[204:207], a[132:135], a[4:7]\n\t" \
  "buffer_load_dwordx4 v9, s[40:43], s84 offen offset:1024 sc0 sc1 lds\n\t" \
  "s_waitcnt lgkmcnt(8)\n\t" \
  "v_mfma_f32_16x16x32_bf16 a[8:11], a[208:211], a[128:131], a[8:11]\n\t" \
  "ds_read_b128 a[144:147], v24 offset:4096\n\t" \
  "s_waitcnt lgkmcnt(7)\n\t" \
  "v_mfma_f32_16x16x32_bf16 a[8:11], a[212:215], a[132:135], a[8:11]\n\t" \
  "ds_read_b128 a[148:151], v25 offset:4096\n\t" \
  "s_waitcnt lgkmcnt(6)\n\t" \
  "v_mfma_f32_16x16x32_bf16 a[12:15], a[216:219], a[128:131], a[12:15]\n\t" \
  "buffer_load_dwordx4 v10, s[40:43], s84 offen offset:2048 sc0 sc1 lds\n\t" \
  "s_waitcnt lgkmcnt(4)\n\t" \
  "v_mfma_f32_16x16x32_bf16 a[12:15], a[220:223], a[132:135], a[12:15]\n\t" \
  "ds_read_b128 a[152:155], v24 offset:6144\n\t" \
  "s_waitcnt lgkmcnt(4)\n\t" \
  "v_mfma_f32_16x16x32_bf16 a[28:31], a[216:219], a[136:139], a[28:31]\n\t" \
  "ds_read_b128 a[156:159], v25 offset:6144\n\t" \
  "s_waitcnt lgkmcnt(4)\n\t" \
  "v_mfma_f32_16x16x32_bf16 a[28:31], a[220:223], a[140:143], a[28:31]\n\t" \
  "buffer_load_dwordx4 v11, s[40:43], s84 offen offset:3072 sc0 sc1 lds\n\t" \
  "v_mfma_f32_16x16x32_bf16 a[24:27], a[208:211], a[136:139], a[24:27]\n\t" \
  "ds_read_b128 a[160:163], v24 offset:8192\n\t" \
  "v_mfma_f32_16x16x32_bf16 a[24:27], a[212:215], a[140:143], a[24:27]\n\t" \
  "ds_read_b128 a[164:167], v25 offset:8192\n\t" \
  "s_add_u32 m0, s27, 32768\n\t" \
  "v_mfma_f32_16x16x32_bf16 a[20:23], a[200:203], a[136:139], a[20:23]\n\t" \
  "buffer_load_dwordx4 v12, s[40:43], s84 offen offset:0 sc0 sc1 lds\n\t" \
  "v_mfma_f32_16x16x32_bf16 a[20:23], a[204:207], a[140:143], a[20:23]\n\t" \
  "ds_read_b128 a[168:171], v24 offset:10240\n\t" \
  "v_mfma_f32_16x16x32_bf16 a[16:19], a[192:195], a[136:139], a[16:19]\n\t" \
  "ds_read_b128 a[172:175], v25 offset:10240\n\t" \
  "v_mfma_f32_16x16x32_bf16 a[16:19], a[196:199], a[140:143], a[16:19]\n\t" \
  "buffer_load_dwordx4 v13, s[40:43], s84 offen offset:1024 sc0 sc1 lds\n\t" \
  "s_waitcnt lgkmcnt(6)\n\t" \
  "v_mfma_f32_16x16x32_bf16 a[32:35], a[192:195], a[144:147], a[32:35]\n\t" \
  "ds_read_b128 a[176:179], v24 offset:12288\n\t" \
  "v_mfma_f32_16x16x32_bf16 a[32:35], a[196:199], a[148:151], a[32:35]\n\t" \
  "ds_read_b128 a[180:183], v25 offset:12288\n\t" \
  "v_mfma_f32_16x16x32_bf16 a[36:39], a[200:203], a[144:147], a[36:39]\n\t" \
  "buffer_load_dwordx4 v14, s[40:43], s84 offen offset:2048 sc0 sc1 lds\n\t" \
  "v_mfma_f32_16x16x32_bf16 a[36:39], a[204:207], a[148:151], a[36:39]\n\t" \
  "ds_read_b128 a[184:187], v24 offset:14336\n\t" \
  "v_mfma_f32_16x16x32_bf16 a[40:43], a[208:211], a[144:147], a[40:43]\n\t" \
  "ds_read_b128 a[188:191], v25 offset:14336\n\t" \
  "v_mfma_f32_16x16x32_bf16 a[40:43], a[212:215], a[148:151], a[40:43]\n\t" \
  "buffer_load_dwordx4 v15, s[40:43], s84 offen offset:3072 sc0 sc1 lds\n\t" \
  "v_mfma_f32_16x16x32_bf16 a[44:47], a[216:219], a[144:147], a[44:47]\n\t" \
  "ds_read_b64_tr_b16 a[224:225], v20 offset:0\n\t" \
  "v_mfma_f32_16x16x32_bf16 a[44:47], a[220:223], a[148:151], a[44:47]\n\t" \
  "ds_read_b64_tr_b16 a[226:227], v20 offset:2048\n\t" \
  "s_waitcnt lgkmcnt(7)\n\t" \
  "v_mfma_f32_16x16x32_bf16 a[60:63], a[216:219], a[152:155], a[60:63]\n\t" \
  "ds_read_b64_tr_b16 a[228:229], v20 offset:16384\n\t" \
  "v_mfma_f32_16x16x32_bf16 a[60:63], a[220:223], a[156:159], a[60:63]\n\t" \
  "ds_read_b64_tr_b16 a[230:231], v20 offset:18432\n\t" \
  "v_mfma_f32_16x16x32_bf16 a[56:59], a[208:211], a[152:155], a[56:59]\n\t" \
  "ds_read_b64_tr_b16 a[232:233], v21 offset:0\n\t" \
  "v_mfma_f32_16x16x32_bf16 a[56:59], a[212:215], a[156:159], a[56:59]\n\t" \
  "ds_read_b64_tr_b16 a[234:235], v21 offset:2048\n\t" \
  "v_mfma_f32_16x16x32_bf16 a[52:55], a[200:203], a[152:155], a[52:55]\n\t" \
  "ds_read_b64_tr_b16 a[236:237], v21 offset:16384\n\t" \
  "v_mfma_f32_16x16x32_bf16 a[52:55], a[204:207], a[156:159], a[52:55]\n\t" \
  "ds_read_b64_tr_b16 a[238:239], v21 offset:18432\n\t" \
  "v_mfma_f32_16x16x32_bf16 a[48:51], a[192:195], a[152:155], a[48:51]\n\t" \
  "ds_read_b64_tr_b16 a[240:241], v22 offset:0\n\t" \
  "v_mfma_f32_16x16x32_bf16 a[48:51], a[196:199], a[156:159], a[48:51]\n\t" \
  "ds_read_b64_tr_b16 a[242:243], v22 offset:2048\n\t" \
  "v_mfma_f32_16x16x32_bf16 a[64:67], a[192:195], a[160:163], a[64:67]\n\t" \
  "ds_read_b64_tr_b16 a[244:245], v22 offset:16384\n\t" \
  "v_mfma_f32_16x16x32_bf16 a[64:67], a[196:199], a[164:167], a[64:67]\n\t" \
  "ds_read_b64_tr_b16 a[246:247], v22 offset:18432\n\t" \
  "v_mfma_f32_16x16x32_bf16 a[68:71], a[200:203], a[160:163], a[68:71]\n\t" \
  "s_mov_b32 s85, s86\n\t" \
  "s_mov_b32 s86, s73\n\t" \
  "ds_read_b64_tr_b16 a[248:249], v23 offset:0\n\t" \
  "v_mfma_f32_16x16x32_bf16 a[68:71], a[204:207], a[164:167], a[68:71]\n\t" \
  "s_lshl_b32 s84, s85, s74\n\t" \
  "s_add_u32 s84, s84, s75\n\t" \
  "ds_read_b64_tr_b16 a[250:251], v23 offset:2048\n\t" \
  "s_waitcnt lgkmcnt(0)\n\t" \
  "s_barrier\n\t" \
  "s_add_u32 m0, s24, 0\n\t" \
  "v_mfma_f32_16x16x32_bf16 a[72:75], a[208:211], a[160:163], a[72:75]\n\t" \
  "ds_read_b64_tr_b16 a[252:253], v23 offset:16384\n\t" \
  "buffer_load_dwordx4 v0, s[40:43], s84 offen offset:0 sc0 sc1 lds\n\t" \
  "v_mfma_f32_16x16x32_bf16 a[72:75], a[212:215], a[164:167], a[72:75]\n\t" \
  "s_lshr_b64 s[10:11], s[8:9], s87\n\t" \
  "s_sext_i32_i8 s10, s10\n\t" \
  "ds_read_b64_tr_b16 a[254:255], v23 offset:18432\n\t" \
  "v_mfma_f32_16x16x32_bf16 a[76:79], a[216:219], a[160:163], a[76:79]\n\t" \
  "s_add_u32 s87, s87, 8\n\t" \
  "s_add_u32 s73, s73, s10\n\t" \
  "v_mfma_f32_16x16x32_bf16 a[76:79], a[220:223], a[164:167], a[76:79]\n\t" \
  "s_cmp_eq_u32 s87, 56\n\t" \
  "s_cselect_b32 s87, 0, s87\n\t" \
  "v_mfma_f32_16x16x32_bf16 a[92:95], a[216:219], a[168:171], a[92:95]\n\t" \
  "v_mfma_f32_16x16x32_bf16 a[92:95], a[220:223], a[172:175], a[92:95]\n\t" \
  "buffer_load_dwordx4 v1, s[40:43], s84 offen offset:1024 sc0 sc1 lds\n\t" \
  "v_mfma_f32_16x16x32_bf16 a[88:91], a[208:211], a[168:171], a[88:91]\n\t" \
  "v_mfma_f32_16x16x32_bf16 a[88:91], a[212:215], a[172:175], a[88:91]\n\t" \
  "v_mfma_f32_16x16x32_bf16 a[84:87], a[200:203], a[168:171], a[84:87]\n\t" \
  "v_mfma_f32_16x16x32_bf16 a[84:87], a[204:207], a[172:175], a[84:87]\n\t" \
  "v_mfma_f32_16x16x32_bf16 a[80:83], a[192:195], a[168:171], a[80:83]\n\t" \
  "buffer_load_dwordx4 v2, s[40:43], s84 offen offset:2048 sc0 sc1 lds\n\t" \
  "v_mfma_f32_16x16x32_bf16 a[80:83], a[196:199], a[172:175], a[80:83]\n\t" \
  "v_mfma_f32_16x16x32_bf16 a[96:99], a[192:195], a[176:179], a[96:99]\n\t" \
  "v_mfma_f32_16x16x32_bf16 a[96:99], a[196:199], a[180:183], a[96:99]\n\t" \
  "v_mfma_f32_16x16x32_bf16 a[100:103], a[200:203], a[176:179], a[100:103]\n\t" \
  "v_mfma_f32_16x16x32_bf16 a[100:103], a[204:207], a[180:183], a[100:103]\n\t" \
  "buffer_load_dwordx4 v3, s[40:43], s84 offen offset:3072 sc0 sc1 lds\n\t" \
  "v_mfma_f32_16x16x32_bf16 a[104:107], a[208:211], a[176:179], a[104:107]\n\t" \
  "v_mfma_f32_16x16x32_bf16 a[104:107], a[212:215], a[180:183], a[104:107]\n\t" \
  "v_mfma_f32_16x16x32_bf16 a[108:111], a[216:219], a[176:179], a[108:111]\n\t" \
  "v_mfma_f32_16x16x32_bf16 a[108:111], a[220:223], a[180:183], a[108:111]\n\t" \
  "s_add_u32 m0, s25, 0\n\t" \
  "v_mfma_f32_16x16x32_bf16 a[124:127], a[216:219], a[184:187], a[124:127]\n\t" \
  "buffer_load_dwordx4 v4, s[40:43], s84 offen offset:0 sc0 sc1 lds\n\t" \
  "v_mfma_f32_16x16x32_bf16 a[124:127], a[220:223], a[188:191], a[124:127]\n\t" \
  "v_mfma_f32_16x16x32_bf16 a[120:123], a[208:211], a[184:187], a[120:123]\n\t" \
  "v_mfma_f32_16x16x32_bf16 a[120:123], a[212:215], a[188:191], a[120:123]\n\t" \
  "v_mfma_f32_16x16x32_bf16 a[116:119], a[200:203], a[184:187], a[116:119]\n\t" \
  "v_mfma_f32_16x16x32_bf16 a[116:119], a[204:207], a[188:191], a[116:119]\n\t" \
  "buffer_load_dwordx4 v5, s[40:43], s84 offen offset:1024 sc0 sc1 lds\n\t" \
  "v_mfma_f32_16x16x32_bf16 a[112:115], a[192:195], a[184:187], a[112:115]\n\t" \
  "v_mfma_f32_16x16x32_bf16 a[112:115], a[196:199], a[188:191], a[112:115]\n\t" \
  "s_waitcnt vmcnt(6)\n\t" \
  "s_barrier\n\t" \
  "v_mfma_f32_16x16x32_bf16 v[128:131], a[224:227], a[128:131], v[128:131]\n\t" \
  "ds_read_b64_tr_b16 a[192:193], v16 offset:32768\n\t" \
  "v_mfma_f32_16x16x32_bf16 v[128:131], a[228:231], a[132:135], v[128:131]\n\t" \
  "ds_read_b64_tr_b16 a[194:195], v16 offset:34816\n\t" \
  "v_mfma_f32_16x16x32_bf16 v[132:135], a[232:235], a[128:131], v[132:135]\n\t" \
  "buffer_load_dwordx4 v6, s[40:43], s84 offen offset:2048 sc0 sc1 lds\n\t" \
  "v_mfma_f32_16x16x32_bf16 v[132:135], a[236:239], a[132:135], v[132:135]\n\t" \
  "buffer_store_dwordx4 v[56:59], v73, s[52:55], s21 offen sc0 sc1\n\t" \
  "v_mfma_f32_16x16x32_bf16 v[136:139], a[240:243], a[128:131], v[136:139]\n\t" \
  "ds_read_b64_tr_b16 a[196:197], v16 offset:49152\n\t" \
  "v_mfma_f32_16x16x32_bf16 v[136:139], a[244:247], a[132:135], v[136:139]\n\t" \
  "ds_read_b64_tr_b16 a[198:199], v16 offset:51200\n\t" \
  "v_mfma_f32_16x16x32_bf16 v[140:143], a[248:251], a[128:131], v[140:143]\n\t" \
  "buffer_store_dword v46, v38, s[48:51], 0 offen sc1\n\t" \
  "s_waitcnt lgkmcnt(4)\n\t" \
  "v_mfma_f32_16x16x32_bf16 v[140:143], a[252:255], a[132:135], v[140:143]\n\t" \
  "buffer_load_dwordx4 v7, s[40:43], s84 offen offset:3072 sc0 sc1 lds\n\t" \
  "v_mfma_f32_16x16x32_bf16 v[156:159], a[248:251], a[136:139], v[156:159]\n\t" \
  "ds_read_b128 a[128:131], v24 offset:32768\n\t" \
  "v_mfma_f32_16x16x32_bf16 v[156:159], a[252:255], a[140:143], v[156:159]\n\t" \
  "ds_read_b128 a[132:135], v25 offset:32768\n\t" \
  "v_mfma_f32_16x16x32_bf16 v[152:155], a[240:243], a[136:139], v[152:155]\n\t" \
  "ds_read_b64_tr_b16 a[200:201], v17 offset:32768\n\t" \
  "v_mfma_f32_16x16x32_bf16 v[152:155], a[244:247], a[140:143], v[152:155]\n\t" \
  "ds_read_b64_tr_b16 a[202:203], v17 offset:34816\n\t" \
  "s_add_u32 m0, s26, 0\n\t" \
  "v_mfma_f32_16x16x32_bf16 v[148:151], a[232:235], a[136:139], v[148:151]\n\t" \
  "buffer_load_dwordx4 v8, s[40:43], s84 offen offset:0 sc0 sc1 lds\n\t" \
  "v_mfma_f32_16x16x32_bf16 v[148:151], a[236:239], a[140:143], v[148:151]\n\t" \
  "ds_read_b64_tr_b16 a[204:205], v17 offset:49152\n\t" \
  "v_mfma_f32_16x16x32_bf16 v[144:147], a[224:227], a[136:139], v[144:147]\n\t" \
  "ds_read_b64_tr_b16 a[206:207], v17 offset:51200\n\t" \
  "v_mfma_f32_16x16x32_bf16 v[144:147], a[228:231], a[140:143], v[144:147]\n\t" \
  "ds_read_b64_tr_b16 a[208:209], v18 offset:32768\n\t" \
  "v_mfma_f32_16x16x32_bf16 v[160:163], a[224:227], a[144:147], v[160:163]\n\t" \
  "ds_read_b128 a[136:139], v24 offset:34816\n\t" \
  "v_mfma_f32_16x16x32_bf16 v[160:163], a[228:231], a[148:151], v[160:163]\n\t" \
  "buffer_load_dwordx4 v9, s[40:43], s84 offen offset:1024 sc0 sc1 lds\n\t" \
  "v_mfma_f32_16x16x32_bf16 v[164:167], a[232:235], a[144:147], v[164:167]\n\t" \
  "ds_read_b128 a[140:143], v25 offset:34816\n\t" \
  "v_mfma_f32_16x16x32_bf16 v[164:167], a[236:239], a[148:151], v[164:167]\n\t" \
  "ds_read_b64_tr_b16 a[210:211], v18 offset:34816\n\t" \
  "v_mfma_f32_16x16x32_bf16 v[168:171], a[240:243], a[144:147], v[168:171]\n\t" \
  "ds_read_b64_tr_b16 a[212:213], v18 offset:49152\n\t" \
  "v_mfma_f32_16x16x32_bf16 v[168:171], a[244:247], a[148:151], v[168:171]\n\t" \
  "ds_read_b64_tr_b16 a[214:215], v18 offset:51200\n\t" \
  "v_mfma_f32_16x16x32_bf16 v[172:175], a[248:251], a[144:147], v[172:175]\n\t" \
  "buffer_load_dwordx4 v10, s[40:43], s84 offen offset:2048 sc0 sc1 lds\n\t" \
  "v_mfma_f32_16x16x32_bf16 v[172:175], a[252:255], a[148:151], v[172:175]\n\t" \
  "ds_read_b64_tr_b16 a[216:217], v19 offset:32768\n\t" \
  "v_mfma_f32_16x16x32_bf16 v[188:191], a[248:251], a[152:155], v[188:191]\n\t" \
  "ds_read_b128 a[144:147], v24 offset:36864\n\t" \
  "v_mfma_f32_16x16x32_bf16 v[188:191], a[252:255], a[156:159], v[188:191]\n\t" \
  "ds_read_b128 a[148:151], v25 offset:36864\n\t" \
  "v_mfma_f32_16x16x32_bf16 v[184:187], a[240:243], a[152:155], v[184:187]\n\t" \
  "ds_read_b64_tr_b16 a[218:219], v19 offset:34816\n\t" \
  "v_mfma_f32_16x16x32_bf16 v[184:187], a[244:247], a[156:159], v[184:187]\n\t" \
  "buffer_load_dwordx4 v11, s[40:43], s84 offen offset:3072 sc0 sc1 lds\n\t" \
  "v_mfma_f32_16x16x32_bf16 v[180:183], a[232:235], a[152:155], v[180:183]\n\t" \
  "ds_read_b64_tr_b16 a[220:221], v19 offset:49152\n\t" \
  "v_mfma_f32_16x16x32_bf16 v[180:183], a[236:239], a[156:159], v[180:183]\n\t" \
  "ds_read_b64_tr_b16 a[222:223], v19 offset:51200\n\t" \
  "v_mfma_f32_16x16x32_bf16 v[176:179], a[224:227], a[152:155], v[176:179]\n\t" \
  "ds_read_b64_tr_b16 v[96:97], v20 offset:32768\n\t" \
  "v_mfma_f32_16x16x32_bf16 v[176:179], a[228:231], a[156:159], v[176:179]\n\t" \
  "ds_read_b64_tr_b16 v[98:99], v20 offset:34816\n\t" \
  "s_add_u32 m0, s27, 0\n\t" \
  "v_mfma_f32_16x16x32_bf16 v[192:195], a[224:227], a[160:163], v[192:195]\n\t" \
  "buffer_load_dwordx4 v12, s[40:43], s84 offen offset:0 sc0 sc1 lds\n\t" \
  "v_mfma_f32_16x16x32_bf16 v[192:195], a[228:231], a[164:167], v[192:195]\n\t" \
  "ds_read_b128 a[152:155], v24 offset:38912\n\t" \
  "v_mfma_f32_16x16x32_bf16 v[196:199], a[232:235], a[160:163], v[196:199]\n\t" \
  "ds_read_b128 a[156:159], v25 offset:38912\n\t" \
  "v_mfma_f32_16x16x32_bf16 v[196:199], a[236:239], a[164:167], v[196:199]\n\t" \
  "ds_read_b64_tr_b16 v[100:101], v20 offset:49152\n\t" \
  "v_mfma_f32_16x16x32_bf16 v[200:203], a[240:243], a[160:163], v[200:203]\n\t" \
  "s_mov_b32 s90, s89\n\t" \
  "ds_read_b64_tr_b16 v[102:103], v20 offset:51200\n\t" \
  "v_mfma_f32_16x16x32_bf16 v[200:203], a[244:247], a[164:167], v[200:203]\n\t" \
  "buffer_load_dwordx4 v13, s[40:43], s84 offen offset:1024 sc0 sc1 lds\n\t" \
  "s_lshl_b32 s89, s73, 9\n\t" \
  "v_mfma_f32_16x16x32_bf16 v[204:207], a[248:251], a[160:163], v[204:207]\n\t" \
  "ds_read_b64_tr_b16 v[104:105], v21 offset:32768\n\t" \
  "v_mfma_f32_16x16x32_bf16 v[204:207], a[252:255], a[164:167], v[204:207]\n\t" \
  "ds_read_b64_tr_b16 v[106:107], v21 offset:34816\n\t" \
  "v_mfma_f32_16x16x32_bf16 v[220:223], a[248:251], a[168:171], v[220:223]\n\t" \
  "ds_read_b128 a[160:163], v24 offset:40960\n\t" \
  "v_mfma_f32_16x16x32_bf16 v[220:223], a[252:255], a[172:175], v[220:223]\n\t" \
  "ds_read_b128 a[164:167], v25 offset:40960\n\t" \
  "v_mfma_f32_16x16x32_bf16 v[216:219], a[240:243], a[168:171], v[216:219]\n\t" \
  "buffer_load_dwordx4 v14, s[40:43], s84 offen offset:2048 sc0 sc1 lds\n\t" \
  "v_mfma_f32_16x16x32_bf16 v[216:219], a[244:247], a[172:175], v[216:219]\n\t" \
  "ds_read_b64_tr_b16 v[108:109], v21 offset:49152\n\t" \
  "v_mfma_f32_16x16x32_bf16 v[212:215], a[232:235], a[168:171], v[212:215]\n\t" \
  "ds_read_b64_tr_b16 v[110:111], v21 offset:51200\n\t" \
  "v_mfma_f32_16x16x32_bf16 v[212:215], a[236:239], a[172:175], v[212:215]\n\t" \
  "ds_read_b64_tr_b16 v[112:113], v22 offset:32768\n\t" \
  "v_mfma_f32_16x16x32_bf16 v[208:211], a[224:227], a[168:171], v[208:211]\n\t" \
  "ds_read_b64_tr_b16 v[114:115], v22 offset:34816\n\t" \
  "v_mfma_f32_16x16x32_bf16 v[208:211], a[228:231], a[172:175], v[208:211]\n\t" \
  "buffer_load_dwordx4 v15, s[40:43], s84 offen offset:3072 sc0 sc1 lds\n\t" \
  "v_mfma_f32_16x16x32_bf16 v[224:227], a[224:227], a[176:179], v[224:227]\n\t" \
  "ds_read_b128 a[168:171], v24 offset:43008\n\t" \
  "v_mfma_f32_16x16x32_bf16 v[224:227], a[228:231], a[180:183], v[224:227]\n\t" \
  "ds_read_b128 a[172:175], v25 offset:43008\n\t" \
  "v_mfma_f32_16x16x32_bf16 v[228:231], a[232:235], a[176:179], v[228:231]\n\t" \
  "buffer_load_dword v92, v27, s[48:51], s89 offen sc0 sc1\n\t" \
  "v_mfma_f32_16x16x32_bf16 v[228:231], a[236:239], a[180:183], v[228:231]\n\t" \
  "ds_read_b64_tr_b16 v[116:117], v22 offset:49152\n\t" \
  "v_mfma_f32_16x16x32_bf16 v[232:235], a[240:243], a[176:179], v[232:235]\n\t" \
  "buffer_store_dwordx4 v[56:59], v74, s[52:55], s21 offen sc0 sc1\n\t" \
  "v_mfma_f32_16x16x32_bf16 v[232:235], a[244:247], a[180:183], v[232:235]\n\t" \
  "ds_read_b64_tr_b16 v[118:119], v22 offset:51200\n\t" \
  "v_mfma_f32_16x16x32_bf16 v[236:239], a[248:251], a[176:179], v[236:239]\n\t" \
  "ds_read_b64_tr_b16 v[120:121], v23 offset:32768\n\t" \
  "v_mfma_f32_16x16x32_bf16 v[236:239], a[252:255], a[180:183], v[236:239]\n\t" \
  "ds_read_b64_tr_b16 v[122:123], v23 offset:34816\n\t" \
  "v_mfma_f32_16x16x32_bf16 v[252:255], a[248:251], a[184:187], v[252:255]\n\t" \
  "ds_read_b128 a[176:179], v24 offset:45056\n\t" \
  "v_mfma_f32_16x16x32_bf16 v[252:255], a[252:255], a[188:191], v[252:255]\n\t" \
  "ds_read_b128 a[180:183], v25 offset:45056\n\t" \
  "v_mfma_f32_16x16x32_bf16 v[248:251], a[240:243], a[184:187], v[248:251]\n\t" \
  "ds_read_b64_tr_b16 v[124:125], v23 offset:49152\n\t" \
  "v_mfma_f32_16x16x32_bf16 v[248:251], a[244:247], a[188:191], v[248:251]\n\t" \
  "ds_read_b64_tr_b16 v[126:127], v23 offset:51200\n\t" \
  "v_mfma_f32_16x16x32_bf16 v[244:247], a[232:235], a[184:187], v[244:247]\n\t" \
  "v_mfma_f32_16x16x32_bf16 v[244:247], a[236:239], a[188:191], v[244:247]\n\t" \
  "v_mfma_f32_16x16x32_bf16 v[240:243], a[224:227], a[184:187], v[240:243]\n\t" \
  "v_mfma_f32_16x16x32_bf16 v[240:243], a[228:231], a[188:191], v[240:243]\n\t" \
  "s_waitcnt lgkmcnt(6)\n\t" \
  "v_mfma_f32_16x16x32_bf16 a[0:3], a[192:195], a[128:131], a[0:3]\n\t" \
  "ds_read_b128 a[184:187], v24 offset:47104\n\t" \
  "v_mfma_f32_16x16x32_bf16 a[0:3], a[196:199], a[132:135], a[0:3]\n\t" \
  "ds_read_b128 a[188:191], v25 offset:47104\n\t" \
  "v_cmp_gt_u32 vcc, s91, v93\n\t" \
  "v_mfma_f32_16x16x32_bf16 a[4:7], a[200:203], a[128:131], a[4:7]\n\t" \
  "v_mfma_f32_16x16x32_bf16 a[4:7], a[204:207], a[132:135], a[4:7]\n\t" \
  "s_cbranch_vccnz Lsp2_%=\n\t" \
  "Lspr2_%=:\n\t" \
  "v_mfma_f32_16x16x32_bf16 a[8:11], a[208:211], a[128:131], a[8:11]\n\t" \
  "s_mov_b32 s85, s86\n\t" \
  "s_mov_b32 s86, s73\n\t" \
  "v_mfma_f32_16x16x32_bf16 a[8:11], a[212:215], a[132:135], a[8:11]\n\t" \
  "s_lshl_b32 s84, s85, s74\n\t" \
  "s_add_u32 s84, s84, s75\n\t" \
  "v_mfma_f32_16x16x32_bf16 a[12:15], a[216:219], a[128:131], a[12:15]\n\t" \
  "s_waitcnt lgkmcnt(0)\n\t" \
  "s_barrier\n\t" \
  "s_add_u32 m0, s24, 32768\n\t" \
  "v_mfma_f32_16x16x32_bf16 a[12:15], a[220:223], a[132:135], a[12:15]\n\t" \
  "buffer_load_dwordx4 v0, s[40:43], s84 offen offset:0 sc0 sc1 lds\n\t" \
  "v_mfma_f32_16x16x32_bf16 a[28:31], a[216:219], a[136:139], a[28:31]\n\t" \
  "v_mfma_f32_16x16x32_bf16 a[28:31], a[220:223], a[140:143], a[28:31]\n\t" \
  "buffer_load_dwordx4 v1, s[40:43], s84 offen offset:1024 sc0 sc1 lds\n\t" \
  "v_mfma_f32_16x16x32_bf16 a[24:27], a[208:211], a[136:139], a[24:27]\n\t" \
  "v_mfma_f32_16x16x32_bf16 a[24:27], a[212:215], a[140:143], a[24:27]\n\t" \
  "buffer_load_dwordx4 v2, s[40:43], s84 offen offset:2048 sc0 sc1 lds\n\t" \
  "v_mfma_f32_16x16x32_bf16 a[20:23], a[200:203], a[136:139], a[20:23]\n\t" \
  "v_mfma_f32_16x16x32_bf16 a[20:23], a[204:207], a[140:143], a[20:23]\n\t" \
  "buffer_load_dwordx4 v3, s[40:43], s84 offen offset:3072 sc0 sc1 lds\n\t" \
  "v_mfma_f32_16x16x32_bf16 a[16:19], a[192:195], a[136:139], a[16:19]\n\t" \
  "v_mfma_f32_16x16x32_bf16 a[16:19], a[196:199], a[140:143], a[16:19]\n\t" \
  "v_mfma_f32_16x16x32_bf16 a[32:35], a[192:195], a[144:147], a[32:35]\n\t" \
  "v_mfma_f32_16x16x32_bf16 a[32:35], a[196:199], a[148:151], a[32:35]\n\t" \
  "v_mfma_f32_16x16x32_bf16 a[36:39], a[200:203], a[144:147], a[36:39]\n\t" \
  "v_mfma_f32_16x16x32_bf16 a[36:39], a[204:207], a[148:151], a[36:39]\n\t" \
  "v_mfma_f32_16x16x32_bf16 a[40:43], a[208:211], a[144:147], a[40:43]\n\t" \
  "v_mfma_f32_16x16x32_bf16 a[40:43], a[212:215], a[148:151], a[40:43]\n\t" \
  "v_mfma_f32_16x16x32_bf16 a[44:47], a[216:219], a[144:147], a[44:47]\n\t" \
  "v_mfma_f32_16x16x32_bf16 a[44:47], a[220:223], a[148:151], a[44:47]\n\t" \
  "v_mfma_f32_16x16x32_bf16 a[60:63], a[216:219], a[152:155], a[60:63]\n\t" \
  "v_mfma_f32_16x16x32_bf16 a[60:63], a[220:223], a[156:159], a[60:63]\n\t" \
  "v_mfma_f32_16x16x32_bf16 a[56:59], a[208:211], a[152:155], a[56:59]\n\t" \
  "v_mfma_f32_16x16x32_bf16 a[56:59], a[212:215], a[156:159], a[56:59]\n\t" \
  "v_mfma_f32_16x16x32_bf16 a[52:55], a[200:203], a[152:155], a[52:55]\n\t" \
  "v_mfma_f32_16x16x32_bf16 a[52:55], a[204:207], a[156:159], a[52:55]\n\t" \
  "v_mfma_f32_16x16x32_bf16 a[48:51], a[192:195], a[152:155], a[48:51]\n\t" \
  "v_mfma_f32_16x16x32_bf16 a[48:51], a[196:199], a[156:159], a[48:51]\n\t" \
  "s_lshr_b64 s[10:11], s[8:9], s87\n\t" \
  "s_sext_i32_i8 s10, s10\n\t" \
  "v_mfma_f32_16x16x32_bf16 a[64:67], a[192:195], a[160:163], a[64:67]\n\t" \
  "s_add_u32 s87, s87, 8\n\t" \
  "s_add_u32 s73, s73, s10\n\t" \
  "v_mfma_f32_16x16x32_bf16 a[64:67], a[196:199], a[164:167], a[64:67]\n\t" \
  "s_cmp_eq_u32 s87, 56\n\t" \
  "s_cselect_b32 s87, 0, s87\n\t" \
  "v_mfma_f32_16x16x32_bf16 a[68:71], a[200:203], a[160:163], a[68:71]\n\t" \
  "v_mfma_f32_16x16x32_bf16 a[68:71], a[204:207], a[164:167], a[68:71]\n\t" \
  "v_mfma_f32_16x16x32_bf16 a[72:75], a[208:211], a[160:163], a[72:75]\n\t" \
  "v_mfma_f32_16x16x32_bf16 a[72:75], a[212:215], a[164:167], a[72:75]\n\t" \
  "v_mfma_f32_16x16x32_bf16 a[76:79], a[216:219], a[160:163], a[76:79]\n\t" \
  "v_mfma_f32_16x16x32_bf16 a[76:79], a[220:223], a[164:167], a[76:79]\n\t" \
  "v_mfma_f32_16x16x32_bf16 a[92:95], a[216:219], a[168:171], a[92:95]\n\t" \
  "v_mfma_f32_16x16x32_bf16 a[92:95], a[220:223], a[172:175], a[92:95]\n\t" \
  "v_mfma_f32_16x16x32_bf16 a[88:91], a[208:211], a[168:171], a[88:91]\n\t" \
  "v_mfma_f32_16x16x32_bf16 a[88:91], a[212:215], a[172:175], a[88:91]\n\t" \
  "v_mfma_f32_16x16x32_bf16 a[84:87], a[200:203], a[168:171], a[84:87]\n\t" \
  "v_mfma_f32_16x16x32_bf16 a[84:87], a[204:207], a[172:175], a[84:87]\n\t" \
  "v_mfma_f32_16x16x32_bf16 a[80:83], a[192:195], a[168:171], a[80:83]\n\t" \
  "v_mfma_f32_16x16x32_bf16 a[80:83], a[196:199], a[172:175], a[80:83]\n\t" \
  "v_mfma_f32_16x16x32_bf16 a[96:99], a[192:195], a[176:179], a[96:99]\n\t" \
  "v_mfma_f32_16x16x32_bf16 a[96:99], a[196:199], a[180:183], a[96:99]\n\t" \
  "v_mfma_f32_16x16x32_bf16 a[100:103], a[200:203], a[176:179], a[100:103]\n\t" \
  "v_mfma_f32_16x16x32_bf16 a[100:103], a[204:207], a[180:183], a[100:103]\n\t" \
  "v_mfma_f32_16x16x32_bf16 a[104:107], a[208:211], a[176:179], a[104:107]\n\t" \
  "v_mfma_f32_16x16x32_bf16 a[104:107], a[212:215], a[180:183], a[104:107]\n\t" \
  "v_mfma_f32_16x16x32_bf16 a[108:111], a[216:219], a[176:179], a[108:111]\n\t" \
  "v_mfma_f32_16x16x32_bf16 a[108:111], a[220:223], a[180:183], a[108:111]\n\t" \
  "s_add_u32 m0, s25, 32768\n\t" \
  "v_mfma_f32_16x16x32_bf16 a[124:127], a[216:219], a[184:187], a[124:127]\n\t" \
  "buffer_load_dwordx4 v4, s[40:43], s84 offen offset:0 sc0 sc1 lds\n\t" \
  "v_mfma_f32_16x16x32_bf16 a[124:127], a[220:223], a[188:191], a[124:127]\n\t" \
  "v_mfma_f32_16x16x32_bf16 a[120:123], a[208:211], a[184:187], a[120:123]\n\t" \
  "v_mfma_f32_16x16x32_bf16 a[120:123], a[212:215], a[188:191], a[120:123]\n\t" \
  "v_mfma_f32_16x16x32_bf16 a[116:119], a[200:203], a[184:187], a[116:119]\n\t" \
  "v_mfma_f32_16x16x32_bf16 a[116:119], a[204:207], a[188:191], a[116:119]\n\t" \
  "buffer_load_dwordx4 v5, s[40:43], s84 offen offset:1024 sc0 sc1 lds\n\t" \
  "v_mfma_f32_16x16x32_bf16 a[112:115], a[192:195], a[184:187], a[112:115]\n\t" \
  "v_mfma_f32_16x16x32_bf16 a[112:115], a[196:199], a[188:191], a[112:115]\n\t" \
  "s_waitcnt vmcnt(7)\n\t" \
  "s_barrier\n\t" \
  "v_mfma_f32_16x16x32_bf16 v[128:131], v[96:99], a[128:131], v[128:131]\n\t" \
  "ds_read_b64_tr_b16 a[192:193], v16 offset:0\n\t" \
  "v_mfma_f32_16x16x32_bf16 v[128:131], v[100:103], a[132:135], v[128:131]\n\t" \
  "ds_read_b64_tr_b16 a[194:195], v16 offset:2048\n\t" \
  "v_mfma_f32_16x16x32_bf16 v[132:135], v[104:107], a[128:131], v[132:135]\n\t" \
  "buffer_load_dwordx4 v6, s[40:43], s84 offen offset:2048 sc0 sc1 lds\n\t" \
  "v_mfma_f32_16x16x32_bf16 v[132:135], v[108:111], a[132:135], v[132:135]\n\t" \
  "ds_read_b64_tr_b16 a[196:197], v16 offset:16384\n\t" \
  "v_mfma_f32_16x16x32_bf16 v[136:139], v[112:115], a[128:131], v[136:139]\n\t" \
  "ds_read_b64_tr_b16 a[198:199], v16 offset:18432\n\t" \
  "v_mfma_f32_16x16x32_bf16 v[136:139], v[116:119], a[132:135], v[136:139]\n\t" \
  "ds_read_b64_tr_b16 a[200:201], v17 offset:0\n\t" \
  "v_mfma_f32_16x16x32_bf16 v[140:143], v[120:123], a[128:131], v[140:143]\n\t" \
  "ds_read_b64_tr_b16 a[202:203], v17 offset:2048\n\t" \
  "v_mfma_f32_16x16x32_bf16 v[140:143], v[124:127], a[132:135], v[140:143]\n\t" \
  "buffer_load_dwordx4 v7, s[40:43], s84 offen offset:3072 sc0 sc1 lds\n\t" \
  "v_mfma_f32_16x16x32_bf16 v[156:159], v[120:123], a[136:139], v[156:159]\n\t" \
  "ds_read_b128 a[128:131], v24 offset:0\n\t" \
  "v_mfma_f32_16x16x32_bf16 v[156:159], v[124:127], a[140:143], v[156:159]\n\t" \
  "ds_read_b128 a[132:135], v25 offset:0\n\t" \
  "v_mfma_f32_16x16x32_bf16 v[152:155], v[112:115], a[136:139], v[152:155]\n\t" \
  "ds_read_b64_tr_b16 a[204:205], v17 offset:16384\n\t" \
  "v_mfma_f32_16x16x32_bf16 v[152:155], v[116:119], a[140:143], v[152:155]\n\t" \
  "ds_read_b64_tr_b16 a[206:207], v17 offset:18432\n\t" \
  "s_add_u32 m0, s26, 32768\n\t" \
  "v_mfma_f32_16x16x32_bf16 v[148:151], v[104:107], a[136:139], v[148:151]\n\t" \
  "buffer_load_dwordx4 v8, s[40:43], s84 offen offset:0 sc0 sc1 lds\n\t" \
  "v_mfma_f32_16x16x32_bf16 v[148:151], v[108:111], a[140:143], v[148:151]\n\t" \
  "ds_read_b64_tr_b16 a[208:209], v18 offset:0\n\t" \
  "v_mfma_f32_16x16x32_bf16 v[144:147], v[96:99], a[136:139], v[144:147]\n\t" \
  "ds_read_b64_tr_b16 a[210:211], v18 offset:2048\n\t" \
  "v_mfma_f32_16x16x32_bf16 v[144:147], v[100:103], a[140:143], v[144:147]\n\t" \
  "ds_read_b64_tr_b16 a[212:213], v18 offset:16384\n\t" \
  "v_mfma_f32_16x16x32_bf16 v[160:163], v[96:99], a[144:147], v[160:163]\n\t" \
  "ds_read_b128 a[136:139], v24 offset:2048\n\t" \
  "v_mfma_f32_16x16x32_bf16 v[160:163], v[100:103], a[148:151], v[160:163]\n\t" \
  "buffer_load_dwordx4 v9, s[40:43], s84 offen offset:1024 sc0 sc1 lds\n\t" \
  "v_mfma_f32_16x16x32_bf16 v[164:167], v[104:107], a[144:147], v[164:167]\n\t" \
  "ds_read_b128 a[140:143], v25 offset:2048\n\t" \
  "v_mfma_f32_16x16x32_bf16 v[164:167], v[108:111], a[148:151], v[164:167]\n\t" \
  "ds_read_b64_tr_b16 a[214:215], v18 offset:18432\n\t" \
  "v_mfma_f32_16x16x32_bf16 v[168:171], v[112:115], a[144:147], v[168:171]\n\t" \
  "ds_read_b64_tr_b16 a[216:217], v19 offset:0\n\t" \
  "v_mfma_f32_16x16x32_bf16 v[168:171], v[116:119], a[148:151], v[168:171]\n\t" \
  "ds_read_b64_tr_b16 a[218:219], v19 offset:2048\n\t" \
  "v_mfma_f32_16x16x32_bf16 v[172:175], v[120:123], a[144:147], v[172:175]\n\t" \
  "buffer_load_dwordx4 v10, s[40:43], s84 offen offset:2048 sc0 sc1 lds\n\t" \
  "v_mfma_f32_16x16x32_bf16 v[172:175], v[124:127], a[148:151], v[172:175]\n\t" \
  "ds_read_b64_tr_b16 a[220:221], v19 offset:16384\n\t" \
  "v_mfma_f32_16x16x32_bf16 v[188:191], v[120:123], a[152:155], v[188:191]\n\t" \
  "ds_read_b128 a[144:147], v24 offset:4096\n\t" \
  "v_mfma_f32_16x16x32_bf16 v[188:191], v[124:127], a[156:159], v[188:191]\n\t" \
  "ds_read_b128 a[148:151], v25 offset:4096\n\t" \
  "v_mfma_f32_16x16x32_bf16 v[184:187], v[112:115], a[152:155], v[184:187]\n\t" \
  "ds_read_b64_tr_b16 a[222:223], v19 offset:18432\n\t" \
  "v_mfma_f32_16x16x32_bf16 v[184:187], v[116:119], a[156:159], v[184:187]\n\t" \
  "buffer_load_dwordx4 v11, s[40:43], s84 offen offset:3072 sc0 sc1 lds\n\t" \
  "v_mfma_f32_16x16x32_bf16 v[180:183], v[104:107], a[152:155], v[180:183]\n\t" \
  "ds_read_b64_tr_b16 a[224:225], v20 offset:0\n\t" \
  "v_mfma_f32_16x16x32_bf16 v[180:183], v[108:111], a[156:159], v[180:183]\n\t" \
  "ds_read_b64_tr_b16 a[226:227], v20 offset:2048\n\t" \
  "v_mfma_f32_16x16x32_bf16 v[176:179], v[96:99], a[152:155], v[176:179]\n\t" \
  "ds_read_b64_tr_b16 a[228:229], v20 offset:16384\n\t" \
  "v_mfma_f32_16x16x32_bf16 v[176:179], v[100:103], a[156:159], v[176:179]\n\t" \
  "ds_read_b64_tr_b16 a[230:231], v20 offset:18432\n\t" \
  "s_add_u32 m0, s27, 32768\n\t" \
  "v_mfma_f32_16x16x32_bf16 v[192:195], v[96:99], a[160:163], v[192:195]\n\t" \
  "buffer_load_dwordx4 v12, s[40:43], s84 offen offset:0 sc0 sc1 lds\n\t" \
  "v_mfma_f32_16x16x32_bf16 v[192:195], v[100:103], a[164:167], v[192:195]\n\t" \
  "ds_read_b128 a[152:155], v24 offset:6144\n\t" \
  "v_mfma_f32_16x16x32_bf16 v[196:199], v[104:107], a[160:163], v[196:199]\n\t" \
  "ds_read_b128 a[156:159], v25 offset:6144\n\t" \
  "v_mfma_f32_16x16x32_bf16 v[196:199], v[108:111], a[164:167], v[196:199]\n\t" \
  "ds_read_b64_tr_b16 a[232:233], v21 offset:0\n\t" \
  "v_mfma_f32_16x16x32_bf16 v[200:203], v[112:115], a[160:163], v[200:203]\n\t" \
  "s_mov_b32 s90, s89\n\t" \
  "ds_read_b64_tr_b16 a[234:235], v21 offset:2048\n\t" \
  "v_mfma_f32_16x16x32_bf16 v[200:203], v[116:119], a[164:167], v[200:203]\n\t" \
  "buffer_load_dwordx4 v13, s[40:43], s84 offen offset:1024 sc0 sc1 lds\n\t" \
  "s_lshl_b32 s89, s73, 9\n\t" \
  "v_mfma_f32_16x16x32_bf16 v[204:207], v[120:123], a[160:163], v[204:207]\n\t" \
  "s_lshr_b32 s20, s21, 8\n\t" \
  "s_lshl_b32 s21, s80, 10\n\t" \
  "ds_read_b64_tr_b16 a[236:237], v21 offset:16384\n\t" \
  "v_mfma_f32_16x16x32_bf16 v[204:207], v[124:127], a[164:167], v[204:207]\n\t" \
  "s_add_u32 s11, s80, s82\n\t" \
  "s_mov_b32 s80, s81\n\t" \
  "ds_read_b64_tr_b16 a[238:239], v21 offset:18432\n\t" \
  "v_mfma_f32_16x16x32_bf16 v[220:223], v[120:123], a[168:171], v[220:223]\n\t" \
  "ds_read_b128 a[160:163], v24 offset:8192\n\t" \
  "s_mov_b32 s81, s11\n\t" \
  "s_sub_u32 s82, 0, s82\n\t" \
  "v_mfma_f32_16x16x32_bf16 v[220:223], v[124:127], a[172:175], v[220:223]\n\t" \
  "ds_read_b128 a[164:167], v25 offset:8192\n\t" \
  "v_mfma_f32_16x16x32_bf16 v[216:219], v[112:115], a[168:171], v[216:219]\n\t" \
  "buffer_load_dwordx4 v14, s[40:43], s84 offen offset:2048 sc0 sc1 lds\n\t" \
  "v_mfma_f32_16x16x32_bf16 v[216:219], v[116:119], a[172:175], v[216:219]\n\t" \
  "ds_read_b64_tr_b16 a[240:241], v22 offset:0\n\t" \
  "v_mfma_f32_16x16x32_bf16 v[212:215], v[104:107], a[168:171], v[212:215]\n\t" \
  "ds_read_b64_tr_b16 a[242:243], v22 offset:2048\n\t" \
  "v_mfma_f32_16x16x32_bf16 v[212:215], v[108:111], a[172:175], v[212:215]\n\t" \
  "ds_read_b64_tr_b16 a[244:245], v22 offset:16384\n\t" \
  "v_mfma_f32_16x16x32_bf16 v[208:211], v[96:99], a[168:171], v[208:211]\n\t" \
  "ds_read_b64_tr_b16 a[246:247], v22 offset:18432\n\t" \
  "v_mfma_f32_16x16x32_bf16 v[208:211], v[100:103], a[172:175], v[208:211]\n\t" \
  "buffer_load_dwordx4 v15, s[40:43], s84 offen offset:3072 sc0 sc1 lds\n\t" \
  "v_mfma_f32_16x16x32_bf16 v[224:227], v[96:99], a[176:179], v[224:227]\n\t" \
  "ds_read_b128 a[168:171], v24 offset:10240\n\t" \
  "v_mfma_f32_16x16x32_bf16 v[224:227], v[100:103], a[180:183], v[224:227]\n\t" \
  "ds_read_b128 a[172:175], v25 offset:10240\n\t" \
  "v_mfma_f32_16x16x32_bf16 v[228:231], v[104:107], a[176:179], v[228:231]\n\t" \
  "buffer_load_dword v93, v27, s[48:51], s89 offen sc0 sc1\n\t" \
  "v_mfma_f32_16x16x32_bf16 v[228:231], v[108:111], a[180:183], v[228:231]\n\t" \
  "ds_read_b64_tr_b16 a[248:249], v23 offset:0\n\t" \
  "v_mfma_f32_16x16x32_bf16 v[232:235], v[112:115], a[176:179], v[232:235]\n\t" \
  "buffer_store_dwordx4 v[60:63], v72, s[52:55], s21 offen sc0 sc1\n\t" \
  "v_mfma_f32_16x16x32_bf16 v[232:235], v[116:119], a[180:183], v[232:235]\n\t" \
  "ds_read_b64_tr_b16 a[250:251], v23 offset:2048\n\t" \
  "v_mfma_f32_16x16x32_bf16 v[236:239], v[120:123], a[176:179], v[236:239]\n\t" \
  "ds_read_b64_tr_b16 a[252:253], v23 offset:16384\n\t" \
  "v_mfma_f32_16x16x32_bf16 v[236:239], v[124:127], a[180:183], v[236:239]\n\t" \
  "buffer_load_dwordx4 v[56:59], v29, s[56:59], s80 offen\n\t" \
  "v_mfma_f32_16x16x32_bf16 v[252:255], v[120:123], a[184:187], v[252:255]\n\t" \
  "ds_read_b128 a[176:179], v24 offset:12288\n\t" \
  "v_mfma_f32_16x16x32_bf16 v[252:255], v[124:127], a[188:191], v[252:255]\n\t" \
  "ds_read_b128 a[180:183], v25 offset:12288\n\t" \
  "v_mfma_f32_16x16x32_bf16 v[248:251], v[112:115], a[184:187], v[248:251]\n\t" \
  "ds_read_b64_tr_b16 a[254:255], v23 offset:18432\n\t" \
  "v_mfma_f32_16x16x32_bf16 v[248:251], v[116:119], a[188:191], v[248:251]\n\t" \
  "v_mfma_f32_16x16x32_bf16 v[244:247], v[104:107], a[184:187], v[244:247]\n\t" \
  "v_mfma_f32_16x16x32_bf16 v[244:247], v[108:111], a[188:191], v[244:247]\n\t" \
  "v_mfma_f32_16x16x32_bf16 v[240:243], v[96:99], a[184:187], v[240:243]\n\t" \
  "v_mfma_f32_16x16x32_bf16 v[240:243], v[100:103], a[188:191], v[240:243]\n\t" \
  "s_waitcnt lgkmcnt(4)\n\t" \
  "v_mfma_f32_16x16x32_bf16 a[0:3], a[192:195], a[128:131], a[0:3]\n\t" \
  "ds_read_b128 a[184:187], v24 offset:14336\n\t" \
  "v_mfma_f32_16x16x32_bf16 a[0:3], a[196:199], a[132:135], a[0:3]\n\t" \
  "ds_read_b128 a[188:191], v25 offset:14336\n\t" \
  "v_cmp_gt_u32 vcc, s91, v92\n\t" \
  "v_mfma_f32_16x16x32_bf16 a[4:7], a[200:203], a[128:131], a[4:7]\n\t" \
  "v_mfma_f32_16x16x32_bf16 a[4:7], a[204:207], a[132:135], a[4:7]\n\t" \
  "s_cbranch_vccnz Lsp3_%=\n\t" \
  "Lspr3_%=:\n\t" \
  "v_mfma_f32_16x16x32_bf16 a[8:11], a[208:211], a[128:131], a[8:11]\n\t" \
  "s_mov_b32 s85, s86\n\t" \
  "s_mov_b32 s86, s73\n\t" \
  "v_mfma_f32_16x16x32_bf16 a[8:11], a[212:215], a[132:135], a[8:11]\n\t" \
  "s_lshl_b32 s84, s85, s74\n\t" \
  "s_add_u32 s84, s84, s75\n\t" \
  "v_mfma_f32_16x16x32_bf16 a[12:15], a[216:219], a[128:131], a[12:15]\n\t" \
  "s_waitcnt lgkmcnt(0)\n\t" \
  "s_barrier\n\t" \
  "s_add_u32 m0, s24, 0\n\t" \
  "v_mfma_f32_16x16x32_bf16 a[12:15], a[220:223], a[132:135], a[12:15]\n\t" \
  "buffer_load_dwordx4 v0, s[40:43], s84 offen offset:0 sc0 sc1 lds\n\t" \
  "v_mfma_f32_16x16x32_bf16 a[28:31], a[216:219], a[136:139], a[28:31]\n\t" \
  "v_mfma_f32_16x16x32_bf16 a[28:31], a[220:223], a[140:143], a[28:31]\n\t" \
  "buffer_load_dwordx4 v1, s[40:43], s84 offen offset:1024 sc0 sc1 lds\n\t" \
  "v_mfma_f32_16x16x32_bf16 a[24:27], a[208:211], a[136:139], a[24:27]\n\t" \
  "v_mfma_f32_16x16x32_bf16 a[24:27], a[212:215], a[140:143], a[24:27]\n\t" \
  "buffer_load_dwordx4 v2, s[40:43], s84 offen offset:2048 sc0 sc1 lds\n\t" \
  "v_mfma_f32_16x16x32_bf16 a[20:23], a[200:203], a[136:139], a[20:23]\n\t" \
  "v_mfma_f32_16x16x32_bf16 a[20:23], a[204:207], a[140:143], a[20:23]\n\t" \
  "buffer_load_dwordx4 v3, s[40:43], s84 offen offset:3072 sc0 sc1 lds\n\t" \
  "v_mfma_f32_16x16x32_bf16 a[16:19], a[192:195], a[136:139], a[16:19]\n\t" \
  "v_mfma_f32_16x16x32_bf16 a[16:19], a[196:199], a[140:143], a[16:19]\n\t" \
  "v_mfma_f32_16x16x32_bf16 a[32:35], a[192:195], a[144:147], a[32:35]\n\t" \
  "v_mfma_f32_16x16x32_bf16 a[32:35], a[196:199], a[148:151], a[32:35]\n\t" \
  "v_mfma_f32_16x16x32_bf16 a[36:39], a[200:203], a[144:147], a[36:39]\n\t" \
  "v_mfma_f32_16x16x32_bf16 a[36:39], a[204:207], a[148:151], a[36:39]\n\t" \
  "v_mfma_f32_16x16x32_bf16 a[40:43], a[208:211], a[144:147], a[40:43]\n\t" \
  "v_mfma_f32_16x16x32_bf16 a[40:43], a[212:215], a[148:151], a[40:43]\n\t" \
  "v_mfma_f32_16x16x32_bf16 a[44:47], a[216:219], a[144:147], a[44:47]\n\t" \
  "v_mfma_f32_16x16x32_bf16 a[44:47], a[220:223], a[148:151], a[44:47]\n\t" \
  "v_mfma_f32_16x16x32_bf16 a[60:63], a[216:219], a[152:155], a[60:63]\n\t" \
  "v_mfma_f32_16x16x32_bf16 a[60:63], a[220:223], a[156:159], a[60:63]\n\t" \
  "v_mfma_f32_16x16x32_bf16 a[56:59], a[208:211], a[152:155], a[56:59]\n\t" \
  "v_mfma_f32_16x16x32_bf16 a[56:59], a[212:215], a[156:159], a[56:59]\n\t" \
  "v_mfma_f32_16x16x32_bf16 a[52:55], a[200:203], a[152:155], a[52:55]\n\t" \
  "v_mfma_f32_16x16x32_bf16 a[52:55], a[204:207], a[156:159], a[52:55]\n\t" \
  "v_mfma_f32_16x16x32_bf16 a[48:51], a[192:195], a[152:155], a[48:51]\n\t" \
  "v_mfma_f32_16x16x32_bf16 a[48:51], a[196:199], a[156:159], a[48:51]\n\t" \
  "s_lshr_b64 s[10:11], s[8:9], s87\n\t" \
  "s_sext_i32_i8 s10, s10\n\t" \
  "v_mfma_f32_16x16x32_bf16 a[64:67], a[192:195], a[160:163], a[64:67]\n\t" \
  "s_add_u32 s87, s87, 8\n\t" \
  "s_add_u32 s73, s73, s10\n\t" \
  "v_mfma_f32_16x16x32_bf16 a[64:67], a[196:199], a[164:167], a[64:67]\n\t" \
  "s_cmp_eq_u32 s87, 56\n\t" \
  "s_cselect_b32 s87, 0, s87\n\t" \
  "v_mfma_f32_16x16x32_bf16 a[68:71], a[200:203], a[160:163], a[68:71]\n\t" \
  "v_mfma_f32_16x16x32_bf16 a[68:71], a[204:207], a[164:167], a[68:71]\n\t" \
  "v_mfma_f32_16x16x32_bf16 a[72:75], a[208:211], a[160:163], a[72:75]\n\t" \
  "v_mfma_f32_16x16x32_bf16 a[72:75], a[212:215], a[164:167], a[72:75]\n\t" \
  "v_mfma_f32_16x16x32_bf16 a[76:79], a[216:219], a[160:163], a[76:79]\n\t" \
  "v_mfma_f32_16x16x32_bf16 a[76:79], a[220:223], a[164:167], a[76:79]\n\t" \
  "v_mfma_f32_16x16x32_bf16 a[92:95], a[216:219], a[168:171], a[92:95]\n\t" \
  "v_mfma_f32_16x16x32_bf16 a[92:95], a[220:223], a[172:175], a[92:95]\n\t" \
  "v_mfma_f32_16x16x32_bf16 a[88:91], a[208:211], a[168:171], a[88:91]\n\t" \
  "v_mfma_f32_16x16x32_bf16 a[88:91], a[212:215], a[172:175], a[88:91]\n\t" \
  "v_mfma_f32_16x16x32_bf16 a[84:87], a[200:203], a[168:171], a[84:87]\n\t" \
  "v_mfma_f32_16x16x32_bf16 a[84:87], a[204:207], a[172:175], a[84:87]\n\t" \
  "v_mfma_f32_16x16x32_bf16 a[80:83], a[192:195], a[168:171], a[80:83]\n\t" \
  "v_mfma_f32_16x16x32_bf16 a[80:83], a[196:199], a[172:175], a[80:83]\n\t" \
  "v_mfma_f32_16x16x32_bf16 a[96:99], a[192:195], a[176:179], a[96:99]\n\t" \
  "v_mfma_f32_16x16x32_bf16 a[96:99], a[196:199], a[180:183], a[96:99]\n\t" \
  "v_mfma_f32_16x16x32_bf16 a[100:103], a[200:203], a[176:179], a[100:103]\n\t" \
  "v_mfma_f32_16x16x32_bf16 a[100:103], a[204:207], a[180:183], a[100:103]\n\t" \
  "v_mfma_f32_16x16x32_bf16 a[104:107], a[208:211], a[176:179], a[104:107]\n\t" \
  "v_mfma_f32_16x16x32_bf16 a[104:107], a[212:215], a[180:183], a[104:107]\n\t" \
  "v_mfma_f32_16x16x32_bf16 a[108:111], a[216:219], a[176:179], a[108:111]\n\t" \
  "v_mfma_f32_16x16x32_bf16 a[108:111], a[220:223], a[180:183], a[108:111]\n\t" \
  "s_add_u32 m0, s25, 0\n\t" \
  "v_mfma_f32_16x16x32_bf16 a[124:127], a[216:219], a[184:187], a[124:127]\n\t" \
  "buffer_load_dwordx4 v4, s[40:43], s84 offen offset:0 sc0 sc1 lds\n\t" \
  "v_mfma_f32_16x16x32_bf16 a[124:127], a[220:223], a[188:191], a[124:127]\n\t" \
  "v_mfma_f32_16x16x32_bf16 a[120:123], a[208:211], a[184:187], a[120:123]\n\t" \
  "v_mfma_f32_16x16x32_bf16 a[120:123], a[212:215], a[188:191], a[120:123]\n\t" \
  "v_mfma_f32_16x16x32_bf16 a[116:119], a[200:203], a[184:187], a[116:119]\n\t" \
  "v_mfma_f32_16x16x32_bf16 a[116:119], a[204:207], a[188:191], a[116:119]\n\t" \
  "buffer_load_dwordx4 v5, s[40:43], s84 offen offset:1024 sc0 sc1 lds\n\t" \
  "v_mfma_f32_16x16x32_bf16 a[112:115], a[192:195], a[184:187], a[112:115]\n\t" \
  "v_mfma_f32_16x16x32_bf16 a[112:115], a[196:199], a[188:191], a[112:115]\n\t" \
  "s_waitcnt vmcnt(8)\n\t" \
  "s_barrier\n\t" \
  "v_mfma_f32_16x16x32_bf16 v[128:131], a[224:227], a[128:131], v[128:131]\n\t" \
  "buffer_store_dword v43, v31, s[64:67], s20 offen sc0 sc1\n\t" \
  "v_mfma_f32_16x16x32_bf16 v[128:131], a[228:231], a[132:135], v[128:131]\n\t" \
  "ds_read_b64_tr_b16 a[192:193], v16 offset:32768\n\t" \
  "v_mfma_f32_16x16x32_bf16 v[132:135], a[232:235], a[128:131], v[132:135]\n\t" \
  "buffer_load_dwordx4 v6, s[40:43], s84 offen offset:2048 sc0 sc1 lds\n\t" \
  "v_mfma_f32_16x16x32_bf16 v[132:135], a[236:239], a[132:135], v[132:135]\n\t" \
  "buffer_store_dwordx4 v[60:63], v73, s[52:55], s21 offen sc0 sc1\n\t" \
  "v_mfma_f32_16x16x32_bf16 v[136:139], a[240:243], a[128:131], v[136:139]\n\t" \
  "ds_read_b64_tr_b16 a[194:195], v16 offset:34816\n\t" \
  "v_mfma_f32_16x16x32_bf16 v[136:139], a[244:247], a[132:135], v[136:139]\n\t" \
  "ds_read_b64_tr_b16 a[196:197], v16 offset:49152\n\t" \
  "v_mfma_f32_16x16x32_bf16 v[140:143], a[248:251], a[128:131], v[140:143]\n\t" \
  "ds_read_b64_tr_b16 a[198:199], v16 offset:51200\n\t" \
  "v_mfma_f32_16x16x32_bf16 v[140:143], a[252:255], a[132:135], v[140:143]\n\t" \
  "buffer_load_dwordx4 v7, s[40:43], s84 offen offset:3072 sc0 sc1 lds\n\t" \
  "v_mfma_f32_16x16x32_bf16 v[156:159], a[248:251], a[136:139], v[156:159]\n\t" \
  "ds_read_b128 a[128:131], v24 offset:32768\n\t" \
  "v_mfma_f32_16x16x32_bf16 v[156:159], a[252:255], a[140:143], v[156:159]\n\t" \
  "ds_read_b128 a[132:135], v25 offset:32768\n\t" \
  "v_mfma_f32_16x16x32_bf16 v[152:155], a[240:243], a[136:139], v[152:155]\n\t" \
  "ds_read_b64_tr_b16 a[200:201], v17 offset:32768\n\t" \
  "v_mfma_f32_16x16x32_bf16 v[152:155], a[244:247], a[140:143], v[152:155]\n\t" \
  "ds_read_b64_tr_b16 a[202:203], v17 offset:34816\n\t" \
  "s_add_u32 m0, s26, 0\n\t" \
  "v_mfma_f32_16x16x32_bf16 v[148:151], a[232:235], a[136:139], v[148:151]\n\t" \
  "buffer_load_dwordx4 v8, s[40:43], s84 offen offset:0 sc0 sc1 lds\n\t" \
  "v_mfma_f32_16x16x32_bf16 v[148:151], a[236:239], a[140:143], v[148:151]\n\t" \
  "ds_read_b64_tr_b16 a[204:205], v17 offset:49152\n\t" \
  "v_mfma_f32_16x16x32_bf16 v[144:147], a[224:227], a[136:139], v[144:147]\n\t" \
  "ds_read_b64_tr_b16 a[206:207], v17 offset:51200\n\t" \
  "v_mfma_f32_16x16x32_bf16 v[144:147], a[228:231], a[140:143], v[144:147]\n\t" \
  "ds_read_b64_tr_b16 a[208:209], v18 offset:32768\n\t" \
  "v_mfma_f32_16x16x32_bf16 v[160:163], a[224:227], a[144:147], v[160:163]\n\t" \
  "ds_read_b128 a[136:139], v24 offset:34816\n\t" \
  "v_mfma_f32_16x16x32_bf16 v[160:163], a[228:231], a[148:151], v[160:163]\n\t" \
  "buffer_load_dwordx4 v9, s[40:43], s84 offen offset:1024 sc0 sc1 lds\n\t" \
  "v_mfma_f32_16x16x32_bf16 v[164:167], a[232:235], a[144:147], v[164:167]\n\t" \
  "ds_read_b128 a[140:143], v25 offset:34816\n\t" \
  "v_mfma_f32_16x16x32_bf16 v[164:167], a[236:239], a[148:151], v[164:167]\n\t" \
  "ds_read_b64_tr_b16 a[210:211], v18 offset:34816\n\t" \
  "v_mfma_f32_16x16x32_bf16 v[168:171], a[240:243], a[144:147], v[168:171]\n\t" \
  "ds_read_b64_tr_b16 a[212:213], v18 offset:49152\n\t" \
  "v_mfma_f32_16x16x32_bf16 v[168:171], a[244:247], a[148:151], v[168:171]\n\t" \
  "ds_read_b64_tr_b16 a[214:215], v18 offset:51200\n\t" \
  "v_mfma_f32_16x16x32_bf16 v[172:175], a[248:251], a[144:147], v[172:175]\n\t" \
  "buffer_load_dwordx4 v10, s[40:43], s84 offen offset:2048 sc0 sc1 lds\n\t" \
  "v_mfma_f32_16x16x32_bf16 v[172:175], a[252:255], a[148:151], v[172:175]\n\t" \
  "ds_read_b64_tr_b16 a[216:217], v19 offset:32768\n\t" \
  "v_mfma_f32_16x16x32_bf16 v[188:191], a[248:251], a[152:155], v[188:191]\n\t" \
  "ds_read_b128 a[144:147], v24 offset:36864\n\t" \
  "v_mfma_f32_16x16x32_bf16 v[188:191], a[252:255], a[156:159], v[188:191]\n\t" \
  "ds_read_b128 a[148:151], v25 offset:36864\n\t" \
  "v_mfma_f32_16x16x32_bf16 v[184:187], a[240:243], a[152:155], v[184:187]\n\t" \
  "ds_read_b64_tr_b16 a[218:219], v19 offset:34816\n\t" \
  "v_mfma_f32_16x16x32_bf16 v[184:187], a[244:247], a[156:159], v[184:187]\n\t" \
  "buffer_load_dwordx4 v11, s[40:43], s84 offen offset:3072 sc0 sc1 lds\n\t" \
  "v_mfma_f32_16x16x32_bf16 v[180:183], a[232:235], a[152:155], v[180:183]\n\t" \
  "ds_read_b64_tr_b16 a[220:221], v19 offset:49152\n\t" \
  "v_mfma_f32_16x16x32_bf16 v[180:183], a[236:239], a[156:159], v[180:183]\n\t" \
  "ds_read_b64_tr_b16 a[222:223], v19 offset:51200\n\t" \
  "v_mfma_f32_16x16x32_bf16 v[176:179], a[224:227], a[152:155], v[176:179]\n\t" \
  "ds_read_b64_tr_b16 v[96:97], v20 offset:32768\n\t" \
  "v_mfma_f32_16x16x32_bf16 v[176:179], a[228:231], a[156:159], v[176:179]\n\t" \
  "ds_read_b64_tr_b16 v[98:99], v20 offset:34816\n\t" \
  "s_add_u32 m0, s27, 0\n\t" \
  "v_mfma_f32_16x16x32_bf16 v[192:195], a[224:227], a[160:163], v[192:195]\n\t" \
  "buffer_load_dwordx4 v12, s[40:43], s84 offen offset:0 sc0 sc1 lds\n\t" \
  "v_mfma_f32_16x16x32_bf16 v[192:195], a[228:231], a[164:167], v[192:195]\n\t" \
  "ds_read_b128 a[152:155], v24 offset:38912\n\t" \
  "v_mfma_f32_16x16x32_bf16 v[196:199], a[232:235], a[160:163], v[196:199]\n\t" \
  "ds_read_b128 a[156:159], v25 offset:38912\n\t" \
  "v_mfma_f32_16x16x32_bf16 v[196:199], a[236:239], a[164:167], v[196:199]\n\t" \
  "ds_read_b64_tr_b16 v[100:101], v20 offset:49152\n\t" \
  "v_mfma_f32_16x16x32_bf16 v[200:203], a[240:243], a[160:163], v[200:203]\n\t" \
  "s_mov_b32 s90, s89\n\t" \
  "ds_read_b64_tr_b16 v[102:103], v20 offset:51200\n\t" \
  "v_mfma_f32_16x16x32_bf16 v[200:203], a[244:247], a[164:167], v[200:203]\n\t" \
  "buffer_load_dwordx4 v13, s[40:43], s84 offen offset:1024 sc0 sc1 lds\n\t" \
  "s_lshl_b32 s89, s73, 9\n\t" \
  "v_mfma_f32_16x16x32_bf16 v[204:207], a[248:251], a[160:163], v[204:207]\n\t" \
  "ds_read_b64_tr_b16 v[104:105], v21 offset:32768\n\t" \
  "v_mfma_f32_16x16x32_bf16 v[204:207], a[252:255], a[164:167], v[204:207]\n\t" \
  "ds_read_b64_tr_b16 v[106:107], v21 offset:34816\n\t" \
  "v_mfma_f32_16x16x32_bf16 v[220:223], a[248:251], a[168:171], v[220:223]\n\t" \
  "ds_read_b128 a[160:163], v24 offset:40960\n\t" \
  "v_mfma_f32_16x16x32_bf16 v[220:223], a[252:255], a[172:175], v[220:223]\n\t" \
  "ds_read_b128 a[164:167], v25 offset:40960\n\t" \
  "v_mfma_f32_16x16x32_bf16 v[216:219], a[240:243], a[168:171], v[216:219]\n\t" \
  "buffer_load_dwordx4 v14, s[40:43], s84 offen offset:2048 sc0 sc1 lds\n\t" \
  "v_mfma_f32_16x16x32_bf16 v[216:219], a[244:247], a[172:175], v[216:219]\n\t" \
  "ds_read_b64_tr_b16 v[108:109], v21 offset:49152\n\t" \
  "v_mfma_f32_16x16x32_bf16 v[212:215], a[232:235], a[168:171], v[212:215]\n\t" \
  "ds_read_b64_tr_b16 v[110:111], v21 offset:51200\n\t" \
  "v_mfma_f32_16x16x32_bf16 v[212:215], a[236:239], a[172:175], v[212:215]\n\t" \
  "ds_read_b64_tr_b16 v[112:113], v22 offset:32768\n\t" \
  "v_mfma_f32_16x16x32_bf16 v[208:211], a[224:227], a[168:171], v[208:211]\n\t" \
  "ds_read_b64_tr_b16 v[114:115], v22 offset:34816\n\t" \
  "v_mfma_f32_16x16x32_bf16 v[208:211], a[228:231], a[172:175], v[208:211]\n\t" \
  "buffer_load_dwordx4 v15, s[40:43], s84 offen offset:3072 sc0 sc1 lds\n\t" \
  "v_mfma_f32_16x16x32_bf16 v[224:227], a[224:227], a[176:179], v[224:227]\n\t" \
  "ds_read_b128 a[168:171], v24 offset:43008\n\t" \
  "v_mfma_f32_16x16x32_bf16 v[224:227], a[228:231], a[180:183], v[224:227]\n\t" \
  "ds_read_b128 a[172:175], v25 offset:43008\n\t" \
  "v_mfma_f32_16x16x32_bf16 v[228:231], a[232:235], a[176:179], v[228:231]\n\t" \
  "buffer_load_dword v92, v27, s[48:51], s89 offen sc0 sc1\n\t" \
  "v_mfma_f32_16x16x32_bf16 v[228:231], a[236:239], a[180:183], v[228:231]\n\t" \
  "ds_read_b64_tr_b16 v[116:117], v22 offset:49152\n\t" \
  "v_mfma_f32_16x16x32_bf16 v[232:235], a[240:243], a[176:179], v[232:235]\n\t" \
  "buffer_store_dwordx4 v[60:63], v74, s[52:55], s21 offen sc0 sc1\n\t" \
  "v_mfma_f32_16x16x32_bf16 v[232:235], a[244:247], a[180:183], v[232:235]\n\t" \
  "ds_read_b64_tr_b16 v[118:119], v22 offset:51200\n\t" \
  "v_mfma_f32_16x16x32_bf16 v[236:239], a[248:251], a[176:179], v[236:239]\n\t" \
  "ds_read_b64_tr_b16 v[120:121], v23 offset:32768\n\t" \
  "v_mfma_f32_16x16x32_bf16 v[236:239], a[252:255], a[180:183], v[236:239]\n\t" \
  "ds_read_b64_tr_b16 v[122:123], v23 offset:34816\n\t" \
  "v_mfma_f32_16x16x32_bf16 v[252:255], a[248:251], a[184:187], v[252:255]\n\t" \
  "ds_read_b128 a[176:179], v24 offset:45056\n\t" \
  "v_mfma_f32_16x16x32_bf16 v[252:255], a[252:255], a[188:191], v[252:255]\n\t" \
  "ds_read_b128 a[180:183], v25 offset:45056\n\t" \
  "v_mfma_f32_16x16x32_bf16 v[248:251], a[240:243], a[184:187], v[248:251]\n\t" \
  "ds_read_b64_tr_b16 v[124:125], v23 offset:49152\n\t" \
  "v_mfma_f32_16x16x32_bf16 v[248:251], a[244:247], a[188:191], v[248:251]\n\t" \
  "ds_read_b64_tr_b16 v[126:127], v23 offset:51200\n\t" \
  "v_mfma_f32_16x16x32_bf16 v[244:247], a[232:235], a[184:187], v[244:247]\n\t" \
  "v_mfma_f32_16x16x32_bf16 v[244:247], a[236:239], a[188:191], v[244:247]\n\t" \
  "v_mfma_f32_16x16x32_bf16 v[240:243], a[224:227], a[184:187], v[240:243]\n\t" \
  "v_mfma_f32_16x16x32_bf16 v[240:243], a[228:231], a[188:191], v[240:243]\n\t" \
  "s_waitcnt lgkmcnt(6)\n\t" \
  "v_mfma_f32_16x16x32_bf16 a[0:3], a[192:195], a[128:131], a[0:3]\n\t" \
  "ds_read_b128 a[184:187], v24 offset:47104\n\t" \
  "v_mfma_f32_16x16x32_bf16 a[0:3], a[196:199], a[132:135], a[0:3]\n\t" \
  "ds_read_b128 a[188:191], v25 offset:47104\n\t" \
  "v_cmp_gt_u32 vcc, s91, v93\n\t" \
  "v_mfma_f32_16x16x32_bf16 a[4:7], a[200:203], a[128:131], a[4:7]\n\t" \
  "v_mfma_f32_16x16x32_bf16 a[4:7], a[204:207], a[132:135], a[4:7]\n\t" \
  "s_cbranch_vccnz Lsp4_%=\n\t" \
  "Lspr4_%=:\n\t" \
  "v_mfma_f32_16x16x32_bf16 a[8:11], a[208:211], a[128:131], a[8:11]\n\t" \
  "s_mov_b32 s85, s86\n\t" \
  "s_mov_b32 s86, s73\n\t" \
  "v_mfma_f32_16x16x32_bf16 a[8:11], a[212:215], a[132:135], a[8:11]\n\t" \
  "s_lshl_b32 s84, s85, s74\n\t" \
  "s_add_u32 s84, s84, s75\n\t" \
  "v_mfma_f32_16x16x32_bf16 a[12:15], a[216:219], a[128:131], a[12:15]\n\t" \
  "s_waitcnt lgkmcnt(0)\n\t" \
  "s_barrier\n\t" \
  "s_add_u32 m0, s24, 32768\n\t" \
  "v_mfma_f32_16x16x32_bf16 a[12:15], a[220:223], a[132:135], a[12:15]\n\t" \
  "buffer_load_dwordx4 v0, s[40:43], s84 offen offset:0 sc0 sc1 lds\n\t" \
  "v_mfma_f32_16x16x32_bf16 a[28:31], a[216:219], a[136:139], a[28:31]\n\t" \
  "v_mfma_f32_16x16x32_bf16 a[28:31], a[220:223], a[140:143], a[28:31]\n\t" \
  "buffer_load_dwordx4 v1, s[40:43], s84 offen offset:1024 sc0 sc1 lds\n\t" \
  "v_mfma_f32_16x16x32_bf16 a[24:27], a[208:211], a[136:139], a[24:27]\n\t" \
  "v_mfma_f32_16x16x32_bf16 a[24:27], a[212:215], a[140:143], a[24:27]\n\t" \
  "buffer_load_dwordx4 v2, s[40:43], s84 offen offset:2048 sc0 sc1 lds\n\t" \
  "v_mfma_f32_16x16x32_bf16 a[20:23], a[200:203], a[136:139], a[20:23]\n\t" \
  "v_mfma_f32_16x16x32_bf16 a[20:23], a[204:207], a[140:143], a[20:23]\n\t" \
  "buffer_load_dwordx4 v3, s[40:43], s84 offen offset:3072 sc0 sc1 lds\n\t" \
  "v_mfma_f32_16x16x32_bf16 a[16:19], a[192:195], a[136:139], a[16:19]\n\t" \
  "v_mfma_f32_16x16x32_bf16 a[16:19], a[196:199], a[140:143], a[16:19]\n\t" \
  "v_mfma_f32_16x16x32_bf16 a[32:35], a[192:195], a[144:147], a[32:35]\n\t" \
  "v_mfma_f32_16x16x32_bf16 a[32:35], a[196:199], a[148:151], a[32:35]\n\t" \
  "v_mfma_f32_16x16x32_bf16 a[36:39], a[200:203], a[144:147], a[36:39]\n\t" \
  "v_mfma_f32_16x16x32_bf16 a[36:39], a[204:207], a[148:151], a[36:39]\n\t" \
  "v_mfma_f32_16x16x32_bf16 a[40:43], a[208:211], a[144:147], a[40:43]\n\t" \
  "v_mfma_f32_16x16x32_bf16 a[40:43], a[212:215], a[148:151], a[40:43]\n\t" \
  "v_mfma_f32_16x16x32_bf16 a[44:47], a[216:219], a[144:147], a[44:47]\n\t" \
  "v_mfma_f32_16x16x32_bf16 a[44:47], a[220:223], a[148:151], a[44:47]\n\t" \
  "v_mfma_f32_16x16x32_bf16 a[60:63], a[216:219], a[152:155], a[60:63]\n\t" \
  "v_mfma_f32_16x16x32_bf16 a[60:63], a[220:223], a[156:159], a[60:63]\n\t" \
  "v_mfma_f32_16x16x32_bf16 a[56:59], a[208:211], a[152:155], a[56:59]\n\t" \
  "v_mfma_f32_16x16x32_bf16 a[56:59], a[212:215], a[156:159], a[56:59]\n\t" \
  "v_mfma_f32_16x16x32_bf16 a[52:55], a[200:203], a[152:155], a[52:55]\n\t" \
  "v_mfma_f32_16x16x32_bf16 a[52:55], a[204:207], a[156:159], a[52:55]\n\t" \
  "v_mfma_f32_16x16x32_bf16 a[48:51], a[192:195], a[152:155], a[48:51]\n\t" \
  "v_mfma_f32_16x16x32_bf16 a[48:51], a[196:199], a[156:159], a[48:51]\n\t" \
  "s_lshr_b64 s[10:11], s[8:9], s87\n\t" \
  "s_sext_i32_i8 s10, s10\n\t" \
  "v_mfma_f32_16x16x32_bf16 a[64:67], a[192:195], a[160:163], a[64:67]\n\t" \
  "s_add_u32 s87, s87, 8\n\t" \
  "s_add_u32 s73, s73, s10\n\t" \
  "v_mfma_f32_16x16x32_bf16 a[64:67], a[196:199], a[164:167], a[64:67]\n\t" \
  "s_cmp_eq_u32 s87, 56\n\t" \
  "s_cselect_b32 s87, 0, s87\n\t" \
  "v_mfma_f32_16x16x32_bf16 a[68:71], a[200:203], a[160:163], a[68:71]\n\t" \
  "v_mfma_f32_16x16x32_bf16 a[68:71], a[204:207], a[164:167], a[68:71]\n\t" \
  "v_mfma_f32_16x16x32_bf16 a[72:75], a[208:211], a[160:163], a[72:75]\n\t" \
  "v_mfma_f32_16x16x32_bf16 a[72:75], a[212:215], a[164:167], a[72:75]\n\t" \
  "v_mfma_f32_16x16x32_bf16 a[76:79], a[216:219], a[160:163], a[76:79]\n\t" \
  "v_mfma_f32_16x16x32_bf16 a[76:79], a[220:223], a[164:167], a[76:79]\n\t" \
  "v_mfma_f32_16x16x32_bf16 a[92:95], a[216:219], a[168:171], a[92:95]\n\t" \
  "v_mfma_f32_16x16x32_bf16 a[92:95], a[220:223], a[172:175], a[92:95]\n\t" \
  "v_mfma_f32_16x16x32_bf16 a[88:91], a[208:211], a[168:171], a[88:91]\n\t" \
  "v_mfma_f32_16x16x32_bf16 a[88:91], a[212:215], a[172:175], a[88:91]\n\t" \
  "v_mfma_f32_16x16x32_bf16 a[84:87], a[200:203], a[168:171], a[84:87]\n\t" \
  "v_mfma_f32_16x16x32_bf16 a[84:87], a[204:207], a[172:175], a[84:87]\n\t" \
  "v_mfma_f32_16x16x32_bf16 a[80:83], a[192:195], a[168:171], a[80:83]\n\t" \
  "v_mfma_f32_16x16x32_bf16 a[80:83], a[196:199], a[172:175], a[80:83]\n\t" \
  "v_mfma_f32_16x16x32_bf16 a[96:99], a[192:195], a[176:179], a[96:99]\n\t" \
  "v_mfma_f32_16x16x32_bf16 a[96:99], a[196:199], a[180:183], a[96:99]\n\t" \
  "v_mfma_f32_16x16x32_bf16 a[100:103], a[200:203], a[176:179], a[100:103]\n\t" \
  "v_mfma_f32_16x16x32_bf16 a[100:103], a[204:207], a[180:183], a[100:103]\n\t" \
  "v_mfma_f32_16x16x32_bf16 a[104:107], a[208:211], a[176:179], a[104:107]\n\t" \
  "v_mfma_f32_16x16x32_bf16 a[104:107], a[212:215], a[180:183], a[104:107]\n\t" \
  "v_mfma_f32_16x16x32_bf16 a[108:111], a[216:219], a[176:179], a[108:111]\n\t" \
  "v_mfma_f32_16x16x32_bf16 a[108:111], a[220:223], a[180:183], a[108:111]\n\t" \
  "s_add_u32 m0, s25, 32768\n\t" \
  "v_mfma_f32_16x16x32_bf16 a[124:127], a[216:219], a[184:187], a[124:127]\n\t" \
  "buffer_load_dwordx4 v4, s[40:43], s84 offen offset:0 sc0 sc1 lds\n\t" \
  "v_mfma_f32_16x16x32_bf16 a[124:127], a[220:223], a[188:191], a[124:127]\n\t" \
  "v_mfma_f32_16x16x32_bf16 a[120:123], a[208:211], a[184:187], a[120:123]\n\t" \
  "v_mfma_f32_16x16x32_bf16 a[120:123], a[212:215], a[188:191], a[120:123]\n\t" \
  "v_mfma_f32_16x16x32_bf16 a[116:119], a[200:203], a[184:187], a[116:119]\n\t" \
  "v_mfma_f32_16x16x32_bf16 a[116:119], a[204:207], a[188:191], a[116:119]\n\t" \
  "buffer_load_dwordx4 v5, s[40:43], s84 offen offset:1024 sc0 sc1 lds\n\t" \
  "v_mfma_f32_16x16x32_bf16 a[112:115], a[192:195], a[184:187], a[112:115]\n\t" \
  "v_mfma_f32_16x16x32_bf16 a[112:115], a[196:199], a[188:191], a[112:115]\n\t" \
  "s_waitcnt vmcnt(7)\n\t" \
  "s_barrier\n\t" \
  "v_mfma_f32_16x16x32_bf16 v[128:131], v[96:99], a[128:131], v[128:131]\n\t" \
  "ds_read_b64_tr_b16 a[192:193], v16 offset:0\n\t" \
  "v_mfma_f32_16x16x32_bf16 v[128:131], v[100:103], a[132:135], v[128:131]\n\t" \
  "ds_read_b64_tr_b16 a[194:195], v16 offset:2048\n\t" \
  "v_mfma_f32_16x16x32_bf16 v[132:135], v[104:107], a[128:131], v[132:135]\n\t" \
  "buffer_load_dwordx4 v6, s[40:43], s84 offen offset:2048 sc0 sc1 lds\n\t" \
  "v_mfma_f32_16x16x32_bf16 v[132:135], v[108:111], a[132:135], v[132:135]\n\t" \
  "ds_read_b64_tr_b16 a[196:197], v16 offset:16384\n\t" \
  "v_mfma_f32_16x16x32_bf16 v[136:139], v[112:115], a[128:131], v[136:139]\n\t" \
  "ds_read_b64_tr_b16 a[198:199], v16 offset:18432\n\t" \
  "v_mfma_f32_16x16x32_bf16 v[136:139], v[116:119], a[132:135], v[136:139]\n\t" \
  "ds_read_b64_tr_b16 a[200:201], v17 offset:0\n\t" \
  "v_mfma_f32_16x16x32_bf16 v[140:143], v[120:123], a[128:131], v[140:143]\n\t" \
  "ds_read_b64_tr_b16 a[202:203], v17 offset:2048\n\t" \
  "v_mfma_f32_16x16x32_bf16 v[140:143], v[124:127], a[132:135], v[140:143]\n\t" \
  "buffer_load_dwordx4 v7, s[40:43], s84 offen offset:3072 sc0 sc1 lds\n\t" \
  "v_mfma_f32_16x16x32_bf16 v[156:159], v[120:123], a[136:139], v[156:159]\n\t" \
  "ds_read_b128 a[128:131], v24 offset:0\n\t" \
  "v_mfma_f32_16x16x32_bf16 v[156:159], v[124:127], a[140:143], v[156:159]\n\t" \
  "ds_read_b128 a[132:135], v25 offset:0\n\t" \
  "v_mfma_f32_16x16x32_bf16 v[152:155], v[112:115], a[136:139], v[152:155]\n\t" \
  "ds_read_b64_tr_b16 a[204:205], v17 offset:16384\n\t" \
  "v_mfma_f32_16x16x32_bf16 v[152:155], v[116:119], a[140:143], v[152:155]\n\t" \
  "ds_read_b64_tr_b16 a[206:207], v17 offset:18432\n\t" \
  "s_add_u32 m0, s26, 32768\n\t" \
  "v_mfma_f32_16x16x32_bf16 v[148:151], v[104:107], a[136:139], v[148:151]\n\t" \
  "buffer_load_dwordx4 v8, s[40:43], s84 offen offset:0 sc0 sc1 lds\n\t" \
  "v_mfma_f32_16x16x32_bf16 v[148:151], v[108:111], a[140:143], v[148:151]\n\t" \
  "ds_read_b64_tr_b16 a[208:209], v18 offset:0\n\t" \
  "v_mfma_f32_16x16x32_bf16 v[144:147], v[96:99], a[136:139], v[144:147]\n\t" \
  "ds_read_b64_tr_b16 a[210:211], v18 offset:2048\n\t" \
  "v_mfma_f32_16x16x32_bf16 v[144:147], v[100:103], a[140:143], v[144:147]\n\t" \
  "ds_read_b64_tr_b16 a[212:213], v18 offset:16384\n\t" \
  "v_mfma_f32_16x16x32_bf16 v[160:163], v[96:99], a[144:147], v[160:163]\n\t" \
  "ds_read_b128 a[136:139], v24 offset:2048\n\t" \
  "v_mfma_f32_16x16x32_bf16 v[160:163], v[100:103], a[148:151], v[160:163]\n\t" \
  "buffer_load_dwordx4 v9, s[40:43], s84 offen offset:1024 sc0 sc1 lds\n\t" \
  "v_mfma_f32_16x16x32_bf16 v[164:167], v[104:107], a[144:147], v[164:167]\n\t" \
  "ds_read_b128 a[140:143], v25 offset:2048\n\t" \
  "v_mfma_f32_16x16x32_bf16 v[164:167], v[108:111], a[148:151], v[164:167]\n\t" \
  "ds_read_b64_tr_b16 a[214:215], v18 offset:18432\n\t" \
  "v_mfma_f32_16x16x32_bf16 v[168:171], v[112:115], a[144:147], v[168:171]\n\t" \
  "ds_read_b64_tr_b16 a[216:217], v19 offset:0\n\t" \
  "v_mfma_f32_16x16x32_bf16 v[168:171], v[116:119], a[148:151], v[168:171]\n\t" \
  "ds_read_b64_tr_b16 a[218:219], v19 offset:2048\n\t" \
  "v_mfma_f32_16x16x32_bf16 v[172:175], v[120:123], a[144:147], v[172:175]\n\t" \
  "buffer_load_dwordx4 v10, s[40:43], s84 offen offset:2048 sc0 sc1 lds\n\t" \
  "v_mfma_f32_16x16x32_bf16 v[172:175], v[124:127], a[148:151], v[172:175]\n\t" \
  "ds_read_b64_tr_b16 a[220:221], v19 offset:16384\n\t" \
  "v_mfma_f32_16x16x32_bf16 v[188:191], v[120:123], a[152:155], v[188:191]\n\t" \
  "ds_read_b128 a[144:147], v24 offset:4096\n\t" \
  "v_mfma_f32_16x16x32_bf16 v[188:191], v[124:127], a[156:159], v[188:191]\n\t" \
  "ds_read_b128 a[148:151], v25 offset:4096\n\t" \
  "v_mfma_f32_16x16x32_bf16 v[184:187], v[112:115], a[152:155], v[184:187]\n\t" \
  "ds_read_b64_tr_b16 a[222:223], v19 offset:18432\n\t" \
  "v_mfma_f32_16x16x32_bf16 v[184:187], v[116:119], a[156:159], v[184:187]\n\t" \
  "buffer_load_dwordx4 v11, s[40:43], s84 offen offset:3072 sc0 sc1 lds\n\t" \
  "v_mfma_f32_16x16x32_bf16 v[180:183], v[104:107], a[152:155], v[180:183]\n\t" \
  "ds_read_b64_tr_b16 a[224:225], v20 offset:0\n\t" \
  "v_mfma_f32_16x16x32_bf16 v[180:183], v[108:111], a[156:159], v[180:183]\n\t" \
  "ds_read_b64_tr_b16 a[226:227], v20 offset:2048\n\t" \
  "v_mfma_f32_16x16x32_bf16 v[176:179], v[96:99], a[152:155], v[176:179]\n\t" \
  "ds_read_b64_tr_b16 a[228:229], v20 offset:16384\n\t" \
  "v_mfma_f32_16x16x32_bf16 v[176:179], v[100:103], a[156:159], v[176:179]\n\t" \
  "ds_read_b64_tr_b16 a[230:231], v20 offset:18432\n\t" \
  "s_add_u32 m0, s27, 32768\n\t" \
  "v_mfma_f32_16x16x32_bf16 v[192:195], v[96:99], a[160:163], v[192:195]\n\t" \
  "buffer_load_dwordx4 v12, s[40:43], s84 offen offset:0 sc0 sc1 lds\n\t" \
  "v_mfma_f32_16x16x32_bf16 v[192:195], v[100:103], a[164:167], v[192:195]\n\t" \
  "ds_read_b128 a[152:155], v24 offset:6144\n\t" \
  "v_mfma_f32_16x16x32_bf16 v[196:199], v[104:107], a[160:163], v[196:199]\n\t" \
  "ds_read_b128 a[156:159], v25 offset:6144\n\t" \
  "v_mfma_f32_16x16x32_bf16 v[196:199], v[108:111], a[164:167], v[196:199]\n\t" \
  "ds_read_b64_tr_b16 a[232:233], v21 offset:0\n\t" \
  "v_mfma_f32_16x16x32_bf16 v[200:203], v[112:115], a[160:163], v[200:203]\n\t" \
  "s_mov_b32 s90, s89\n\t" \
  "ds_read_b64_tr_b16 a[234:235], v21 offset:2048\n\t" \
  "v_mfma_f32_16x16x32_bf16 v[200:203], v[116:119], a[164:167], v[200:203]\n\t" \
  "buffer_load_dwordx4 v13, s[40:43], s84 offen offset:1024 sc0 sc1 lds\n\t" \
  "s_lshl_b32 s89, s73, 9\n\t" \
  "v_mfma_f32_16x16x32_bf16 v[204:207], v[120:123], a[160:163], v[204:207]\n\t" \
  "s_lshr_b32 s20, s21, 8\n\t" \
  "s_lshl_b32 s21, s80, 10\n\t" \
  "ds_read_b64_tr_b16 a[236:237], v21 offset:16384\n\t" \
  "v_mfma_f32_16x16x32_bf16 v[204:207], v[124:127], a[164:167], v[204:207]\n\t" \
  "s_add_u32 s11, s80, s82\n\t" \
  "s_mov_b32 s80, s81\n\t" \
  "ds_read_b64_tr_b16 a[238:239], v21 offset:18432\n\t" \
  "v_mfma_f32_16x16x32_bf16 v[220:223], v[120:123], a[168:171], v[220:223]\n\t" \
  "ds_read_b128 a[160:163], v24 offset:8192\n\t" \
  "s_mov_b32 s81, s11\n\t" \
  "s_sub_u32 s82, 0, s82\n\t" \
  "v_mfma_f32_16x16x32_bf16 v[220:223], v[124:127], a[172:175], v[220:223]\n\t" \
  "ds_read_b128 a[164:167], v25 offset:8192\n\t" \
  "v_mfma_f32_16x16x32_bf16 v[216:219], v[112:115], a[168:171], v[216:219]\n\t" \
  "buffer_load_dwordx4 v14, s[40:43], s84 offen offset:2048 sc0 sc1 lds\n\t" \
  "v_mfma_f32_16x16x32_bf16 v[216:219], v[116:119], a[172:175], v[216:219]\n\t" \
  "ds_read_b64_tr_b16 a[240:241], v22 offset:0\n\t" \
  "v_mfma_f32_16x16x32_bf16 v[212:215], v[104:107], a[168:171], v[212:215]\n\t" \
  "ds_read_b64_tr_b16 a[242:243], v22 offset:2048\n\t" \
  "v_mfma_f32_16x16x32_bf16 v[212:215], v[108:111], a[172:175], v[212:215]\n\t" \
  "ds_read_b64_tr_b16 a[244:245], v22 offset:16384\n\t" \
  "v_mfma_f32_16x16x32_bf16 v[208:211], v[96:99], a[168:171], v[208:211]\n\t" \
  "ds_read_b64_tr_b16 a[246:247], v22 offset:18432\n\t" \
  "v_mfma_f32_16x16x32_bf16 v[208:211], v[100:103], a[172:175], v[208:211]\n\t" \
  "buffer_load_dwordx4 v15, s[40:43], s84 offen offset:3072 sc0 sc1 lds\n\t" \
  "v_mfma_f32_16x16x32_bf16 v[224:227], v[96:99], a[176:179], v[224:227]\n\t" \
  "ds_read_b128 a[168:171], v24 offset:10240\n\t" \
  "v_mfma_f32_16x16x32_bf16 v[224:227], v[100:103], a[180:183], v[224:227]\n\t" \
  "ds_read_b128 a[172:175], v25 offset:10240\n\t" \
  "v_mfma_f32_16x16x32_bf16 v[228:231], v[104:107], a[176:179], v[228:231]\n\t" \
  "buffer_load_dword v94, v39, s[48:51], 0 offen sc1\n\t" \
  "v_mfma_f32_16x16x32_bf16 v[228:231], v[108:111], a[180:183], v[228:231]\n\t" \
  "ds_read_b64_tr_b16 a[248:249], v23 offset:0\n\t" \
  "v_mfma_f32_16x16x32_bf16 v[232:235], v[112:115], a[176:179], v[232:235]\n\t" \
  "buffer_load_dword v93, v27, s[48:51], s89 offen sc0 sc1\n\t" \
  "v_mfma_f32_16x16x32_bf16 v[232:235], v[116:119], a[180:183], v[232:235]\n\t" \
  "ds_read_b64_tr_b16 a[250:251], v23 offset:2048\n\t" \
  "v_mfma_f32_16x16x32_bf16 v[236:239], v[120:123], a[176:179], v[236:239]\n\t" \
  "buffer_store_dwordx4 v[56:59], v72, s[52:55], s21 offen sc0 sc1\n\t" \
  "v_mfma_f32_16x16x32_bf16 v[236:239], v[124:127], a[180:183], v[236:239]\n\t" \
  "ds_read_b64_tr_b16 a[252:253], v23 offset:16384\n\t" \
  "v_mfma_f32_16x16x32_bf16 v[252:255], v[120:123], a[184:187], v[252:255]\n\t" \
  "ds_read_b128 a[176:179], v24 offset:12288\n\t" \
  "v_mfma_f32_16x16x32_bf16 v[252:255], v[124:127], a[188:191], v[252:255]\n\t" \
  "ds_read_b128 a[180:183], v25 offset:12288\n\t" \
  "v_mfma_f32_16x16x32_bf16 v[248:251], v[112:115], a[184:187], v[248:251]\n\t" \
  "ds_read_b64_tr_b16 a[254:255], v23 offset:18432\n\t" \
  "v_mfma_f32_16x16x32_bf16 v[248:251], v[116:119], a[188:191], v[248:251]\n\t" \
  "v_mfma_f32_16x16x32_bf16 v[244:247], v[104:107], a[184:187], v[244:247]\n\t" \
  "v_mfma_f32_16x16x32_bf16 v[244:247], v[108:111], a[188:191], v[244:247]\n\t" \
  "v_mfma_f32_16x16x32_bf16 v[240:243], v[96:99], a[184:187], v[240:243]\n\t" \
  "v_mfma_f32_16x16x32_bf16 v[240:243], v[100:103], a[188:191], v[240:243]\n\t" \
  "s_mov_b32 s95, 8\n\t" \
  "LloopC_%=:\n\t" \
  "s_waitcnt lgkmcnt(4)\n\t" \
  "v_mfma_f32_16x16x32_bf16 a[0:3], a[192:195], a[128:131], a[0:3]\n\t" \
  "ds_read_b128 a[184:187], v24 offset:14336\n\t" \
  "v_mfma_f32_16x16x32_bf16 a[0:3], a[196:199], a[132:135], a[0:3]\n\t" \
  "ds_read_b128 a[188:191], v25 offset:14336\n\t" \
  "v_cmp_gt_u32 vcc, s91, v92\n\t" \
  "v_mfma_f32_16x16x32_bf16 a[4:7], a[200:203], a[128:131], a[4:7]\n\t" \
  "v_mfma_f32_16x16x32_bf16 a[4:7], a[204:207], a[132:135], a[4:7]\n\t" \
  "s_cbranch_vccnz Lsp5_%=\n\t" \
  "Lspr5_%=:\n\t" \
  "v_mfma_f32_16x16x32_bf16 a[8:11], a[208:211], a[128:131], a[8:11]\n\t" \
  "s_mov_b32 s85, s86\n\t" \
  "s_mov_b32 s86, s73\n\t" \
  "v_mfma_f32_16x16x32_bf16 a[8:11], a[212:215], a[132:135], a[8:11]\n\t" \
  "s_lshl_b32 s84, s85, s74\n\t" \
  "s_add_u32 s84, s84, s75\n\t" \
  "v_mfma_f32_16x16x32_bf16 a[12:15], a[216:219], a[128:131], a[12:15]\n\t" \
  "s_waitcnt lgkmcnt(0)\n\t" \
  "s_barrier\n\t" \
  "s_add_u32 m0, s24, 0\n\t" \
  "v_mfma_f32_16x16x32_bf16 a[12:15], a[220:223], a[132:135], a[12:15]\n\t" \
  "buffer_load_dwordx4 v0, s[40:43], s84 offen offset:0 sc0 sc1 lds\n\t" \
  "v_mfma_f32_16x16x32_bf16 a[28:31], a[216:219], a[136:139], a[28:31]\n\t" \
  "v_mfma_f32_16x16x32_bf16 a[28:31], a[220:223], a[140:143], a[28:31]\n\t" \
  "buffer_load_dwordx4 v1, s[40:43], s84 offen offset:1024 sc0 sc1 lds\n\t" \
  "v_mfma_f32_16x16x32_bf16 a[24:27], a[208:211], a[136:139], a[24:27]\n\t" \
  "v_mfma_f32_16x16x32_bf16 a[24:27], a[212:215], a[140:143], a[24:27]\n\t" \
  "buffer_load_dwordx4 v2, s[40:43], s84 offen offset:2048 sc0 sc1 lds\n\t" \
  "v_mfma_f32_16x16x32_bf16 a[20:23], a[200:203], a[136:139], a[20:23]\n\t" \
  "v_mfma_f32_16x16x32_bf16 a[20:23], a[204:207], a[140:143], a[20:23]\n\t" \
  "buffer_load_dwordx4 v3, s[40:43], s84 offen offset:3072 sc0 sc1 lds\n\t" \
  "v_mfma_f32_16x16x32_bf16 a[16:19], a[192:195], a[136:139], a[16:19]\n\t" \
  "v_mfma_f32_16x16x32_bf16 a[16:19], a[196:199], a[140:143], a[16:19]\n\t" \
  "v_mfma_f32_16x16x32_bf16 a[32:35], a[192:195], a[144:147], a[32:35]\n\t" \
  "v_mfma_f32_16x16x32_bf16 a[32:35], a[196:199], a[148:151], a[32:35]\n\t" \
  "v_mfma_f32_16x16x32_bf16 a[36:39], a[200:203], a[144:147], a[36:39]\n\t" \
  "v_mfma_f32_16x16x32_bf16 a[36:39], a[204:207], a[148:151], a[36:39]\n\t" \
  "s_cmp_eq_u32 s95, 8\n\t" \
  "s_cbranch_scc1 Lswu_%=\n\t" \
  "Lswur_%=:\n\t" \
  "v_mfma_f32_16x16x32_bf16 a[40:43], a[208:211], a[144:147], a[40:43]\n\t" \
  "v_mfma_f32_16x16x32_bf16 a[40:43], a[212:215], a[148:151], a[40:43]\n\t" \
  "v_mfma_f32_16x16x32_bf16 a[44:47], a[216:219], a[144:147], a[44:47]\n\t" \
  "v_mfma_f32_16x16x32_bf16 a[44:47], a[220:223], a[148:151], a[44:47]\n\t" \
  "v_mfma_f32_16x16x32_bf16 a[60:63], a[216:219], a[152:155], a[60:63]\n\t" \
  "v_mfma_f32_16x16x32_bf16 a[60:63], a[220:223], a[156:159], a[60:63]\n\t" \
  "v_mfma_f32_16x16x32_bf16 a[56:59], a[208:211], a[152:155], a[56:59]\n\t" \
  "v_mfma_f32_16x16x32_bf16 a[56:59], a[212:215], a[156:159], a[56:59]\n\t" \
  "v_mfma_f32_16x16x32_bf16 a[52:55], a[200:203], a[152:155], a[52:55]\n\t" \
  "v_mfma_f32_16x16x32_bf16 a[52:55], a[204:207], a[156:159], a[52:55]\n\t" \
  "v_mfma_f32_16x16x32_bf16 a[48:51], a[192:195], a[152:155], a[48:51]\n\t" \
  "v_mfma_f32_16x16x32_bf16 a[48:51], a[196:199], a[156:159], a[48:51]\n\t" \
  "s_lshr_b64 s[10:11], s[8:9], s87\n\t" \
  "s_sext_i32_i8 s10, s10\n\t" \
  "v_mfma_f32_16x16x32_bf16 a[64:67], a[192:195], a[160:163], a[64:67]\n\t" \
  "s_add_u32 s87, s87, 8\n\t" \
  "s_add_u32 s73, s73, s10\n\t" \
  "v_mfma_f32_16x16x32_bf16 a[64:67], a[196:199], a[164:167], a[64:67]\n\t" \
  "s_cmp_eq_u32 s87, 56\n\t" \
  "s_cselect_b32 s87, 0, s87\n\t" \
  "v_mfma_f32_16x16x32_bf16 a[68:71], a[200:203], a[160:163], a[68:71]\n\t" \
  "v_mfma_f32_16x16x32_bf16 a[68:71], a[204:207], a[164:167], a[68:71]\n\t" \
  "v_mfma_f32_16x16x32_bf16 a[72:75], a[208:211], a[160:163], a[72:75]\n\t" \
  "v_mfma_f32_16x16x32_bf16 a[72:75], a[212:215], a[164:167], a[72:75]\n\t" \
  "v_mfma_f32_16x16x32_bf16 a[76:79], a[216:219], a[160:163], a[76:79]\n\t" \
  "v_mfma_f32_16x16x32_bf16 a[76:79], a[220:223], a[164:167], a[76:79]\n\t" \
  "v_mfma_f32_16x16x32_bf16 a[92:95], a[216:219], a[168:171], a[92:95]\n\t" \
  "v_mfma_f32_16x16x32_bf16 a[92:95], a[220:223], a[172:175], a[92:95]\n\t" \
  "v_mfma_f32_16x16x32_bf16 a[88:91], a[208:211], a[168:171], a[88:91]\n\t" \
  "v_mfma_f32_16x16x32_bf16 a[88:91], a[212:215], a[172:175], a[88:91]\n\t" \
  "v_mfma_f32_16x16x32_bf16 a[84:87], a[200:203], a[168:171], a[84:87]\n\t" \
  "v_mfma_f32_16x16x32_bf16 a[84:87], a[204:207], a[172:175], a[84:87]\n\t" \
  "v_mfma_f32_16x16x32_bf16 a[80:83], a[192:195], a[168:171], a[80:83]\n\t" \
  "v_mfma_f32_16x16x32_bf16 a[80:83], a[196:199], a[172:175], a[80:83]\n\t" \
  "v_mfma_f32_16x16x32_bf16 a[96:99], a[192:195], a[176:179], a[96:99]\n\t" \
  "v_mfma_f32_16x16x32_bf16 a[96:99], a[196:199], a[180:183], a[96:99]\n\t" \
  "v_mfma_f32_16x16x32_bf16 a[100:103], a[200:203], a[176:179], a[100:103]\n\t" \
  "v_mfma_f32_16x16x32_bf16 a[100:103], a[204:207], a[180:183], a[100:103]\n\t" \
  "v_mfma_f32_16x16x32_bf16 a[104:107], a[208:211], a[176:179], a[104:107]\n\t" \
  "v_mfma_f32_16x16x32_bf16 a[104:107], a[212:215], a[180:183], a[104:107]\n\t" \
  "v_mfma_f32_16x16x32_bf16 a[108:111], a[216:219], a[176:179], a[108:111]\n\t" \
  "v_mfma_f32_16x16x32_bf16 a[108:111], a[220:223], a[180:183], a[108:111]\n\t" \
  "s_add_u32 m0, s25, 0\n\t" \
  "v_mfma_f32_16x16x32_bf16 a[124:127], a[216:219], a[184:187], a[124:127]\n\t" \
  "buffer_load_dwordx4 v4, s[40:43], s84 offen offset:0 sc0 sc1 lds\n\t" \
  "v_mfma_f32_16x16x32_bf16 a[124:127], a[220:223], a[188:191], a[124:127]\n\t" \
  "v_mfma_f32_16x16x32_bf16 a[120:123], a[208:211], a[184:187], a[120:123]\n\t" \
  "v_mfma_f32_16x16x32_bf16 a[120:123], a[212:215], a[188:191], a[120:123]\n\t" \
  "v_mfma_f32_16x16x32_bf16 a[116:119], a[200:203], a[184:187], a[116:119]\n\t" \
  "v_mfma_f32_16x16x32_bf16 a[116:119], a[204:207], a[188:191], a[116:119]\n\t" \
  "buffer_load_dwordx4 v5, s[40:43], s84 offen offset:1024 sc0 sc1 lds\n\t" \
  "v_mfma_f32_16x16x32_bf16 a[112:115], a[192:195], a[184:187], a[112:115]\n\t" \
  "v_mfma_f32_16x16x32_bf16 a[112:115], a[196:199], a[188:191], a[112:115]\n\t" \
  "s_waitcnt vmcnt(7)\n\t" \
  "s_barrier\n\t" \
  "v_mfma_f32_16x16x32_bf16 v[128:131], a[224:227], a[128:131], v[128:131]\n\t" \
  "ds_read_b64_tr_b16 a[192:193], v16 offset:32768\n\t" \
  "v_mfma_f32_16x16x32_bf16 v[128:131], a[228:231], a[132:135], v[128:131]\n\t" \
  "ds_read_b64_tr_b16 a[194:195], v16 offset:34816\n\t" \
  "v_mfma_f32_16x16x32_bf16 v[132:135], a[232:235], a[128:131], v[132:135]\n\t" \
  "buffer_load_dwordx4 v6, s[40:43], s84 offen offset:2048 sc0 sc1 lds\n\t" \
  "v_mfma_f32_16x16x32_bf16 v[132:135], a[236:239], a[132:135], v[132:135]\n\t" \
  "ds_read_b64_tr_b16 a[196:197], v16 offset:49152\n\t" \
  "v_mfma_f32_16x16x32_bf16 v[136:139], a[240:243], a[128:131], v[136:139]\n\t" \
  "ds_read_b64_tr_b16 a[198:199], v16 offset:51200\n\t" \
  "v_mfma_f32_16x16x32_bf16 v[136:139], a[244:247], a[132:135], v[136:139]\n\t" \
  "ds_read_b64_tr_b16 a[200:201], v17 offset:32768\n\t" \
  "v_mfma_f32_16x16x32_bf16 v[140:143], a[248:251], a[128:131], v[140:143]\n\t" \
  "ds_read_b64_tr_b16 a[202:203], v17 offset:34816\n\t" \
  "v_mfma_f32_16x16x32_bf16 v[140:143], a[252:255], a[132:135], v[140:143]\n\t" \
  "buffer_load_dwordx4 v7, s[40:43], s84 offen offset:3072 sc0 sc1 lds\n\t" \
  "v_mfma_f32_16x16x32_bf16 v[156:159], a[248:251], a[136:139], v[156:159]\n\t" \
  "ds_read_b128 a[128:131], v24 offset:32768\n\t" \
  "v_mfma_f32_16x16x32_bf16 v[156:159], a[252:255], a[140:143], v[156:159]\n\t" \
  "ds_read_b128 a[132:135], v25 offset:32768\n\t" \
  "v_mfma_f32_16x16x32_bf16 v[152:155], a[240:243], a[136:139], v[152:155]\n\t" \
  "ds_read_b64_tr_b16 a[204:205], v17 offset:49152\n\t" \
  "v_mfma_f32_16x16x32_bf16 v[152:155], a[244:247], a[140:143], v[152:155]\n\t" \
  "ds_read_b64_tr_b16 a[206:207], v17 offset:51200\n\t" \
  "s_add_u32 m0, s26, 0\n\t" \
  "v_mfma_f32_16x16x32_bf16 v[148:151], a[232:235], a[136:139], v[148:151]\n\t" \
  "buffer_load_dwordx4 v8, s[40:43], s84 offen offset:0 sc0 sc1 lds\n\t" \
  "v_mfma_f32_16x16x32_bf16 v[148:151], a[236:239], a[140:143], v[148:151]\n\t" \
  "ds_read_b64_tr_b16 a[208:209], v18 offset:32768\n\t" \
  "v_mfma_f32_16x16x32_bf16 v[144:147], a[224:227], a[136:139], v[144:147]\n\t" \
  "ds_read_b64_tr_b16 a[210:211], v18 offset:34816\n\t" \
  "v_mfma_f32_16x16x32_bf16 v[144:147], a[228:231], a[140:143], v[144:147]\n\t" \
  "ds_read_b64_tr_b16 a[212:213], v18 offset:49152\n\t" \
  "v_mfma_f32_16x16x32_bf16 v[160:163], a[224:227], a[144:147], v[160:163]\n\t" \
  "ds_read_b128 a[136:139], v24 offset:34816\n\t" \
  "v_mfma_f32_16x16x32_bf16 v[160:163], a[228:231], a[148:151], v[160:163]\n\t" \
  "buffer_load_dwordx4 v9, s[40:43], s84 offen offset:1024 sc0 sc1 lds\n\t" \
  "v_mfma_f32_16x16x32_bf16 v[164:167], a[232:235], a[144:147], v[164:167]\n\t" \
  "ds_read_b128 a[140:143], v25 offset:34816\n\t" \
  "v_mfma_f32_16x16x32_bf16 v[164:167], a[236:239], a[148:151], v[164:167]\n\t" \
  "ds_read_b64_tr_b16 a[214:215], v18 offset:51200\n\t" \
  "v_mfma_f32_16x16x32_bf16 v[168:171], a[240:243], a[144:147], v[168:171]\n\t" \
  "ds_read_b64_tr_b16 a[216:217], v19 offset:32768\n\t" \
  "v_mfma_f32_16x16x32_bf16 v[168:171], a[244:247], a[148:151], v[168:171]\n\t" \
  "ds_read_b64_tr_b16 a[218:219], v19 offset:34816\n\t" \
  "v_mfma_f32_16x16x32_bf16 v[172:175], a[248:251], a[144:147], v[172:175]\n\t" \
  "buffer_load_dwordx4 v10, s[40:43], s84 offen offset:2048 sc0 sc1 lds\n\t" \
  "v_mfma_f32_16x16x32_bf16 v[172:175], a[252:255], a[148:151], v[172:175]\n\t" \
  "ds_read_b64_tr_b16 a[220:221], v19 offset:49152\n\t" \
  "v_mfma_f32_16x16x32_bf16 v[188:191], a[248:251], a[152:155], v[188:191]\n\t" \
  "ds_read_b128 a[144:147], v24 offset:36864\n\t" \
  "v_mfma_f32_16x16x32_bf16 v[188:191], a[252:255], a[156:159], v[188:191]\n\t" \
  "ds_read_b128 a[148:151], v25 offset:36864\n\t" \
  "v_mfma_f32_16x16x32_bf16 v[184:187], a[240:243], a[152:155], v[184:187]\n\t" \
  "ds_read_b64_tr_b16 a[222:223], v19 offset:51200\n\t" \
  "v_mfma_f32_16x16x32_bf16 v[184:187], a[244:247], a[156:159], v[184:187]\n\t" \
  "buffer_load_dwordx4 v11, s[40:43], s84 offen offset:3072 sc0 sc1 lds\n\t" \
  "v_mfma_f32_16x16x32_bf16 v[180:183], a[232:235], a[152:155], v[180:183]\n\t" \
  "ds_read_b64_tr_b16 v[96:97], v20 offset:32768\n\t" \
  "v_mfma_f32_16x16x32_bf16 v[180:183], a[236:239], a[156:159], v[180:183]\n\t" \
  "ds_read_b64_tr_b16 v[98:99], v20 offset:34816\n\t" \
  "v_mfma_f32_16x16x32_bf16 v[176:179], a[224:227], a[152:155], v[176:179]\n\t" \
  "ds_read_b64_tr_b16 v[100:101], v20 offset:49152\n\t" \
  "v_mfma_f32_16x16x32_bf16 v[176:179], a[228:231], a[156:159], v[176:179]\n\t" \
  "ds_read_b64_tr_b16 v[102:103], v20 offset:51200\n\t" \
  "s_add_u32 m0, s27, 0\n\t" \
  "v_mfma_f32_16x16x32_bf16 v[192:195], a[224:227], a[160:163], v[192:195]\n\t" \
  "buffer_load_dwordx4 v12, s[40:43], s84 offen offset:0 sc0 sc1 lds\n\t" \
  "v_mfma_f32_16x16x32_bf16 v[192:195], a[228:231], a[164:167], v[192:195]\n\t" \
  "ds_read_b128 a[152:155], v24 offset:38912\n\t" \
  "v_mfma_f32_16x16x32_bf16 v[196:199], a[232:235], a[160:163], v[196:199]\n\t" \
  "ds_read_b128 a[156:159], v25 offset:38912\n\t" \
  "v_mfma_f32_16x16x32_bf16 v[196:199], a[236:239], a[164:167], v[196:199]\n\t" \
  "ds_read_b64_tr_b16 v[104:105], v21 offset:32768\n\t" \
  "v_mfma_f32_16x16x32_bf16 v[200:203], a[240:243], a[160:163], v[200:203]\n\t" \
  "s_mov_b32 s90, s89\n\t" \
  "ds_read_b64_tr_b16 v[106:107], v21 offset:34816\n\t" \
  "v_mfma_f32_16x16x32_bf16 v[200:203], a[244:247], a[164:167], v[200:203]\n\t" \
  "buffer_load_dwordx4 v13, s[40:43], s84 offen offset:1024 sc0 sc1 lds\n\t" \
  "s_lshl_b32 s89, s73, 9\n\t" \
  "v_mfma_f32_16x16x32_bf16 v[204:207], a[248:251], a[160:163], v[204:207]\n\t" \
  "ds_read_b64_tr_b16 v[108:109], v21 offset:49152\n\t" \
  "v_mfma_f32_16x16x32_bf16 v[204:207], a[252:255], a[164:167], v[204:207]\n\t" \
  "ds_read_b64_tr_b16 v[110:111], v21 offset:51200\n\t" \
  "v_mfma_f32_16x16x32_bf16 v[220:223], a[248:251], a[168:171], v[220:223]\n\t" \
  "ds_read_b128 a[160:163], v24 offset:40960\n\t" \
  "v_mfma_f32_16x16x32_bf16 v[220:223], a[252:255], a[172:175], v[220:223]\n\t" \
  "ds_read_b128 a[164:167], v25 offset:40960\n\t" \
  "v_mfma_f32_16x16x32_bf16 v[216:219], a[240:243], a[168:171], v[216:219]\n\t" \
  "buffer_load_dwordx4 v14, s[40:43], s84 offen offset:2048 sc0 sc1 lds\n\t" \
  "v_mfma_f32_16x16x32_bf16 v[216:219], a[244:247], a[172:175], v[216:219]\n\t" \
  "ds_read_b64_tr_b16 v[112:113], v22 offset:32768\n\t" \
  "v_mfma_f32_16x16x32_bf16 v[212:215], a[232:235], a[168:171], v[212:215]\n\t" \
  "ds_read_b64_tr_b16 v[114:115], v22 offset:34816\n\t" \
  "v_mfma_f32_16x16x32_bf16 v[212:215], a[236:239], a[172:175], v[212:215]\n\t" \
  "ds_read_b64_tr_b16 v[116:117], v22 offset:49152\n\t" \
  "v_mfma_f32_16x16x32_bf16 v[208:211], a[224:227], a[168:171], v[208:211]\n\t" \
  "ds_read_b64_tr_b16 v[118:119], v22 offset:51200\n\t" \
  "v_mfma_f32_16x16x32_bf16 v[208:211], a[228:231], a[172:175], v[208:211]\n\t" \
  "buffer_load_dwordx4 v15, s[40:43], s84 offen offset:3072 sc0 sc1 lds\n\t" \
  "v_mfma_f32_16x16x32_bf16 v[224:227], a[224:227], a[176:179], v[224:227]\n\t" \
  "ds_read_b128 a[168:171], v24 offset:43008\n\t" \
  "v_mfma_f32_16x16x32_bf16 v[224:227], a[228:231], a[180:183], v[224:227]\n\t" \
  "ds_read_b128 a[172:175], v25 offset:43008\n\t" \
  "v_mfma_f32_16x16x32_bf16 v[228:231], a[232:235], a[176:179], v[228:231]\n\t" \
  "buffer_load_dword v92, v27, s[48:51], s89 offen sc0 sc1\n\t" \
  "v_mfma_f32_16x16x32_bf16 v[228:231], a[236:239], a[180:183], v[228:231]\n\t" \
  "ds_read_b64_tr_b16 v[120:121], v23 offset:32768\n\t" \
  "v_mfma_f32_16x16x32_bf16 v[232:235], a[240:243], a[176:179], v[232:235]\n\t" \
  "buffer_store_dwordx4 v[56:59], v73, s[52:55], s21 offen sc0 sc1\n\t" \
  "v_mfma_f32_16x16x32_bf16 v[232:235], a[244:247], a[180:183], v[232:235]\n\t" \
  "ds_read_b64_tr_b16 v[122:123], v23 offset:34816\n\t" \
  "v_mfma_f32_16x16x32_bf16 v[236:239], a[248:251], a[176:179], v[236:239]\n\t" \
  "ds_read_b64_tr_b16 v[124:125], v23 offset:49152\n\t" \
  "v_mfma_f32_16x16x32_bf16 v[236:239], a[252:255], a[180:183], v[236:239]\n\t" \
  "buffer_store_dword v43, v31, s[64:67], s20 offen sc0 sc1\n\t" \
  "v_mfma_f32_16x16x32_bf16 v[252:255], a[248:251], a[184:187], v[252:255]\n\t" \
  "ds_read_b128 a[176:179], v24 offset:45056\n\t" \
  "v_mfma_f32_16x16x32_bf16 v[252:255], a[252:255], a[188:191], v[252:255]\n\t" \
  "ds_read_b128 a[180:183], v25 offset:45056\n\t" \
  "v_mfma_f32_16x16x32_bf16 v[248:251], a[240:243], a[184:187], v[248:251]\n\t" \
  "buffer_load_dwordx4 v[60:63], v29, s[56:59], s80 offen\n\t" \
  "v_mfma_f32_16x16x32_bf16 v[248:251], a[244:247], a[188:191], v[248:251]\n\t" \
  "ds_read_b64_tr_b16 v[126:127], v23 offset:51200\n\t" \
  "v_mfma_f32_16x16x32_bf16 v[244:247], a[232:235], a[184:187], v[244:247]\n\t" \
  "v_mfma_f32_16x16x32_bf16 v[244:247], a[236:239], a[188:191], v[244:247]\n\t" \
  "v_mfma_f32_16x16x32_bf16 v[240:243], a[224:227], a[184:187], v[240:243]\n\t" \
  "v_mfma_f32_16x16x32_bf16 v[240:243], a[228:231], a[188:191], v[240:243]\n\t" \
  "s_waitcnt lgkmcnt(4)\n\t" \
  "v_mfma_f32_16x16x32_bf16 a[0:3], a[192:195], a[128:131], a[0:3]\n\t" \
  "ds_read_b128 a[184:187], v24 offset:47104\n\t" \
  "v_mfma_f32_16x16x32_bf16 a[0:3], a[196:199], a[132:135], a[0:3]\n\t" \
  "ds_read_b128 a[188:191], v25 offset:47104\n\t" \
  "v_cmp_gt_u32 vcc, s91, v93\n\t" \
  "v_mfma_f32_16x16x32_bf16 a[4:7], a[200:203], a[128:131], a[4:7]\n\t" \
  "v_mfma_f32_16x16x32_bf16 a[4:7], a[204:207], a[132:135], a[4:7]\n\t" \
  "s_cbranch_vccnz Lsp6_%=\n\t" \
  "Lspr6_%=:\n\t" \
  "v_mfma_f32_16x16x32_bf16 a[8:11], a[208:211], a[128:131], a[8:11]\n\t" \
  "s_mov_b32 s85, s86\n\t" \
  "s_mov_b32 s86, s73\n\t" \
  "v_mfma_f32_16x16x32_bf16 a[8:11], a[212:215], a[132:135], a[8:11]\n\t" \
  "s_lshl_b32 s84, s85, s74\n\t" \
  "s_add_u32 s84, s84, s75\n\t" \
  "v_mfma_f32_16x16x32_bf16 a[12:15], a[216:219], a[128:131], a[12:15]\n\t" \
  "s_waitcnt lgkmcnt(0)\n\t" \
  "s_barrier\n\t" \
  "s_add_u32 m0, s24, 32768\n\t" \
  "v_mfma_f32_16x16x32_bf16 a[12:15], a[220:223], a[132:135], a[12:15]\n\t" \
  "buffer_load_dwordx4 v0, s[40:43], s84 offen offset:0 sc0 sc1 lds\n\t" \
  "v_mfma_f32_16x16x32_bf16 a[28:31], a[216:219], a[136:139], a[28:31]\n\t" \
  "v_mfma_f32_16x16x32_bf16 a[28:31], a[220:223], a[140:143], a[28:31]\n\t" \
  "buffer_load_dwordx4 v1, s[40:43], s84 offen offset:1024 sc0 sc1 lds\n\t" \
  "v_mfma_f32_16x16x32_bf16 a[24:27], a[208:211], a[136:139], a[24:27]\n\t" \
  "v_mfma_f32_16x16x32_bf16 a[24:27], a[212:215], a[140:143], a[24:27]\n\t" \
  "buffer_load_dwordx4 v2, s[40:43], s84 offen offset:2048 sc0 sc1 lds\n\t" \
  "v_mfma_f32_16x16x32_bf16 a[20:23], a[200:203], a[136:139], a[20:23]\n\t" \
  "v_mfma_f32_16x16x32_bf16 a[20:23], a[204:207], a[140:143], a[20:23]\n\t" \
  "buffer_load_dwordx4 v3, s[40:43], s84 offen offset:3072 sc0 sc1 lds\n\t" \
  "v_mfma_f32_16x16x32_bf16 a[16:19], a[192:195], a[136:139], a[16:19]\n\t" \
  "v_mfma_f32_16x16x32_bf16 a[16:19], a[196:199], a[140:143], a[16:19]\n\t" \
  "v_mfma_f32_16x16x32_bf16 a[32:35], a[192:195], a[144:147], a[32:35]\n\t" \
  "v_mfma_f32_16x16x32_bf16 a[32:35], a[196:199], a[148:151], a[32:35]\n\t" \
  "v_mfma_f32_16x16x32_bf16 a[36:39], a[200:203], a[144:147], a[36:39]\n\t" \
  "v_mfma_f32_16x16x32_bf16 a[36:39], a[204:207], a[148:151], a[36:39]\n\t" \
  "v_mfma_f32_16x16x32_bf16 a[40:43], a[208:211], a[144:147], a[40:43]\n\t" \
  "v_mfma_f32_16x16x32_bf16 a[40:43], a[212:215], a[148:151], a[40:43]\n\t" \
  "v_mfma_f32_16x16x32_bf16 a[44:47], a[216:219], a[144:147], a[44:47]\n\t" \
  "v_mfma_f32_16x16x32_bf16 a[44:47], a[220:223], a[148:151], a[44:47]\n\t" \
  "v_mfma_f32_16x16x32_bf16 a[60:63], a[216:219], a[152:155], a[60:63]\n\t" \
  "v_mfma_f32_16x16x32_bf16 a[60:63], a[220:223], a[156:159], a[60:63]\n\t" \
  "v_mfma_f32_16x16x32_bf16 a[56:59], a[208:211], a[152:155], a[56:59]\n\t" \
  "v_mfma_f32_16x16x32_bf16 a[56:59], a[212:215], a[156:159], a[56:59]\n\t" \
  "v_mfma_f32_16x16x32_bf16 a[52:55], a[200:203], a[152:155], a[52:55]\n\t" \
  "v_mfma_f32_16x16x32_bf16 a[52:55], a[204:207], a[156:159], a[52:55]\n\t" \
  "v_mfma_f32_16x16x32_bf16 a[48:51], a[192:195], a[152:155], a[48:51]\n\t" \
  "v_mfma_f32_16x16x32_bf16 a[48:51], a[196:199], a[156:159], a[48:51]\n\t" \
  "s_lshr_b64 s[10:11], s[8:9], s87\n\t" \
  "s_sext_i32_i8 s10, s10\n\t" \
  "v_mfma_f32_16x16x32_bf16 a[64:67], a[192:195], a[160:163], a[64:67]\n\t" \
  "s_add_u32 s87, s87, 8\n\t" \
  "s_add_u32 s73, s73, s10\n\t" \
  "v_mfma_f32_16x16x32_bf16 a[64:67], a[196:199], a[164:167], a[64:67]\n\t" \
  "s_cmp_eq_u32 s87, 56\n\t" \
  "s_cselect_b32 s87, 0, s87\n\t" \
  "v_mfma_f32_16x16x32_bf16 a[68:71], a[200:203], a[160:163], a[68:71]\n\t" \
  "v_mfma_f32_16x16x32_bf16 a[68:71], a[204:207], a[164:167], a[68:71]\n\t" \
  "v_mfma_f32_16x16x32_bf16 a[72:75], a[208:211], a[160:163], a[72:75]\n\t" \
  "v_mfma_f32_16x16x32_bf16 a[72:75], a[212:215], a[164:167], a[72:75]\n\t" \
  "v_mfma_f32_16x16x32_bf16 a[76:79], a[216:219], a[160:163], a[76:79]\n\t" \
  "v_mfma_f32_16x16x32_bf16 a[76:79], a[220:223], a[164:167], a[76:79]\n\t" \
  "v_mfma_f32_16x16x32_bf16 a[92:95], a[216:219], a[168:171], a[92:95]\n\t" \
  "v_mfma_f32_16x16x32_bf16 a[92:95], a[220:223], a[172:175], a[92:95]\n\t" \
  "v_mfma_f32_16x16x32_bf16 a[88:91], a[208:211], a[168:171], a[88:91]\n\t" \
  "v_mfma_f32_16x16x32_bf16 a[88:91], a[212:215], a[172:175], a[88:91]\n\t" \
  "v_mfma_f32_16x16x32_bf16 a[84:87], a[200:203], a[168:171], a[84:87]\n\t" \
  "v_mfma_f32_16x16x32_bf16 a[84:87], a[204:207], a[172:175], a[84:87]\n\t" \
  "v_mfma_f32_16x16x32_bf16 a[80:83], a[192:195], a[168:171], a[80:83]\n\t" \
  "v_mfma_f32_16x16x32_bf16 a[80:83], a[196:199], a[172:175], a[80:83]\n\t" \
  "v_mfma_f32_16x16x32_bf16 a[96:99], a[192:195], a[176:179], a[96:99]\n\t" \
  "v_mfma_f32_16x16x32_bf16 a[96:99], a[196:199], a[180:183], a[96:99]\n\t" \
  "v_mfma_f32_16x16x32_bf16 a[100:103], a[200:203], a[176:179], a[100:103]\n\t" \
  "v_mfma_f32_16x16x32_bf16 a[100:103], a[204:207], a[180:183], a[100:103]\n\t" \
  "v_mfma_f32_16x16x32_bf16 a[104:107], a[208:211], a[176:179], a[104:107]\n\t" \
  "v_mfma_f32_16x16x32_bf16 a[104:107], a[212:215], a[180:183], a[104:107]\n\t" \
  "v_mfma_f32_16x16x32_bf16 a[108:111], a[216:219], a[176:179], a[108:111]\n\t" \
  "v_mfma_f32_16x16x32_bf16 a[108:111], a[220:223], a[180:183], a[108:111]\n\t" \
  "s_add_u32 m0, s25, 32768\n\t" \
  "v_mfma_f32_16x16x32_bf16 a[124:127], a[216:219], a[184:187], a[124:127]\n\t" \
  "buffer_load_dwordx4 v4, s[40:43], s84 offen offset:0 sc0 sc1 lds\n\t" \
  "v_mfma_f32_16x16x32_bf16 a[124:127], a[220:223], a[188:191], a[124:127]\n\t" \
  "v_mfma_f32_16x16x32_bf16 a[120:123], a[208:211], a[184:187], a[120:123]\n\t" \
  "v_mfma_f32_16x16x32_bf16 a[120:123], a[212:215], a[188:191], a[120:123]\n\t" \
  "v_mfma_f32_16x16x32_bf16 a[116:119], a[200:203], a[184:187], a[116:119]\n\t" \
  "v_mfma_f32_16x16x32_bf16 a[116:119], a[204:207], a[188:191], a[116:119]\n\t" \
  "buffer_load_dwordx4 v5, s[40:43], s84 offen offset:1024 sc0 sc1 lds\n\t" \
  "v_mfma_f32_16x16x32_bf16 a[112:115], a[192:195], a[184:187], a[112:115]\n\t" \
  "v_mfma_f32_16x16x32_bf16 a[112:115], a[196:199], a[188:191], a[112:115]\n\t" \
  "s_waitcnt vmcnt(9)\n\t" \
  "s_barrier\n\t" \
  "v_mfma_f32_16x16x32_bf16 v[128:131], v[96:99], a[128:131], v[128:131]\n\t" \
  "ds_read_b64_tr_b16 a[192:193], v16 offset:0\n\t" \
  "v_mfma_f32_16x16x32_bf16 v[128:131], v[100:103], a[132:135], v[128:131]\n\t" \
  "ds_read_b64_tr_b16 a[194:195], v16 offset:2048\n\t" \
  "v_mfma_f32_16x16x32_bf16 v[132:135], v[104:107], a[128:131], v[132:135]\n\t" \
  "buffer_load_dwordx4 v6, s[40:43], s84 offen offset:2048 sc0 sc1 lds\n\t" \
  "v_mfma_f32_16x16x32_bf16 v[132:135], v[108:111], a[132:135], v[132:135]\n\t" \
  "ds_read_b64_tr_b16 a[196:197], v16 offset:16384\n\t" \
  "v_mfma_f32_16x16x32_bf16 v[136:139], v[112:115], a[128:131], v[136:139]\n\t" \
  "ds_read_b64_tr_b16 a[198:199], v16 offset:18432\n\t" \
  "v_mfma_f32_16x16x32_bf16 v[136:139], v[116:119], a[132:135], v[136:139]\n\t" \
  "ds_read_b64_tr_b16 a[200:201], v17 offset:0\n\t" \
  "v_mfma_f32_16x16x32_bf16 v[140:143], v[120:123], a[128:131], v[140:143]\n\t" \
  "ds_read_b64_tr_b16 a[202:203], v17 offset:2048\n\t" \
  "v_mfma_f32_16x16x32_bf16 v[140:143], v[124:127], a[132:135], v[140:143]\n\t" \
  "buffer_load_dwordx4 v7, s[40:43], s84 offen offset:3072 sc0 sc1 lds\n\t" \
  "v_mfma_f32_16x16x32_bf16 v[156:159], v[120:123], a[136:139], v[156:159]\n\t" \
  "ds_read_b128 a[128:131], v24 offset:0\n\t" \
  "v_mfma_f32_16x16x32_bf16 v[156:159], v[124:127], a[140:143], v[156:159]\n\t" \
  "ds_read_b128 a[132:135], v25 offset:0\n\t" \
  "v_mfma_f32_16x16x32_bf16 v[152:155], v[112:115], a[136:139], v[152:155]\n\t" \
  "ds_read_b64_tr_b16 a[204:205], v17 offset:16384\n\t" \
  "v_mfma_f32_16x16x32_bf16 v[152:155], v[116:119], a[140:143], v[152:155]\n\t" \
  "ds_read_b64_tr_b16 a[206:207], v17 offset:18432\n\t" \
  "s_add_u32 m0, s26, 32768\n\t" \
  "v_mfma_f32_16x16x32_bf16 v[148:151], v[104:107], a[136:139], v[148:151]\n\t" \
  "buffer_load_dwordx4 v8, s[40:43], s84 offen offset:0 sc0 sc1 lds\n\t" \
  "v_mfma_f32_16x16x32_bf16 v[148:151], v[108:111], a[140:143], v[148:151]\n\t" \
  "ds_read_b64_tr_b16 a[208:209], v18 offset:0\n\t" \
  "v_mfma_f32_16x16x32_bf16 v[144:147], v[96:99], a[136:139], v[144:147]\n\t" \
  "ds_read_b64_tr_b16 a[210:211], v18 offset:2048\n\t" \
  "v_mfma_f32_16x16x32_bf16 v[144:147], v[100:103], a[140:143], v[144:147]\n\t" \
  "ds_read_b64_tr_b16 a[212:213], v18 offset:16384\n\t" \
  "v_mfma_f32_16x16x32_bf16 v[160:163], v[96:99], a[144:147], v[160:163]\n\t" \
  "ds_read_b128 a[136:139], v24 offset:2048\n\t" \
  "v_mfma_f32_16x16x32_bf16 v[160:163], v[100:103], a[148:151], v[160:163]\n\t" \
  "buffer_load_dwordx4 v9, s[40:43], s84 offen offset:1024 sc0 sc1 lds\n\t" \
  "v_mfma_f32_16x16x32_bf16 v[164:167], v[104:107], a[144:147], v[164:167]\n\t" \
  "ds_read_b128 a[140:143], v25 offset:2048\n\t" \
  "v_mfma_f32_16x16x32_bf16 v[164:167], v[108:111], a[148:151], v[164:167]\n\t" \
  "ds_read_b64_tr_b16 a[214:215], v18 offset:18432\n\t" \
  "v_mfma_f32_16x16x32_bf16 v[168:171], v[112:115], a[144:147], v[168:171]\n\t" \
  "ds_read_b64_tr_b16 a[216:217], v19 offset:0\n\t" \
  "v_mfma_f32_16x16x32_bf16 v[168:171], v[116:119], a[148:151], v[168:171]\n\t" \
  "ds_read_b64_tr_b16 a[218:219], v19 offset:2048\n\t" \
  "v_mfma_f32_16x16x32_bf16 v[172:175], v[120:123], a[144:147], v[172:175]\n\t" \
  "buffer_load_dwordx4 v10, s[40:43], s84 offen offset:2048 sc0 sc1 lds\n\t" \
  "v_mfma_f32_16x16x32_bf16 v[172:175], v[124:127], a[148:151], v[172:175]\n\t" \
  "ds_read_b64_tr_b16 a[220:221], v19 offset:16384\n\t" \
  "v_mfma_f32_16x16x32_bf16 v[188:191], v[120:123], a[152:155], v[188:191]\n\t" \
  "ds_read_b128 a[144:147], v24 offset:4096\n\t" \
  "v_mfma_f32_16x16x32_bf16 v[188:191], v[124:127], a[156:159], v[188:191]\n\t" \
  "ds_read_b128 a[148:151], v25 offset:4096\n\t" \
  "v_mfma_f32_16x16x32_bf16 v[184:187], v[112:115], a[152:155], v[184:187]\n\t" \
  "ds_read_b64_tr_b16 a[222:223], v19 offset:18432\n\t" \
  "v_mfma_f32_16x16x32_bf16 v[184:187], v[116:119], a[156:159], v[184:187]\n\t" \
  "buffer_load_dwordx4 v11, s[40:43], s84 offen offset:3072 sc0 sc1 lds\n\t" \
  "v_mfma_f32_16x16x32_bf16 v[180:183], v[104:107], a[152:155], v[180:183]\n\t" \
  "ds_read_b64_tr_b16 a[224:225], v20 offset:0\n\t" \
  "v_mfma_f32_16x16x32_bf16 v[180:183], v[108:111], a[156:159], v[180:183]\n\t" \
  "ds_read_b64_tr_b16 a[226:227], v20 offset:2048\n\t" \
  "v_mfma_f32_16x16x32_bf16 v[176:179], v[96:99], a[152:155], v[176:179]\n\t" \
  "ds_read_b64_tr_b16 a[228:229], v20 offset:16384\n\t" \
  "v_mfma_f32_16x16x32_bf16 v[176:179], v[100:103], a[156:159], v[176:179]\n\t" \
  "ds_read_b64_tr_b16 a[230:231], v20 offset:18432\n\t" \
  "s_add_u32 m0, s27, 32768\n\t" \
  "v_mfma_f32_16x16x32_bf16 v[192:195], v[96:99], a[160:163], v[192:195]\n\t" \
  "buffer_load_dwordx4 v12, s[40:43], s84 offen offset:0 sc0 sc1 lds\n\t" \
  "v_mfma_f32_16x16x32_bf16 v[192:195], v[100:103], a[164:167], v[192:195]\n\t" \
  "ds_read_b128 a[152:155], v24 offset:6144\n\t" \
  "v_mfma_f32_16x16x32_bf16 v[196:199], v[104:107], a[160:163], v[196:199]\n\t" \
  "ds_read_b128 a[156:159], v25 offset:6144\n\t" \
  "v_mfma_f32_16x16x32_bf16 v[196:199], v[108:111], a[164:167], v[196:199]\n\t" \
  "ds_read_b64_tr_b16 a[232:233], v21 offset:0\n\t" \
  "v_mfma_f32_16x16x32_bf16 v[200:203], v[112:115], a[160:163], v[200:203]\n\t" \
  "s_mov_b32 s90, s89\n\t" \
  "ds_read_b64_tr_b16 a[234:235], v21 offset:2048\n\t" \
  "v_mfma_f32_16x16x32_bf16 v[200:203], v[116:119], a[164:167], v[200:203]\n\t" \
  "buffer_load_dwordx4 v13, s[40:43], s84 offen offset:1024 sc0 sc1 lds\n\t" \
  "s_lshl_b32 s89, s73, 9\n\t" \
  "v_mfma_f32_16x16x32_bf16 v[204:207], v[120:123], a[160:163], v[204:207]\n\t" \
  "ds_read_b64_tr_b16 a[236:237], v21 offset:16384\n\t" \
  "v_mfma_f32_16x16x32_bf16 v[204:207], v[124:127], a[164:167], v[204:207]\n\t" \
  "ds_read_b64_tr_b16 a[238:239], v21 offset:18432\n\t" \
  "v_mfma_f32_16x16x32_bf16 v[220:223], v[120:123], a[168:171], v[220:223]\n\t" \
  "ds_read_b128 a[160:163], v24 offset:8192\n\t" \
  "v_mfma_f32_16x16x32_bf16 v[220:223], v[124:127], a[172:175], v[220:223]\n\t" \
  "ds_read_b128 a[164:167], v25 offset:8192\n\t" \
  "v_mfma_f32_16x16x32_bf16 v[216:219], v[112:115], a[168:171], v[216:219]\n\t" \
  "buffer_load_dwordx4 v14, s[40:43], s84 offen offset:2048 sc0 sc1 lds\n\t" \
  "v_mfma_f32_16x16x32_bf16 v[216:219], v[116:119], a[172:175], v[216:219]\n\t" \
  "ds_read_b64_tr_b16 a[240:241], v22 offset:0\n\t" \
  "v_mfma_f32_16x16x32_bf16 v[212:215], v[104:107], a[168:171], v[212:215]\n\t" \
  "ds_read_b64_tr_b16 a[242:243], v22 offset:2048\n\t" \
  "v_mfma_f32_16x16x32_bf16 v[212:215], v[108:111], a[172:175], v[212:215]\n\t" \
  "ds_read_b64_tr_b16 a[244:245], v22 offset:16384\n\t" \
  "v_mfma_f32_16x16x32_bf16 v[208:211], v[96:99], a[168:171], v[208:211]\n\t" \
  "ds_read_b64_tr_b16 a[246:247], v22 offset:18432\n\t" \
  "v_mfma_f32_16x16x32_bf16 v[208:211], v[100:103], a[172:175], v[208:211]\n\t" \
  "buffer_load_dwordx4 v15, s[40:43], s84 offen offset:3072 sc0 sc1 lds\n\t" \
  "v_mfma_f32_16x16x32_bf16 v[224:227], v[96:99], a[176:179], v[224:227]\n\t" \
  "ds_read_b128 a[168:171], v24 offset:10240\n\t" \
  "v_mfma_f32_16x16x32_bf16 v[224:227], v[100:103], a[180:183], v[224:227]\n\t" \
  "ds_read_b128 a[172:175], v25 offset:10240\n\t" \
  "v_mfma_f32_16x16x32_bf16 v[228:231], v[104:107], a[176:179], v[228:231]\n\t" \
  "buffer_load_dword v93, v27, s[48:51], s89 offen sc0 sc1\n\t" \
  "v_mfma_f32_16x16x32_bf16 v[228:231], v[108:111], a[180:183], v[228:231]\n\t" \
  "ds_read_b64_tr_b16 a[248:249], v23 offset:0\n\t" \
  "v_mfma_f32_16x16x32_bf16 v[232:235], v[112:115], a[176:179], v[232:235]\n\t" \
  "buffer_store_dwordx4 v[56:59], v74, s[52:55], s21 offen sc0 sc1\n\t" \
  "v_mfma_f32_16x16x32_bf16 v[232:235], v[116:119], a[180:183], v[232:235]\n\t" \
  "ds_read_b64_tr_b16 a[250:251], v23 offset:2048\n\t" \
  "v_mfma_f32_16x16x32_bf16 v[236:239], v[120:123], a[176:179], v[236:239]\n\t" \
  "ds_read_b64_tr_b16 a[252:253], v23 offset:16384\n\t" \
  "v_mfma_f32_16x16x32_bf16 v[236:239], v[124:127], a[180:183], v[236:239]\n\t" \
  "ds_read_b64_tr_b16 a[254:255], v23 offset:18432\n\t" \
  "v_mfma_f32_16x16x32_bf16 v[252:255], v[120:123], a[184:187], v[252:255]\n\t" \
  "ds_read_b128 a[176:179], v24 offset:12288\n\t" \
  "v_mfma_f32_16x16x32_bf16 v[252:255], v[124:127], a[188:191], v[252:255]\n\t" \
  "ds_read_b128 a[180:183], v25 offset:12288\n\t" \
  "v_mfma_f32_16x16x32_bf16 v[248:251], v[112:115], a[184:187], v[248:251]\n\t" \
  "v_mfma_f32_16x16x32_bf16 v[248:251], v[116:119], a[188:191], v[248:251]\n\t" \
  "v_mfma_f32_16x16x32_bf16 v[244:247], v[104:107], a[184:187], v[244:247]\n\t" \
  "v_mfma_f32_16x16x32_bf16 v[244:247], v[108:111], a[188:191], v[244:247]\n\t" \
  "v_mfma_f32_16x16x32_bf16 v[240:243], v[96:99], a[184:187], v[240:243]\n\t" \
  "v_mfma_f32_16x16x32_bf16 v[240:243], v[100:103], a[188:191], v[240:243]\n\t" \
  "s_waitcnt lgkmcnt(4)\n\t" \
  "v_mfma_f32_16x16x32_bf16 a[0:3], a[192:195], a[128:131], a[0:3]\n\t" \
  "ds_read_b128 a[184:187], v24 offset:14336\n\t" \
  "s_cmp_eq_u32 s95, 8\n\t" \
  "s_cbranch_scc1 Lswx_%=\n\t" \
  "Lswxr_%=:\n\t" \
  "v_mfma_f32_16x16x32_bf16 a[0:3], a[196:199], a[132:135], a[0:3]\n\t" \
  "ds_read_b128 a[188:191], v25 offset:14336\n\t" \
  "v_cmp_gt_u32 vcc, s91, v92\n\t" \
  "v_mfma_f32_16x16x32_bf16 a[4:7], a[200:203], a[128:131], a[4:7]\n\t" \
  "v_mfma_f32_16x16x32_bf16 a[4:7], a[204:207], a[132:135], a[4:7]\n\t" \
  "s_cbranch_vccnz Lsp7_%=\n\t" \
  "Lspr7_%=:\n\t" \
  "v_mfma_f32_16x16x32_bf16 a[8:11], a[208:211], a[128:131], a[8:11]\n\t" \
  "s_mov_b32 s85, s86\n\t" \
  "s_mov_b32 s86, s73\n\t" \
  "v_mfma_f32_16x16x32_bf16 a[8:11], a[212:215], a[132:135], a[8:11]\n\t" \
  "s_lshl_b32 s84, s85, s74\n\t" \
  "s_add_u32 s84, s84, s75\n\t" \
  "v_mfma_f32_16x16x32_bf16 a[12:15], a[216:219], a[128:131], a[12:15]\n\t" \
  "s_waitcnt lgkmcnt(0)\n\t" \
  "s_barrier\n\t" \
  "s_add_u32 m0, s24, 0\n\t" \
  "v_mfma_f32_16x16x32_bf16 a[12:15], a[220:223], a[132:135], a[12:15]\n\t" \
  "buffer_load_dwordx4 v0, s[40:43], s84 offen offset:0 sc0 sc1 lds\n\t" \
  "v_mfma_f32_16x16x32_bf16 a[28:31], a[216:219], a[136:139], a[28:31]\n\t" \
  "v_mfma_f32_16x16x32_bf16 a[28:31], a[220:223], a[140:143], a[28:31]\n\t" \
  "buffer_load_dwordx4 v1, s[40:43], s84 offen offset:1024 sc0 sc1 lds\n\t" \
  "v_mfma_f32_16x16x32_bf16 a[24:27], a[208:211], a[136:139], a[24:27]\n\t" \
  "v_mfma_f32_16x16x32_bf16 a[24:27], a[212:215], a[140:143], a[24:27]\n\t" \
  "buffer_load_dwordx4 v2, s[40:43], s84 offen offset:2048 sc0 sc1 lds\n\t" \
  "v_mfma_f32_16x16x32_bf16 a[20:23], a[200:203], a[136:139], a[20:23]\n\t" \
  "v_mfma_f32_16x16x32_bf16 a[20:23], a[204:207], a[140:143], a[20:23]\n\t" \
  "buffer_load_dwordx4 v3, s[40:43], s84 offen offset:3072 sc0 sc1 lds\n\t" \
  "v_mfma_f32_16x16x32_bf16 a[16:19], a[192:195], a[136:139], a[16:19]\n\t" \
  "v_mfma_f32_16x16x32_bf16 a[16:19], a[196:199], a[140:143], a[16:19]\n\t" \
  "v_mfma_f32_16x16x32_bf16 a[32:35], a[192:195], a[144:147], a[32:35]\n\t" \
  "v_mfma_f32_16x16x32_bf16 a[32:35], a[196:199], a[148:151], a[32:35]\n\t" \
  "v_mfma_f32_16x16x32_bf16 a[36:39], a[200:203], a[144:147], a[36:39]\n\t" \
  "v_mfma_f32_16x16x32_bf16 a[36:39], a[204:207], a[148:151], a[36:39]\n\t" \
  "v_mfma_f32_16x16x32_bf16 a[40:43], a[208:211], a[144:147], a[40:43]\n\t" \
  "v_mfma_f32_16x16x32_bf16 a[40:43], a[212:215], a[148:151], a[40:43]\n\t" \
  "v_mfma_f32_16x16x32_bf16 a[44:47], a[216:219], a[144:147], a[44:47]\n\t" \
  "v_mfma_f32_16x16x32_bf16 a[44:47], a[220:223], a[148:151], a[44:47]\n\t" \
  "v_mfma_f32_16x16x32_bf16 a[60:63], a[216:219], a[152:155], a[60:63]\n\t" \
  "v_mfma_f32_16x16x32_bf16 a[60:63], a[220:223], a[156:159], a[60:63]\n\t" \
  "v_mfma_f32_16x16x32_bf16 a[56:59], a[208:211], a[152:155], a[56:59]\n\t" \
  "v_mfma_f32_16x16x32_bf16 a[56:59], a[212:215], a[156:159], a[56:59]\n\t" \
  "v_mfma_f32_16x16x32_bf16 a[52:55], a[200:203], a[152:155], a[52:55]\n\t" \
  "v_mfma_f32_16x16x32_bf16 a[52:55], a[204:207], a[156:159], a[52:55]\n\t" \
  "v_mfma_f32_16x16x32_bf16 a[48:51], a[192:195], a[152:155], a[48:51]\n\t" \
  "v_mfma_f32_16x16x32_bf16 a[48:51], a[196:199], a[156:159], a[48:51]\n\t" \
  "s_lshr_b64 s[10:11], s[8:9], s87\n\t" \
  "s_sext_i32_i8 s10, s10\n\t" \
  "v_mfma_f32_16x16x32_bf16 a[64:67], a[192:195], a[160:163], a[64:67]\n\t" \
  "s_add_u32 s87, s87, 8\n\t" \
  "s_add_u32 s73, s73, s10\n\t" \
  "v_mfma_f32_16x16x32_bf16 a[64:67], a[196:199], a[164:167], a[64:67]\n\t" \
  "s_cmp_eq_u32 s87, 56\n\t" \
  "s_cselect_b32 s87, 0, s87\n\t" \
  "v_mfma_f32_16x16x32_bf16 a[68:71], a[200:203], a[160:163], a[68:71]\n\t" \
  "v_mfma_f32_16x16x32_bf16 a[68:71], a[204:207], a[164:167], a[68:71]\n\t" \
  "v_mfma_f32_16x16x32_bf16 a[72:75], a[208:211], a[160:163], a[72:75]\n\t" \
  "v_mfma_f32_16x16x32_bf16 a[72:75], a[212:215], a[164:167], a[72:75]\n\t" \
  "v_mfma_f32_16x16x32_bf16 a[76:79], a[216:219], a[160:163], a[76:79]\n\t" \
  "v_mfma_f32_16x16x32_bf16 a[76:79], a[220:223], a[164:167], a[76:79]\n\t" \
  "v_mfma_f32_16x16x32_bf16 a[92:95], a[216:219], a[168:171], a[92:95]\n\t" \
  "v_mfma_f32_16x16x32_bf16 a[92:95], a[220:223], a[172:175], a[92:95]\n\t" \
  "v_mfma_f32_16x16x32_bf16 a[88:91], a[208:211], a[168:171], a[88:91]\n\t" \
  "v_mfma_f32_16x16x32_bf16 a[88:91], a[212:215], a[172:175], a[88:91]\n\t" \
  "v_mfma_f32_16x16x32_bf16 a[84:87], a[200:203], a[168:171], a[84:87]\n\t" \
  "v_mfma_f32_16x16x32_bf16 a[84:87], a[204:207], a[172:175], a[84:87]\n\t" \
  "v_mfma_f32_16x16x32_bf16 a[80:83], a[192:195], a[168:171], a[80:83]\n\t" \
  "v_mfma_f32_16x16x32_bf16 a[80:83], a[196:199], a[172:175], a[80:83]\n\t" \
  "v_mfma_f32_16x16x32_bf16 a[96:99], a[192:195], a[176:179], a[96:99]\n\t" \
  "v_mfma_f32_16x16x32_bf16 a[96:99], a[196:199], a[180:183], a[96:99]\n\t" \
  "v_mfma_f32_16x16x32_bf16 a[100:103], a[200:203], a[176:179], a[100:103]\n\t" \
  "v_mfma_f32_16x16x32_bf16 a[100:103], a[204:207], a[180:183], a[100:103]\n\t" \
  "v_mfma_f32_16x16x32_bf16 a[104:107], a[208:211], a[176:179], a[104:107]\n\t" \
  "v_mfma_f32_16x16x32_bf16 a[104:107], a[212:215], a[180:183], a[104:107]\n\t" \
  "v_mfma_f32_16x16x32_bf16 a[108:111], a[216:219], a[176:179], a[108:111]\n\t" \
  "v_mfma_f32_16x16x32_bf16 a[108:111], a[220:223], a[180:183], a[108:111]\n\t" \
  "s_add_u32 m0, s25, 0\n\t" \
  "v_mfma_f32_16x16x32_bf16 a[124:127], a[216:219], a[184:187], a[124:127]\n\t" \
  "buffer_load_dwordx4 v4, s[40:43], s84 offen offset:0 sc0 sc1 lds\n\t" \
  "v_mfma_f32_16x16x32_bf16 a[124:127], a[220:223], a[188:191], a[124:127]\n\t" \
  "v_mfma_f32_16x16x32_bf16 a[120:123], a[208:211], a[184:187], a[120:123]\n\t" \
  "v_mfma_f32_16x16x32_bf16 a[120:123], a[212:215], a[188:191], a[120:123]\n\t" \
  "v_mfma_f32_16x16x32_bf16 a[116:119], a[200:203], a[184:187], a[116:119]\n\t" \
  "v_mfma_f32_16x16x32_bf16 a[116:119], a[204:207], a[188:191], a[116:119]\n\t" \
  "buffer_load_dwordx4 v5, s[40:43], s84 offen offset:1024 sc0 sc1 lds\n\t" \
  "v_mfma_f32_16x16x32_bf16 a[112:115], a[192:195], a[184:187], a[112:115]\n\t" \
  "v_mfma_f32_16x16x32_bf16 a[112:115], a[196:199], a[188:191], a[112:115]\n\t" \
  "s_waitcnt vmcnt(7)\n\t" \
  "s_barrier\n\t" \
  "v_mfma_f32_16x16x32_bf16 v[128:131], a[224:227], a[128:131], v[128:131]\n\t" \
  "ds_read_b64_tr_b16 a[192:193], v16 offset:32768\n\t" \
  "v_mfma_f32_16x16x32_bf16 v[128:131], a[228:231], a[132:135], v[128:131]\n\t" \
  "ds_read_b64_tr_b16 a[194:195], v16 offset:34816\n\t" \
  "v_mfma_f32_16x16x32_bf16 v[132:135], a[232:235], a[128:131], v[132:135]\n\t" \
  "buffer_load_dwordx4 v6, s[40:43], s84 offen offset:2048 sc0 sc1 lds\n\t" \
  "v_mfma_f32_16x16x32_bf16 v[132:135], a[236:239], a[132:135], v[132:135]\n\t" \
  "ds_read_b64_tr_b16 a[196:197], v16 offset:49152\n\t" \
  "v_mfma_f32_16x16x32_bf16 v[136:139], a[240:243], a[128:131], v[136:139]\n\t" \
  "ds_read_b64_tr_b16 a[198:199], v16 offset:51200\n\t" \
  "v_mfma_f32_16x16x32_bf16 v[136:139], a[244:247], a[132:135], v[136:139]\n\t" \
  "ds_read_b64_tr_b16 a[200:201], v17 offset:32768\n\t" \
  "v_mfma_f32_16x16x32_bf16 v[140:143], a[248:251], a[128:131], v[140:143]\n\t" \
  "ds_read_b64_tr_b16 a[202:203], v17 offset:34816\n\t" \
  "v_mfma_f32_16x16x32_bf16 v[140:143], a[252:255], a[132:135], v[140:143]\n\t" \
  "buffer_load_dwordx4 v7, s[40:43], s84 offen offset:3072 sc0 sc1 lds\n\t" \
  "v_mfma_f32_16x16x32_bf16 v[156:159], a[248:251], a[136:139], v[156:159]\n\t" \
  "ds_read_b128 a[128:131], v24 offset:32768\n\t" \
  "v_mfma_f32_16x16x32_bf16 v[156:159], a[252:255], a[140:143], v[156:159]\n\t" \
  "ds_read_b128 a[132:135], v25 offset:32768\n\t" \
  "v_mfma_f32_16x16x32_bf16 v[152:155], a[240:243], a[136:139], v[152:155]\n\t" \
  "ds_read_b64_tr_b16 a[204:205], v17 offset:49152\n\t" \
  "v_mfma_f32_16x16x32_bf16 v[152:155], a[244:247], a[140:143], v[152:155]\n\t" \
  "ds_read_b64_tr_b16 a[206:207], v17 offset:51200\n\t" \
  "s_add_u32 m0, s26, 0\n\t" \
  "v_mfma_f32_16x16x32_bf16 v[148:151], a[232:235], a[136:139], v[148:151]\n\t" \
  "buffer_load_dwordx4 v8, s[40:43], s84 offen offset:0 sc0 sc1 lds\n\t" \
  "v_mfma_f32_16x16x32_bf16 v[148:151], a[236:239], a[140:143], v[148:151]\n\t" \
  "ds_read_b64_tr_b16 a[208:209], v18 offset:32768\n\t" \
  "v_mfma_f32_16x16x32_bf16 v[144:147], a[224:227], a[136:139], v[144:147]\n\t" \
  "ds_read_b64_tr_b16 a[210:211], v18 offset:34816\n\t" \
  "v_mfma_f32_16x16x32_bf16 v[144:147], a[228:231], a[140:143], v[144:147]\n\t" \
  "ds_read_b64_tr_b16 a[212:213], v18 offset:49152\n\t" \
  "v_mfma_f32_16x16x32_bf16 v[160:163], a[224:227], a[144:147], v[160:163]\n\t" \
  "ds_read_b128 a[136:139], v24 offset:34816\n\t" \
  "v_mfma_f32_16x16x32_bf16 v[160:163], a[228:231], a[148:151], v[160:163]\n\t" \
  "buffer_load_dwordx4 v9, s[40:43], s84 offen offset:1024 sc0 sc1 lds\n\t" \
  "v_mfma_f32_16x16x32_bf16 v[164:167], a[232:235], a[144:147], v[164:167]\n\t" \
  "ds_read_b128 a[140:143], v25 offset:34816\n\t" \
  "v_mfma_f32_16x16x32_bf16 v[164:167], a[236:239], a[148:151], v[164:167]\n\t" \
  "ds_read_b64_tr_b16 a[214:215], v18 offset:51200\n\t" \
  "v_mfma_f32_16x16x32_bf16 v[168:171], a[240:243], a[144:147], v[168:171]\n\t" \
  "ds_read_b64_tr_b16 a[216:217], v19 offset:32768\n\t" \
  "v_mfma_f32_16x16x32_bf16 v[168:171], a[244:247], a[148:151], v[168:171]\n\t" \
  "ds_read_b64_tr_b16 a[218:219], v19 offset:34816\n\t" \
  "v_mfma_f32_16x16x32_bf16 v[172:175], a[248:251], a[144:147], v[172:175]\n\t" \
  "buffer_load_dwordx4 v10, s[40:43], s84 offen offset:2048 sc0 sc1 lds\n\t" \
  "v_mfma_f32_16x16x32_bf16 v[172:175], a[252:255], a[148:151], v[172:175]\n\t" \
  "ds_read_b64_tr_b16 a[220:221], v19 offset:49152\n\t" \
  "v_mfma_f32_16x16x32_bf16 v[188:191], a[248:251], a[152:155], v[188:191]\n\t" \
  "ds_read_b128 a[144:147], v24 offset:36864\n\t" \
  "v_mfma_f32_16x16x32_bf16 v[188:191], a[252:255], a[156:159], v[188:191]\n\t" \
  "ds_read_b128 a[148:151], v25 offset:36864\n\t" \
  "v_mfma_f32_16x16x32_bf16 v[184:187], a[240:243], a[152:155], v[184:187]\n\t" \
  "ds_read_b64_tr_b16 a[222:223], v19 offset:51200\n\t" \
  "v_mfma_f32_16x16x32_bf16 v[184:187], a[244:247], a[156:159], v[184:187]\n\t" \
  "buffer_load_dwordx4 v11, s[40:43], s84 offen offset:3072 sc0 sc1 lds\n\t" \
  "v_mfma_f32_16x16x32_bf16 v[180:183], a[232:235], a[152:155], v[180:183]\n\t" \
  "ds_read_b64_tr_b16 v[96:97], v20 offset:32768\n\t" \
  "v_mfma_f32_16x16x32_bf16 v[180:183], a[236:239], a[156:159], v[180:183]\n\t" \
  "ds_read_b64_tr_b16 v[98:99], v20 offset:34816\n\t" \
  "v_mfma_f32_16x16x32_bf16 v[176:179], a[224:227], a[152:155], v[176:179]\n\t" \
  "ds_read_b64_tr_b16 v[100:101], v20 offset:49152\n\t" \
  "v_mfma_f32_16x16x32_bf16 v[176:179], a[228:231], a[156:159], v[176:179]\n\t" \
  "ds_read_b64_tr_b16 v[102:103], v20 offset:51200\n\t" \
  "s_add_u32 m0, s27, 0\n\t" \
  "v_mfma_f32_16x16x32_bf16 v[192:195], a[224:227], a[160:163], v[192:195]\n\t" \
  "buffer_load_dwordx4 v12, s[40:43], s84 offen offset:0 sc0 sc1 lds\n\t" \
  "v_mfma_f32_16x16x32_bf16 v[192:195], a[228:231], a[164:167], v[192:195]\n\t" \
  "ds_read_b128 a[152:155], v24 offset:38912\n\t" \
  "v_mfma_f32_16x16x32_bf16 v[196:199], a[232:235], a[160:163], v[196:199]\n\t" \
  "ds_read_b128 a[156:159], v25 offset:38912\n\t" \
  "v_mfma_f32_16x16x32_bf16 v[196:199], a[236:239], a[164:167], v[196:199]\n\t" \
  "ds_read_b64_tr_b16 v[104:105], v21 offset:32768\n\t" \
  "v_mfma_f32_16x16x32_bf16 v[200:203], a[240:243], a[160:163], v[200:203]\n\t" \
  "s_mov_b32 s90, s89\n\t" \
  "ds_read_b64_tr_b16 v[106:107], v21 offset:34816\n\t" \
  "v_mfma_f32_16x16x32_bf16 v[200:203], a[244:247], a[164:167], v[200:203]\n\t" \
  "buffer_load_dwordx4 v13, s[40:43], s84 offen offset:1024 sc0 sc1 lds\n\t" \
  "s_lshl_b32 s89, s73, 9\n\t" \
  "v_mfma_f32_16x16x32_bf16 v[204:207], a[248:251], a[160:163], v[204:207]\n\t" \
  "s_lshr_b32 s20, s21, 8\n\t" \
  "s_lshl_b32 s21, s80, 10\n\t" \
  "ds_read_b64_tr_b16 v[108:109], v21 offset:49152\n\t" \
  "v_mfma_f32_16x16x32_bf16 v[204:207], a[252:255], a[164:167], v[204:207]\n\t" \
  "s_add_u32 s11, s80, s82\n\t" \
  "s_mov_b32 s80, s81\n\t" \
  "ds_read_b64_tr_b16 v[110:111], v21 offset:51200\n\t" \
  "v_mfma_f32_16x16x32_bf16 v[220:223], a[248:251], a[168:171], v[220:223]\n\t" \
  "ds_read_b128 a[160:163], v24 offset:40960\n\t" \
  "s_mov_b32 s81, s11\n\t" \
  "s_sub_u32 s82, 0, s82\n\t" \
  "v_mfma_f32_16x16x32_bf16 v[220:223], a[252:255], a[172:175], v[220:223]\n\t" \
  "ds_read_b128 a[164:167], v25 offset:40960\n\t" \
  "v_mfma_f32_16x16x32_bf16 v[216:219], a[240:243], a[168:171], v[216:219]\n\t" \
  "buffer_load_dwordx4 v14, s[40:43], s84 offen offset:2048 sc0 sc1 lds\n\t" \
  "v_mfma_f32_16x16x32_bf16 v[216:219], a[244:247], a[172:175], v[216:219]\n\t" \
  "ds_read_b64_tr_b16 v[112:113], v22 offset:32768\n\t" \
  "v_mfma_f32_16x16x32_bf16 v[212:215], a[232:235], a[168:171], v[212:215]\n\t" \
  "ds_read_b64_tr_b16 v[114:115], v22 offset:34816\n\t" \
  "v_mfma_f32_16x16x32_bf16 v[212:215], a[236:239], a[172:175], v[212:215]\n\t" \
  "ds_read_b64_tr_b16 v[116:117], v22 offset:49152\n\t" \
  "v_mfma_f32_16x16x32_bf16 v[208:211], a[224:227], a[168:171], v[208:211]\n\t" \
  "ds_read_b64_tr_b16 v[118:119], v22 offset:51200\n\t" \
  "v_mfma_f32_16x16x32_bf16 v[208:211], a[228:231], a[172:175], v[208:211]\n\t" \
  "buffer_load_dwordx4 v15, s[40:43], s84 offen offset:3072 sc0 sc1 lds\n\t" \
  "v_mfma_f32_16x16x32_bf16 v[224:227], a[224:227], a[176:179], v[224:227]\n\t" \
  "ds_read_b128 a[168:171], v24 offset:43008\n\t" \
  "v_mfma_f32_16x16x32_bf16 v[224:227], a[228:231], a[180:183], v[224:227]\n\t" \
  "ds_read_b128 a[172:175], v25 offset:43008\n\t" \
  "v_mfma_f32_16x16x32_bf16 v[228:231], a[232:235], a[176:179], v[228:231]\n\t" \
  "buffer_load_dword v92, v27, s[48:51], s89 offen sc0 sc1\n\t" \
  "v_mfma_f32_16x16x32_bf16 v[228:231], a[236:239], a[180:183], v[228:231]\n\t" \
  "ds_read_b64_tr_b16 v[120:121], v23 offset:32768\n\t" \
  "v_mfma_f32_16x16x32_bf16 v[232:235], a[240:243], a[176:179], v[232:235]\n\t" \
  "buffer_store_dwordx4 v[60:63], v72, s[52:55], s21 offen sc0 sc1\n\t" \
  "v_mfma_f32_16x16x32_bf16 v[232:235], a[244:247], a[180:183], v[232:235]\n\t" \
  "ds_read_b64_tr_b16 v[122:123], v23 offset:34816\n\t" \
  "v_mfma_f32_16x16x32_bf16 v[236:239], a[248:251], a[176:179], v[236:239]\n\t" \
  "ds_read_b64_tr_b16 v[124:125], v23 offset:49152\n\t" \
  "v_mfma_f32_16x16x32_bf16 v[236:239], a[252:255], a[180:183], v[236:239]\n\t" \
  "ds_read_b64_tr_b16 v[126:127], v23 offset:51200\n\t" \
  "v_mfma_f32_16x16x32_bf16 v[252:255], a[248:251], a[184:187], v[252:255]\n\t" \
  "ds_read_b128 a[176:179], v24 offset:45056\n\t" \
  "v_mfma_f32_16x16x32_bf16 v[252:255], a[252:255], a[188:191], v[252:255]\n\t" \
  "ds_read_b128 a[180:183], v25 offset:45056\n\t" \
  "v_mfma_f32_16x16x32_bf16 v[248:251], a[240:243], a[184:187], v[248:251]\n\t" \
  "v_mfma_f32_16x16x32_bf16 v[248:251], a[244:247], a[188:191], v[248:251]\n\t" \
  "v_mfma_f32_16x16x32_bf16 v[244:247], a[232:235], a[184:187], v[244:247]\n\t" \
  "v_mfma_f32_16x16x32_bf16 v[244:247], a[236:239], a[188:191], v[244:247]\n\t" \
  "v_mfma_f32_16x16x32_bf16 v[240:243], a[224:227], a[184:187], v[240:243]\n\t" \
  "v_mfma_f32_16x16x32_bf16 v[240:243], a[228:231], a[188:191], v[240:243]\n\t" \
  "s_waitcnt lgkmcnt(4)\n\t" \
  "v_mfma_f32_16x16x32_bf16 a[0:3], a[192:195], a[128:131], a[0:3]\n\t" \
  "ds_read_b128 a[184:187], v24 offset:47104\n\t" \
  "v_mfma_f32_16x16x32_bf16 a[0:3], a[196:199], a[132:135], a[0:3]\n\t" \
  "ds_read_b128 a[188:191], v25 offset:47104\n\t" \
  "v_cmp_gt_u32 vcc, s91, v93\n\t" \
  "v_mfma_f32_16x16x32_bf16 a[4:7], a[200:203], a[128:131], a[4:7]\n\t" \
  "v_mfma_f32_16x16x32_bf16 a[4:7], a[204:207], a[132:135], a[4:7]\n\t" \
  "s_cbranch_vccnz Lsp8_%=\n\t" \
  "Lspr8_%=:\n\t" \
  "v_mfma_f32_16x16x32_bf16 a[8:11], a[208:211], a[128:131], a[8:11]\n\t" \
  "s_mov_b32 s85, s86\n\t" \
  "s_mov_b32 s86, s73\n\t" \
  "v_mfma_f32_16x16x32_bf16 a[8:11], a[212:215], a[132:135], a[8:11]\n\t" \
  "s_lshl_b32 s84, s85, s74\n\t" \
  "s_add_u32 s84, s84, s75\n\t" \
  "v_mfma_f32_16x16x32_bf16 a[12:15], a[216:219], a[128:131], a[12:15]\n\t" \
  "s_waitcnt lgkmcnt(0)\n\t" \
  "s_barrier\n\t" \
  "s_add_u32 m0, s24, 32768\n\t" \
  "v_mfma_f32_16x16x32_bf16 a[12:15], a[220:223], a[132:135], a[12:15]\n\t" \
  "buffer_load_dwordx4 v0, s[40:43], s84 offen offset:0 sc0 sc1 lds\n\t" \
  "v_mfma_f32_16x16x32_bf16 a[28:31], a[216:219], a[136:139], a[28:31]\n\t" \
  "v_mfma_f32_16x16x32_bf16 a[28:31], a[220:223], a[140:143], a[28:31]\n\t" \
  "buffer_load_dwordx4 v1, s[40:43], s84 offen offset:1024 sc0 sc1 lds\n\t" \
  "v_mfma_f32_16x16x32_bf16 a[24:27], a[208:211], a[136:139], a[24:27]\n\t" \
  "v_mfma_f32_16x16x32_bf16 a[24:27], a[212:215], a[140:143], a[24:27]\n\t" \
  "buffer_load_dwordx4 v2, s[40:43], s84 offen offset:2048 sc0 sc1 lds\n\t" \
  "v_mfma_f32_16x16x32_bf16 a[20:23], a[200:203], a[136:139], a[20:23]\n\t" \
  "v_mfma_f32_16x16x32_bf16 a[20:23], a[204:207], a[140:143], a[20:23]\n\t" \
  "buffer_load_dwordx4 v3, s[40:43], s84 offen offset:3072 sc0 sc1 lds\n\t" \
  "v_mfma_f32_16x16x32_bf16 a[16:19], a[192:195], a[136:139], a[16:19]\n\t" \
  "v_mfma_f32_16x16x32_bf16 a[16:19], a[196:199], a[140:143], a[16:19]\n\t" \
  "v_mfma_f32_16x16x32_bf16 a[32:35], a[192:195], a[144:147], a[32:35]\n\t" \
  "v_mfma_f32_16x16x32_bf16 a[32:35], a[196:199], a[148:151], a[32:35]\n\t" \
  "v_mfma_f32_16x16x32_bf16 a[36:39], a[200:203], a[144:147], a[36:39]\n\t" \
  "v_mfma_f32_16x16x32_bf16 a[36:39], a[204:207], a[148:151], a[36:39]\n\t" \
  "v_mfma_f32_16x16x32_bf16 a[40:43], a[208:211], a[144:147], a[40:43]\n\t" \
  "v_mfma_f32_16x16x32_bf16 a[40:43], a[212:215], a[148:151], a[40:43]\n\t" \
  "v_mfma_f32_16x16x32_bf16 a[44:47], a[216:219], a[144:147], a[44:47]\n\t" \
  "v_mfma_f32_16x16x32_bf16 a[44:47], a[220:223], a[148:151], a[44:47]\n\t" \
  "v_mfma_f32_16x16x32_bf16 a[60:63], a[216:219], a[152:155], a[60:63]\n\t" \
  "v_mfma_f32_16x16x32_bf16 a[60:63], a[220:223], a[156:159], a[60:63]\n\t" \
  "v_mfma_f32_16x16x32_bf16 a[56:59], a[208:211], a[152:155], a[56:59]\n\t" \
  "v_mfma_f32_16x16x32_bf16 a[56:59], a[212:215], a[156:159], a[56:59]\n\t" \
  "v_mfma_f32_16x16x32_bf16 a[52:55], a[200:203], a[152:155], a[52:55]\n\t" \
  "v_mfma_f32_16x16x32_bf16 a[52:55], a[204:207], a[156:159], a[52:55]\n\t" \
  "v_mfma_f32_16x16x32_bf16 a[48:51], a[192:195], a[152:155], a[48:51]\n\t" \
  "v_mfma_f32_16x16x32_bf16 a[48:51], a[196:199], a[156:159], a[48:51]\n\t" \
  "s_lshr_b64 s[10:11], s[8:9], s87\n\t" \
  "s_sext_i32_i8 s10, s10\n\t" \
  "v_mfma_f32_16x16x32_bf16 a[64:67], a[192:195], a[160:163], a[64:67]\n\t" \
  "s_add_u32 s87, s87, 8\n\t" \
  "s_add_u32 s73, s73, s10\n\t" \
  "v_mfma_f32_16x16x32_bf16 a[64:67], a[196:199], a[164:167], a[64:67]\n\t" \
  "s_cmp_eq_u32 s87, 56\n\t" \
  "s_cselect_b32 s87, 0, s87\n\t" \
  "v_mfma_f32_16x16x32_bf16 a[68:71], a[200:203], a[160:163], a[68:71]\n\t" \
  "v_mfma_f32_16x16x32_bf16 a[68:71], a[204:207], a[164:167], a[68:71]\n\t" \
  "v_mfma_f32_16x16x32_bf16 a[72:75], a[208:211], a[160:163], a[72:75]\n\t" \
  "v_mfma_f32_16x16x32_bf16 a[72:75], a[212:215], a[164:167], a[72:75]\n\t" \
  "v_mfma_f32_16x16x32_bf16 a[76:79], a[216:219], a[160:163], a[76:79]\n\t" \
  "v_mfma_f32_16x16x32_bf16 a[76:79], a[220:223], a[164:167], a[76:79]\n\t" \
  "v_mfma_f32_16x16x32_bf16 a[92:95], a[216:219], a[168:171], a[92:95]\n\t" \
  "v_mfma_f32_16x16x32_bf16 a[92:95], a[220:223], a[172:175], a[92:95]\n\t" \
  "v_mfma_f32_16x16x32_bf16 a[88:91], a[208:211], a[168:171], a[88:91]\n\t" \
  "v_mfma_f32_16x16x32_bf16 a[88:91], a[212:215], a[172:175], a[88:91]\n\t" \
  "v_mfma_f32_16x16x32_bf16 a[84:87], a[200:203], a[168:171], a[84:87]\n\t" \
  "v_mfma_f32_16x16x32_bf16 a[84:87], a[204:207], a[172:175], a[84:87]\n\t" \
  "v_mfma_f32_16x16x32_bf16 a[80:83], a[192:195], a[168:171], a[80:83]\n\t" \
  "v_mfma_f32_16x16x32_bf16 a[80:83], a[196:199], a[172:175], a[80:83]\n\t" \
  "v_mfma_f32_16x16x32_bf16 a[96:99], a[192:195], a[176:179], a[96:99]\n\t" \
  "v_mfma_f32_16x16x32_bf16 a[96:99], a[196:199], a[180:183], a[96:99]\n\t" \
  "v_mfma_f32_16x16x32_bf16 a[100:103], a[200:203], a[176:179], a[100:103]\n\t" \
  "v_mfma_f32_16x16x32_bf16 a[100:103], a[204:207], a[180:183], a[100:103]\n\t" \
  "v_mfma_f32_16x16x32_bf16 a[104:107], a[208:211], a[176:179], a[104:107]\n\t" \
  "v_mfma_f32_16x16x32_bf16 a[104:107], a[212:215], a[180:183], a[104:107]\n\t" \
  "v_mfma_f32_16x16x32_bf16 a[108:111], a[216:219], a[176:179], a[108:111]\n\t" \
  "v_mfma_f32_16x16x32_bf16 a[108:111], a[220:223], a[180:183], a[108:111]\n\t" \
  "s_add_u32 m0, s25, 32768\n\t" \
  "v_mfma_f32_16x16x32_bf16 a[124:127], a[216:219], a[184:187], a[124:127]\n\t" \
  "buffer_load_dwordx4 v4, s[40:43], s84 offen offset:0 sc0 sc1 lds\n\t" \
  "v_mfma_f32_16x16x32_bf16 a[124:127], a[220:223], a[188:191], a[124:127]\n\t" \
  "v_mfma_f32_16x16x32_bf16 a[120:123], a[208:211], a[184:187], a[120:123]\n\t" \
  "v_mfma_f32_16x16x32_bf16 a[120:123], a[212:215], a[188:191], a[120:123]\n\t" \
  "v_mfma_f32_16x16x32_bf16 a[116:119], a[200:203], a[184:187], a[116:119]\n\t" \
  "v_mfma_f32_16x16x32_bf16 a[116:119], a[204:207], a[188:191], a[116:119]\n\t" \
  "buffer_load_dwordx4 v5, s[40:43], s84 offen offset:1024 sc0 sc1 lds\n\t" \
  "v_mfma_f32_16x16x32_bf16 a[112:115], a[192:195], a[184:187], a[112:115]\n\t" \
  "v_mfma_f32_16x16x32_bf16 a[112:115], a[196:199], a[188:191], a[112:115]\n\t" \
  "s_waitcnt vmcnt(7)\n\t" \
  "s_barrier\n\t" \
  "v_mfma_f32_16x16x32_bf16 v[128:131], v[96:99], a[128:131], v[128:131]\n\t" \
  "ds_read_b64_tr_b16 a[192:193], v16 offset:0\n\t" \
  "v_mfma_f32_16x16x32_bf16 v[128:131], v[100:103], a[132:135], v[128:131]\n\t" \
  "ds_read_b64_tr_b16 a[194:195], v16 offset:2048\n\t" \
  "v_mfma_f32_16x16x32_bf16 v[132:135], v[104:107], a[128:131], v[132:135]\n\t" \
  "buffer_load_dwordx4 v6, s[40:43], s84 offen offset:2048 sc0 sc1 lds\n\t" \
  "v_mfma_f32_16x16x32_bf16 v[132:135], v[108:111], a[132:135], v[132:135]\n\t" \
  "ds_read_b64_tr_b16 a[196:197], v16 offset:16384\n\t" \
  "v_mfma_f32_16x16x32_bf16 v[136:139], v[112:115], a[128:131], v[136:139]\n\t" \
  "ds_read_b64_tr_b16 a[198:199], v16 offset:18432\n\t" \
  "v_mfma_f32_16x16x32_bf16 v[136:139], v[116:119], a[132:135], v[136:139]\n\t" \
  "ds_read_b64_tr_b16 a[200:201], v17 offset:0\n\t" \
  "v_mfma_f32_16x16x32_bf16 v[140:143], v[120:123], a[128:131], v[140:143]\n\t" \
  "ds_read_b64_tr_b16 a[202:203], v17 offset:2048\n\t" \
  "v_mfma_f32_16x16x32_bf16 v[140:143], v[124:127], a[132:135], v[140:143]\n\t" \
  "buffer_load_dwordx4 v7, s[40:43], s84 offen offset:3072 sc0 sc1 lds\n\t" \
  "v_mfma_f32_16x16x32_bf16 v[156:159], v[120:123], a[136:139], v[156:159]\n\t" \
  "ds_read_b128 a[128:131], v24 offset:0\n\t" \
  "v_mfma_f32_16x16x32_bf16 v[156:159], v[124:127], a[140:143], v[156:159]\n\t" \
  "ds_read_b128 a[132:135], v25 offset:0\n\t" \
  "v_mfma_f32_16x16x32_bf16 v[152:155], v[112:115], a[136:139], v[152:155]\n\t" \
  "ds_read_b64_tr_b16 a[204:205], v17 offset:16384\n\t" \
  "v_mfma_f32_16x16x32_bf16 v[152:155], v[116:119], a[140:143], v[152:155]\n\t" \
  "ds_read_b64_tr_b16 a[206:207], v17 offset:18432\n\t" \
  "s_add_u32 m0, s26, 32768\n\t" \
  "v_mfma_f32_16x16x32_bf16 v[148:151], v[104:107], a[136:139], v[148:151]\n\t" \
  "buffer_load_dwordx4 v8, s[40:43], s84 offen offset:0 sc0 sc1 lds\n\t" \
  "v_mfma_f32_16x16x32_bf16 v[148:151], v[108:111], a[140:143], v[148:151]\n\t" \
  "ds_read_b64_tr_b16 a[208:209], v18 offset:0\n\t" \
  "v_mfma_f32_16x16x32_bf16 v[144:147], v[96:99], a[136:139], v[144:147]\n\t" \
  "ds_read_b64_tr_b16 a[210:211], v18 offset:2048\n\t" \
  "v_mfma_f32_16x16x32_bf16 v[144:147], v[100:103], a[140:143], v[144:147]\n\t" \
  "ds_read_b64_tr_b16 a[212:213], v18 offset:16384\n\t" \
  "v_mfma_f32_16x16x32_bf16 v[160:163], v[96:99], a[144:147], v[160:163]\n\t" \
  "ds_read_b128 a[136:139], v24 offset:2048\n\t" \
  "v_mfma_f32_16x16x32_bf16 v[160:163], v[100:103], a[148:151], v[160:163]\n\t" \
  "buffer_load_dwordx4 v9, s[40:43], s84 offen offset:1024 sc0 sc1 lds\n\t" \
  "v_mfma_f32_16x16x32_bf16 v[164:167], v[104:107], a[144:147], v[164:167]\n\t" \
  "ds_read_b128 a[140:143], v25 offset:2048\n\t" \
  "v_mfma_f32_16x16x32_bf16 v[164:167], v[108:111], a[148:151], v[164:167]\n\t" \
  "ds_read_b64_tr_b16 a[214:215], v18 offset:18432\n\t" \
  "v_mfma_f32_16x16x32_bf16 v[168:171], v[112:115], a[144:147], v[168:171]\n\t" \
  "ds_read_b64_tr_b16 a[216:217], v19 offset:0\n\t" \
  "v_mfma_f32_16x16x32_bf16 v[168:171], v[116:119], a[148:151], v[168:171]\n\t" \
  "ds_read_b64_tr_b16 a[218:219], v19 offset:2048\n\t" \
  "v_mfma_f32_16x16x32_bf16 v[172:175], v[120:123], a[144:147], v[172:175]\n\t" \
  "buffer_load_dwordx4 v10, s[40:43], s84 offen offset:2048 sc0 sc1 lds\n\t" \
  "v_mfma_f32_16x16x32_bf16 v[172:175], v[124:127], a[148:151], v[172:175]\n\t" \
  "ds_read_b64_tr_b16 a[220:221], v19 offset:16384\n\t" \
  "v_mfma_f32_16x16x32_bf16 v[188:191], v[120:123], a[152:155], v[188:191]\n\t" \
  "ds_read_b128 a[144:147], v24 offset:4096\n\t" \
  "v_mfma_f32_16x16x32_bf16 v[188:191], v[124:127], a[156:159], v[188:191]\n\t" \
  "ds_read_b128 a[148:151], v25 offset:4096\n\t" \
  "v_mfma_f32_16x16x32_bf16 v[184:187], v[112:115], a[152:155], v[184:187]\n\t" \
  "ds_read_b64_tr_b16 a[222:223], v19 offset:18432\n\t" \
  "v_mfma_f32_16x16x32_bf16 v[184:187], v[116:119], a[156:159], v[184:187]\n\t" \
  "buffer_load_dwordx4 v11, s[40:43], s84 offen offset:3072 sc0 sc1 lds\n\t" \
  "v_mfma_f32_16x16x32_bf16 v[180:183], v[104:107], a[152:155], v[180:183]\n\t" \
  "ds_read_b64_tr_b16 a[224:225], v20 offset:0\n\t" \
  "v_mfma_f32_16x16x32_bf16 v[180:183], v[108:111], a[156:159], v[180:183]\n\t" \
  "ds_read_b64_tr_b16 a[226:227], v20 offset:2048\n\t" \
  "v_mfma_f32_16x16x32_bf16 v[176:179], v[96:99], a[152:155], v[176:179]\n\t" \
  "ds_read_b64_tr_b16 a[228:229], v20 offset:16384\n\t" \
  "v_mfma_f32_16x16x32_bf16 v[176:179], v[100:103], a[156:159], v[176:179]\n\t" \
  "ds_read_b64_tr_b16 a[230:231], v20 offset:18432\n\t" \
  "s_add_u32 m0, s27, 32768\n\t" \
  "v_mfma_f32_16x16x32_bf16 v[192:195], v[96:99], a[160:163], v[192:195]\n\t" \
  "buffer_load_dwordx4 v12, s[40:43], s84 offen offset:0 sc0 sc1 lds\n\t" \
  "v_mfma_f32_16x16x32_bf16 v[192:195], v[100:103], a[164:167], v[192:195]\n\t" \
  "ds_read_b128 a[152:155], v24 offset:6144\n\t" \
  "v_mfma_f32_16x16x32_bf16 v[196:199], v[104:107], a[160:163], v[196:199]\n\t" \
  "ds_read_b128 a[156:159], v25 offset:6144\n\t" \
  "v_mfma_f32_16x16x32_bf16 v[196:199], v[108:111], a[164:167], v[196:199]\n\t" \
  "ds_read_b64_tr_b16 a[232:233], v21 offset:0\n\t" \
  "v_mfma_f32_16x16x32_bf16 v[200:203], v[112:115], a[160:163], v[200:203]\n\t" \
  "s_mov_b32 s90, s89\n\t" \
  "ds_read_b64_tr_b16 a[234:235], v21 offset:2048\n\t" \
  "v_mfma_f32_16x16x32_bf16 v[200:203], v[116:119], a[164:167], v[200:203]\n\t" \
  "buffer_load_dwordx4 v13, s[40:43], s84 offen offset:1024 sc0 sc1 lds\n\t" \
  "s_lshl_b32 s89, s73, 9\n\t" \
  "v_mfma_f32_16x16x32_bf16 v[204:207], v[120:123], a[160:163], v[204:207]\n\t" \
  "ds_read_b64_tr_b16 a[236:237], v21 offset:16384\n\t" \
  "v_mfma_f32_16x16x32_bf16 v[204:207], v[124:127], a[164:167], v[204:207]\n\t" \
  "ds_read_b64_tr_b16 a[238:239], v21 offset:18432\n\t" \
  "v_mfma_f32_16x16x32_bf16 v[220:223], v[120:123], a[168:171], v[220:223]\n\t" \
  "ds_read_b128 a[160:163], v24 offset:8192\n\t" \
  "v_mfma_f32_16x16x32_bf16 v[220:223], v[124:127], a[172:175], v[220:223]\n\t" \
  "ds_read_b128 a[164:167], v25 offset:8192\n\t" \
  "v_mfma_f32_16x16x32_bf16 v[216:219], v[112:115], a[168:171], v[216:219]\n\t" \
  "buffer_load_dwordx4 v14, s[40:43], s84 offen offset:2048 sc0 sc1 lds\n\t" \
  "v_mfma_f32_16x16x32_bf16 v[216:219], v[116:119], a[172:175], v[216:219]\n\t" \
  "ds_read_b64_tr_b16 a[240:241], v22 offset:0\n\t" \
  "v_mfma_f32_16x16x32_bf16 v[212:215], v[104:107], a[168:171], v[212:215]\n\t" \
  "ds_read_b64_tr_b16 a[242:243], v22 offset:2048\n\t" \
  "v_mfma_f32_16x16x32_bf16 v[212:215], v[108:111], a[172:175], v[212:215]\n\t" \
  "ds_read_b64_tr_b16 a[244:245], v22 offset:16384\n\t" \
  "v_mfma_f32_16x16x32_bf16 v[208:211], v[96:99], a[168:171], v[208:211]\n\t" \
  "ds_read_b64_tr_b16 a[246:247], v22 offset:18432\n\t" \
  "v_mfma_f32_16x16x32_bf16 v[208:211], v[100:103], a[172:175], v[208:211]\n\t" \
  "buffer_load_dwordx4 v15, s[40:43], s84 offen offset:3072 sc0 sc1 lds\n\t" \
  "v_mfma_f32_16x16x32_bf16 v[224:227], v[96:99], a[176:179], v[224:227]\n\t" \
  "ds_read_b128 a[168:171], v24 offset:10240\n\t" \
  "v_mfma_f32_16x16x32_bf16 v[224:227], v[100:103], a[180:183], v[224:227]\n\t" \
  "ds_read_b128 a[172:175], v25 offset:10240\n\t" \
  "v_mfma_f32_16x16x32_bf16 v[228:231], v[104:107], a[176:179], v[228:231]\n\t" \
  "buffer_load_dword v93, v27, s[48:51], s89 offen sc0 sc1\n\t" \
  "v_mfma_f32_16x16x32_bf16 v[228:231], v[108:111], a[180:183], v[228:231]\n\t" \
  "ds_read_b64_tr_b16 a[248:249], v23 offset:0\n\t" \
  "v_mfma_f32_16x16x32_bf16 v[232:235], v[112:115], a[176:179], v[232:235]\n\t" \
  "buffer_store_dwordx4 v[60:63], v73, s[52:55], s21 offen sc0 sc1\n\t" \
  "v_mfma_f32_16x16x32_bf16 v[232:235], v[116:119], a[180:183], v[232:235]\n\t" \
  "ds_read_b64_tr_b16 a[250:251], v23 offset:2048\n\t" \
  "v_mfma_f32_16x16x32_bf16 v[236:239], v[120:123], a[176:179], v[236:239]\n\t" \
  "ds_read_b64_tr_b16 a[252:253], v23 offset:16384\n\t" \
  "v_mfma_f32_16x16x32_bf16 v[236:239], v[124:127], a[180:183], v[236:239]\n\t" \
  "buffer_store_dword v43, v31, s[64:67], s20 offen sc0 sc1\n\t" \
  "v_mfma_f32_16x16x32_bf16 v[252:255], v[120:123], a[184:187], v[252:255]\n\t" \
  "ds_read_b128 a[176:179], v24 offset:12288\n\t" \
  "v_mfma_f32_16x16x32_bf16 v[252:255], v[124:127], a[188:191], v[252:255]\n\t" \
  "ds_read_b128 a[180:183], v25 offset:12288\n\t" \
  "v_mfma_f32_16x16x32_bf16 v[248:251], v[112:115], a[184:187], v[248:251]\n\t" \
  "buffer_load_dwordx4 v[56:59], v29, s[56:59], s80 offen\n\t" \
  "v_mfma_f32_16x16x32_bf16 v[248:251], v[116:119], a[188:191], v[248:251]\n\t" \
  "ds_read_b64_tr_b16 a[254:255], v23 offset:18432\n\t" \
  "v_mfma_f32_16x16x32_bf16 v[244:247], v[104:107], a[184:187], v[244:247]\n\t" \
  "v_mfma_f32_16x16x32_bf16 v[244:247], v[108:111], a[188:191], v[244:247]\n\t" \
  "v_mfma_f32_16x16x32_bf16 v[240:243], v[96:99], a[184:187], v[240:243]\n\t" \
  "v_mfma_f32_16x16x32_bf16 v[240:243], v[100:103], a[188:191], v[240:243]\n\t" \
  "s_waitcnt lgkmcnt(4)\n\t" \
  "v_mfma_f32_16x16x32_bf16 a[0:3], a[192:195], a[128:131], a[0:3]\n\t" \
  "ds_read_b128 a[184:187], v24 offset:14336\n\t" \
  "v_mfma_f32_16x16x32_bf16 a[0:3], a[196:199], a[132:135], a[0:3]\n\t" \
  "ds_read_b128 a[188:191], v25 offset:14336\n\t" \
  "v_cmp_gt_u32 vcc, s91, v92\n\t" \
  "v_mfma_f32_16x16x32_bf16 a[4:7], a[200:203], a[128:131], a[4:7]\n\t" \
  "v_mfma_f32_16x16x32_bf16 a[4:7], a[204:207], a[132:135], a[4:7]\n\t" \
  "s_cbranch_vccnz Lsp9_%=\n\t" \
  "Lspr9_%=:\n\t" \
  "v_mfma_f32_16x16x32_bf16 a[8:11], a[208:211], a[128:131], a[8:11]\n\t" \
  "s_mov_b32 s85, s86\n\t" \
  "s_mov_b32 s86, s73\n\t" \
  "v_mfma_f32_16x16x32_bf16 a[8:11], a[212:215], a[132:135], a[8:11]\n\t" \
  "s_lshl_b32 s84, s85, s74\n\t" \
  "s_add_u32 s84, s84, s75\n\t" \
  "v_mfma_f32_16x16x32_bf16 a[12:15], a[216:219], a[128:131], a[12:15]\n\t" \
  "s_waitcnt lgkmcnt(0)\n\t" \
  "s_barrier\n\t" \
  "s_add_u32 m0, s24, 0\n\t" \
  "v_mfma_f32_16x16x32_bf16 a[12:15], a[220:223], a[132:135], a[12:15]\n\t" \
  "buffer_load_dwordx4 v0, s[40:43], s84 offen offset:0 sc0 sc1 lds\n\t" \
  "v_mfma_f32_16x16x32_bf16 a[28:31], a[216:219], a[136:139], a[28:31]\n\t" \
  "v_mfma_f32_16x16x32_bf16 a[28:31], a[220:223], a[140:143], a[28:31]\n\t" \
  "buffer_load_dwordx4 v1, s[40:43], s84 offen offset:1024 sc0 sc1 lds\n\t" \
  "v_mfma_f32_16x16x32_bf16 a[24:27], a[208:211], a[136:139], a[24:27]\n\t" \
  "v_mfma_f32_16x16x32_bf16 a[24:27], a[212:215], a[140:143], a[24:27]\n\t" \
  "buffer_load_dwordx4 v2, s[40:43], s84 offen offset:2048 sc0 sc1 lds\n\t" \
  "v_mfma_f32_16x16x32_bf16 a[20:23], a[200:203], a[136:139], a[20:23]\n\t" \
  "v_mfma_f32_16x16x32_bf16 a[20:23], a[204:207], a[140:143], a[20:23]\n\t" \
  "buffer_load_dwordx4 v3, s[40:43], s84 offen offset:3072 sc0 sc1 lds\n\t" \
  "v_mfma_f32_16x16x32_bf16 a[16:19], a[192:195], a[136:139], a[16:19]\n\t" \
  "v_mfma_f32_16x16x32_bf16 a[16:19], a[196:199], a[140:143], a[16:19]\n\t" \
  "v_mfma_f32_16x16x32_bf16 a[32:35], a[192:195], a[144:147], a[32:35]\n\t" \
  "v_mfma_f32_16x16x32_bf16 a[32:35], a[196:199], a[148:151], a[32:35]\n\t" \
  "v_mfma_f32_16x16x32_bf16 a[36:39], a[200:203], a[144:147], a[36:39]\n\t" \
  "v_mfma_f32_16x16x32_bf16 a[36:39], a[204:207], a[148:151], a[36:39]\n\t" \
  "v_mfma_f32_16x16x32_bf16 a[40:43], a[208:211], a[144:147], a[40:43]\n\t" \
  "v_mfma_f32_16x16x32_bf16 a[40:43], a[212:215], a[148:151], a[40:43]\n\t" \
  "v_mfma_f32_16x16x32_bf16 a[44:47], a[216:219], a[144:147], a[44:47]\n\t" \
  "v_mfma_f32_16x16x32_bf16 a[44:47], a[220:223], a[148:151], a[44:47]\n\t" \
  "v_mfma_f32_16x16x32_bf16 a[60:63], a[216:219], a[152:155], a[60:63]\n\t" \
  "v_mfma_f32_16x16x32_bf16 a[60:63], a[220:223], a[156:159], a[60:63]\n\t" \
  "v_mfma_f32_16x16x32_bf16 a[56:59], a[208:211], a[152:155], a[56:59]\n\t" \
  "v_mfma_f32_16x16x32_bf16 a[56:59], a[212:215], a[156:159], a[56:59]\n\t" \
  "v_mfma_f32_16x16x32_bf16 a[52:55], a[200:203], a[152:155], a[52:55]\n\t" \
  "v_mfma_f32_16x16x32_bf16 a[52:55], a[204:207], a[156:159], a[52:55]\n\t" \
  "v_mfma_f32_16x16x32_bf16 a[48:51], a[192:195], a[152:155], a[48:51]\n\t" \
  "v_mfma_f32_16x16x32_bf16 a[48:51], a[196:199], a[156:159], a[48:51]\n\t" \
  "s_lshr_b64 s[10:11], s[8:9], s87\n\t" \
  "s_sext_i32_i8 s10, s10\n\t" \
  "v_mfma_f32_16x16x32_bf16 a[64:67], a[192:195], a[160:163], a[64:67]\n\t" \
  "s_add_u32 s87, s87, 8\n\t" \
  "s_add_u32 s73, s73, s10\n\t" \
  "v_mfma_f32_16x16x32_bf16 a[64:67], a[196:199], a[164:167], a[64:67]\n\t" \
  "s_cmp_eq_u32 s87, 56\n\t" \
  "s_cselect_b32 s87, 0, s87\n\t" \
  "v_mfma_f32_16x16x32_bf16 a[68:71], a[200:203], a[160:163], a[68:71]\n\t" \
  "v_mfma_f32_16x16x32_bf16 a[68:71], a[204:207], a[164:167], a[68:71]\n\t" \
  "v_mfma_f32_16x16x32_bf16 a[72:75], a[208:211], a[160:163], a[72:75]\n\t" \
  "v_mfma_f32_16x16x32_bf16 a[72:75], a[212:215], a[164:167], a[72:75]\n\t" \
  "v_mfma_f32_16x16x32_bf16 a[76:79], a[216:219], a[160:163], a[76:79]\n\t" \
  "v_mfma_f32_16x16x32_bf16 a[76:79], a[220:223], a[164:167], a[76:79]\n\t" \
  "v_mfma_f32_16x16x32_bf16 a[92:95], a[216:219], a[168:171], a[92:95]\n\t" \
  "v_mfma_f32_16x16x32_bf16 a[92:95], a[220:223], a[172:175], a[92:95]\n\t" \
  "v_mfma_f32_16x16x32_bf16 a[88:91], a[208:211], a[168:171], a[88:91]\n\t" \
  "v_mfma_f32_16x16x32_bf16 a[88:91], a[212:215], a[172:175], a[88:91]\n\t" \
  "v_mfma_f32_16x16x32_bf16 a[84:87], a[200:203], a[168:171], a[84:87]\n\t" \
  "v_mfma_f32_16x16x32_bf16 a[84:87], a[204:207], a[172:175], a[84:87]\n\t" \
  "v_mfma_f32_16x16x32_bf16 a[80:83], a[192:195], a[168:171], a[80:83]\n\t" \
  "v_mfma_f32_16x16x32_bf16 a[80:83], a[196:199], a[172:175], a[80:83]\n\t" \
  "v_mfma_f32_16x16x32_bf16 a[96:99], a[192:195], a[176:179], a[96:99]\n\t" \
  "v_mfma_f32_16x16x32_bf16 a[96:99], a[196:199], a[180:183], a[96:99]\n\t" \
  "v_mfma_f32_16x16x32_bf16 a[100:103], a[200:203], a[176:179], a[100:103]\n\t" \
  "v_mfma_f32_16x16x32_bf16 a[100:103], a[204:207], a[180:183], a[100:103]\n\t" \
  "v_mfma_f32_16x16x32_bf16 a[104:107], a[208:211], a[176:179], a[104:107]\n\t" \
  "v_mfma_f32_16x16x32_bf16 a[104:107], a[212:215], a[180:183], a[104:107]\n\t" \
  "v_mfma_f32_16x16x32_bf16 a[108:111], a[216:219], a[176:179], a[108:111]\n\t" \
  "v_mfma_f32_16x16x32_bf16 a[108:111], a[220:223], a[180:183], a[108:111]\n\t" \
  "s_add_u32 m0, s25, 0\n\t" \
  "v_mfma_f32_16x16x32_bf16 a[124:127], a[216:219], a[184:187], a[124:127]\n\t" \
  "buffer_load_dwordx4 v4, s[40:43], s84 offen offset:0 sc0 sc1 lds\n\t" \
  "v_mfma_f32_16x16x32_bf16 a[124:127], a[220:223], a[188:191], a[124:127]\n\t" \
  "v_mfma_f32_16x16x32_bf16 a[120:123], a[208:211], a[184:187], a[120:123]\n\t" \
  "v_mfma_f32_16x16x32_bf16 a[120:123], a[212:215], a[188:191], a[120:123]\n\t" \
  "v_mfma_f32_16x16x32_bf16 a[116:119], a[200:203], a[184:187], a[116:119]\n\t" \
  "v_mfma_f32_16x16x32_bf16 a[116:119], a[204:207], a[188:191], a[116:119]\n\t" \
  "buffer_load_dwordx4 v5, s[40:43], s84 offen offset:1024 sc0 sc1 lds\n\t" \
  "v_mfma_f32_16x16x32_bf16 a[112:115], a[192:195], a[184:187], a[112:115]\n\t" \
  "v_mfma_f32_16x16x32_bf16 a[112:115], a[196:199], a[188:191], a[112:115]\n\t" \
  "s_waitcnt vmcnt(9)\n\t" \
  "s_barrier\n\t" \
  "v_mfma_f32_16x16x32_bf16 v[128:131], a[224:227], a[128:131], v[128:131]\n\t" \
  "ds_read_b64_tr_b16 a[192:193], v16 offset:32768\n\t" \
  "v_mfma_f32_16x16x32_bf16 v[128:131], a[228:231], a[132:135], v[128:131]\n\t" \
  "ds_read_b64_tr_b16 a[194:195], v16 offset:34816\n\t" \
  "v_mfma_f32_16x16x32_bf16 v[132:135], a[232:235], a[128:131], v[132:135]\n\t" \
  "buffer_load_dwordx4 v6, s[40:43], s84 offen offset:2048 sc0 sc1 lds\n\t" \
  "v_mfma_f32_16x16x32_bf16 v[132:135], a[236:239], a[132:135], v[132:135]\n\t" \
  "ds_read_b64_tr_b16 a[196:197], v16 offset:49152\n\t" \
  "v_mfma_f32_16x16x32_bf16 v[136:139], a[240:243], a[128:131], v[136:139]\n\t" \
  "ds_read_b64_tr_b16 a[198:199], v16 offset:51200\n\t" \
  "v_mfma_f32_16x16x32_bf16 v[136:139], a[244:247], a[132:135], v[136:139]\n\t" \
  "ds_read_b64_tr_b16 a[200:201], v17 offset:32768\n\t" \
  "v_mfma_f32_16x16x32_bf16 v[140:143], a[248:251], a[128:131], v[140:143]\n\t" \
  "ds_read_b64_tr_b16 a[202:203], v17 offset:34816\n\t" \
  "v_mfma_f32_16x16x32_bf16 v[140:143], a[252:255], a[132:135], v[140:143]\n\t" \
  "buffer_load_dwordx4 v7, s[40:43], s84 offen offset:3072 sc0 sc1 lds\n\t" \
  "v_mfma_f32_16x16x32_bf16 v[156:159], a[248:251], a[136:139], v[156:159]\n\t" \
  "ds_read_b128 a[128:131], v24 offset:32768\n\t" \
  "v_mfma_f32_16x16x32_bf16 v[156:159], a[252:255], a[140:143], v[156:159]\n\t" \
  "ds_read_b128 a[132:135], v25 offset:32768\n\t" \
  "v_mfma_f32_16x16x32_bf16 v[152:155], a[240:243], a[136:139], v[152:155]\n\t" \
  "ds_read_b64_tr_b16 a[204:205], v17 offset:49152\n\t" \
  "v_mfma_f32_16x16x32_bf16 v[152:155], a[244:247], a[140:143], v[152:155]\n\t" \
  "ds_read_b64_tr_b16 a[206:207], v17 offset:51200\n\t" \
  "s_add_u32 m0, s26, 0\n\t" \
  "v_mfma_f32_16x16x32_bf16 v[148:151], a[232:235], a[136:139], v[148:151]\n\t" \
  "buffer_load_dwordx4 v8, s[40:43], s84 offen offset:0 sc0 sc1 lds\n\t" \
  "v_mfma_f32_16x16x32_bf16 v[148:151], a[236:239], a[140:143], v[148:151]\n\t" \
  "ds_read_b64_tr_b16 a[208:209], v18 offset:32768\n\t" \
  "v_mfma_f32_16x16x32_bf16 v[144:147], a[224:227], a[136:139], v[144:147]\n\t" \
  "ds_read_b64_tr_b16 a[210:211], v18 offset:34816\n\t" \
  "v_mfma_f32_16x16x32_bf16 v[144:147], a[228:231], a[140:143], v[144:147]\n\t" \
  "ds_read_b64_tr_b16 a[212:213], v18 offset:49152\n\t" \
  "v_mfma_f32_16x16x32_bf16 v[160:163], a[224:227], a[144:147], v[160:163]\n\t" \
  "ds_read_b128 a[136:139], v24 offset:34816\n\t" \
  "v_mfma_f32_16x16x32_bf16 v[160:163], a[228:231], a[148:151], v[160:163]\n\t" \
  "buffer_load_dwordx4 v9, s[40:43], s84 offen offset:1024 sc0 sc1 lds\n\t" \
  "v_mfma_f32_16x16x32_bf16 v[164:167], a[232:235], a[144:147], v[164:167]\n\t" \
  "ds_read_b128 a[140:143], v25 offset:34816\n\t" \
  "v_mfma_f32_16x16x32_bf16 v[164:167], a[236:239], a[148:151], v[164:167]\n\t" \
  "ds_read_b64_tr_b16 a[214:215], v18 offset:51200\n\t" \
  "v_mfma_f32_16x16x32_bf16 v[168:171], a[240:243], a[144:147], v[168:171]\n\t" \
  "ds_read_b64_tr_b16 a[216:217], v19 offset:32768\n\t" \
  "v_mfma_f32_16x16x32_bf16 v[168:171], a[244:247], a[148:151], v[168:171]\n\t" \
  "ds_read_b64_tr_b16 a[218:219], v19 offset:34816\n\t" \
  "v_mfma_f32_16x16x32_bf16 v[172:175], a[248:251], a[144:147], v[172:175]\n\t" \
  "buffer_load_dwordx4 v10, s[40:43], s84 offen offset:2048 sc0 sc1 lds\n\t" \
  "v_mfma_f32_16x16x32_bf16 v[172:175], a[252:255], a[148:151], v[172:175]\n\t" \
  "ds_read_b64_tr_b16 a[220:221], v19 offset:49152\n\t" \
  "v_mfma_f32_16x16x32_bf16 v[188:191], a[248:251], a[152:155], v[188:191]\n\t" \
  "ds_read_b128 a[144:147], v24 offset:36864\n\t" \
  "v_mfma_f32_16x16x32_bf16 v[188:191], a[252:255], a[156:159], v[188:191]\n\t" \
  "ds_read_b128 a[148:151], v25 offset:36864\n\t" \
  "v_mfma_f32_16x16x32_bf16 v[184:187], a[240:243], a[152:155], v[184:187]\n\t" \
  "ds_read_b64_tr_b16 a[222:223], v19 offset:51200\n\t" \
  "v_mfma_f32_16x16x32_bf16 v[184:187], a[244:247], a[156:159], v[184:187]\n\t" \
  "buffer_load_dwordx4 v11, s[40:43], s84 offen offset:3072 sc0 sc1 lds\n\t" \
  "v_mfma_f32_16x16x32_bf16 v[180:183], a[232:235], a[152:155], v[180:183]\n\t" \
  "ds_read_b64_tr_b16 v[96:97], v20 offset:32768\n\t" \
  "v_mfma_f32_16x16x32_bf16 v[180:183], a[236:239], a[156:159], v[180:183]\n\t" \
  "ds_read_b64_tr_b16 v[98:99], v20 offset:34816\n\t" \
  "v_mfma_f32_16x16x32_bf16 v[176:179], a[224:227], a[152:155], v[176:179]\n\t" \
  "ds_read_b64_tr_b16 v[100:101], v20 offset:49152\n\t" \
  "v_mfma_f32_16x16x32_bf16 v[176:179], a[228:231], a[156:159], v[176:179]\n\t" \
  "ds_read_b64_tr_b16 v[102:103], v20 offset:51200\n\t" \
  "s_add_u32 m0, s27, 0\n\t" \
  "v_mfma_f32_16x16x32_bf16 v[192:195], a[224:227], a[160:163], v[192:195]\n\t" \
  "buffer_load_dwordx4 v12, s[40:43], s84 offen offset:0 sc0 sc1 lds\n\t" \
  "v_mfma_f32_16x16x32_bf16 v[192:195], a[228:231], a[164:167], v[192:195]\n\t" \
  "ds_read_b128 a[152:155], v24 offset:38912\n\t" \
  "v_mfma_f32_16x16x32_bf16 v[196:199], a[232:235], a[160:163], v[196:199]\n\t" \
  "ds_read_b128 a[156:159], v25 offset:38912\n\t" \
  "v_mfma_f32_16x16x32_bf16 v[196:199], a[236:239], a[164:167], v[196:199]\n\t" \
  "ds_read_b64_tr_b16 v[104:105], v21 offset:32768\n\t" \
  "v_mfma_f32_16x16x32_bf16 v[200:203], a[240:243], a[160:163], v[200:203]\n\t" \
  "s_mov_b32 s90, s89\n\t" \
  "ds_read_b64_tr_b16 v[106:107], v21 offset:34816\n\t" \
  "v_mfma_f32_16x16x32_bf16 v[200:203], a[244:247], a[164:167], v[200:203]\n\t" \
  "buffer_load_dwordx4 v13, s[40:43], s84 offen offset:1024 sc0 sc1 lds\n\t" \
  "s_lshl_b32 s89, s73, 9\n\t" \
  "v_mfma_f32_16x16x32_bf16 v[204:207], a[248:251], a[160:163], v[204:207]\n\t" \
  "ds_read_b64_tr_b16 v[108:109], v21 offset:49152\n\t" \
  "v_mfma_f32_16x16x32_bf16 v[204:207], a[252:255], a[164:167], v[204:207]\n\t" \
  "ds_read_b64_tr_b16 v[110:111], v21 offset:51200\n\t" \
  "v_mfma_f32_16x16x32_bf16 v[220:223], a[248:251], a[168:171], v[220:223]\n\t" \
  "ds_read_b128 a[160:163], v24 offset:40960\n\t" \
  "v_mfma_f32_16x16x32_bf16 v[220:223], a[252:255], a[172:175], v[220:223]\n\t" \
  "ds_read_b128 a[164:167], v25 offset:40960\n\t" \
  "v_mfma_f32_16x16x32_bf16 v[216:219], a[240:243], a[168:171], v[216:219]\n\t" \
  "buffer_load_dwordx4 v14, s[40:43], s84 offen offset:2048 sc0 sc1 lds\n\t" \
  "v_mfma_f32_16x16x32_bf16 v[216:219], a[244:247], a[172:175], v[216:219]\n\t" \
  "ds_read_b64_tr_b16 v[112:113], v22 offset:32768\n\t" \
  "v_mfma_f32_16x16x32_bf16 v[212:215], a[232:235], a[168:171], v[212:215]\n\t" \
  "ds_read_b64_tr_b16 v[114:115], v22 offset:34816\n\t" \
  "v_mfma_f32_16x16x32_bf16 v[212:215], a[236:239], a[172:175], v[212:215]\n\t" \
  "ds_read_b64_tr_b16 v[116:117], v22 offset:49152\n\t" \
  "v_mfma_f32_16x16x32_bf16 v[208:211], a[224:227], a[168:171], v[208:211]\n\t" \
  "ds_read_b64_tr_b16 v[118:119], v22 offset:51200\n\t" \
  "v_mfma_f32_16x16x32_bf16 v[208:211], a[228:231], a[172:175], v[208:211]\n\t" \
  "buffer_load_dwordx4 v15, s[40:43], s84 offen offset:3072 sc0 sc1 lds\n\t" \
  "v_mfma_f32_16x16x32_bf16 v[224:227], a[224:227], a[176:179], v[224:227]\n\t" \
  "ds_read_b128 a[168:171], v24 offset:43008\n\t" \
  "v_mfma_f32_16x16x32_bf16 v[224:227], a[228:231], a[180:183], v[224:227]\n\t" \
  "ds_read_b128 a[172:175], v25 offset:43008\n\t" \
  "v_mfma_f32_16x16x32_bf16 v[228:231], a[232:235], a[176:179], v[228:231]\n\t" \
  "buffer_load_dword v92, v27, s[48:51], s89 offen sc0 sc1\n\t" \
  "v_mfma_f32_16x16x32_bf16 v[228:231], a[236:239], a[180:183], v[228:231]\n\t" \
  "ds_read_b64_tr_b16 v[120:121], v23 offset:32768\n\t" \
  "v_mfma_f32_16x16x32_bf16 v[232:235], a[240:243], a[176:179], v[232:235]\n\t" \
  "buffer_store_dwordx4 v[60:63], v74, s[52:55], s21 offen sc0 sc1\n\t" \
  "v_mfma_f32_16x16x32_bf16 v[232:235], a[244:247], a[180:183], v[232:235]\n\t" \
  "ds_read_b64_tr_b16 v[122:123], v23 offset:34816\n\t" \
  "v_mfma_f32_16x16x32_bf16 v[236:239], a[248:251], a[176:179], v[236:239]\n\t" \
  "ds_read_b64_tr_b16 v[124:125], v23 offset:49152\n\t" \
  "v_mfma_f32_16x16x32_bf16 v[236:239], a[252:255], a[180:183], v[236:239]\n\t" \
  "ds_read_b64_tr_b16 v[126:127], v23 offset:51200\n\t" \
  "v_mfma_f32_16x16x32_bf16 v[252:255], a[248:251], a[184:187], v[252:255]\n\t" \
  "ds_read_b128 a[176:179], v24 offset:45056\n\t" \
  "v_mfma_f32_16x16x32_bf16 v[252:255], a[252:255], a[188:191], v[252:255]\n\t" \
  "ds_read_b128 a[180:183], v25 offset:45056\n\t" \
  "v_mfma_f32_16x16x32_bf16 v[248:251], a[240:243], a[184:187], v[248:251]\n\t" \
  "v_mfma_f32_16x16x32_bf16 v[248:251], a[244:247], a[188:191], v[248:251]\n\t" \
  "v_mfma_f32_16x16x32_bf16 v[244:247], a[232:235], a[184:187], v[244:247]\n\t" \
  "v_mfma_f32_16x16x32_bf16 v[244:247], a[236:239], a[188:191], v[244:247]\n\t" \
  "v_mfma_f32_16x16x32_bf16 v[240:243], a[224:227], a[184:187], v[240:243]\n\t" \
  "v_mfma_f32_16x16x32_bf16 v[240:243], a[228:231], a[188:191], v[240:243]\n\t" \
  "s_waitcnt lgkmcnt(4)\n\t" \
  "v_mfma_f32_16x16x32_bf16 a[0:3], a[192:195], a[128:131], a[0:3]\n\t" \
  "ds_read_b128 a[184:187], v24 offset:47104\n\t" \
  "v_mfma_f32_16x16x32_bf16 a[0:3], a[196:199], a[132:135], a[0:3]\n\t" \
  "ds_read_b128 a[188:191], v25 offset:47104\n\t" \
  "v_cmp_gt_u32 vcc, s91, v93\n\t" \
  "v_mfma_f32_16x16x32_bf16 a[4:7], a[200:203], a[128:131], a[4:7]\n\t" \
  "v_mfma_f32_16x16x32_bf16 a[4:7], a[204:207], a[132:135], a[4:7]\n\t" \
  "s_cbranch_vccnz Lsp10_%=\n\t" \
  "Lspr10_%=:\n\t" \
  "v_mfma_f32_16x16x32_bf16 a[8:11], a[208:211], a[128:131], a[8:11]\n\t" \
  "s_mov_b32 s85, s86\n\t" \
  "s_mov_b32 s86, s73\n\t" \
  "v_mfma_f32_16x16x32_bf16 a[8:11], a[212:215], a[132:135], a[8:11]\n\t" \
  "s_lshl_b32 s84, s85, s74\n\t" \
  "s_add_u32 s84, s84, s75\n\t" \
  "v_mfma_f32_16x16x32_bf16 a[12:15], a[216:219], a[128:131], a[12:15]\n\t" \
  "s_waitcnt lgkmcnt(0)\n\t" \
  "s_barrier\n\t" \
  "s_add_u32 m0, s24, 32768\n\t" \
  "v_mfma_f32_16x16x32_bf16 a[12:15], a[220:223], a[132:135], a[12:15]\n\t" \
  "buffer_load_dwordx4 v0, s[40:43], s84 offen offset:0 sc0 sc1 lds\n\t" \
  "v_mfma_f32_16x16x32_bf16 a[28:31], a[216:219], a[136:139], a[28:31]\n\t" \
  "v_mfma_f32_16x16x32_bf16 a[28:31], a[220:223], a[140:143], a[28:31]\n\t" \
  "buffer_load_dwordx4 v1, s[40:43], s84 offen offset:1024 sc0 sc1 lds\n\t" \
  "v_mfma_f32_16x16x32_bf16 a[24:27], a[208:211], a[136:139], a[24:27]\n\t" \
  "v_mfma_f32_16x16x32_bf16 a[24:27], a[212:215], a[140:143], a[24:27]\n\t" \
  "buffer_load_dwordx4 v2, s[40:43], s84 offen offset:2048 sc0 sc1 lds\n\t" \
  "v_mfma_f32_16x16x32_bf16 a[20:23], a[200:203], a[136:139], a[20:23]\n\t" \
  "v_mfma_f32_16x16x32_bf16 a[20:23], a[204:207], a[140:143], a[20:23]\n\t" \
  "buffer_load_dwordx4 v3, s[40:43], s84 offen offset:3072 sc0 sc1 lds\n\t" \
  "v_mfma_f32_16x16x32_bf16 a[16:19], a[192:195], a[136:139], a[16:19]\n\t" \
  "v_mfma_f32_16x16x32_bf16 a[16:19], a[196:199], a[140:143], a[16:19]\n\t" \
  "v_mfma_f32_16x16x32_bf16 a[32:35], a[192:195], a[144:147], a[32:35]\n\t" \
  "v_mfma_f32_16x16x32_bf16 a[32:35], a[196:199], a[148:151], a[32:35]\n\t" \
  "v_mfma_f32_16x16x32_bf16 a[36:39], a[200:203], a[144:147], a[36:39]\n\t" \
  "v_mfma_f32_16x16x32_bf16 a[36:39], a[204:207], a[148:151], a[36:39]\n\t" \
  "v_mfma_f32_16x16x32_bf16 a[40:43], a[208:211], a[144:147], a[40:43]\n\t" \
  "v_mfma_f32_16x16x32_bf16 a[40:43], a[212:215], a[148:151], a[40:43]\n\t" \
  "v_mfma_f32_16x16x32_bf16 a[44:47], a[216:219], a[144:147], a[44:47]\n\t" \
  "v_mfma_f32_16x16x32_bf16 a[44:47], a[220:223], a[148:151], a[44:47]\n\t" \
  "v_mfma_f32_16x16x32_bf16 a[60:63], a[216:219], a[152:155], a[60:63]\n\t" \
  "v_mfma_f32_16x16x32_bf16 a[60:63], a[220:223], a[156:159], a[60:63]\n\t" \
  "v_mfma_f32_16x16x32_bf16 a[56:59], a[208:211], a[152:155], a[56:59]\n\t" \
  "v_mfma_f32_16x16x32_bf16 a[56:59], a[212:215], a[156:159], a[56:59]\n\t" \
  "v_mfma_f32_16x16x32_bf16 a[52:55], a[200:203], a[152:155], a[52:55]\n\t" \
  "v_mfma_f32_16x16x32_bf16 a[52:55], a[204:207], a[156:159], a[52:55]\n\t" \
  "v_mfma_f32_16x16x32_bf16 a[48:51], a[192:195], a[152:155], a[48:51]\n\t" \
  "v_mfma_f32_16x16x32_bf16 a[48:51], a[196:199], a[156:159], a[48:51]\n\t" \
  "s_lshr_b64 s[10:11], s[8:9], s87\n\t" \
  "s_sext_i32_i8 s10, s10\n\t" \
  "v_mfma_f32_16x16x32_bf16 a[64:67], a[192:195], a[160:163], a[64:67]\n\t" \
  "s_add_u32 s87, s87, 8\n\t" \
  "s_add_u32 s73, s73, s10\n\t" \
  "v_mfma_f32_16x16x32_bf16 a[64:67], a[196:199], a[164:167], a[64:67]\n\t" \
  "s_cmp_eq_u32 s87, 56\n\t" \
  "s_cselect_b32 s87, 0, s87\n\t" \
  "v_mfma_f32_16x16x32_bf16 a[68:71], a[200:203], a[160:163], a[68:71]\n\t" \
  "v_mfma_f32_16x16x32_bf16 a[68:71], a[204:207], a[164:167], a[68:71]\n\t" \
  "v_mfma_f32_16x16x32_bf16 a[72:75], a[208:211], a[160:163], a[72:75]\n\t" \
  "v_mfma_f32_16x16x32_bf16 a[72:75], a[212:215], a[164:167], a[72:75]\n\t" \
  "v_mfma_f32_16x16x32_bf16 a[76:79], a[216:219], a[160:163], a[76:79]\n\t" \
  "v_mfma_f32_16x16x32_bf16 a[76:79], a[220:223], a[164:167], a[76:79]\n\t" \
  "v_mfma_f32_16x16x32_bf16 a[92:95], a[216:219], a[168:171], a[92:95]\n\t" \
  "v_mfma_f32_16x16x32_bf16 a[92:95], a[220:223], a[172:175], a[92:95]\n\t" \
  "v_mfma_f32_16x16x32_bf16 a[88:91], a[208:211], a[168:171], a[88:91]\n\t" \
  "v_mfma_f32_16x16x32_bf16 a[88:91], a[212:215], a[172:175], a[88:91]\n\t" \
  "v_mfma_f32_16x16x32_bf16 a[84:87], a[200:203], a[168:171], a[84:87]\n\t" \
  "v_mfma_f32_16x16x32_bf16 a[84:87], a[204:207], a[172:175], a[84:87]\n\t" \
  "v_mfma_f32_16x16x32_bf16 a[80:83], a[192:195], a[168:171], a[80:83]\n\t" \
  "v_mfma_f32_16x16x32_bf16 a[80:83], a[196:199], a[172:175], a[80:83]\n\t" \
  "v_mfma_f32_16x16x32_bf16 a[96:99], a[192:195], a[176:179], a[96:99]\n\t" \
  "v_mfma_f32_16x16x32_bf16 a[96:99], a[196:199], a[180:183], a[96:99]\n\t" \
  "v_mfma_f32_16x16x32_bf16 a[100:103], a[200:203], a[176:179], a[100:103]\n\t" \
  "v_mfma_f32_16x16x32_bf16 a[100:103], a[204:207], a[180:183], a[100:103]\n\t" \
  "v_mfma_f32_16x16x32_bf16 a[104:107], a[208:211], a[176:179], a[104:107]\n\t" \
  "v_mfma_f32_16x16x32_bf16 a[104:107], a[212:215], a[180:183], a[104:107]\n\t" \
  "v_mfma_f32_16x16x32_bf16 a[108:111], a[216:219], a[176:179], a[108:111]\n\t" \
  "v_mfma_f32_16x16x32_bf16 a[108:111], a[220:223], a[180:183], a[108:111]\n\t" \
  "s_add_u32 m0, s25, 32768\n\t" \
  "v_mfma_f32_16x16x32_bf16 a[124:127], a[216:219], a[184:187], a[124:127]\n\t" \
  "buffer_load_dwordx4 v4, s[40:43], s84 offen offset:0 sc0 sc1 lds\n\t" \
  "v_mfma_f32_16x16x32_bf16 a[124:127], a[220:223], a[188:191], a[124:127]\n\t" \
  "v_mfma_f32_16x16x32_bf16 a[120:123], a[208:211], a[184:187], a[120:123]\n\t" \
  "v_mfma_f32_16x16x32_bf16 a[120:123], a[212:215], a[188:191], a[120:123]\n\t" \
  "v_mfma_f32_16x16x32_bf16 a[116:119], a[200:203], a[184:187], a[116:119]\n\t" \
  "v_mfma_f32_16x16x32_bf16 a[116:119], a[204:207], a[188:191], a[116:119]\n\t" \
  "buffer_load_dwordx4 v5, s[40:43], s84 offen offset:1024 sc0 sc1 lds\n\t" \
  "v_mfma_f32_16x16x32_bf16 a[112:115], a[192:195], a[184:187], a[112:115]\n\t" \
  "v_mfma_f32_16x16x32_bf16 a[112:115], a[196:199], a[188:191], a[112:115]\n\t" \
  "s_waitcnt vmcnt(7)\n\t" \
  "s_barrier\n\t" \
  "v_mfma_f32_16x16x32_bf16 v[128:131], v[96:99], a[128:131], v[128:131]\n\t" \
  "ds_read_b64_tr_b16 a[192:193], v16 offset:0\n\t" \
  "v_mfma_f32_16x16x32_bf16 v[128:131], v[100:103], a[132:135], v[128:131]\n\t" \
  "ds_read_b64_tr_b16 a[194:195], v16 offset:2048\n\t" \
  "v_mfma_f32_16x16x32_bf16 v[132:135], v[104:107], a[128:131], v[132:135]\n\t" \
  "buffer_load_dwordx4 v6, s[40:43], s84 offen offset:2048 sc0 sc1 lds\n\t" \
  "v_mfma_f32_16x16x32_bf16 v[132:135], v[108:111], a[132:135], v[132:135]\n\t" \
  "ds_read_b64_tr_b16 a[196:197], v16 offset:16384\n\t" \
  "v_mfma_f32_16x16x32_bf16 v[136:139], v[112:115], a[128:131], v[136:139]\n\t" \
  "ds_read_b64_tr_b16 a[198:199], v16 offset:18432\n\t" \
  "v_mfma_f32_16x16x32_bf16 v[136:139], v[116:119], a[132:135], v[136:139]\n\t" \
  "ds_read_b64_tr_b16 a[200:201], v17 offset:0\n\t" \
  "v_mfma_f32_16x16x32_bf16 v[140:143], v[120:123], a[128:131], v[140:143]\n\t" \
  "ds_read_b64_tr_b16 a[202:203], v17 offset:2048\n\t" \
  "v_mfma_f32_16x16x32_bf16 v[140:143], v[124:127], a[132:135], v[140:143]\n\t" \
  "buffer_load_dwordx4 v7, s[40:43], s84 offen offset:3072 sc0 sc1 lds\n\t" \
  "v_mfma_f32_16x16x32_bf16 v[156:159], v[120:123], a[136:139], v[156:159]\n\t" \
  "ds_read_b128 a[128:131], v24 offset:0\n\t" \
  "v_mfma_f32_16x16x32_bf16 v[156:159], v[124:127], a[140:143], v[156:159]\n\t" \
  "ds_read_b128 a[132:135], v25 offset:0\n\t" \
  "v_mfma_f32_16x16x32_bf16 v[152:155], v[112:115], a[136:139], v[152:155]\n\t" \
  "ds_read_b64_tr_b16 a[204:205], v17 offset:16384\n\t" \
  "v_mfma_f32_16x16x32_bf16 v[152:155], v[116:119], a[140:143], v[152:155]\n\t" \
  "ds_read_b64_tr_b16 a[206:207], v17 offset:18432\n\t" \
  "s_add_u32 m0, s26, 32768\n\t" \
  "v_mfma_f32_16x16x32_bf16 v[148:151], v[104:107], a[136:139], v[148:151]\n\t" \
  "buffer_load_dwordx4 v8, s[40:43], s84 offen offset:0 sc0 sc1 lds\n\t" \
  "v_mfma_f32_16x16x32_bf16 v[148:151], v[108:111], a[140:143], v[148:151]\n\t" \
  "ds_read_b64_tr_b16 a[208:209], v18 offset:0\n\t" \
  "v_mfma_f32_16x16x32_bf16 v[144:147], v[96:99], a[136:139], v[144:147]\n\t" \
  "ds_read_b64_tr_b16 a[210:211], v18 offset:2048\n\t" \
  "v_mfma_f32_16x16x32_bf16 v[144:147], v[100:103], a[140:143], v[144:147]\n\t" \
  "ds_read_b64_tr_b16 a[212:213], v18 offset:16384\n\t" \
  "v_mfma_f32_16x16x32_bf16 v[160:163], v[96:99], a[144:147], v[160:163]\n\t" \
  "ds_read_b128 a[136:139], v24 offset:2048\n\t" \
  "v_mfma_f32_16x16x32_bf16 v[160:163], v[100:103], a[148:151], v[160:163]\n\t" \
  "buffer_load_dwordx4 v9, s[40:43], s84 offen offset:1024 sc0 sc1 lds\n\t" \
  "v_mfma_f32_16x16x32_bf16 v[164:167], v[104:107], a[144:147], v[164:167]\n\t" \
  "ds_read_b128 a[140:143], v25 offset:2048\n\t" \
  "v_mfma_f32_16x16x32_bf16 v[164:167], v[108:111], a[148:151], v[164:167]\n\t" \
  "ds_read_b64_tr_b16 a[214:215], v18 offset:18432\n\t" \
  "v_mfma_f32_16x16x32_bf16 v[168:171], v[112:115], a[144:147], v[168:171]\n\t" \
  "ds_read_b64_tr_b16 a[216:217], v19 offset:0\n\t" \
  "v_mfma_f32_16x16x32_bf16 v[168:171], v[116:119], a[148:151], v[168:171]\n\t" \
  "ds_read_b64_tr_b16 a[218:219], v19 offset:2048\n\t" \
  "v_mfma_f32_16x16x32_bf16 v[172:175], v[120:123], a[144:147], v[172:175]\n\t" \
  "buffer_load_dwordx4 v10, s[40:43], s84 offen offset:2048 sc0 sc1 lds\n\t" \
  "v_mfma_f32_16x16x32_bf16 v[172:175], v[124:127], a[148:151], v[172:175]\n\t" \
  "ds_read_b64_tr_b16 a[220:221], v19 offset:16384\n\t" \
  "v_mfma_f32_16x16x32_bf16 v[188:191], v[120:123], a[152:155], v[188:191]\n\t" \
  "ds_read_b128 a[144:147], v24 offset:4096\n\t" \
  "v_mfma_f32_16x16x32_bf16 v[188:191], v[124:127], a[156:159], v[188:191]\n\t" \
  "ds_read_b128 a[148:151], v25 offset:4096\n\t" \
  "v_mfma_f32_16x16x32_bf16 v[184:187], v[112:115], a[152:155], v[184:187]\n\t" \
  "s_cmp_le_u32 s95, 2\n\t" \
  "s_cselect_b32 s54, 0, s54\n\t" \
  "s_cselect_b32 s58, 0, s58\n\t" \
  "ds_read_b64_tr_b16 a[222:223], v19 offset:18432\n\t" \
  "v_mfma_f32_16x16x32_bf16 v[184:187], v[116:119], a[156:159], v[184:187]\n\t" \
  "buffer_load_dwordx4 v11, s[40:43], s84 offen offset:3072 sc0 sc1 lds\n\t" \
  "v_mfma_f32_16x16x32_bf16 v[180:183], v[104:107], a[152:155], v[180:183]\n\t" \
  "ds_read_b64_tr_b16 a[224:225], v20 offset:0\n\t" \
  "v_mfma_f32_16x16x32_bf16 v[180:183], v[108:111], a[156:159], v[180:183]\n\t" \
  "ds_read_b64_tr_b16 a[226:227], v20 offset:2048\n\t" \
  "v_mfma_f32_16x16x32_bf16 v[176:179], v[96:99], a[152:155], v[176:179]\n\t" \
  "ds_read_b64_tr_b16 a[228:229], v20 offset:16384\n\t" \
  "v_mfma_f32_16x16x32_bf16 v[176:179], v[100:103], a[156:159], v[176:179]\n\t" \
  "ds_read_b64_tr_b16 a[230:231], v20 offset:18432\n\t" \
  "s_add_u32 m0, s27, 32768\n\t" \
  "v_mfma_f32_16x16x32_bf16 v[192:195], v[96:99], a[160:163], v[192:195]\n\t" \
  "buffer_load_dwordx4 v12, s[40:43], s84 offen offset:0 sc0 sc1 lds\n\t" \
  "v_mfma_f32_16x16x32_bf16 v[192:195], v[100:103], a[164:167], v[192:195]\n\t" \
  "ds_read_b128 a[152:155], v24 offset:6144\n\t" \
  "v_mfma_f32_16x16x32_bf16 v[196:199], v[104:107], a[160:163], v[196:199]\n\t" \
  "ds_read_b128 a[156:159], v25 offset:6144\n\t" \
  "v_mfma_f32_16x16x32_bf16 v[196:199], v[108:111], a[164:167], v[196:199]\n\t" \
  "ds_read_b64_tr_b16 a[232:233], v21 offset:0\n\t" \
  "v_mfma_f32_16x16x32_bf16 v[200:203], v[112:115], a[160:163], v[200:203]\n\t" \
  "s_mov_b32 s90, s89\n\t" \
  "ds_read_b64_tr_b16 a[234:235], v21 offset:2048\n\t" \
  "v_mfma_f32_16x16x32_bf16 v[200:203], v[116:119], a[164:167], v[200:203]\n\t" \
  "buffer_load_dwordx4 v13, s[40:43], s84 offen offset:1024 sc0 sc1 lds\n\t" \
  "s_lshl_b32 s89, s73, 9\n\t" \
  "v_mfma_f32_16x16x32_bf16 v[204:207], v[120:123], a[160:163], v[204:207]\n\t" \
  "s_lshr_b32 s20, s21, 8\n\t" \
  "s_lshl_b32 s21, s80, 10\n\t" \
  "ds_read_b64_tr_b16 a[236:237], v21 offset:16384\n\t" \
  "v_mfma_f32_16x16x32_bf16 v[204:207], v[124:127], a[164:167], v[204:207]\n\t" \
  "s_add_u32 s11, s80, s82\n\t" \
  "s_mov_b32 s80, s81\n\t" \
  "ds_read_b64_tr_b16 a[238:239], v21 offset:18432\n\t" \
  "v_mfma_f32_16x16x32_bf16 v[220:223], v[120:123], a[168:171], v[220:223]\n\t" \
  "ds_read_b128 a[160:163], v24 offset:8192\n\t" \
  "s_mov_b32 s81, s11\n\t" \
  "s_sub_u32 s82, 0, s82\n\t" \
  "v_mfma_f32_16x16x32_bf16 v[220:223], v[124:127], a[172:175], v[220:223]\n\t" \
  "ds_read_b128 a[164:167], v25 offset:8192\n\t" \
  "v_mfma_f32_16x16x32_bf16 v[216:219], v[112:115], a[168:171], v[216:219]\n\t" \
  "buffer_load_dwordx4 v14, s[40:43], s84 offen offset:2048 sc0 sc1 lds\n\t" \
  "v_mfma_f32_16x16x32_bf16 v[216:219], v[116:119], a[172:175], v[216:219]\n\t" \
  "ds_read_b64_tr_b16 a[240:241], v22 offset:0\n\t" \
  "v_mfma_f32_16x16x32_bf16 v[212:215], v[104:107], a[168:171], v[212:215]\n\t" \
  "ds_read_b64_tr_b16 a[242:243], v22 offset:2048\n\t" \
  "v_mfma_f32_16x16x32_bf16 v[212:215], v[108:111], a[172:175], v[212:215]\n\t" \
  "ds_read_b64_tr_b16 a[244:245], v22 offset:16384\n\t" \
  "v_mfma_f32_16x16x32_bf16 v[208:211], v[96:99], a[168:171], v[208:211]\n\t" \
  "ds_read_b64_tr_b16 a[246:247], v22 offset:18432\n\t" \
  "v_mfma_f32_16x16x32_bf16 v[208:211], v[100:103], a[172:175], v[208:211]\n\t" \
  "buffer_load_dwordx4 v15, s[40:43], s84 offen offset:3072 sc0 sc1 lds\n\t" \
  "v_mfma_f32_16x16x32_bf16 v[224:227], v[96:99], a[176:179], v[224:227]\n\t" \
  "ds_read_b128 a[168:171], v24 offset:10240\n\t" \
  "v_mfma_f32_16x16x32_bf16 v[224:227], v[100:103], a[180:183], v[224:227]\n\t" \
  "ds_read_b128 a[172:175], v25 offset:10240\n\t" \
  "v_mfma_f32_16x16x32_bf16 v[228:231], v[104:107], a[176:179], v[228:231]\n\t" \
  "buffer_load_dword v93, v27, s[48:51], s89 offen sc0 sc1\n\t" \
  "v_mfma_f32_16x16x32_bf16 v[228:231], v[108:111], a[180:183], v[228:231]\n\t" \
  "ds_read_b64_tr_b16 a[248:249], v23 offset:0\n\t" \
  "v_mfma_f32_16x16x32_bf16 v[232:235], v[112:115], a[176:179], v[232:235]\n\t" \
  "buffer_store_dwordx4 v[56:59], v72, s[52:55], s21 offen sc0 sc1\n\t" \
  "v_mfma_f32_16x16x32_bf16 v[232:235], v[116:119], a[180:183], v[232:235]\n\t" \
  "ds_read_b64_tr_b16 a[250:251], v23 offset:2048\n\t" \
  "v_mfma_f32_16x16x32_bf16 v[236:239], v[120:123], a[176:179], v[236:239]\n\t" \
  "ds_read_b64_tr_b16 a[252:253], v23 offset:16384\n\t" \
  "v_mfma_f32_16x16x32_bf16 v[236:239], v[124:127], a[180:183], v[236:239]\n\t" \
  "ds_read_b64_tr_b16 a[254:255], v23 offset:18432\n\t" \
  "v_mfma_f32_16x16x32_bf16 v[252:255], v[120:123], a[184:187], v[252:255]\n\t" \
  "ds_read_b128 a[176:179], v24 offset:12288\n\t" \
  "v_mfma_f32_16x16x32_bf16 v[252:255], v[124:127], a[188:191], v[252:255]\n\t" \
  "ds_read_b128 a[180:183], v25 offset:12288\n\t" \
  "v_mfma_f32_16x16x32_bf16 v[248:251], v[112:115], a[184:187], v[248:251]\n\t" \
  "v_mfma_f32_16x16x32_bf16 v[248:251], v[116:119], a[188:191], v[248:251]\n\t" \
  "v_mfma_f32_16x16x32_bf16 v[244:247], v[104:107], a[184:187], v[244:247]\n\t" \
  "v_mfma_f32_16x16x32_bf16 v[244:247], v[108:111], a[188:191], v[244:247]\n\t" \
  "v_mfma_f32_16x16x32_bf16 v[240:243], v[96:99], a[184:187], v[240:243]\n\t" \
  "v_mfma_f32_16x16x32_bf16 v[240:243], v[100:103], a[188:191], v[240:243]\n\t" \
  "s_sub_u32 s95, s95, 1\n\t" \
  "s_cmp_lg_u32 s95, 0\n\t" \
  "s_cbranch_scc1 LloopC_%=\n\t" \
  "Ldec_%=:\n\t" \
  "v_readfirstlane_b32 s10, v94\n\t" \
  "s_nop 0\n\t" \
  "s_lshr_b32 s11, s10, 4\n\t" \
  "s_cmp_lg_u32 s11, s92\n\t" \
  "s_cbranch_scc1 Ldsp_%=\n\t" \
  "s_and_b32 s11, s93, 1\n\t" \
  "s_sub_u32 s11, 5, s11\n\t" \
  "s_xor_b32 s10, s10, s93\n\t" \
  "s_bitcmp1_b32 s10, 0\n\t" \
  "s_cselect_b32 s95, s11, s71\n\t" \
  "Ldecr_%=:\n\t" \
  "LloopP_%=:\n\t" \
  "s_waitcnt lgkmcnt(3)\n\t" \
  "v_mfma_f32_16x16x32_bf16 a[0:3], a[192:195], a[128:131], a[0:3]\n\t" \
  "ds_read_b128 a[184:187], v24 offset:14336\n\t" \
  "v_mfma_f32_16x16x32_bf16 a[0:3], a[196:199], a[132:135], a[0:3]\n\t" \
  "ds_read_b128 a[188:191], v25 offset:14336\n\t" \
  "v_cmp_gt_u32 vcc, s91, v92\n\t" \
  "v_mfma_f32_16x16x32_bf16 a[4:7], a[200:203], a[128:131], a[4:7]\n\t" \
  "v_mfma_f32_16x16x32_bf16 a[4:7], a[204:207], a[132:135], a[4:7]\n\t" \
  "s_cbranch_vccnz Lsp11_%=\n\t" \
  "Lspr11_%=:\n\t" \
  "v_mfma_f32_16x16x32_bf16 a[8:11], a[208:211], a[128:131], a[8:11]\n\t" \
  "s_mov_b32 s85, s86\n\t" \
  "s_mov_b32 s86, s73\n\t" \
  "v_mfma_f32_16x16x32_bf16 a[8:11], a[212:215], a[132:135], a[8:11]\n\t" \
  "s_lshl_b32 s84, s85, s74\n\t" \
  "s_add_u32 s84, s84, s75\n\t" \
  "v_mfma_f32_16x16x32_bf16 a[12:15], a[216:219], a[128:131], a[12:15]\n\t" \
  "s_waitcnt lgkmcnt(0)\n\t" \
  "s_barrier\n\t" \
  "s_add_u32 m0, s24, 0\n\t" \
  "v_mfma_f32_16x16x32_bf16 a[12:15], a[220:223], a[132:135], a[12:15]\n\t" \
  "buffer_load_dwordx4 v0, s[40:43], s84 offen offset:0 sc0 sc1 lds\n\t" \
  "v_mfma_f32_16x16x32_bf16 a[28:31], a[216:219], a[136:139], a[28:31]\n\t" \
  "v_mfma_f32_16x16x32_bf16 a[28:31], a[220:223], a[140:143], a[28:31]\n\t" \
  "buffer_load_dwordx4 v1, s[40:43], s84 offen offset:1024 sc0 sc1 lds\n\t" \
  "v_mfma_f32_16x16x32_bf16 a[24:27], a[208:211], a[136:139], a[24:27]\n\t" \
  "v_mfma_f32_16x16x32_bf16 a[24:27], a[212:215], a[140:143], a[24:27]\n\t" \
  "buffer_load_dwordx4 v2, s[40:43], s84 offen offset:2048 sc0 sc1 lds\n\t" \
  "v_mfma_f32_16x16x32_bf16 a[20:23], a[200:203], a[136:139], a[20:23]\n\t" \
  "v_mfma_f32_16x16x32_bf16 a[20:23], a[204:207], a[140:143], a[20:23]\n\t" \
  "buffer_load_dwordx4 v3, s[40:43], s84 offen offset:3072 sc0 sc1 lds\n\t" \
  "v_mfma_f32_16x16x32_bf16 a[16:19], a[192:195], a[136:139], a[16:19]\n\t" \
  "v_mfma_f32_16x16x32_bf16 a[16:19], a[196:199], a[140:143], a[16:19]\n\t" \
  "v_mfma_f32_16x16x32_bf16 a[32:35], a[192:195], a[144:147], a[32:35]\n\t" \
  "v_mfma_f32_16x16x32_bf16 a[32:35], a[196:199], a[148:151], a[32:35]\n\t" \
  "v_mfma_f32_16x16x32_bf16 a[36:39], a[200:203], a[144:147], a[36:39]\n\t" \
  "v_mfma_f32_16x16x32_bf16 a[36:39], a[204:207], a[148:151], a[36:39]\n\t" \
  "v_mfma_f32_16x16x32_bf16 a[40:43], a[208:211], a[144:147], a[40:43]\n\t" \
  "v_mfma_f32_16x16x32_bf16 a[40:43], a[212:215], a[148:151], a[40:43]\n\t" \
  "v_mfma_f32_16x16x32_bf16 a[44:47], a[216:219], a[144:147], a[44:47]\n\t" \
  "v_mfma_f32_16x16x32_bf16 a[44:47], a[220:223], a[148:151], a[44:47]\n\t" \
  "v_mfma_f32_16x16x32_bf16 a[60:63], a[216:219], a[152:155], a[60:63]\n\t" \
  "v_mfma_f32_16x16x32_bf16 a[60:63], a[220:223], a[156:159], a[60:63]\n\t" \
  "v_mfma_f32_16x16x32_bf16 a[56:59], a[208:211], a[152:155], a[56:59]\n\t" \
  "v_mfma_f32_16x16x32_bf16 a[56:59], a[212:215], a[156:159], a[56:59]\n\t" \
  "v_mfma_f32_16x16x32_bf16 a[52:55], a[200:203], a[152:155], a[52:55]\n\t" \
  "v_mfma_f32_16x16x32_bf16 a[52:55], a[204:207], a[156:159], a[52:55]\n\t" \
  "v_mfma_f32_16x16x32_bf16 a[48:51], a[192:195], a[152:155], a[48:51]\n\t" \
  "v_mfma_f32_16x16x32_bf16 a[48:51], a[196:199], a[156:159], a[48:51]\n\t" \
  "s_lshr_b64 s[10:11], s[8:9], s87\n\t" \
  "s_sext_i32_i8 s10, s10\n\t" \
  "v_mfma_f32_16x16x32_bf16 a[64:67], a[192:195], a[160:163], a[64:67]\n\t" \
  "s_add_u32 s87, s87, 8\n\t" \
  "s_add_u32 s73, s73, s10\n\t" \
  "v_mfma_f32_16x16x32_bf16 a[64:67], a[196:199], a[164:167], a[64:67]\n\t" \
  "s_cmp_eq_u32 s87, 56\n\t" \
  "s_cselect_b32 s87, 0, s87\n\t" \
  "v_mfma_f32_16x16x32_bf16 a[68:71], a[200:203], a[160:163], a[68:71]\n\t" \
  "v_mfma_f32_16x16x32_bf16 a[68:71], a[204:207], a[164:167], a[68:71]\n\t" \
  "v_mfma_f32_16x16x32_bf16 a[72:75], a[208:211], a[160:163], a[72:75]\n\t" \
  "v_mfma_f32_16x16x32_bf16 a[72:75], a[212:215], a[164:167], a[72:75]\n\t" \
  "v_mfma_f32_16x16x32_bf16 a[76:79], a[216:219], a[160:163], a[76:79]\n\t" \
  "v_mfma_f32_16x16x32_bf16 a[76:79], a[220:223], a[164:167], a[76:79]\n\t" \
  "v_mfma_f32_16x16x32_bf16 a[92:95], a[216:219], a[168:171], a[92:95]\n\t" \
  "v_mfma_f32_16x16x32_bf16 a[92:95], a[220:223], a[172:175], a[92:95]\n\t" \
  "v_mfma_f32_16x16x32_bf16 a[88:91], a[208:211], a[168:171], a[88:91]\n\t" \
  "v_mfma_f32_16x16x32_bf16 a[88:91], a[212:215], a[172:175], a[88:91]\n\t" \
  "v_mfma_f32_16x16x32_bf16 a[84:87], a[200:203], a[168:171], a[84:87]\n\t" \
  "v_mfma_f32_16x16x32_bf16 a[84:87], a[204:207], a[172:175], a[84:87]\n\t" \
  "v_mfma_f32_16x16x32_bf16 a[80:83], a[192:195], a[168:171], a[80:83]\n\t" \
  "v_mfma_f32_16x16x32_bf16 a[80:83], a[196:199], a[172:175], a[80:83]\n\t" \
  "v_mfma_f32_16x16x32_bf16 a[96:99], a[192:195], a[176:179], a[96:99]\n\t" \
  "v_mfma_f32_16x16x32_bf16 a[96:99], a[196:199], a[180:183], a[96:99]\n\t" \
  "v_mfma_f32_16x16x32_bf16 a[100:103], a[200:203], a[176:179], a[100:103]\n\t" \
  "v_mfma_f32_16x16x32_bf16 a[100:103], a[204:207], a[180:183], a[100:103]\n\t" \
  "v_mfma_f32_16x16x32_bf16 a[104:107], a[208:211], a[176:179], a[104:107]\n\t" \
  "v_mfma_f32_16x16x32_bf16 a[104:107], a[212:215], a[180:183], a[104:107]\n\t" \
  "v_mfma_f32_16x16x32_bf16 a[108:111], a[216:219], a[176:179], a[108:111]\n\t" \
  "v_mfma_f32_16x16x32_bf16 a[108:111], a[220:223], a[180:183], a[108:111]\n\t" \
  "s_add_u32 m0, s25, 0\n\t" \
  "v_mfma_f32_16x16x32_bf16 a[124:127], a[216:219], a[184:187], a[124:127]\n\t" \
  "buffer_load_dwordx4 v4, s[40:43], s84 offen offset:0 sc0 sc1 lds\n\t" \
  "v_mfma_f32_16x16x32_bf16 a[124:127], a[220:223], a[188:191], a[124:127]\n\t" \
  "v_mfma_f32_16x16x32_bf16 a[120:123], a[208:211], a[184:187], a[120:123]\n\t" \
  "v_mfma_f32_16x16x32_bf16 a[120:123], a[212:215], a[188:191], a[120:123]\n\t" \
  "v_mfma_f32_16x16x32_bf16 a[116:119], a[200:203], a[184:187], a[116:119]\n\t" \
  "v_mfma_f32_16x16x32_bf16 a[116:119], a[204:207], a[188:191], a[116:119]\n\t" \
  "buffer_load_dwordx4 v5, s[40:43], s84 offen offset:1024 sc0 sc1 lds\n\t" \
  "v_mfma_f32_16x16x32_bf16 a[112:115], a[192:195], a[184:187], a[112:115]\n\t" \
  "v_mfma_f32_16x16x32_bf16 a[112:115], a[196:199], a[188:191], a[112:115]\n\t" \
  "s_waitcnt vmcnt(6)\n\t" \
  "s_barrier\n\t" \
  "v_mfma_f32_16x16x32_bf16 v[128:131], a[224:227], a[128:131], v[128:131]\n\t" \
  "ds_read_b64_tr_b16 a[192:193], v16 offset:32768\n\t" \
  "v_mfma_f32_16x16x32_bf16 v[128:131], a[228:231], a[132:135], v[128:131]\n\t" \
  "ds_read_b64_tr_b16 a[194:195], v16 offset:34816\n\t" \
  "v_mfma_f32_16x16x32_bf16 v[132:135], a[232:235], a[128:131], v[132:135]\n\t" \
  "buffer_load_dwordx4 v6, s[40:43], s84 offen offset:2048 sc0 sc1 lds\n\t" \
  "v_mfma_f32_16x16x32_bf16 v[132:135], a[236:239], a[132:135], v[132:135]\n\t" \
  "ds_read_b64_tr_b16 a[196:197], v16 offset:49152\n\t" \
  "v_mfma_f32_16x16x32_bf16 v[136:139], a[240:243], a[128:131], v[136:139]\n\t" \
  "ds_read_b64_tr_b16 a[198:199], v16 offset:51200\n\t" \
  "v_mfma_f32_16x16x32_bf16 v[136:139], a[244:247], a[132:135], v[136:139]\n\t" \
  "ds_read_b64_tr_b16 a[200:201], v17 offset:32768\n\t" \
  "v_mfma_f32_16x16x32_bf16 v[140:143], a[248:251], a[128:131], v[140:143]\n\t" \
  "ds_read_b64_tr_b16 a[202:203], v17 offset:34816\n\t" \
  "v_mfma_f32_16x16x32_bf16 v[140:143], a[252:255], a[132:135], v[140:143]\n\t" \
  "buffer_load_dwordx4 v7, s[40:43], s84 offen offset:3072 sc0 sc1 lds\n\t" \
  "v_mfma_f32_16x16x32_bf16 v[156:159], a[248:251], a[136:139], v[156:159]\n\t" \
  "ds_read_b128 a[128:131], v24 offset:32768\n\t" \
  "v_mfma_f32_16x16x32_bf16 v[156:159], a[252:255], a[140:143], v[156:159]\n\t" \
  "ds_read_b128 a[132:135], v25 offset:32768\n\t" \
  "v_mfma_f32_16x16x32_bf16 v[152:155], a[240:243], a[136:139], v[152:155]\n\t" \
  "ds_read_b64_tr_b16 a[204:205], v17 offset:49152\n\t" \
  "v_mfma_f32_16x16x32_bf16 v[152:155], a[244:247], a[140:143], v[152:155]\n\t" \
  "ds_read_b64_tr_b16 a[206:207], v17 offset:51200\n\t" \
  "s_add_u32 m0, s26, 0\n\t" \
  "v_mfma_f32_16x16x32_bf16 v[148:151], a[232:235], a[136:139], v[148:151]\n\t" \
  "buffer_load_dwordx4 v8, s[40:43], s84 offen offset:0 sc0 sc1 lds\n\t" \
  "v_mfma_f32_16x16x32_bf16 v[148:151], a[236:239], a[140:143], v[148:151]\n\t" \
  "ds_read_b64_tr_b16 a[208:209], v18 offset:32768\n\t" \
  "v_mfma_f32_16x16x32_bf16 v[144:147], a[224:227], a[136:139], v[144:147]\n\t" \
  "ds_read_b64_tr_b16 a[210:211], v18 offset:34816\n\t" \
  "v_mfma_f32_16x16x32_bf16 v[144:147], a[228:231], a[140:143], v[144:147]\n\t" \
  "ds_read_b64_tr_b16 a[212:213], v18 offset:49152\n\t" \
  "v_mfma_f32_16x16x32_bf16 v[160:163], a[224:227], a[144:147], v[160:163]\n\t" \
  "ds_read_b128 a[136:139], v24 offset:34816\n\t" \
  "v_mfma_f32_16x16x32_bf16 v[160:163], a[228:231], a[148:151], v[160:163]\n\t" \
  "buffer_load_dwordx4 v9, s[40:43], s84 offen offset:1024 sc0 sc1 lds\n\t" \
  "v_mfma_f32_16x16x32_bf16 v[164:167], a[232:235], a[144:147], v[164:167]\n\t" \
  "ds_read_b128 a[140:143], v25 offset:34816\n\t" \
  "v_mfma_f32_16x16x32_bf16 v[164:167], a[236:239], a[148:151], v[164:167]\n\t" \
  "ds_read_b64_tr_b16 a[214:215], v18 offset:51200\n\t" \
  "v_mfma_f32_16x16x32_bf16 v[168:171], a[240:243], a[144:147], v[168:171]\n\t" \
  "ds_read_b64_tr_b16 a[216:217], v19 offset:32768\n\t" \
  "v_mfma_f32_16x16x32_bf16 v[168:171], a[244:247], a[148:151], v[168:171]\n\t" \
  "ds_read_b64_tr_b16 a[218:219], v19 offset:34816\n\t" \
  "v_mfma_f32_16x16x32_bf16 v[172:175], a[248:251], a[144:147], v[172:175]\n\t" \
  "buffer_load_dwordx4 v10, s[40:43], s84 offen offset:2048 sc0 sc1 lds\n\t" \
  "v_mfma_f32_16x16x32_bf16 v[172:175], a[252:255], a[148:151], v[172:175]\n\t" \
  "ds_read_b64_tr_b16 a[220:221], v19 offset:49152\n\t" \
  "v_mfma_f32_16x16x32_bf16 v[188:191], a[248:251], a[152:155], v[188:191]\n\t" \
  "ds_read_b128 a[144:147], v24 offset:36864\n\t" \
  "v_mfma_f32_16x16x32_bf16 v[188:191], a[252:255], a[156:159], v[188:191]\n\t" \
  "ds_read_b128 a[148:151], v25 offset:36864\n\t" \
  "v_mfma_f32_16x16x32_bf16 v[184:187], a[240:243], a[152:155], v[184:187]\n\t" \
  "ds_read_b64_tr_b16 a[222:223], v19 offset:51200\n\t" \
  "v_mfma_f32_16x16x32_bf16 v[184:187], a[244:247], a[156:159], v[184:187]\n\t" \
  "buffer_load_dwordx4 v11, s[40:43], s84 offen offset:3072 sc0 sc1 lds\n\t" \
  "v_mfma_f32_16x16x32_bf16 v[180:183], a[232:235], a[152:155], v[180:183]\n\t" \
  "ds_read_b64_tr_b16 v[96:97], v20 offset:32768\n\t" \
  "v_mfma_f32_16x16x32_bf16 v[180:183], a[236:239], a[156:159], v[180:183]\n\t" \
  "ds_read_b64_tr_b16 v[98:99], v20 offset:34816\n\t" \
  "v_mfma_f32_16x16x32_bf16 v[176:179], a[224:227], a[152:155], v[176:179]\n\t" \
  "ds_read_b64_tr_b16 v[100:101], v20 offset:49152\n\t" \
  "v_mfma_f32_16x16x32_bf16 v[176:179], a[228:231], a[156:159], v[176:179]\n\t" \
  "ds_read_b64_tr_b16 v[102:103], v20 offset:51200\n\t" \
  "s_add_u32 m0, s27, 0\n\t" \
  "v_mfma_f32_16x16x32_bf16 v[192:195], a[224:227], a[160:163], v[192:195]\n\t" \
  "buffer_load_dwordx4 v12, s[40:43], s84 offen offset:0 sc0 sc1 lds\n\t" \
  "v_mfma_f32_16x16x32_bf16 v[192:195], a[228:231], a[164:167], v[192:195]\n\t" \
  "ds_read_b128 a[152:155], v24 offset:38912\n\t" \
  "v_mfma_f32_16x16x32_bf16 v[196:199], a[232:235], a[160:163], v[196:199]\n\t" \
  "ds_read_b128 a[156:159], v25 offset:38912\n\t" \
  "v_mfma_f32_16x16x32_bf16 v[196:199], a[236:239], a[164:167], v[196:199]\n\t" \
  "ds_read_b64_tr_b16 v[104:105], v21 offset:32768\n\t" \
  "v_mfma_f32_16x16x32_bf16 v[200:203], a[240:243], a[160:163], v[200:203]\n\t" \
  "s_mov_b32 s90, s89\n\t" \
  "ds_read_b64_tr_b16 v[106:107], v21 offset:34816\n\t" \
  "v_mfma_f32_16x16x32_bf16 v[200:203], a[244:247], a[164:167], v[200:203]\n\t" \
  "buffer_load_dwordx4 v13, s[40:43], s84 offen offset:1024 sc0 sc1 lds\n\t" \
  "s_lshl_b32 s89, s73, 9\n\t" \
  "v_mfma_f32_16x16x32_bf16 v[204:207], a[248:251], a[160:163], v[204:207]\n\t" \
  "ds_read_b64_tr_b16 v[108:109], v21 offset:49152\n\t" \
  "v_mfma_f32_16x16x32_bf16 v[204:207], a[252:255], a[164:167], v[204:207]\n\t" \
  "ds_read_b64_tr_b16 v[110:111], v21 offset:51200\n\t" \
  "v_mfma_f32_16x16x32_bf16 v[220:223], a[248:251], a[168:171], v[220:223]\n\t" \
  "ds_read_b128 a[160:163], v24 offset:40960\n\t" \
  "v_mfma_f32_16x16x32_bf16 v[220:223], a[252:255], a[172:175], v[220:223]\n\t" \
  "ds_read_b128 a[164:167], v25 offset:40960\n\t" \
  "v_mfma_f32_16x16x32_bf16 v[216:219], a[240:243], a[168:171], v[216:219]\n\t" \
  "buffer_load_dwordx4 v14, s[40:43], s84 offen offset:2048 sc0 sc1 lds\n\t" \
  "v_mfma_f32_16x16x32_bf16 v[216:219], a[244:247], a[172:175], v[216:219]\n\t" \
  "ds_read_b64_tr_b16 v[112:113], v22 offset:32768\n\t" \
  "v_mfma_f32_16x16x32_bf16 v[212:215], a[232:235], a[168:171], v[212:215]\n\t" \
  "ds_read_b64_tr_b16 v[114:115], v22 offset:34816\n\t" \
  "v_mfma_f32_16x16x32_bf16 v[212:215], a[236:239], a[172:175], v[212:215]\n\t" \
  "ds_read_b64_tr_b16 v[116:117], v22 offset:49152\n\t" \
  "v_mfma_f32_16x16x32_bf16 v[208:211], a[224:227], a[168:171], v[208:211]\n\t" \
  "ds_read_b64_tr_b16 v[118:119], v22 offset:51200\n\t" \
  "v_mfma_f32_16x16x32_bf16 v[208:211], a[228:231], a[172:175], v[208:211]\n\t" \
  "buffer_load_dwordx4 v15, s[40:43], s84 offen offset:3072 sc0 sc1 lds\n\t" \
  "v_mfma_f32_16x16x32_bf16 v[224:227], a[224:227], a[176:179], v[224:227]\n\t" \
  "ds_read_b128 a[168:171], v24 offset:43008\n\t" \
  "v_mfma_f32_16x16x32_bf16 v[224:227], a[228:231], a[180:183], v[224:227]\n\t" \
  "ds_read_b128 a[172:175], v25 offset:43008\n\t" \
  "v_mfma_f32_16x16x32_bf16 v[228:231], a[232:235], a[176:179], v[228:231]\n\t" \
  "buffer_load_dword v92, v27, s[48:51], s89 offen sc0 sc1\n\t" \
  "v_mfma_f32_16x16x32_bf16 v[228:231], a[236:239], a[180:183], v[228:231]\n\t" \
  "ds_read_b64_tr_b16 v[120:121], v23 offset:32768\n\t" \
  "v_mfma_f32_16x16x32_bf16 v[232:235], a[240:243], a[176:179], v[232:235]\n\t" \
  "ds_read_b64_tr_b16 v[122:123], v23 offset:34816\n\t" \
  "v_mfma_f32_16x16x32_bf16 v[232:235], a[244:247], a[180:183], v[232:235]\n\t" \
  "ds_read_b64_tr_b16 v[124:125], v23 offset:49152\n\t" \
  "v_mfma_f32_16x16x32_bf16 v[236:239], a[248:251], a[176:179], v[236:239]\n\t" \
  "ds_read_b64_tr_b16 v[126:127], v23 offset:51200\n\t" \
  "v_mfma_f32_16x16x32_bf16 v[236:239], a[252:255], a[180:183], v[236:239]\n\t" \
  "v_mfma_f32_16x16x32_bf16 v[252:255], a[248:251], a[184:187], v[252:255]\n\t" \
  "ds_read_b128 a[176:179], v24 offset:45056\n\t" \
  "v_mfma_f32_16x16x32_bf16 v[252:255], a[252:255], a[188:191], v[252:255]\n\t" \
  "ds_read_b128 a[180:183], v25 offset:45056\n\t" \
  "v_mfma_f32_16x16x32_bf16 v[248:251], a[240:243], a[184:187], v[248:251]\n\t" \
  "v_mfma_f32_16x16x32_bf16 v[248:251], a[244:247], a[188:191], v[248:251]\n\t" \
  "v_mfma_f32_16x16x32_bf16 v[244:247], a[232:235], a[184:187], v[244:247]\n\t" \
  "v_mfma_f32_16x16x32_bf16 v[244:247], a[236:239], a[188:191], v[244:247]\n\t" \
  "v_mfma_f32_16x16x32_bf16 v[240:243], a[224:227], a[184:187], v[240:243]\n\t" \
  "v_mfma_f32_16x16x32_bf16 v[240:243], a[228:231], a[188:191], v[240:243]\n\t" \
  "s_waitcnt lgkmcnt(3)\n\t" \
  "v_mfma_f32_16x16x32_bf16 a[0:3], a[192:195], a[128:131], a[0:3]\n\t" \
  "ds_read_b128 a[184:187], v24 offset:47104\n\t" \
  "v_mfma_f32_16x16x32_bf16 a[0:3], a[196:199], a[132:135], a[0:3]\n\t" \
  "ds_read_b128 a[188:191], v25 offset:47104\n\t" \
  "v_cmp_gt_u32 vcc, s91, v93\n\t" \
  "v_mfma_f32_16x16x32_bf16 a[4:7], a[200:203], a[128:131], a[4:7]\n\t" \
  "v_mfma_f32_16x16x32_bf16 a[4:7], a[204:207], a[132:135], a[4:7]\n\t" \
  "s_cbranch_vccnz Lsp12_%=\n\t" \
  "Lspr12_%=:\n\t" \
  "v_mfma_f32_16x16x32_bf16 a[8:11], a[208:211], a[128:131], a[8:11]\n\t" \
  "s_mov_b32 s85, s86\n\t" \
  "s_mov_b32 s86, s73\n\t" \
  "v_mfma_f32_16x16x32_bf16 a[8:11], a[212:215], a[132:135], a[8:11]\n\t" \
  "s_lshl_b32 s84, s85, s74\n\t" \
  "s_add_u32 s84, s84, s75\n\t" \
  "v_mfma_f32_16x16x32_bf16 a[12:15], a[216:219], a[128:131], a[12:15]\n\t" \
  "s_waitcnt lgkmcnt(0)\n\t" \
  "s_barrier\n\t" \
  "s_add_u32 m0, s24, 32768\n\t" \
  "v_mfma_f32_16x16x32_bf16 a[12:15], a[220:223], a[132:135], a[12:15]\n\t" \
  "buffer_load_dwordx4 v0, s[40:43], s84 offen offset:0 sc0 sc1 lds\n\t" \
  "v_mfma_f32_16x16x32_bf16 a[28:31], a[216:219], a[136:139], a[28:31]\n\t" \
  "v_mfma_f32_16x16x32_bf16 a[28:31], a[220:223], a[140:143], a[28:31]\n\t" \
  "buffer_load_dwordx4 v1, s[40:43], s84 offen offset:1024 sc0 sc1 lds\n\t" \
  "v_mfma_f32_16x16x32_bf16 a[24:27], a[208:211], a[136:139], a[24:27]\n\t" \
  "v_mfma_f32_16x16x32_bf16 a[24:27], a[212:215], a[140:143], a[24:27]\n\t" \
  "buffer_load_dwordx4 v2, s[40:43], s84 offen offset:2048 sc0 sc1 lds\n\t" \
  "v_mfma_f32_16x16x32_bf16 a[20:23], a[200:203], a[136:139], a[20:23]\n\t" \
  "v_mfma_f32_16x16x32_bf16 a[20:23], a[204:207], a[140:143], a[20:23]\n\t" \
  "buffer_load_dwordx4 v3, s[40:43], s84 offen offset:3072 sc0 sc1 lds\n\t" \
  "v_mfma_f32_16x16x32_bf16 a[16:19], a[192:195], a[136:139], a[16:19]\n\t" \
  "v_mfma_f32_16x16x32_bf16 a[16:19], a[196:199], a[140:143], a[16:19]\n\t" \
  "v_mfma_f32_16x16x32_bf16 a[32:35], a[192:195], a[144:147], a[32:35]\n\t" \
  "v_mfma_f32_16x16x32_bf16 a[32:35], a[196:199], a[148:151], a[32:35]\n\t" \
  "v_mfma_f32_16x16x32_bf16 a[36:39], a[200:203], a[144:147], a[36:39]\n\t" \
  "v_mfma_f32_16x16x32_bf16 a[36:39], a[204:207], a[148:151], a[36:39]\n\t" \
  "v_mfma_f32_16x16x32_bf16 a[40:43], a[208:211], a[144:147], a[40:43]\n\t" \
  "v_mfma_f32_16x16x32_bf16 a[40:43], a[212:215], a[148:151], a[40:43]\n\t" \
  "v_mfma_f32_16x16x32_bf16 a[44:47], a[216:219], a[144:147], a[44:47]\n\t" \
  "v_mfma_f32_16x16x32_bf16 a[44:47], a[220:223], a[148:151], a[44:47]\n\t" \
  "v_mfma_f32_16x16x32_bf16 a[60:63], a[216:219], a[152:155], a[60:63]\n\t" \
  "v_mfma_f32_16x16x32_bf16 a[60:63], a[220:223], a[156:159], a[60:63]\n\t" \
  "v_mfma_f32_16x16x32_bf16 a[56:59], a[208:211], a[152:155], a[56:59]\n\t" \
  "v_mfma_f32_16x16x32_bf16 a[56:59], a[212:215], a[156:159], a[56:59]\n\t" \
  "v_mfma_f32_16x16x32_bf16 a[52:55], a[200:203], a[152:155], a[52:55]\n\t" \
  "v_mfma_f32_16x16x32_bf16 a[52:55], a[204:207], a[156:159], a[52:55]\n\t" \
  "v_mfma_f32_16x16x32_bf16 a[48:51], a[192:195], a[152:155], a[48:51]\n\t" \
  "v_mfma_f32_16x16x32_bf16 a[48:51], a[196:199], a[156:159], a[48:51]\n\t" \
  "s_lshr_b64 s[10:11], s[8:9], s87\n\t" \
  "s_sext_i32_i8 s10, s10\n\t" \
  "v_mfma_f32_16x16x32_bf16 a[64:67], a[192:195], a[160:163], a[64:67]\n\t" \
  "s_add_u32 s87, s87, 8\n\t" \
  "s_add_u32 s73, s73, s10\n\t" \
  "v_mfma_f32_16x16x32_bf16 a[64:67], a[196:199], a[164:167], a[64:67]\n\t" \
  "s_cmp_eq_u32 s87, 56\n\t" \
  "s_cselect_b32 s87, 0, s87\n\t" \
  "v_mfma_f32_16x16x32_bf16 a[68:71], a[200:203], a[160:163], a[68:71]\n\t" \
  "v_mfma_f32_16x16x32_bf16 a[68:71], a[204:207], a[164:167], a[68:71]\n\t" \
  "v_mfma_f32_16x16x32_bf16 a[72:75], a[208:211], a[160:163], a[72:75]\n\t" \
  "v_mfma_f32_16x16x32_bf16 a[72:75], a[212:215], a[164:167], a[72:75]\n\t" \
  "v_mfma_f32_16x16x32_bf16 a[76:79], a[216:219], a[160:163], a[76:79]\n\t" \
  "v_mfma_f32_16x16x32_bf16 a[76:79], a[220:223], a[164:167], a[76:79]\n\t" \
  "v_mfma_f32_16x16x32_bf16 a[92:95], a[216:219], a[168:171], a[92:95]\n\t" \
  "v_mfma_f32_16x16x32_bf16 a[92:95], a[220:223], a[172:175], a[92:95]\n\t" \
  "v_mfma_f32_16x16x32_bf16 a[88:91], a[208:211], a[168:171], a[88:91]\n\t" \
  "v_mfma_f32_16x16x32_bf16 a[88:91], a[212:215], a[172:175], a[88:91]\n\t" \
  "v_mfma_f32_16x16x32_bf16 a[84:87], a[200:203], a[168:171], a[84:87]\n\t" \
  "v_mfma_f32_16x16x32_bf16 a[84:87], a[204:207], a[172:175], a[84:87]\n\t" \
  "v_mfma_f32_16x16x32_bf16 a[80:83], a[192:195], a[168:171], a[80:83]\n\t" \
  "v_mfma_f32_16x16x32_bf16 a[80:83], a[196:199], a[172:175], a[80:83]\n\t" \
  "v_mfma_f32_16x16x32_bf16 a[96:99], a[192:195], a[176:179], a[96:99]\n\t" \
  "v_mfma_f32_16x16x32_bf16 a[96:99], a[196:199], a[180:183], a[96:99]\n\t" \
  "v_mfma_f32_16x16x32_bf16 a[100:103], a[200:203], a[176:179], a[100:103]\n\t" \
  "v_mfma_f32_16x16x32_bf16 a[100:103], a[204:207], a[180:183], a[100:103]\n\t" \
  "v_mfma_f32_16x16x32_bf16 a[104:107], a[208:211], a[176:179], a[104:107]\n\t" \
  "v_mfma_f32_16x16x32_bf16 a[104:107], a[212:215], a[180:183], a[104:107]\n\t" \
  "v_mfma_f32_16x16x32_bf16 a[108:111], a[216:219], a[176:179], a[108:111]\n\t" \
  "v_mfma_f32_16x16x32_bf16 a[108:111], a[220:223], a[180:183], a[108:111]\n\t" \
  "s_add_u32 m0, s25, 32768\n\t" \
  "v_mfma_f32_16x16x32_bf16 a[124:127], a[216:219], a[184:187], a[124:127]\n\t" \
  "buffer_load_dwordx4 v4, s[40:43], s84 offen offset:0 sc0 sc1 lds\n\t" \
  "v_mfma_f32_16x16x32_bf16 a[124:127], a[220:223], a[188:191], a[124:127]\n\t" \
  "v_mfma_f32_16x16x32_bf16 a[120:123], a[208:211], a[184:187], a[120:123]\n\t" \
  "v_mfma_f32_16x16x32_bf16 a[120:123], a[212:215], a[188:191], a[120:123]\n\t" \
  "v_mfma_f32_16x16x32_bf16 a[116:119], a[200:203], a[184:187], a[116:119]\n\t" \
  "v_mfma_f32_16x16x32_bf16 a[116:119], a[204:207], a[188:191], a[116:119]\n\t" \
  "buffer_load_dwordx4 v5, s[40:43], s84 offen offset:1024 sc0 sc1 lds\n\t" \
  "v_mfma_f32_16x16x32_bf16 a[112:115], a[192:195], a[184:187], a[112:115]\n\t" \
  "v_mfma_f32_16x16x32_bf16 a[112:115], a[196:199], a[188:191], a[112:115]\n\t" \
  "s_waitcnt vmcnt(6)\n\t" \
  "s_barrier\n\t" \
  "v_mfma_f32_16x16x32_bf16 v[128:131], v[96:99], a[128:131], v[128:131]\n\t" \
  "ds_read_b64_tr_b16 a[192:193], v16 offset:0\n\t" \
  "v_mfma_f32_16x16x32_bf16 v[128:131], v[100:103], a[132:135], v[128:131]\n\t" \
  "ds_read_b64_tr_b16 a[194:195], v16 offset:2048\n\t" \
  "v_mfma_f32_16x16x32_bf16 v[132:135], v[104:107], a[128:131], v[132:135]\n\t" \
  "buffer_load_dwordx4 v6, s[40:43], s84 offen offset:2048 sc0 sc1 lds\n\t" \
  "v_mfma_f32_16x16x32_bf16 v[132:135], v[108:111], a[132:135], v[132:135]\n\t" \
  "ds_read_b64_tr_b16 a[196:197], v16 offset:16384\n\t" \
  "v_mfma_f32_16x16x32_bf16 v[136:139], v[112:115], a[128:131], v[136:139]\n\t" \
  "ds_read_b64_tr_b16 a[198:199], v16 offset:18432\n\t" \
  "v_mfma_f32_16x16x32_bf16 v[136:139], v[116:119], a[132:135], v[136:139]\n\t" \
  "ds_read_b64_tr_b16 a[200:201], v17 offset:0\n\t" \
  "v_mfma_f32_16x16x32_bf16 v[140:143], v[120:123], a[128:131], v[140:143]\n\t" \
  "ds_read_b64_tr_b16 a[202:203], v17 offset:2048\n\t" \
  "v_mfma_f32_16x16x32_bf16 v[140:143], v[124:127], a[132:135], v[140:143]\n\t" \
  "buffer_load_dwordx4 v7, s[40:43], s84 offen offset:3072 sc0 sc1 lds\n\t" \
  "v_mfma_f32_16x16x32_bf16 v[156:159], v[120:123], a[136:139], v[156:159]\n\t" \
  "ds_read_b128 a[128:131], v24 offset:0\n\t" \
  "v_mfma_f32_16x16x32_bf16 v[156:159], v[124:127], a[140:143], v[156:159]\n\t" \
  "ds_read_b128 a[132:135], v25 offset:0\n\t" \
  "v_mfma_f32_16x16x32_bf16 v[152:155], v[112:115], a[136:139], v[152:155]\n\t" \
  "ds_read_b64_tr_b16 a[204:205], v17 offset:16384\n\t" \
  "v_mfma_f32_16x16x32_bf16 v[152:155], v[116:119], a[140:143], v[152:155]\n\t" \
  "ds_read_b64_tr_b16 a[206:207], v17 offset:18432\n\t" \
  "s_add_u32 m0, s26, 32768\n\t" \
  "v_mfma_f32_16x16x32_bf16 v[148:151], v[104:107], a[136:139], v[148:151]\n\t" \
  "buffer_load_dwordx4 v8, s[40:43], s84 offen offset:0 sc0 sc1 lds\n\t" \
  "v_mfma_f32_16x16x32_bf16 v[148:151], v[108:111], a[140:143], v[148:151]\n\t" \
  "ds_read_b64_tr_b16 a[208:209], v18 offset:0\n\t" \
  "v_mfma_f32_16x16x32_bf16 v[144:147], v[96:99], a[136:139], v[144:147]\n\t" \
  "ds_read_b64_tr_b16 a[210:211], v18 offset:2048\n\t" \
  "v_mfma_f32_16x16x32_bf16 v[144:147], v[100:103], a[140:143], v[144:147]\n\t" \
  "ds_read_b64_tr_b16 a[212:213], v18 offset:16384\n\t" \
  "v_mfma_f32_16x16x32_bf16 v[160:163], v[96:99], a[144:147], v[160:163]\n\t" \
  "ds_read_b128 a[136:139], v24 offset:2048\n\t" \
  "v_mfma_f32_16x16x32_bf16 v[160:163], v[100:103], a[148:151], v[160:163]\n\t" \
  "buffer_load_dwordx4 v9, s[40:43], s84 offen offset:1024 sc0 sc1 lds\n\t" \
  "v_mfma_f32_16x16x32_bf16 v[164:167], v[104:107], a[144:147], v[164:167]\n\t" \
  "ds_read_b128 a[140:143], v25 offset:2048\n\t" \
  "v_mfma_f32_16x16x32_bf16 v[164:167], v[108:111], a[148:151], v[164:167]\n\t" \
  "ds_read_b64_tr_b16 a[214:215], v18 offset:18432\n\t" \
  "v_mfma_f32_16x16x32_bf16 v[168:171], v[112:115], a[144:147], v[168:171]\n\t" \
  "ds_read_b64_tr_b16 a[216:217], v19 offset:0\n\t" \
  "v_mfma_f32_16x16x32_bf16 v[168:171], v[116:119], a[148:151], v[168:171]\n\t" \
  "ds_read_b64_tr_b16 a[218:219], v19 offset:2048\n\t" \
  "v_mfma_f32_16x16x32_bf16 v[172:175], v[120:123], a[144:147], v[172:175]\n\t" \
  "buffer_load_dwordx4 v10, s[40:43], s84 offen offset:2048 sc0 sc1 lds\n\t" \
  "v_mfma_f32_16x16x32_bf16 v[172:175], v[124:127], a[148:151], v[172:175]\n\t" \
  "ds_read_b64_tr_b16 a[220:221], v19 offset:16384\n\t" \
  "v_mfma_f32_16x16x32_bf16 v[188:191], v[120:123], a[152:155], v[188:191]\n\t" \
  "ds_read_b128 a[144:147], v24 offset:4096\n\t" \
  "v_mfma_f32_16x16x32_bf16 v[188:191], v[124:127], a[156:159], v[188:191]\n\t" \
  "ds_read_b128 a[148:151], v25 offset:4096\n\t" \
  "v_mfma_f32_16x16x32_bf16 v[184:187], v[112:115], a[152:155], v[184:187]\n\t" \
  "ds_read_b64_tr_b16 a[222:223], v19 offset:18432\n\t" \
  "v_mfma_f32_16x16x32_bf16 v[184:187], v[116:119], a[156:159], v[184:187]\n\t" \
  "buffer_load_dwordx4 v11, s[40:43], s84 offen offset:3072 sc0 sc1 lds\n\t" \
  "v_mfma_f32_16x16x32_bf16 v[180:183], v[104:107], a[152:155], v[180:183]\n\t" \
  "ds_read_b64_tr_b16 a[224:225], v20 offset:0\n\t" \
  "v_mfma_f32_16x16x32_bf16 v[180:183], v[108:111], a[156:159], v[180:183]\n\t" \
  "ds_read_b64_tr_b16 a[226:227], v20 offset:2048\n\t" \
  "v_mfma_f32_16x16x32_bf16 v[176:179], v[96:99], a[152:155], v[176:179]\n\t" \
  "ds_read_b64_tr_b16 a[228:229], v20 offset:16384\n\t" \
  "v_mfma_f32_16x16x32_bf16 v[176:179], v[100:103], a[156:159], v[176:179]\n\t" \
  "ds_read_b64_tr_b16 a[230:231], v20 offset:18432\n\t" \
  "s_add_u32 m0, s27, 32768\n\t" \
  "v_mfma_f32_16x16x32_bf16 v[192:195], v[96:99], a[160:163], v[192:195]\n\t" \
  "buffer_load_dwordx4 v12, s[40:43], s84 offen offset:0 sc0 sc1 lds\n\t" \
  "v_mfma_f32_16x16x32_bf16 v[192:195], v[100:103], a[164:167], v[192:195]\n\t" \
  "ds_read_b128 a[152:155], v24 offset:6144\n\t" \
  "v_mfma_f32_16x16x32_bf16 v[196:199], v[104:107], a[160:163], v[196:199]\n\t" \
  "ds_read_b128 a[156:159], v25 offset:6144\n\t" \
  "v_mfma_f32_16x16x32_bf16 v[196:199], v[108:111], a[164:167], v[196:199]\n\t" \
  "ds_read_b64_tr_b16 a[232:233], v21 offset:0\n\t" \
  "v_mfma_f32_16x16x32_bf16 v[200:203], v[112:115], a[160:163], v[200:203]\n\t" \
  "s_mov_b32 s90, s89\n\t" \
  "ds_read_b64_tr_b16 a[234:235], v21 offset:2048\n\t" \
  "v_mfma_f32_16x16x32_bf16 v[200:203], v[116:119], a[164:167], v[200:203]\n\t" \
  "buffer_load_dwordx4 v13, s[40:43], s84 offen offset:1024 sc0 sc1 lds\n\t" \
  "s_lshl_b32 s89, s73, 9\n\t" \
  "v_mfma_f32_16x16x32_bf16 v[204:207], v[120:123], a[160:163], v[204:207]\n\t" \
  "ds_read_b64_tr_b16 a[236:237], v21 offset:16384\n\t" \
  "v_mfma_f32_16x16x32_bf16 v[204:207], v[124:127], a[164:167], v[204:207]\n\t" \
  "ds_read_b64_tr_b16 a[238:239], v21 offset:18432\n\t" \
  "v_mfma_f32_16x16x32_bf16 v[220:223], v[120:123], a[168:171], v[220:223]\n\t" \
  "ds_read_b128 a[160:163], v24 offset:8192\n\t" \
  "v_mfma_f32_16x16x32_bf16 v[220:223], v[124:127], a[172:175], v[220:223]\n\t" \
  "ds_read_b128 a[164:167], v25 offset:8192\n\t" \
  "v_mfma_f32_16x16x32_bf16 v[216:219], v[112:115], a[168:171], v[216:219]\n\t" \
  "buffer_load_dwordx4 v14, s[40:43], s84 offen offset:2048 sc0 sc1 lds\n\t" \
  "v_mfma_f32_16x16x32_bf16 v[216:219], v[116:119], a[172:175], v[216:219]\n\t" \
  "ds_read_b64_tr_b16 a[240:241], v22 offset:0\n\t" \
  "v_mfma_f32_16x16x32_bf16 v[212:215], v[104:107], a[168:171], v[212:215]\n\t" \
  "ds_read_b64_tr_b16 a[242:243], v22 offset:2048\n\t" \
  "v_mfma_f32_16x16x32_bf16 v[212:215], v[108:111], a[172:175], v[212:215]\n\t" \
  "ds_read_b64_tr_b16 a[244:245], v22 offset:16384\n\t" \
  "v_mfma_f32_16x16x32_bf16 v[208:211], v[96:99], a[168:171], v[208:211]\n\t" \
  "ds_read_b64_tr_b16 a[246:247], v22 offset:18432\n\t" \
  "v_mfma_f32_16x16x32_bf16 v[208:211], v[100:103], a[172:175], v[208:211]\n\t" \
  "buffer_load_dwordx4 v15, s[40:43], s84 offen offset:3072 sc0 sc1 lds\n\t" \
  "v_mfma_f32_16x16x32_bf16 v[224:227], v[96:99], a[176:179], v[224:227]\n\t" \
  "ds_read_b128 a[168:171], v24 offset:10240\n\t" \
  "v_mfma_f32_16x16x32_bf16 v[224:227], v[100:103], a[180:183], v[224:227]\n\t" \
  "ds_read_b128 a[172:175], v25 offset:10240\n\t" \
  "v_mfma_f32_16x16x32_bf16 v[228:231], v[104:107], a[176:179], v[228:231]\n\t" \
  "buffer_load_dword v93, v27, s[48:51], s89 offen sc0 sc1\n\t" \
  "v_mfma_f32_16x16x32_bf16 v[228:231], v[108:111], a[180:183], v[228:231]\n\t" \
  "ds_read_b64_tr_b16 a[248:249], v23 offset:0\n\t" \
  "v_mfma_f32_16x16x32_bf16 v[232:235], v[112:115], a[176:179], v[232:235]\n\t" \
  "ds_read_b64_tr_b16 a[250:251], v23 offset:2048\n\t" \
  "v_mfma_f32_16x16x32_bf16 v[232:235], v[116:119], a[180:183], v[232:235]\n\t" \
  "ds_read_b64_tr_b16 a[252:253], v23 offset:16384\n\t" \
  "v_mfma_f32_16x16x32_bf16 v[236:239], v[120:123], a[176:179], v[236:239]\n\t" \
  "ds_read_b64_tr_b16 a[254:255], v23 offset:18432\n\t" \
  "v_mfma_f32_16x16x32_bf16 v[236:239], v[124:127], a[180:183], v[236:239]\n\t" \
  "v_mfma_f32_16x16x32_bf16 v[252:255], v[120:123], a[184:187], v[252:255]\n\t" \
  "ds_read_b128 a[176:179], v24 offset:12288\n\t" \
  "v_mfma_f32_16x16x32_bf16 v[252:255], v[124:127], a[188:191], v[252:255]\n\t" \
  "ds_read_b128 a[180:183], v25 offset:12288\n\t" \
  "v_mfma_f32_16x16x32_bf16 v[248:251], v[112:115], a[184:187], v[248:251]\n\t" \
  "v_mfma_f32_16x16x32_bf16 v[248:251], v[116:119], a[188:191], v[248:251]\n\t" \
  "v_mfma_f32_16x16x32_bf16 v[244:247], v[104:107], a[184:187], v[244:247]\n\t" \
  "v_mfma_f32_16x16x32_bf16 v[244:247], v[108:111], a[188:191], v[244:247]\n\t" \
  "v_mfma_f32_16x16x32_bf16 v[240:243], v[96:99], a[184:187], v[240:243]\n\t" \
  "v_mfma_f32_16x16x32_bf16 v[240:243], v[100:103], a[188:191], v[240:243]\n\t" \
  "s_sub_u32 s95, s95, 1\n\t" \
  "s_cmp_lg_u32 s95, 0\n\t" \
  "s_cbranch_scc1 LloopP_%=\n\t" \
  "Ltail1_%=:\n\t" \
  "s_waitcnt lgkmcnt(3)\n\t" \
  "v_mfma_f32_16x16x32_bf16 a[0:3], a[192:195], a[128:131], a[0:3]\n\t" \
  "ds_read_b128 a[184:187], v24 offset:14336\n\t" \
  "v_mfma_f32_16x16x32_bf16 a[0:3], a[196:199], a[132:135], a[0:3]\n\t" \
  "ds_read_b128 a[188:191], v25 offset:14336\n\t" \
  "v_cmp_gt_u32 vcc, s91, v92\n\t" \
  "v_mfma_f32_16x16x32_bf16 a[4:7], a[200:203], a[128:131], a[4:7]\n\t" \
  "v_mfma_f32_16x16x32_bf16 a[4:7], a[204:207], a[132:135], a[4:7]\n\t" \
  "s_cbranch_vccnz LspH0_%=\n\t" \
  "LsprH0_%=:\n\t" \
  "v_mfma_f32_16x16x32_bf16 a[8:11], a[208:211], a[128:131], a[8:11]\n\t" \
  "s_lshl_b32 s84, s86, s74\n\t" \
  "s_add_u32 s84, s84, s75\n\t" \
  "v_mfma_f32_16x16x32_bf16 a[8:11], a[212:215], a[132:135], a[8:11]\n\t" \
  "s_mov_b32 s18, 0\n\t" \
  "v_mfma_f32_16x16x32_bf16 a[12:15], a[216:219], a[128:131], a[12:15]\n\t" \
  "s_cmp_lt_u32 s72, 2\n\t" \
  "s_cselect_b32 s18, 0x20000, s18\n\t" \
  "s_waitcnt lgkmcnt(0)\n\t" \
  "s_barrier\n\t" \
  "s_add_u32 m0, s24, s18\n\t" \
  "v_mfma_f32_16x16x32_bf16 a[12:15], a[220:223], a[132:135], a[12:15]\n\t" \
  "buffer_load_dwordx4 v0, s[40:43], s84 offen offset:0 sc0 sc1 lds\n\t" \
  "v_mfma_f32_16x16x32_bf16 a[28:31], a[216:219], a[136:139], a[28:31]\n\t" \
  "v_mfma_f32_16x16x32_bf16 a[28:31], a[220:223], a[140:143], a[28:31]\n\t" \
  "v_mfma_f32_16x16x32_bf16 a[24:27], a[208:211], a[136:139], a[24:27]\n\t" \
  "v_mfma_f32_16x16x32_bf16 a[24:27], a[212:215], a[140:143], a[24:27]\n\t" \
  "buffer_load_dwordx4 v1, s[40:43], s84 offen offset:1024 sc0 sc1 lds\n\t" \
  "v_mfma_f32_16x16x32_bf16 a[20:23], a[200:203], a[136:139], a[20:23]\n\t" \
  "v_mfma_f32_16x16x32_bf16 a[20:23], a[204:207], a[140:143], a[20:23]\n\t" \
  "v_mfma_f32_16x16x32_bf16 a[16:19], a[192:195], a[136:139], a[16:19]\n\t" \
  "v_add_u32 v82, 0x20000, v24\n\t" \
  "v_mfma_f32_16x16x32_bf16 a[16:19], a[196:199], a[140:143], a[16:19]\n\t" \
  "v_add_u32 v83, 0x20000, v25\n\t" \
  "buffer_load_dwordx4 v2, s[40:43], s84 offen offset:2048 sc0 sc1 lds\n\t" \
  "v_mfma_f32_16x16x32_bf16 a[32:35], a[192:195], a[144:147], a[32:35]\n\t" \
  "v_mfma_f32_16x16x32_bf16 a[32:35], a[196:199], a[148:151], a[32:35]\n\t" \
  "v_mfma_f32_16x16x32_bf16 a[36:39], a[200:203], a[144:147], a[36:39]\n\t" \
  "v_mfma_f32_16x16x32_bf16 a[36:39], a[204:207], a[148:151], a[36:39]\n\t" \
  "buffer_load_dwordx4 v3, s[40:43], s84 offen offset:3072 sc0 sc1 lds\n\t" \
  "v_mfma_f32_16x16x32_bf16 a[40:43], a[208:211], a[144:147], a[40:43]\n\t" \
  "v_mfma_f32_16x16x32_bf16 a[40:43], a[212:215], a[148:151], a[40:43]\n\t" \
  "v_mfma_f32_16x16x32_bf16 a[44:47], a[216:219], a[144:147], a[44:47]\n\t" \
  "s_add_u32 m0, s25, s18\n\t" \
  "v_mfma_f32_16x16x32_bf16 a[44:47], a[220:223], a[148:151], a[44:47]\n\t" \
  "buffer_load_dwordx4 v4, s[40:43], s84 offen offset:0 sc0 sc1 lds\n\t" \
  "v_mfma_f32_16x16x32_bf16 a[60:63], a[216:219], a[152:155], a[60:63]\n\t" \
  "v_mfma_f32_16x16x32_bf16 a[60:63], a[220:223], a[156:159], a[60:63]\n\t" \
  "v_mfma_f32_16x16x32_bf16 a[56:59], a[208:211], a[152:155], a[56:59]\n\t" \
  "v_mfma_f32_16x16x32_bf16 a[56:59], a[212:215], a[156:159], a[56:59]\n\t" \
  "buffer_load_dwordx4 v5, s[40:43], s84 offen offset:1024 sc0 sc1 lds\n\t" \
  "v_mfma_f32_16x16x32_bf16 a[52:55], a[200:203], a[152:155], a[52:55]\n\t" \
  "v_mfma_f32_16x16x32_bf16 a[52:55], a[204:207], a[156:159], a[52:55]\n\t" \
  "v_mfma_f32_16x16x32_bf16 a[48:51], a[192:195], a[152:155], a[48:51]\n\t" \
  "v_mfma_f32_16x16x32_bf16 a[48:51], a[196:199], a[156:159], a[48:51]\n\t" \
  "buffer_load_dwordx4 v6, s[40:43], s84 offen offset:2048 sc0 sc1 lds\n\t" \
  "v_mfma_f32_16x16x32_bf16 a[64:67], a[192:195], a[160:163], a[64:67]\n\t" \
  "v_mfma_f32_16x16x32_bf16 a[64:67], a[196:199], a[164:167], a[64:67]\n\t" \
  "v_mfma_f32_16x16x32_bf16 a[68:71], a[200:203], a[160:163], a[68:71]\n\t" \
  "v_mfma_f32_16x16x32_bf16 a[68:71], a[204:207], a[164:167], a[68:71]\n\t" \
  "buffer_load_dwordx4 v7, s[40:43], s84 offen offset:3072 sc0 sc1 lds\n\t" \
  "v_mfma_f32_16x16x32_bf16 a[72:75], a[208:211], a[160:163], a[72:75]\n\t" \
  "v_mfma_f32_16x16x32_bf16 a[72:75], a[212:215], a[164:167], a[72:75]\n\t" \
  "v_mfma_f32_16x16x32_bf16 a[76:79], a[216:219], a[160:163], a[76:79]\n\t" \
  "s_add_u32 m0, s26, s18\n\t" \
  "v_mfma_f32_16x16x32_bf16 a[76:79], a[220:223], a[164:167], a[76:79]\n\t" \
  "buffer_load_dwordx4 v8, s[40:43], s84 offen offset:0 sc0 sc1 lds\n\t" \
  "v_mfma_f32_16x16x32_bf16 a[92:95], a[216:219], a[168:171], a[92:95]\n\t" \
  "s_waitcnt vmcnt(9)\n\t" \
  "s_barrier\n\t" \
  "v_mfma_f32_16x16x32_bf16 a[92:95], a[220:223], a[172:175], a[92:95]\n\t" \
  "ds_read_b128 a[128:131], v24 offset:32768\n\t" \
  "ds_read_b128 a[136:139], v24 offset:34816\n\t" \
  "v_mfma_f32_16x16x32_bf16 a[88:91], a[208:211], a[168:171], a[88:91]\n\t" \
  "ds_read_b128 a[132:135], v25 offset:32768\n\t" \
  "ds_read_b128 a[140:143], v25 offset:34816\n\t" \
  "v_mfma_f32_16x16x32_bf16 a[88:91], a[212:215], a[172:175], a[88:91]\n\t" \
  "buffer_load_dwordx4 v9, s[40:43], s84 offen offset:1024 sc0 sc1 lds\n\t" \
  "ds_read_b64_tr_b16 v[96:97], v16 offset:32768\n\t" \
  "v_mfma_f32_16x16x32_bf16 a[84:87], a[200:203], a[168:171], a[84:87]\n\t" \
  "ds_read_b64_tr_b16 v[98:99], v16 offset:34816\n\t" \
  "ds_read_b128 a[144:147], v24 offset:36864\n\t" \
  "v_mfma_f32_16x16x32_bf16 a[84:87], a[204:207], a[172:175], a[84:87]\n\t" \
  "ds_read_b64_tr_b16 v[100:101], v16 offset:49152\n\t" \
  "ds_read_b128 a[148:151], v25 offset:36864\n\t" \
  "v_mfma_f32_16x16x32_bf16 a[80:83], a[192:195], a[168:171], a[80:83]\n\t" \
  "ds_read_b64_tr_b16 v[102:103], v16 offset:51200\n\t" \
  "ds_read_b128 a[152:155], v24 offset:38912\n\t" \
  "v_mfma_f32_16x16x32_bf16 a[80:83], a[196:199], a[172:175], a[80:83]\n\t" \
  "buffer_load_dwordx4 v10, s[40:43], s84 offen offset:2048 sc0 sc1 lds\n\t" \
  "ds_read_b64_tr_b16 v[104:105], v17 offset:32768\n\t" \
  "v_mfma_f32_16x16x32_bf16 a[96:99], a[192:195], a[176:179], a[96:99]\n\t" \
  "ds_read_b64_tr_b16 v[106:107], v17 offset:34816\n\t" \
  "ds_read_b128 a[156:159], v25 offset:38912\n\t" \
  "v_mfma_f32_16x16x32_bf16 a[96:99], a[196:199], a[180:183], a[96:99]\n\t" \
  "ds_read_b64_tr_b16 v[108:109], v17 offset:49152\n\t" \
  "ds_read_b128 a[160:163], v24 offset:40960\n\t" \
  "v_mfma_f32_16x16x32_bf16 a[100:103], a[200:203], a[176:179], a[100:103]\n\t" \
  "ds_read_b64_tr_b16 v[110:111], v17 offset:51200\n\t" \
  "ds_read_b128 a[164:167], v25 offset:40960\n\t" \
  "v_mfma_f32_16x16x32_bf16 a[100:103], a[204:207], a[180:183], a[100:103]\n\t" \
  "buffer_load_dwordx4 v11, s[40:43], s84 offen offset:3072 sc0 sc1 lds\n\t" \
  "ds_read_b64_tr_b16 v[112:113], v18 offset:32768\n\t" \
  "v_mfma_f32_16x16x32_bf16 a[104:107], a[208:211], a[176:179], a[104:107]\n\t" \
  "ds_read_b64_tr_b16 v[114:115], v18 offset:34816\n\t" \
  "ds_read_b128 a[168:171], v24 offset:43008\n\t" \
  "v_mfma_f32_16x16x32_bf16 a[104:107], a[212:215], a[180:183], a[104:107]\n\t" \
  "ds_read_b64_tr_b16 v[116:117], v18 offset:49152\n\t" \
  "ds_read_b128 a[172:175], v25 offset:43008\n\t" \
  "v_mfma_f32_16x16x32_bf16 a[108:111], a[216:219], a[176:179], a[108:111]\n\t" \
  "ds_read_b64_tr_b16 v[118:119], v18 offset:51200\n\t" \
  "s_add_u32 m0, s27, s18\n\t" \
  "v_mfma_f32_16x16x32_bf16 a[108:111], a[220:223], a[180:183], a[108:111]\n\t" \
  "buffer_load_dwordx4 v12, s[40:43], s84 offen offset:0 sc0 sc1 lds\n\t" \
  "ds_read_b64_tr_b16 v[120:121], v19 offset:32768\n\t" \
  "v_mfma_f32_16x16x32_bf16 a[124:127], a[216:219], a[184:187], a[124:127]\n\t" \
  "ds_read_b64_tr_b16 v[122:123], v19 offset:34816\n\t" \
  "ds_read_b128 a[176:179], v24 offset:45056\n\t" \
  "v_mfma_f32_16x16x32_bf16 a[124:127], a[220:223], a[188:191], a[124:127]\n\t" \
  "ds_read_b64_tr_b16 v[124:125], v19 offset:49152\n\t" \
  "ds_read_b128 a[180:183], v25 offset:45056\n\t" \
  "v_mfma_f32_16x16x32_bf16 a[120:123], a[208:211], a[184:187], a[120:123]\n\t" \
  "ds_read_b64_tr_b16 v[126:127], v19 offset:51200\n\t" \
  "v_mfma_f32_16x16x32_bf16 a[120:123], a[212:215], a[188:191], a[120:123]\n\t" \
  "buffer_load_dwordx4 v13, s[40:43], s84 offen offset:1024 sc0 sc1 lds\n\t" \
  "v_mfma_f32_16x16x32_bf16 a[116:119], a[200:203], a[184:187], a[116:119]\n\t" \
  "v_mfma_f32_16x16x32_bf16 a[116:119], a[204:207], a[188:191], a[116:119]\n\t" \
  "v_mfma_f32_16x16x32_bf16 a[112:115], a[192:195], a[184:187], a[112:115]\n\t" \
  "v_mfma_f32_16x16x32_bf16 a[112:115], a[196:199], a[188:191], a[112:115]\n\t" \
  "buffer_load_dwordx4 v14, s[40:43], s84 offen offset:2048 sc0 sc1 lds\n\t" \
  "s_waitcnt lgkmcnt(7)\n\t" \
  "v_mfma_f32_16x16x32_bf16 a[0:3], v[96:99], a[128:131], a[0:3]\n\t" \
  "ds_read_b128 a[184:187], v24 offset:47104\n\t" \
  "v_mfma_f32_16x16x32_bf16 a[0:3], v[100:103], a[132:135], a[0:3]\n\t" \
  "ds_read_b128 a[188:191], v25 offset:47104\n\t" \
  "v_mfma_f32_16x16x32_bf16 a[4:7], v[104:107], a[128:131], a[4:7]\n\t" \
  "v_mfma_f32_16x16x32_bf16 a[4:7], v[108:111], a[132:135], a[4:7]\n\t" \
  "buffer_load_dwordx4 v15, s[40:43], s84 offen offset:3072 sc0 sc1 lds\n\t" \
  "v_mfma_f32_16x16x32_bf16 a[8:11], v[112:115], a[128:131], a[8:11]\n\t" \
  "s_waitcnt lgkmcnt(2)\n\t" \
  "v_mfma_f32_16x16x32_bf16 a[8:11], v[116:119], a[132:135], a[8:11]\n\t" \
  "v_mfma_f32_16x16x32_bf16 a[12:15], v[120:123], a[128:131], a[12:15]\n\t" \
  "v_mfma_f32_16x16x32_bf16 a[12:15], v[124:127], a[132:135], a[12:15]\n\t" \
  "v_mfma_f32_16x16x32_bf16 a[28:31], v[120:123], a[136:139], a[28:31]\n\t" \
  "v_mfma_f32_16x16x32_bf16 a[28:31], v[124:127], a[140:143], a[28:31]\n\t" \
  "v_mfma_f32_16x16x32_bf16 a[24:27], v[112:115], a[136:139], a[24:27]\n\t" \
  "v_mfma_f32_16x16x32_bf16 a[24:27], v[116:119], a[140:143], a[24:27]\n\t" \
  "v_mfma_f32_16x16x32_bf16 a[20:23], v[104:107], a[136:139], a[20:23]\n\t" \
  "v_mfma_f32_16x16x32_bf16 a[20:23], v[108:111], a[140:143], a[20:23]\n\t" \
  "v_mfma_f32_16x16x32_bf16 a[16:19], v[96:99], a[136:139], a[16:19]\n\t" \
  "v_mfma_f32_16x16x32_bf16 a[16:19], v[100:103], a[140:143], a[16:19]\n\t" \
  "v_mfma_f32_16x16x32_bf16 a[32:35], v[96:99], a[144:147], a[32:35]\n\t" \
  "v_mfma_f32_16x16x32_bf16 a[32:35], v[100:103], a[148:151], a[32:35]\n\t" \
  "v_mfma_f32_16x16x32_bf16 a[36:39], v[104:107], a[144:147], a[36:39]\n\t" \
  "v_mfma_f32_16x16x32_bf16 a[36:39], v[108:111], a[148:151], a[36:39]\n\t" \
  "v_mfma_f32_16x16x32_bf16 a[40:43], v[112:115], a[144:147], a[40:43]\n\t" \
  "v_mfma_f32_16x16x32_bf16 a[40:43], v[116:119], a[148:151], a[40:43]\n\t" \
  "v_mfma_f32_16x16x32_bf16 a[44:47], v[120:123], a[144:147], a[44:47]\n\t" \
  "v_mfma_f32_16x16x32_bf16 a[44:47], v[124:127], a[148:151], a[44:47]\n\t" \
  "v_mfma_f32_16x16x32_bf16 a[60:63], v[120:123], a[152:155], a[60:63]\n\t" \
  "v_mfma_f32_16x16x32_bf16 a[60:63], v[124:127], a[156:159], a[60:63]\n\t" \
  "v_mfma_f32_16x16x32_bf16 a[56:59], v[112:115], a[152:155], a[56:59]\n\t" \
  "v_mfma_f32_16x16x32_bf16 a[56:59], v[116:119], a[156:159], a[56:59]\n\t" \
  "v_mfma_f32_16x16x32_bf16 a[52:55], v[104:107], a[152:155], a[52:55]\n\t" \
  "v_mfma_f32_16x16x32_bf16 a[52:55], v[108:111], a[156:159], a[52:55]\n\t" \
  "v_mfma_f32_16x16x32_bf16 a[48:51], v[96:99], a[152:155], a[48:51]\n\t" \
  "v_mfma_f32_16x16x32_bf16 a[48:51], v[100:103], a[156:159], a[48:51]\n\t" \
  "v_mfma_f32_16x16x32_bf16 a[64:67], v[96:99], a[160:163], a[64:67]\n\t" \
  "v_mfma_f32_16x16x32_bf16 a[64:67], v[100:103], a[164:167], a[64:67]\n\t" \
  "v_mfma_f32_16x16x32_bf16 a[68:71], v[104:107], a[160:163], a[68:71]\n\t" \
  "v_mfma_f32_16x16x32_bf16 a[68:71], v[108:111], a[164:167], a[68:71]\n\t" \
  "v_mfma_f32_16x16x32_bf16 a[72:75], v[112:115], a[160:163], a[72:75]\n\t" \
  "s_waitcnt vmcnt(0)\n\t" \
  "s_barrier\n\t" \
  "v_mfma_f32_16x16x32_bf16 a[72:75], v[116:119], a[164:167], a[72:75]\n\t" \
  "ds_read_b128 a[128:131], v82 offset:0\n\t" \
  "ds_read_b128 a[136:139], v82 offset:2048\n\t" \
  "v_mfma_f32_16x16x32_bf16 a[76:79], v[120:123], a[160:163], a[76:79]\n\t" \
  "ds_read_b128 a[132:135], v83 offset:0\n\t" \
  "ds_read_b128 a[140:143], v83 offset:2048\n\t" \
  "v_mfma_f32_16x16x32_bf16 a[76:79], v[124:127], a[164:167], a[76:79]\n\t" \
  "ds_read_b64_tr_b16 a[192:193], v16 offset:0\n\t" \
  "ds_read_b128 a[144:147], v82 offset:4096\n\t" \
  "v_mfma_f32_16x16x32_bf16 a[92:95], v[120:123], a[168:171], a[92:95]\n\t" \
  "ds_read_b64_tr_b16 a[194:195], v16 offset:2048\n\t" \
  "ds_read_b128 a[148:151], v83 offset:4096\n\t" \
  "v_mfma_f32_16x16x32_bf16 a[92:95], v[124:127], a[172:175], a[92:95]\n\t" \
  "ds_read_b64_tr_b16 a[196:197], v16 offset:16384\n\t" \
  "ds_read_b128 a[152:155], v82 offset:6144\n\t" \
  "v_mfma_f32_16x16x32_bf16 a[88:91], v[112:115], a[168:171], a[88:91]\n\t" \
  "ds_read_b64_tr_b16 a[198:199], v16 offset:18432\n\t" \
  "ds_read_b128 a[156:159], v83 offset:6144\n\t" \
  "v_mfma_f32_16x16x32_bf16 a[88:91], v[116:119], a[172:175], a[88:91]\n\t" \
  "ds_read_b64_tr_b16 a[200:201], v17 offset:0\n\t" \
  "ds_read_b128 a[160:163], v82 offset:8192\n\t" \
  "v_mfma_f32_16x16x32_bf16 a[84:87], v[104:107], a[168:171], a[84:87]\n\t" \
  "ds_read_b64_tr_b16 a[202:203], v17 offset:2048\n\t" \
  "ds_read_b128 a[164:167], v83 offset:8192\n\t" \
  "v_mfma_f32_16x16x32_bf16 a[84:87], v[108:111], a[172:175], a[84:87]\n\t" \
  "ds_read_b64_tr_b16 a[204:205], v17 offset:16384\n\t" \
  "v_mfma_f32_16x16x32_bf16 a[80:83], v[96:99], a[168:171], a[80:83]\n\t" \
  "ds_read_b64_tr_b16 a[206:207], v17 offset:18432\n\t" \
  "v_mfma_f32_16x16x32_bf16 a[80:83], v[100:103], a[172:175], a[80:83]\n\t" \
  "ds_read_b64_tr_b16 a[208:209], v18 offset:0\n\t" \
  "v_mfma_f32_16x16x32_bf16 a[96:99], v[96:99], a[176:179], a[96:99]\n\t" \
  "ds_read_b64_tr_b16 a[210:211], v18 offset:2048\n\t" \
  "ds_read_b128 a[168:171], v82 offset:10240\n\t" \
  "v_mfma_f32_16x16x32_bf16 a[96:99], v[100:103], a[180:183], a[96:99]\n\t" \
  "ds_read_b64_tr_b16 a[212:213], v18 offset:16384\n\t" \
  "ds_read_b128 a[172:175], v83 offset:10240\n\t" \
  "v_mfma_f32_16x16x32_bf16 a[100:103], v[104:107], a[176:179], a[100:103]\n\t" \
  "ds_read_b64_tr_b16 a[214:215], v18 offset:18432\n\t" \
  "v_mfma_f32_16x16x32_bf16 a[100:103], v[108:111], a[180:183], a[100:103]\n\t" \
  "ds_read_b64_tr_b16 a[216:217], v19 offset:0\n\t" \
  "v_mfma_f32_16x16x32_bf16 a[104:107], v[112:115], a[176:179], a[104:107]\n\t" \
  "ds_read_b64_tr_b16 a[218:219], v19 offset:2048\n\t" \
  "v_mfma_f32_16x16x32_bf16 a[104:107], v[116:119], a[180:183], a[104:107]\n\t" \
  "ds_read_b64_tr_b16 a[220:221], v19 offset:16384\n\t" \
  "v_mfma_f32_16x16x32_bf16 a[108:111], v[120:123], a[176:179], a[108:111]\n\t" \
  "ds_read_b64_tr_b16 a[222:223], v19 offset:18432\n\t" \
  "v_mfma_f32_16x16x32_bf16 a[108:111], v[124:127], a[180:183], a[108:111]\n\t" \
  "s_waitcnt lgkmcnt(11)\n\t" \
  "v_mfma_f32_16x16x32_bf16 a[124:127], v[120:123], a[184:187], a[124:127]\n\t" \
  "ds_read_b128 a[176:179], v82 offset:12288\n\t" \
  "v_mfma_f32_16x16x32_bf16 a[124:127], v[124:127], a[188:191], a[124:127]\n\t" \
  "ds_read_b128 a[180:183], v83 offset:12288\n\t" \
  "v_mfma_f32_16x16x32_bf16 a[120:123], v[112:115], a[184:187], a[120:123]\n\t" \
  "v_mfma_f32_16x16x32_bf16 a[120:123], v[116:119], a[188:191], a[120:123]\n\t" \
  "v_mfma_f32_16x16x32_bf16 a[116:119], v[104:107], a[184:187], a[116:119]\n\t" \
  "v_mfma_f32_16x16x32_bf16 a[116:119], v[108:111], a[188:191], a[116:119]\n\t" \
  "v_mfma_f32_16x16x32_bf16 a[112:115], v[96:99], a[184:187], a[112:115]\n\t" \
  "v_mfma_f32_16x16x32_bf16 a[112:115], v[100:103], a[188:191], a[112:115]\n\t" \
  "v_mfma_f32_16x16x32_bf16 a[0:3], a[192:195], a[128:131], a[0:3]\n\t" \
  "ds_read_b128 a[184:187], v82 offset:14336\n\t" \
  "v_mfma_f32_16x16x32_bf16 a[0:3], a[196:199], a[132:135], a[0:3]\n\t" \
  "ds_read_b128 a[188:191], v83 offset:14336\n\t" \
  "v_mfma_f32_16x16x32_bf16 a[4:7], a[200:203], a[128:131], a[4:7]\n\t" \
  "ds_read_b64_tr_b16 v[96:97], v20 offset:0\n\t" \
  "s_waitcnt lgkmcnt(4)\n\t" \
  "v_mfma_f32_16x16x32_bf16 a[4:7], a[204:207], a[132:135], a[4:7]\n\t" \
  "v_mfma_f32_16x16x32_bf16 a[8:11], a[208:211], a[128:131], a[8:11]\n\t" \
  "ds_read_b64_tr_b16 v[98:99], v20 offset:2048\n\t" \
  "v_mfma_f32_16x16x32_bf16 a[8:11], a[212:215], a[132:135], a[8:11]\n\t" \
  "v_mfma_f32_16x16x32_bf16 a[12:15], a[216:219], a[128:131], a[12:15]\n\t" \
  "ds_read_b64_tr_b16 v[100:101], v20 offset:16384\n\t" \
  "v_mfma_f32_16x16x32_bf16 a[12:15], a[220:223], a[132:135], a[12:15]\n\t" \
  "v_mfma_f32_16x16x32_bf16 a[28:31], a[216:219], a[136:139], a[28:31]\n\t" \
  "ds_read_b64_tr_b16 v[102:103], v20 offset:18432\n\t" \
  "v_mfma_f32_16x16x32_bf16 a[28:31], a[220:223], a[140:143], a[28:31]\n\t" \
  "v_mfma_f32_16x16x32_bf16 a[24:27], a[208:211], a[136:139], a[24:27]\n\t" \
  "ds_read_b64_tr_b16 v[104:105], v21 offset:0\n\t" \
  "v_mfma_f32_16x16x32_bf16 a[24:27], a[212:215], a[140:143], a[24:27]\n\t" \
  "v_mfma_f32_16x16x32_bf16 a[20:23], a[200:203], a[136:139], a[20:23]\n\t" \
  "ds_read_b64_tr_b16 v[106:107], v21 offset:2048\n\t" \
  "v_accvgpr_read_b32 v0, a0\n\t" \
  "v_accvgpr_read_b32 v1, a1\n\t" \
  "v_accvgpr_read_b32 v2, a2\n\t" \
  "v_mfma_f32_16x16x32_bf16 a[20:23], a[204:207], a[140:143], a[20:23]\n\t" \
  "v_accvgpr_read_b32 v3, a3\n\t" \
  "v_accvgpr_read_b32 v4, a4\n\t" \
  "v_accvgpr_read_b32 v5, a5\n\t" \
  "v_mfma_f32_16x16x32_bf16 a[16:19], a[192:195], a[136:139], a[16:19]\n\t" \
  "ds_read_b64_tr_b16 v[108:109], v21 offset:16384\n\t" \
  "v_accvgpr_read_b32 v6, a6\n\t" \
  "v_accvgpr_read_b32 v7, a7\n\t" \
  "v_cvt_pk_bf16_f32 v60, v0, v1\n\t" \
  "v_mfma_f32_16x16x32_bf16 a[16:19], a[196:199], a[140:143], a[16:19]\n\t" \
  "v_cvt_pk_bf16_f32 v61, v2, v3\n\t" \
  "v_cvt_pk_bf16_f32 v62, v4, v5\n\t" \
  "v_cvt_pk_bf16_f32 v63, v6, v7\n\t" \
  "v_mfma_f32_16x16x32_bf16 a[32:35], a[192:195], a[144:147], a[32:35]\n\t" \
  "ds_read_b64_tr_b16 v[110:111], v21 offset:18432\n\t" \
  "v_accvgpr_read_b32 v8, a8\n\t" \
  "v_accvgpr_read_b32 v9, a9\n\t" \
  "v_accvgpr_read_b32 v10, a10\n\t" \
  "v_mfma_f32_16x16x32_bf16 a[32:35], a[196:199], a[148:151], a[32:35]\n\t" \
  "v_accvgpr_read_b32 v11, a11\n\t" \
  "v_accvgpr_read_b32 v12, a12\n\t" \
  "v_accvgpr_read_b32 v13, a13\n\t" \
  "v_mfma_f32_16x16x32_bf16 a[36:39], a[200:203], a[144:147], a[36:39]\n\t" \
  "ds_read_b64_tr_b16 v[112:113], v22 offset:0\n\t" \
  "v_accvgpr_read_b32 v14, a14\n\t" \
  "v_accvgpr_read_b32 v15, a15\n\t" \
  "v_cvt_pk_bf16_f32 v64, v8, v9\n\t" \
  "v_mfma_f32_16x16x32_bf16 a[36:39], a[204:207], a[148:151], a[36:39]\n\t" \
  "v_cvt_pk_bf16_f32 v65, v10, v11\n\t" \
  "v_cvt_pk_bf16_f32 v66, v12, v13\n\t" \
  "v_cvt_pk_bf16_f32 v67, v14, v15\n\t" \
  "s_mov_b32 s10, 0\n\t" \
  "v_mfma_f32_16x16x32_bf16 a[40:43], a[208:211], a[144:147], a[40:43]\n\t" \
  "ds_read_b64_tr_b16 v[114:115], v22 offset:2048\n\t" \
  "v_accvgpr_read_b32 v0, a16\n\t" \
  "v_accvgpr_read_b32 v1, a17\n\t" \
  "v_accvgpr_read_b32 v2, a18\n\t" \
  "v_mfma_f32_16x16x32_bf16 a[40:43], a[212:215], a[148:151], a[40:43]\n\t" \
  "buffer_store_dwordx4 v[60:63], v32, s[60:63], s10 offen sc1\n\t" \
  "v_accvgpr_read_b32 v3, a19\n\t" \
  "v_accvgpr_read_b32 v4, a20\n\t" \
  "v_accvgpr_read_b32 v5, a21\n\t" \
  "v_mfma_f32_16x16x32_bf16 a[44:47], a[216:219], a[144:147], a[44:47]\n\t" \
  "ds_read_b64_tr_b16 v[116:117], v22 offset:16384\n\t" \
  "v_accvgpr_read_b32 v6, a22\n\t" \
  "v_accvgpr_read_b32 v7, a23\n\t" \
  "v_cvt_pk_bf16_f32 v68, v0, v1\n\t" \
  "v_mfma_f32_16x16x32_bf16 a[44:47], a[220:223], a[148:151], a[44:47]\n\t" \
  "v_cvt_pk_bf16_f32 v69, v2, v3\n\t" \
  "v_cvt_pk_bf16_f32 v70, v4, v5\n\t" \
  "v_cvt_pk_bf16_f32 v71, v6, v7\n\t" \
  "s_mov_b32 s10, 131072\n\t" \
  "v_mfma_f32_16x16x32_bf16 a[60:63], a[216:219], a[152:155], a[60:63]\n\t" \
  "ds_read_b64_tr_b16 v[118:119], v22 offset:18432\n\t" \
  "v_accvgpr_read_b32 v8, a24\n\t" \
  "v_accvgpr_read_b32 v9, a25\n\t" \
  "v_accvgpr_read_b32 v10, a26\n\t" \
  "v_mfma_f32_16x16x32_bf16 a[60:63], a[220:223], a[156:159], a[60:63]\n\t" \
  "buffer_store_dwordx4 v[64:67], v32, s[60:63], s10 offen sc1\n\t" \
  "v_accvgpr_read_b32 v11, a27\n\t" \
  "v_accvgpr_read_b32 v12, a28\n\t" \
  "v_accvgpr_read_b32 v13, a29\n\t" \
  "v_mfma_f32_16x16x32_bf16 a[56:59], a[208:211], a[152:155], a[56:59]\n\t" \
  "ds_read_b64_tr_b16 v[120:121], v23 offset:0\n\t" \
  "v_accvgpr_read_b32 v14, a30\n\t" \
  "v_accvgpr_read_b32 v15, a31\n\t" \
  "v_cvt_pk_bf16_f32 v60, v8, v9\n\t" \
  "v_mfma_f32_16x16x32_bf16 a[56:59], a[212:215], a[156:159], a[56:59]\n\t" \
  "v_cvt_pk_bf16_f32 v61, v10, v11\n\t" \
  "v_cvt_pk_bf16_f32 v62, v12, v13\n\t" \
  "v_cvt_pk_bf16_f32 v63, v14, v15\n\t" \
  "s_mov_b32 s10, 262144\n\t" \
  "v_mfma_f32_16x16x32_bf16 a[52:55], a[200:203], a[152:155], a[52:55]\n\t" \
  "ds_read_b64_tr_b16 v[122:123], v23 offset:2048\n\t" \
  "v_accvgpr_read_b32 v0, a32\n\t" \
  "v_accvgpr_read_b32 v1, a33\n\t" \
  "v_accvgpr_read_b32 v2, a34\n\t" \
  "v_mfma_f32_16x16x32_bf16 a[52:55], a[204:207], a[156:159], a[52:55]\n\t" \
  "buffer_store_dwordx4 v[68:71], v32, s[60:63], s10 offen sc1\n\t" \
  "v_accvgpr_read_b32 v3, a35\n\t" \
  "v_accvgpr_read_b32 v4, a36\n\t" \
  "v_accvgpr_read_b32 v5, a37\n\t" \
  "v_mfma_f32_16x16x32_bf16 a[48:51], a[192:195], a[152:155], a[48:51]\n\t" \
  "ds_read_b64_tr_b16 v[124:125], v23 offset:16384\n\t" \
  "v_accvgpr_read_b32 v6, a38\n\t" \
  "v_accvgpr_read_b32 v7, a39\n\t" \
  "v_cvt_pk_bf16_f32 v64, v0, v1\n\t" \
  "v_mfma_f32_16x16x32_bf16 a[48:51], a[196:199], a[156:159], a[48:51]\n\t" \
  "v_cvt_pk_bf16_f32 v65, v2, v3\n\t" \
  "v_cvt_pk_bf16_f32 v66, v4, v5\n\t" \
  "v_cvt_pk_bf16_f32 v67, v6, v7\n\t" \
  "s_mov_b32 s10, 393216\n\t" \
  "v_mfma_f32_16x16x32_bf16 a[64:67], a[192:195], a[160:163], a[64:67]\n\t" \
  "ds_read_b64_tr_b16 v[126:127], v23 offset:18432\n\t" \
  "v_accvgpr_read_b32 v8, a40\n\t" \
  "v_accvgpr_read_b32 v9, a41\n\t" \
  "v_accvgpr_read_b32 v10, a42\n\t" \
  "v_mfma_f32_16x16x32_bf16 a[64:67], a[196:199], a[164:167], a[64:67]\n\t" \
  "buffer_store_dwordx4 v[60:63], v32, s[60:63], s10 offen sc1\n\t" \
  "v_accvgpr_read_b32 v11, a43\n\t" \
  "v_accvgpr_read_b32 v12, a44\n\t" \
  "v_accvgpr_read_b32 v13, a45\n\t" \
  "v_mfma_f32_16x16x32_bf16 a[68:71], a[200:203], a[160:163], a[68:71]\n\t" \
  "v_accvgpr_read_b32 v14, a46\n\t" \
  "v_accvgpr_read_b32 v15, a47\n\t" \
  "v_cvt_pk_bf16_f32 v68, v8, v9\n\t" \
  "v_mfma_f32_16x16x32_bf16 a[68:71], a[204:207], a[164:167], a[68:71]\n\t" \
  "v_cvt_pk_bf16_f32 v69, v10, v11\n\t" \
  "v_cvt_pk_bf16_f32 v70, v12, v13\n\t" \
  "v_cvt_pk_bf16_f32 v71, v14, v15\n\t" \
  "s_mov_b32 s10, 524288\n\t" \
  "ds_read_b128 a[32:35], v24 offset:0\n\t" \
  "v_mfma_f32_16x16x32_bf16 a[72:75], a[208:211], a[160:163], a[72:75]\n\t" \
  "buffer_store_dwordx4 v[64:67], v32, s[60:63], s10 offen sc1\n\t" \
  "v_accvgpr_read_b32 v0, a48\n\t" \
  "v_accvgpr_read_b32 v1, a49\n\t" \
  "v_accvgpr_read_b32 v2, a50\n\t" \
  "v_mfma_f32_16x16x32_bf16 a[72:75], a[212:215], a[164:167], a[72:75]\n\t" \
  "ds_read_b128 a[36:39], v25 offset:0\n\t" \
  "v_accvgpr_read_b32 v3, a51\n\t" \
  "v_accvgpr_read_b32 v4, a52\n\t" \
  "v_accvgpr_read_b32 v5, a53\n\t" \
  "v_mfma_f32_16x16x32_bf16 a[76:79], a[216:219], a[160:163], a[76:79]\n\t" \
  "ds_read_b128 a[40:43], v24 offset:32768\n\t" \
  "v_accvgpr_read_b32 v6, a54\n\t" \
  "v_accvgpr_read_b32 v7, a55\n\t" \
  "v_cvt_pk_bf16_f32 v60, v0, v1\n\t" \
  "v_mfma_f32_16x16x32_bf16 a[76:79], a[220:223], a[164:167], a[76:79]\n\t" \
  "ds_read_b128 a[44:47], v25 offset:32768\n\t" \
  "v_cvt_pk_bf16_f32 v61, v2, v3\n\t" \
  "v_cvt_pk_bf16_f32 v62, v4, v5\n\t" \
  "v_cvt_pk_bf16_f32 v63, v6, v7\n\t" \
  "s_mov_b32 s10, 655360\n\t" \
  "v_mfma_f32_16x16x32_bf16 a[92:95], a[216:219], a[168:171], a[92:95]\n\t" \
  "buffer_store_dwordx4 v[68:71], v32, s[60:63], s10 offen sc1\n\t" \
  "v_accvgpr_read_b32 v8, a56\n\t" \
  "v_accvgpr_read_b32 v9, a57\n\t" \
  "v_accvgpr_read_b32 v10, a58\n\t" \
  "v_mfma_f32_16x16x32_bf16 a[92:95], a[220:223], a[172:175], a[92:95]\n\t" \
  "v_accvgpr_read_b32 v11, a59\n\t" \
  "v_accvgpr_read_b32 v12, a60\n\t" \
  "v_accvgpr_read_b32 v13, a61\n\t" \
  "v_mfma_f32_16x16x32_bf16 a[88:91], a[208:211], a[168:171], a[88:91]\n\t" \
  "v_accvgpr_read_b32 v14, a62\n\t" \
  "v_accvgpr_read_b32 v15, a63\n\t" \
  "v_cvt_pk_bf16_f32 v64, v8, v9\n\t" \
  "v_mfma_f32_16x16x32_bf16 a[88:91], a[212:215], a[172:175], a[88:91]\n\t" \
  "v_cvt_pk_bf16_f32 v65, v10, v11\n\t" \
  "v_cvt_pk_bf16_f32 v66, v12, v13\n\t" \
  "v_cvt_pk_bf16_f32 v67, v14, v15\n\t" \
  "s_mov_b32 s10, 786432\n\t" \
  "ds_read_b128 a[48:51], v24 offset:2048\n\t" \
  "v_mfma_f32_16x16x32_bf16 a[84:87], a[200:203], a[168:171], a[84:87]\n\t" \
  "buffer_store_dwordx4 v[60:63], v32, s[60:63], s10 offen sc1\n\t" \
  "v_accvgpr_read_b32 v0, a64\n\t" \
  "v_accvgpr_read_b32 v1, a65\n\t" \
  "v_accvgpr_read_b32 v2, a66\n\t" \
  "v_mfma_f32_16x16x32_bf16 a[84:87], a[204:207], a[172:175], a[84:87]\n\t" \
  "ds_read_b128 a[52:55], v25 offset:2048\n\t" \
  "v_accvgpr_read_b32 v3, a67\n\t" \
  "v_accvgpr_read_b32 v4, a68\n\t" \
  "v_accvgpr_read_b32 v5, a69\n\t" \
  "v_mfma_f32_16x16x32_bf16 a[80:83], a[192:195], a[168:171], a[80:83]\n\t" \
  "ds_read_b128 a[56:59], v24 offset:34816\n\t" \
  "v_accvgpr_read_b32 v6, a70\n\t" \
  "v_accvgpr_read_b32 v7, a71\n\t" \
  "v_cvt_pk_bf16_f32 v68, v0, v1\n\t" \
  "v_mfma_f32_16x16x32_bf16 a[80:83], a[196:199], a[172:175], a[80:83]\n\t" \
  "ds_read_b128 a[60:63], v25 offset:34816\n\t" \
  "v_cvt_pk_bf16_f32 v69, v2, v3\n\t" \
  "v_cvt_pk_bf16_f32 v70, v4, v5\n\t" \
  "v_cvt_pk_bf16_f32 v71, v6, v7\n\t" \
  "s_mov_b32 s10, 917504\n\t" \
  "v_mfma_f32_16x16x32_bf16 a[96:99], a[192:195], a[176:179], a[96:99]\n\t" \
  "buffer_store_dwordx4 v[64:67], v32, s[60:63], s10 offen sc1\n\t" \
  "v_accvgpr_read_b32 v8, a72\n\t" \
  "v_accvgpr_read_b32 v9, a73\n\t" \
  "v_accvgpr_read_b32 v10, a74\n\t" \
  "s_waitcnt lgkmcnt(5)\n\t" \
  "v_mfma_f32_16x16x32_bf16 a[96:99], a[196:199], a[180:183], a[96:99]\n\t" \
  "v_accvgpr_read_b32 v11, a75\n\t" \
  "v_accvgpr_read_b32 v12, a76\n\t" \
  "v_accvgpr_read_b32 v13, a77\n\t" \
  "v_mfma_f32_16x16x32_bf16 a[100:103], a[200:203], a[176:179], a[100:103]\n\t" \
  "v_accvgpr_read_b32 v14, a78\n\t" \
  "v_accvgpr_read_b32 v15, a79\n\t" \
  "v_cvt_pk_bf16_f32 v60, v8, v9\n\t" \
  "v_mfma_f32_16x16x32_bf16 a[100:103], a[204:207], a[180:183], a[100:103]\n\t" \
  "v_cvt_pk_bf16_f32 v61, v10, v11\n\t" \
  "v_cvt_pk_bf16_f32 v62, v12, v13\n\t" \
  "v_cvt_pk_bf16_f32 v63, v14, v15\n\t" \
  "s_mov_b32 s10, 1048576\n\t" \
  "v_mfma_f32_16x16x32_bf16 a[104:107], a[208:211], a[176:179], a[104:107]\n\t" \
  "buffer_store_dwordx4 v[68:71], v32, s[60:63], s10 offen sc1\n\t" \
  "v_accvgpr_read_b32 v0, a80\n\t" \
  "v_accvgpr_read_b32 v1, a81\n\t" \
  "v_accvgpr_read_b32 v2, a82\n\t" \
  "v_mfma_f32_16x16x32_bf16 a[104:107], a[212:215], a[180:183], a[104:107]\n\t" \
  "v_accvgpr_read_b32 v3, a83\n\t" \
  "v_accvgpr_read_b32 v4, a84\n\t" \
  "v_accvgpr_read_b32 v5, a85\n\t" \
  "v_mfma_f32_16x16x32_bf16 a[108:111], a[216:219], a[176:179], a[108:111]\n\t" \
  "v_accvgpr_read_b32 v6, a86\n\t" \
  "v_accvgpr_read_b32 v7, a87\n\t" \
  "v_cvt_pk_bf16_f32 v64, v0, v1\n\t" \
  "v_mfma_f32_16x16x32_bf16 a[108:111], a[220:223], a[180:183], a[108:111]\n\t" \
  "v_cvt_pk_bf16_f32 v65, v2, v3\n\t" \
  "v_cvt_pk_bf16_f32 v66, v4, v5\n\t" \
  "v_cvt_pk_bf16_f32 v67, v6, v7\n\t" \
  "s_mov_b32 s10, 1179648\n\t" \
  "v_mfma_f32_16x16x32_bf16 a[124:127], a[216:219], a[184:187], a[124:127]\n\t" \
  "buffer_store_dwordx4 v[60:63], v32, s[60:63], s10 offen sc1\n\t" \
  "v_accvgpr_read_b32 v8, a88\n\t" \
  "v_accvgpr_read_b32 v9, a89\n\t" \
  "v_accvgpr_read_b32 v10, a90\n\t" \
  "v_mfma_f32_16x16x32_bf16 a[124:127], a[220:223], a[188:191], a[124:127]\n\t" \
  "v_accvgpr_read_b32 v11, a91\n\t" \
  "v_accvgpr_read_b32 v12, a92\n\t" \
  "v_accvgpr_read_b32 v13, a93\n\t" \
  "v_mfma_f32_16x16x32_bf16 a[120:123], a[208:211], a[184:187], a[120:123]\n\t" \
  "v_accvgpr_read_b32 v14, a94\n\t" \
  "v_accvgpr_read_b32 v15, a95\n\t" \
  "v_cvt_pk_bf16_f32 v68, v8, v9\n\t" \
  "v_mfma_f32_16x16x32_bf16 a[120:123], a[212:215], a[188:191], a[120:123]\n\t" \
  "v_cvt_pk_bf16_f32 v69, v10, v11\n\t" \
  "v_cvt_pk_bf16_f32 v70, v12, v13\n\t" \
  "v_cvt_pk_bf16_f32 v71, v14, v15\n\t" \
  "s_mov_b32 s10, 1310720\n\t" \
  "v_mfma_f32_16x16x32_bf16 a[116:119], a[200:203], a[184:187], a[116:119]\n\t" \
  "buffer_store_dwordx4 v[64:67], v32, s[60:63], s10 offen sc1\n\t" \
  "v_accvgpr_read_b32 v0, a96\n\t" \
  "v_accvgpr_read_b32 v1, a97\n\t" \
  "v_accvgpr_read_b32 v2, a98\n\t" \
  "v_mfma_f32_16x16x32_bf16 a[116:119], a[204:207], a[188:191], a[116:119]\n\t" \
  "v_accvgpr_read_b32 v3, a99\n\t" \
  "v_accvgpr_read_b32 v4, a100\n\t" \
  "v_accvgpr_read_b32 v5, a101\n\t" \
  "v_mfma_f32_16x16x32_bf16 a[112:115], a[192:195], a[184:187], a[112:115]\n\t" \
  "v_accvgpr_read_b32 v6, a102\n\t" \
  "v_accvgpr_read_b32 v7, a103\n\t" \
  "v_cvt_pk_bf16_f32 v60, v0, v1\n\t" \
  "s_waitcnt lgkmcnt(0)\n\t" \
  "s_barrier\n\t" \
  "v_mfma_f32_16x16x32_bf16 a[112:115], a[196:199], a[188:191], a[112:115]\n\t" \
  "ds_read_b64_tr_b16 a[192:193], v20 offset:32768\n\t" \
  "v_cvt_pk_bf16_f32 v61, v2, v3\n\t" \
  "v_cvt_pk_bf16_f32 v62, v4, v5\n\t" \
  "v_cvt_pk_bf16_f32 v63, v6, v7\n\t" \
  "s_mov_b32 s10, 1441792\n\t" \
  "v_mfma_f32_16x16x32_bf16 v[128:131], v[96:99], a[128:131], v[128:131]\n\t" \
  "ds_read_b64_tr_b16 a[194:195], v20 offset:34816\n\t" \
  "v_mfma_f32_16x16x32_bf16 v[128:131], v[100:103], a[132:135], v[128:131]\n\t" \
  "ds_read_b64_tr_b16 a[196:197], v20 offset:49152\n\t" \
  "v_mfma_f32_16x16x32_bf16 v[128:131], a[224:227], a[32:35], v[128:131]\n\t" \
  "ds_read_b64_tr_b16 a[198:199], v20 offset:51200\n\t" \
  "v_mov_b32 v27, v37\n\t" \
  "s_mov_b32 s90, 0\n\t" \
  "s_lshl_b32 s91, s92, 1\n\t" \
  "v_mfma_f32_16x16x32_bf16 v[128:131], a[228:231], a[36:39], v[128:131]\n\t" \
  "ds_read_b64_tr_b16 a[200:201], v21 offset:32768\n\t" \
  "v_mfma_f32_16x16x32_bf16 v[132:135], v[104:107], a[128:131], v[132:135]\n\t" \
  "ds_read_b64_tr_b16 a[202:203], v21 offset:34816\n\t" \
  "v_mfma_f32_16x16x32_bf16 v[132:135], v[108:111], a[132:135], v[132:135]\n\t" \
  "ds_read_b64_tr_b16 a[204:205], v21 offset:49152\n\t" \
  "v_mfma_f32_16x16x32_bf16 v[132:135], a[232:235], a[32:35], v[132:135]\n\t" \
  "ds_read_b64_tr_b16 a[206:207], v21 offset:51200\n\t" \
  "v_mfma_f32_16x16x32_bf16 v[132:135], a[236:239], a[36:39], v[132:135]\n\t" \
  "ds_read_b64_tr_b16 a[208:209], v22 offset:32768\n\t" \
  "v_mfma_f32_16x16x32_bf16 v[136:139], v[112:115], a[128:131], v[136:139]\n\t" \
  "ds_read_b64_tr_b16 a[210:211], v22 offset:34816\n\t" \
  "v_mfma_f32_16x16x32_bf16 v[136:139], v[116:119], a[132:135], v[136:139]\n\t" \
  "ds_read_b64_tr_b16 a[212:213], v22 offset:49152\n\t" \
  "v_mfma_f32_16x16x32_bf16 v[136:139], a[240:243], a[32:35], v[136:139]\n\t" \
  "ds_read_b64_tr_b16 a[214:215], v22 offset:51200\n\t" \
  "v_mfma_f32_16x16x32_bf16 v[136:139], a[244:247], a[36:39], v[136:139]\n\t" \
  "ds_read_b64_tr_b16 a[216:217], v23 offset:32768\n\t" \
  "v_mfma_f32_16x16x32_bf16 v[140:143], v[120:123], a[128:131], v[140:143]\n\t" \
  "ds_read_b64_tr_b16 a[218:219], v23 offset:34816\n\t" \
  "v_accvgpr_read_b32 v8, a104\n\t" \
  "v_accvgpr_read_b32 v9, a105\n\t" \
  "v_accvgpr_read_b32 v10, a106\n\t" \
  "v_mfma_f32_16x16x32_bf16 v[140:143], v[124:127], a[132:135], v[140:143]\n\t" \
  "ds_read_b64_tr_b16 a[220:221], v23 offset:49152\n\t" \
  "v_accvgpr_read_b32 v11, a107\n\t" \
  "v_accvgpr_read_b32 v12, a108\n\t" \
  "v_accvgpr_read_b32 v13, a109\n\t" \
  "v_mfma_f32_16x16x32_bf16 v[140:143], a[248:251], a[32:35], v[140:143]\n\t" \
  "ds_read_b64_tr_b16 a[222:223], v23 offset:51200\n\t" \
  "v_accvgpr_read_b32 v14, a110\n\t" \
  "v_accvgpr_read_b32 v15, a111\n\t" \
  "v_cvt_pk_bf16_f32 v64, v8, v9\n\t" \
  "v_mfma_f32_16x16x32_bf16 v[140:143], a[252:255], a[36:39], v[140:143]\n\t" \
  "buffer_store_dwordx4 v[68:71], v32, s[60:63], s10 offen sc1\n\t" \
  "v_cvt_pk_bf16_f32 v65, v10, v11\n\t" \
  "v_cvt_pk_bf16_f32 v66, v12, v13\n\t" \
  "v_cvt_pk_bf16_f32 v67, v14, v15\n\t" \
  "s_mov_b32 s10, 1572864\n\t" \
  "s_waitcnt lgkmcnt(9)\n\t" \
  "v_mfma_f32_16x16x32_bf16 v[128:131], a[192:195], a[40:43], v[128:131]\n\t" \
  "buffer_store_dwordx4 v[60:63], v32, s[60:63], s10 offen sc1\n\t" \
  "v_accvgpr_read_b32 v0, a112\n\t" \
  "v_accvgpr_read_b32 v1, a113\n\t" \
  "v_accvgpr_read_b32 v2, a114\n\t" \
  "v_mfma_f32_16x16x32_bf16 v[128:131], a[196:199], a[44:47], v[128:131]\n\t" \
  "v_accvgpr_read_b32 v3, a115\n\t" \
  "v_accvgpr_read_b32 v4, a116\n\t" \
  "v_accvgpr_read_b32 v5, a117\n\t" \
  "v_mfma_f32_16x16x32_bf16 v[132:135], a[200:203], a[40:43], v[132:135]\n\t" \
  "v_accvgpr_read_b32 v6, a118\n\t" \
  "v_accvgpr_read_b32 v7, a119\n\t" \
  "v_cvt_pk_bf16_f32 v68, v0, v1\n\t" \
  "s_waitcnt lgkmcnt(6)\n\t" \
  "v_mfma_f32_16x16x32_bf16 v[132:135], a[204:207], a[44:47], v[132:135]\n\t" \
  "v_cvt_pk_bf16_f32 v69, v2, v3\n\t" \
  "v_cvt_pk_bf16_f32 v70, v4, v5\n\t" \
  "v_cvt_pk_bf16_f32 v71, v6, v7\n\t" \
  "s_mov_b32 s10, 1703936\n\t" \
  "v_mfma_f32_16x16x32_bf16 v[136:139], a[208:211], a[40:43], v[136:139]\n\t" \
  "buffer_store_dwordx4 v[64:67], v32, s[60:63], s10 offen sc1\n\t" \
  "v_accvgpr_read_b32 v8, a120\n\t" \
  "v_accvgpr_read_b32 v9, a121\n\t" \
  "v_accvgpr_read_b32 v10, a122\n\t" \
  "s_waitcnt lgkmcnt(4)\n\t" \
  "v_mfma_f32_16x16x32_bf16 v[136:139], a[212:215], a[44:47], v[136:139]\n\t" \
  "v_accvgpr_read_b32 v11, a123\n\t" \
  "v_accvgpr_read_b32 v12, a124\n\t" \
  "v_accvgpr_read_b32 v13, a125\n\t" \
  "s_waitcnt lgkmcnt(2)\n\t" \
  "v_mfma_f32_16x16x32_bf16 v[140:143], a[216:219], a[40:43], v[140:143]\n\t" \
  "v_accvgpr_read_b32 v14, a126\n\t" \
  "v_accvgpr_read_b32 v15, a127\n\t" \
  "v_cvt_pk_bf16_f32 v60, v8, v9\n\t" \
  "s_waitcnt lgkmcnt(0)\n\t" \
  "v_mfma_f32_16x16x32_bf16 v[140:143], a[220:223], a[44:47], v[140:143]\n\t" \
  "v_cvt_pk_bf16_f32 v61, v10, v11\n\t" \
  "v_cvt_pk_bf16_f32 v62, v12, v13\n\t" \
  "v_cvt_pk_bf16_f32 v63, v14, v15\n\t" \
  "s_mov_b32 s10, 1835008\n\t" \
  "v_mfma_f32_16x16x32_bf16 v[144:147], v[96:99], a[136:139], v[144:147]\n\t" \
  "ds_read_b128 a[32:35], v24 offset:4096\n\t" \
  "v_mfma_f32_16x16x32_bf16 v[144:147], v[100:103], a[140:143], v[144:147]\n\t" \
  "ds_read_b128 a[36:39], v25 offset:4096\n\t" \
  "v_mfma_f32_16x16x32_bf16 v[144:147], a[224:227], a[48:51], v[144:147]\n\t" \
  "ds_read_b128 a[40:43], v24 offset:36864\n\t" \
  "v_mfma_f32_16x16x32_bf16 v[144:147], a[228:231], a[52:55], v[144:147]\n\t" \
  "ds_read_b128 a[44:47], v25 offset:36864\n\t" \
  "v_mfma_f32_16x16x32_bf16 v[148:151], v[104:107], a[136:139], v[148:151]\n\t" \
  "buffer_store_dwordx4 v[68:71], v32, s[60:63], s10 offen sc1\n\t" \
  "s_mov_b32 s10, 1966080\n\t" \
  "v_mfma_f32_16x16x32_bf16 v[148:151], v[108:111], a[140:143], v[148:151]\n\t" \
  "buffer_store_dwordx4 v[60:63], v32, s[60:63], s10 offen sc1\n\t" \
  "v_mfma_f32_16x16x32_bf16 v[148:151], a[232:235], a[48:51], v[148:151]\n\t" \
  "v_mfma_f32_16x16x32_bf16 v[148:151], a[236:239], a[52:55], v[148:151]\n\t" \
  "v_mfma_f32_16x16x32_bf16 v[152:155], v[112:115], a[136:139], v[152:155]\n\t" \
  "v_mfma_f32_16x16x32_bf16 v[152:155], v[116:119], a[140:143], v[152:155]\n\t" \
  "v_mfma_f32_16x16x32_bf16 v[152:155], a[240:243], a[48:51], v[152:155]\n\t" \
  "v_mfma_f32_16x16x32_bf16 v[152:155], a[244:247], a[52:55], v[152:155]\n\t" \
  "v_mfma_f32_16x16x32_bf16 v[156:159], v[120:123], a[136:139], v[156:159]\n\t" \
  "v_mfma_f32_16x16x32_bf16 v[156:159], v[124:127], a[140:143], v[156:159]\n\t" \
  "v_mfma_f32_16x16x32_bf16 v[156:159], a[248:251], a[48:51], v[156:159]\n\t" \
  "v_mfma_f32_16x16x32_bf16 v[156:159], a[252:255], a[52:55], v[156:159]\n\t" \
  "v_mfma_f32_16x16x32_bf16 v[144:147], a[192:195], a[56:59], v[144:147]\n\t" \
  "v_mfma_f32_16x16x32_bf16 v[144:147], a[196:199], a[60:63], v[144:147]\n\t" \
  "v_mfma_f32_16x16x32_bf16 v[148:151], a[200:203], a[56:59], v[148:151]\n\t" \
  "v_mfma_f32_16x16x32_bf16 v[148:151], a[204:207], a[60:63], v[148:151]\n\t" \
  "v_mfma_f32_16x16x32_bf16 v[152:155], a[208:211], a[56:59], v[152:155]\n\t" \
  "v_mfma_f32_16x16x32_bf16 v[152:155], a[212:215], a[60:63], v[152:155]\n\t" \
  "v_mfma_f32_16x16x32_bf16 v[156:159], a[216:219], a[56:59], v[156:159]\n\t" \
  "v_mfma_f32_16x16x32_bf16 v[156:159], a[220:223], a[60:63], v[156:159]\n\t" \
  "v_mfma_f32_16x16x32_bf16 v[160:163], v[96:99], a[144:147], v[160:163]\n\t" \
  "ds_read_b128 a[48:51], v24 offset:6144\n\t" \
  "v_mfma_f32_16x16x32_bf16 v[160:163], v[100:103], a[148:151], v[160:163]\n\t" \
  "ds_read_b128 a[52:55], v25 offset:6144\n\t" \
  "s_waitcnt lgkmcnt(2)\n\t" \
  "v_mfma_f32_16x16x32_bf16 v[160:163], a[224:227], a[32:35], v[160:163]\n\t" \
  "ds_read_b128 a[56:59], v24 offset:38912\n\t" \
  "v_mfma_f32_16x16x32_bf16 v[160:163], a[228:231], a[36:39], v[160:163]\n\t" \
  "ds_read_b128 a[60:63], v25 offset:38912\n\t" \
  "v_mfma_f32_16x16x32_bf16 v[164:167], v[104:107], a[144:147], v[164:167]\n\t" \
  "v_mfma_f32_16x16x32_bf16 v[164:167], v[108:111], a[148:151], v[164:167]\n\t" \
  "v_mfma_f32_16x16x32_bf16 v[164:167], a[232:235], a[32:35], v[164:167]\n\t" \
  "v_mfma_f32_16x16x32_bf16 v[164:167], a[236:239], a[36:39], v[164:167]\n\t" \
  "v_mfma_f32_16x16x32_bf16 v[168:171], v[112:115], a[144:147], v[168:171]\n\t" \
  "v_mfma_f32_16x16x32_bf16 v[168:171], v[116:119], a[148:151], v[168:171]\n\t" \
  "v_mfma_f32_16x16x32_bf16 v[168:171], a[240:243], a[32:35], v[168:171]\n\t" \
  "v_mfma_f32_16x16x32_bf16 v[168:171], a[244:247], a[36:39], v[168:171]\n\t" \
  "v_mfma_f32_16x16x32_bf16 v[172:175], v[120:123], a[144:147], v[172:175]\n\t" \
  "v_mfma_f32_16x16x32_bf16 v[172:175], v[124:127], a[148:151], v[172:175]\n\t" \
  "v_mfma_f32_16x16x32_bf16 v[172:175], a[248:251], a[32:35], v[172:175]\n\t" \
  "v_mfma_f32_16x16x32_bf16 v[172:175], a[252:255], a[36:39], v[172:175]\n\t" \
  "v_mfma_f32_16x16x32_bf16 v[160:163], a[192:195], a[40:43], v[160:163]\n\t" \
  "v_mfma_f32_16x16x32_bf16 v[160:163], a[196:199], a[44:47], v[160:163]\n\t" \
  "v_mfma_f32_16x16x32_bf16 v[164:167], a[200:203], a[40:43], v[164:167]\n\t" \
  "v_mfma_f32_16x16x32_bf16 v[164:167], a[204:207], a[44:47], v[164:167]\n\t" \
  "v_mfma_f32_16x16x32_bf16 v[168:171], a[208:211], a[40:43], v[168:171]\n\t" \
  "v_mfma_f32_16x16x32_bf16 v[168:171], a[212:215], a[44:47], v[168:171]\n\t" \
  "v_mfma_f32_16x16x32_bf16 v[172:175], a[216:219], a[40:43], v[172:175]\n\t" \
  "v_mfma_f32_16x16x32_bf16 v[172:175], a[220:223], a[44:47], v[172:175]\n\t" \
  "v_mfma_f32_16x16x32_bf16 v[176:179], v[96:99], a[152:155], v[176:179]\n\t" \
  "ds_read_b128 a[32:35], v24 offset:8192\n\t" \
  "v_mfma_f32_16x16x32_bf16 v[176:179], v[100:103], a[156:159], v[176:179]\n\t" \
  "ds_read_b128 a[36:39], v25 offset:8192\n\t" \
  "s_waitcnt lgkmcnt(2)\n\t" \
  "v_mfma_f32_16x16x32_bf16 v[176:179], a[224:227], a[48:51], v[176:179]\n\t" \
  "ds_read_b128 a[40:43], v24 offset:40960\n\t" \
  "v_mfma_f32_16x16x32_bf16 v[176:179], a[228:231], a[52:55], v[176:179]\n\t" \
  "ds_read_b128 a[44:47], v25 offset:40960\n\t" \
  "v_mfma_f32_16x16x32_bf16 v[180:183], v[104:107], a[152:155], v[180:183]\n\t" \
  "v_mfma_f32_16x16x32_bf16 v[180:183], v[108:111], a[156:159], v[180:183]\n\t" \
  "v_mfma_f32_16x16x32_bf16 v[180:183], a[232:235], a[48:51], v[180:183]\n\t" \
  "s_lshl_b32 s11, s92, 1\n\t" \
  "s_add_u32 s11, s11, 1\n\t" \
  "v_mov_b32 v46, s11\n\t" \
  "v_mfma_f32_16x16x32_bf16 v[180:183], a[236:239], a[52:55], v[180:183]\n\t" \
  "s_waitcnt vmcnt(0)\n\t" \
  "buffer_store_dword v46, v36, s[48:51], 0 offen sc1\n\t" \
  "v_mfma_f32_16x16x32_bf16 v[184:187], v[112:115], a[152:155], v[184:187]\n\t" \
  "v_mfma_f32_16x16x32_bf16 v[184:187], v[116:119], a[156:159], v[184:187]\n\t" \
  "v_mfma_f32_16x16x32_bf16 v[184:187], a[240:243], a[48:51], v[184:187]\n\t" \
  "v_mfma_f32_16x16x32_bf16 v[184:187], a[244:247], a[52:55], v[184:187]\n\t" \
  "v_mfma_f32_16x16x32_bf16 v[188:191], v[120:123], a[152:155], v[188:191]\n\t" \
  "v_mfma_f32_16x16x32_bf16 v[188:191], v[124:127], a[156:159], v[188:191]\n\t" \
  "v_mfma_f32_16x16x32_bf16 v[188:191], a[248:251], a[48:51], v[188:191]\n\t" \
  "v_mfma_f32_16x16x32_bf16 v[188:191], a[252:255], a[52:55], v[188:191]\n\t" \
  "v_mfma_f32_16x16x32_bf16 v[176:179], a[192:195], a[56:59], v[176:179]\n\t" \
  "v_mfma_f32_16x16x32_bf16 v[176:179], a[196:199], a[60:63], v[176:179]\n\t" \
  "v_mfma_f32_16x16x32_bf16 v[180:183], a[200:203], a[56:59], v[180:183]\n\t" \
  "v_mfma_f32_16x16x32_bf16 v[180:183], a[204:207], a[60:63], v[180:183]\n\t" \
  "v_mfma_f32_16x16x32_bf16 v[184:187], a[208:211], a[56:59], v[184:187]\n\t" \
  "v_mfma_f32_16x16x32_bf16 v[184:187], a[212:215], a[60:63], v[184:187]\n\t" \
  "v_mfma_f32_16x16x32_bf16 v[188:191], a[216:219], a[56:59], v[188:191]\n\t" \
  "v_mfma_f32_16x16x32_bf16 v[188:191], a[220:223], a[60:63], v[188:191]\n\t" \
  "v_mfma_f32_16x16x32_bf16 v[192:195], v[96:99], a[160:163], v[192:195]\n\t" \
  "ds_read_b128 a[48:51], v24 offset:10240\n\t" \
  "v_mfma_f32_16x16x32_bf16 v[192:195], v[100:103], a[164:167], v[192:195]\n\t" \
  "ds_read_b128 a[52:55], v25 offset:10240\n\t" \
  "s_waitcnt lgkmcnt(2)\n\t" \
  "v_mfma_f32_16x16x32_bf16 v[192:195], a[224:227], a[32:35], v[192:195]\n\t" \
  "ds_read_b128 a[56:59], v24 offset:43008\n\t" \
  "v_mfma_f32_16x16x32_bf16 v[192:195], a[228:231], a[36:39], v[192:195]\n\t" \
  "ds_read_b128 a[60:63], v25 offset:43008\n\t" \
  "v_mfma_f32_16x16x32_bf16 v[196:199], v[104:107], a[160:163], v[196:199]\n\t" \
  "v_mfma_f32_16x16x32_bf16 v[196:199], v[108:111], a[164:167], v[196:199]\n\t" \
  "v_mfma_f32_16x16x32_bf16 v[196:199], a[232:235], a[32:35], v[196:199]\n\t" \
  "v_mfma_f32_16x16x32_bf16 v[196:199], a[236:239], a[36:39], v[196:199]\n\t" \
  "v_mfma_f32_16x16x32_bf16 v[200:203], v[112:115], a[160:163], v[200:203]\n\t" \
  "buffer_load_dword v93, v27, s[48:51], s90 offen sc0 sc1\n\t" \
  "v_mfma_f32_16x16x32_bf16 v[200:203], v[116:119], a[164:167], v[200:203]\n\t" \
  "v_mfma_f32_16x16x32_bf16 v[200:203], a[240:243], a[32:35], v[200:203]\n\t" \
  "v_mfma_f32_16x16x32_bf16 v[200:203], a[244:247], a[36:39], v[200:203]\n\t" \
  "v_mfma_f32_16x16x32_bf16 v[204:207], v[120:123], a[160:163], v[204:207]\n\t" \
  "v_mfma_f32_16x16x32_bf16 v[204:207], v[124:127], a[164:167], v[204:207]\n\t" \
  "v_mfma_f32_16x16x32_bf16 v[204:207], a[248:251], a[32:35], v[204:207]\n\t" \
  "v_mfma_f32_16x16x32_bf16 v[204:207], a[252:255], a[36:39], v[204:207]\n\t" \
  "v_mfma_f32_16x16x32_bf16 v[192:195], a[192:195], a[40:43], v[192:195]\n\t" \
  "v_mfma_f32_16x16x32_bf16 v[192:195], a[196:199], a[44:47], v[192:195]\n\t" \
  "v_mfma_f32_16x16x32_bf16 v[196:199], a[200:203], a[40:43], v[196:199]\n\t" \
  "v_mfma_f32_16x16x32_bf16 v[196:199], a[204:207], a[44:47], v[196:199]\n\t" \
  "v_mfma_f32_16x16x32_bf16 v[200:203], a[208:211], a[40:43], v[200:203]\n\t" \
  "v_mfma_f32_16x16x32_bf16 v[200:203], a[212:215], a[44:47], v[200:203]\n\t" \
  "v_mfma_f32_16x16x32_bf16 v[204:207], a[216:219], a[40:43], v[204:207]\n\t" \
  "v_mfma_f32_16x16x32_bf16 v[204:207], a[220:223], a[44:47], v[204:207]\n\t" \
  "v_mfma_f32_16x16x32_bf16 v[208:211], v[96:99], a[168:171], v[208:211]\n\t" \
  "ds_read_b128 a[32:35], v24 offset:12288\n\t" \
  "v_mfma_f32_16x16x32_bf16 v[208:211], v[100:103], a[172:175], v[208:211]\n\t" \
  "ds_read_b128 a[36:39], v25 offset:12288\n\t" \
  "s_waitcnt lgkmcnt(2)\n\t" \
  "v_mfma_f32_16x16x32_bf16 v[208:211], a[224:227], a[48:51], v[208:211]\n\t" \
  "ds_read_b128 a[40:43], v24 offset:45056\n\t" \
  "v_mfma_f32_16x16x32_bf16 v[208:211], a[228:231], a[52:55], v[208:211]\n\t" \
  "ds_read_b128 a[44:47], v25 offset:45056\n\t" \
  "v_mfma_f32_16x16x32_bf16 v[212:215], v[104:107], a[168:171], v[212:215]\n\t" \
  "v_mfma_f32_16x16x32_bf16 v[212:215], v[108:111], a[172:175], v[212:215]\n\t" \
  "v_mfma_f32_16x16x32_bf16 v[212:215], a[232:235], a[48:51], v[212:215]\n\t" \
  "v_mfma_f32_16x16x32_bf16 v[212:215], a[236:239], a[52:55], v[212:215]\n\t" \
  "v_mfma_f32_16x16x32_bf16 v[216:219], v[112:115], a[168:171], v[216:219]\n\t" \
  "v_mfma_f32_16x16x32_bf16 v[216:219], v[116:119], a[172:175], v[216:219]\n\t" \
  "v_mfma_f32_16x16x32_bf16 v[216:219], a[240:243], a[48:51], v[216:219]\n\t" \
  "v_mfma_f32_16x16x32_bf16 v[216:219], a[244:247], a[52:55], v[216:219]\n\t" \
  "v_mfma_f32_16x16x32_bf16 v[220:223], v[120:123], a[168:171], v[220:223]\n\t" \
  "v_mfma_f32_16x16x32_bf16 v[220:223], v[124:127], a[172:175], v[220:223]\n\t" \
  "v_mfma_f32_16x16x32_bf16 v[220:223], a[248:251], a[48:51], v[220:223]\n\t" \
  "s_add_u32 s91, s91, 1\n\t" \
  "s_waitcnt vmcnt(0)\n\t" \
  "v_cmp_gt_u32 vcc, s91, v93\n\t" \
  "s_cbranch_vccnz LspT01_%=\n\t" \
  "LsprT01_%=:\n\t" \
  "v_mfma_f32_16x16x32_bf16 v[220:223], a[252:255], a[52:55], v[220:223]\n\t" \
  "buffer_load_dword v45, v40, s[48:51], 0 offen sc1\n\t" \
  "v_mfma_f32_16x16x32_bf16 v[208:211], a[192:195], a[56:59], v[208:211]\n\t" \
  "s_mov_b32 s18, 0\n\t" \
  "buffer_load_dwordx4 a[0:3], v34, s[60:63], s18 offen sc1\n\t" \
  "v_mfma_f32_16x16x32_bf16 v[208:211], a[196:199], a[60:63], v[208:211]\n\t" \
  "v_mfma_f32_16x16x32_bf16 v[212:215], a[200:203], a[56:59], v[212:215]\n\t" \
  "s_mov_b32 s18, 131072\n\t" \
  "buffer_load_dwordx4 a[4:7], v34, s[60:63], s18 offen sc1\n\t" \
  "v_mfma_f32_16x16x32_bf16 v[212:215], a[204:207], a[60:63], v[212:215]\n\t" \
  "v_mfma_f32_16x16x32_bf16 v[216:219], a[208:211], a[56:59], v[216:219]\n\t" \
  "s_mov_b32 s18, 262144\n\t" \
  "buffer_load_dwordx4 a[8:11], v34, s[60:63], s18 offen sc1\n\t" \
  "v_mfma_f32_16x16x32_bf16 v[216:219], a[212:215], a[60:63], v[216:219]\n\t" \
  "v_mfma_f32_16x16x32_bf16 v[220:223], a[216:219], a[56:59], v[220:223]\n\t" \
  "s_mov_b32 s18, 393216\n\t" \
  "buffer_load_dwordx4 a[12:15], v34, s[60:63], s18 offen sc1\n\t" \
  "v_mfma_f32_16x16x32_bf16 v[220:223], a[220:223], a[60:63], v[220:223]\n\t" \
  "v_mfma_f32_16x16x32_bf16 v[224:227], v[96:99], a[176:179], v[224:227]\n\t" \
  "ds_read_b128 a[48:51], v24 offset:14336\n\t" \
  "v_mfma_f32_16x16x32_bf16 v[224:227], v[100:103], a[180:183], v[224:227]\n\t" \
  "ds_read_b128 a[52:55], v25 offset:14336\n\t" \
  "s_waitcnt lgkmcnt(2)\n\t" \
  "v_mfma_f32_16x16x32_bf16 v[224:227], a[224:227], a[32:35], v[224:227]\n\t" \
  "ds_read_b128 a[56:59], v24 offset:47104\n\t" \
  "v_mfma_f32_16x16x32_bf16 v[224:227], a[228:231], a[36:39], v[224:227]\n\t" \
  "ds_read_b128 a[60:63], v25 offset:47104\n\t" \
  "v_mfma_f32_16x16x32_bf16 v[228:231], v[104:107], a[176:179], v[228:231]\n\t" \
  "s_mov_b32 s18, 524288\n\t" \
  "buffer_load_dwordx4 a[16:19], v34, s[60:63], s18 offen sc1\n\t" \
  "v_mfma_f32_16x16x32_bf16 v[228:231], v[108:111], a[180:183], v[228:231]\n\t" \
  "s_mov_b32 s18, 655360\n\t" \
  "buffer_load_dwordx4 a[20:23], v34, s[60:63], s18 offen sc1\n\t" \
  "v_mfma_f32_16x16x32_bf16 v[228:231], a[232:235], a[32:35], v[228:231]\n\t" \
  "s_mov_b32 s18, 786432\n\t" \
  "buffer_load_dwordx4 a[24:27], v34, s[60:63], s18 offen sc1\n\t" \
  "v_mfma_f32_16x16x32_bf16 v[228:231], a[236:239], a[36:39], v[228:231]\n\t" \
  "s_mov_b32 s18, 917504\n\t" \
  "buffer_load_dwordx4 a[28:31], v34, s[60:63], s18 offen sc1\n\t" \
  "v_mfma_f32_16x16x32_bf16 v[232:235], v[112:115], a[176:179], v[232:235]\n\t" \
  "s_mov_b32 s18, 1048576\n\t" \
  "buffer_load_dwordx4 a[64:67], v34, s[60:63], s18 offen sc1\n\t" \
  "v_mfma_f32_16x16x32_bf16 v[232:235], v[116:119], a[180:183], v[232:235]\n\t" \
  "v_mfma_f32_16x16x32_bf16 v[232:235], a[240:243], a[32:35], v[232:235]\n\t" \
  "s_mov_b32 s18, 1179648\n\t" \
  "buffer_load_dwordx4 a[68:71], v34, s[60:63], s18 offen sc1\n\t" \
  "v_mfma_f32_16x16x32_bf16 v[232:235], a[244:247], a[36:39], v[232:235]\n\t" \
  "v_mfma_f32_16x16x32_bf16 v[236:239], v[120:123], a[176:179], v[236:239]\n\t" \
  "s_mov_b32 s18, 1310720\n\t" \
  "buffer_load_dwordx4 a[72:75], v34, s[60:63], s18 offen sc1\n\t" \
  "v_mfma_f32_16x16x32_bf16 v[236:239], v[124:127], a[180:183], v[236:239]\n\t" \
  "v_mfma_f32_16x16x32_bf16 v[236:239], a[248:251], a[32:35], v[236:239]\n\t" \
  "s_mov_b32 s18, 1441792\n\t" \
  "buffer_load_dwordx4 a[76:79], v34, s[60:63], s18 offen sc1\n\t" \
  "v_mfma_f32_16x16x32_bf16 v[236:239], a[252:255], a[36:39], v[236:239]\n\t" \
  "v_mfma_f32_16x16x32_bf16 v[224:227], a[192:195], a[40:43], v[224:227]\n\t" \
  "s_mov_b32 s18, 1572864\n\t" \
  "buffer_load_dwordx4 a[80:83], v34, s[60:63], s18 offen sc1\n\t" \
  "v_mfma_f32_16x16x32_bf16 v[224:227], a[196:199], a[44:47], v[224:227]\n\t" \
  "v_mfma_f32_16x16x32_bf16 v[228:231], a[200:203], a[40:43], v[228:231]\n\t" \
  "s_mov_b32 s18, 1703936\n\t" \
  "buffer_load_dwordx4 a[84:87], v34, s[60:63], s18 offen sc1\n\t" \
  "v_mfma_f32_16x16x32_bf16 v[228:231], a[204:207], a[44:47], v[228:231]\n\t" \
  "v_mfma_f32_16x16x32_bf16 v[232:235], a[208:211], a[40:43], v[232:235]\n\t" \
  "s_mov_b32 s18, 1835008\n\t" \
  "buffer_load_dwordx4 a[88:91], v34, s[60:63], s18 offen sc1\n\t" \
  "v_mfma_f32_16x16x32_bf16 v[232:235], a[212:215], a[44:47], v[232:235]\n\t" \
  "v_mfma_f32_16x16x32_bf16 v[236:239], a[216:219], a[40:43], v[236:239]\n\t" \
  "s_mov_b32 s18, 1966080\n\t" \
  "buffer_load_dwordx4 a[92:95], v34, s[60:63], s18 offen sc1\n\t" \
  "v_mfma_f32_16x16x32_bf16 v[236:239], a[220:223], a[44:47], v[236:239]\n\t" \
  "v_mfma_f32_16x16x32_bf16 v[240:243], v[96:99], a[184:187], v[240:243]\n\t" \
  "v_mfma_f32_16x16x32_bf16 v[240:243], v[100:103], a[188:191], v[240:243]\n\t" \
  "s_waitcnt lgkmcnt(0)\n\t" \
  "v_mfma_f32_16x16x32_bf16 v[240:243], a[224:227], a[48:51], v[240:243]\n\t" \
  "v_mfma_f32_16x16x32_bf16 v[240:243], a[228:231], a[52:55], v[240:243]\n\t" \
  "v_mfma_f32_16x16x32_bf16 v[244:247], v[104:107], a[184:187], v[244:247]\n\t" \
  "v_mfma_f32_16x16x32_bf16 v[244:247], v[108:111], a[188:191], v[244:247]\n\t" \
  "v_mfma_f32_16x16x32_bf16 v[244:247], a[232:235], a[48:51], v[244:247]\n\t" \
  "v_mfma_f32_16x16x32_bf16 v[244:247], a[236:239], a[52:55], v[244:247]\n\t" \
  "v_mfma_f32_16x16x32_bf16 v[248:251], v[112:115], a[184:187], v[248:251]\n\t" \
  "v_mfma_f32_16x16x32_bf16 v[248:251], v[116:119], a[188:191], v[248:251]\n\t" \
  "v_mfma_f32_16x16x32_bf16 v[248:251], a[240:243], a[48:51], v[248:251]\n\t" \
  "v_mfma_f32_16x16x32_bf16 v[248:251], a[244:247], a[52:55], v[248:251]\n\t" \
  "v_mfma_f32_16x16x32_bf16 v[252:255], v[120:123], a[184:187], v[252:255]\n\t" \
  "v_mfma_f32_16x16x32_bf16 v[252:255], v[124:127], a[188:191], v[252:255]\n\t" \
  "v_mfma_f32_16x16x32_bf16 v[252:255], a[248:251], a[48:51], v[252:255]\n\t" \
  "v_mfma_f32_16x16x32_bf16 v[252:255], a[252:255], a[52:55], v[252:255]\n\t" \
  "v_mfma_f32_16x16x32_bf16 v[240:243], a[192:195], a[56:59], v[240:243]\n\t" \
  "v_mfma_f32_16x16x32_bf16 v[240:243], a[196:199], a[60:63], v[240:243]\n\t" \
  "v_mfma_f32_16x16x32_bf16 v[244:247], a[200:203], a[56:59], v[244:247]\n\t" \
  "v_mfma_f32_16x16x32_bf16 v[244:247], a[204:207], a[60:63], v[244:247]\n\t" \
  "v_mfma_f32_16x16x32_bf16 v[248:251], a[208:211], a[56:59], v[248:251]\n\t" \
  "v_mfma_f32_16x16x32_bf16 v[248:251], a[212:215], a[60:63], v[248:251]\n\t" \
  "v_mfma_f32_16x16x32_bf16 v[252:255], a[216:219], a[56:59], v[252:255]\n\t" \
  "v_mfma_f32_16x16x32_bf16 v[252:255], a[220:223], a[60:63], v[252:255]\n\t" \
  "s_waitcnt vmcnt(14)\n\t" \
  "v_cmp_eq_u32 vcc, s92, v45\n\t" \
  "s_cbranch_vccnz Lnan01_%=\n\t" \
  "Lnanr01_%=:\n\t" \
  "v_mfma_f32_16x16x32_bf16 v[128:131], v[48:51], a[0:3], v[128:131]\n\t" \
  "v_mfma_f32_16x16x32_bf16 v[132:135], v[52:55], a[0:3], v[132:135]\n\t" \
  "v_mfma_f32_16x16x32_bf16 v[136:139], v[48:51], a[4:7], v[136:139]\n\t" \
  "v_mfma_f32_16x16x32_bf16 v[140:143], v[52:55], a[4:7], v[140:143]\n\t" \
  "s_waitcnt vmcnt(12)\n\t" \
  "v_mfma_f32_16x16x32_bf16 v[144:147], v[48:51], a[8:11], v[144:147]\n\t" \
  "v_mfma_f32_16x16x32_bf16 v[148:151], v[52:55], a[8:11], v[148:151]\n\t" \
  "v_mfma_f32_16x16x32_bf16 v[152:155], v[48:51], a[12:15], v[152:155]\n\t" \
  "v_mfma_f32_16x16x32_bf16 v[156:159], v[52:55], a[12:15], v[156:159]\n\t" \
  "s_waitcnt vmcnt(10)\n\t" \
  "v_mfma_f32_16x16x32_bf16 v[160:163], v[48:51], a[16:19], v[160:163]\n\t" \
  "v_cvt_pk_bf16_f32 v0, v128, v129\n\t" \
  "v_cvt_pk_bf16_f32 v1, v130, v131\n\t" \
  "v_cvt_pk_bf16_f32 v2, v132, v133\n\t" \
  "v_mfma_f32_16x16x32_bf16 v[164:167], v[52:55], a[16:19], v[164:167]\n\t" \
  "v_cvt_pk_bf16_f32 v3, v134, v135\n\t" \
  "v_cvt_pk_bf16_f32 v4, v136, v137\n\t" \
  "v_cvt_pk_bf16_f32 v5, v138, v139\n\t" \
  "v_mfma_f32_16x16x32_bf16 v[168:171], v[48:51], a[20:23], v[168:171]\n\t" \
  "v_cvt_pk_bf16_f32 v6, v140, v141\n\t" \
  "v_cvt_pk_bf16_f32 v7, v142, v143\n\t" \
  "ds_write_b64 v77, v[0:1] offset:0\n\t" \
  "v_mfma_f32_16x16x32_bf16 v[172:175], v[52:55], a[20:23], v[172:175]\n\t" \
  "ds_write_b64 v78, v[2:3] offset:0\n\t" \
  "s_waitcnt vmcnt(8)\n\t" \
  "v_mfma_f32_16x16x32_bf16 v[176:179], v[48:51], a[24:27], v[176:179]\n\t" \
  "ds_write_b64 v79, v[4:5] offset:0\n\t" \
  "v_mfma_f32_16x16x32_bf16 v[180:183], v[52:55], a[24:27], v[180:183]\n\t" \
  "ds_write_b64 v80, v[6:7] offset:0\n\t" \
  "v_mfma_f32_16x16x32_bf16 v[184:187], v[48:51], a[28:31], v[184:187]\n\t" \
  "ds_read_b128 v[16:19], v81 offset:0\n\t" \
  "v_mfma_f32_16x16x32_bf16 v[188:191], v[52:55], a[28:31], v[188:191]\n\t" \
  "ds_read_b128 v[20:23], v81 offset:1024\n\t" \
  "v_cvt_pk_bf16_f32 v8, v144, v145\n\t" \
  "v_cvt_pk_bf16_f32 v9, v146, v147\n\t" \
  "v_cvt_pk_bf16_f32 v10, v148, v149\n\t" \
  "s_waitcnt vmcnt(6)\n\t" \
  "v_mfma_f32_16x16x32_bf16 v[192:195], v[48:51], a[64:67], v[192:195]\n\t" \
  "v_cvt_pk_bf16_f32 v11, v150, v151\n\t" \
  "v_cvt_pk_bf16_f32 v12, v152, v153\n\t" \
  "v_cvt_pk_bf16_f32 v13, v154, v155\n\t" \
  "v_mfma_f32_16x16x32_bf16 v[196:199], v[52:55], a[64:67], v[196:199]\n\t" \
  "v_cvt_pk_bf16_f32 v14, v156, v157\n\t" \
  "v_cvt_pk_bf16_f32 v15, v158, v159\n\t" \
  "s_waitcnt lgkmcnt(0)\n\t" \
  "s_mov_b32 s10, 0\n\t" \
  "s_mov_b32 s11, 131072\n\t" \
  "buffer_store_dwordx4 v[16:19], v34, s[60:63], s10 offen sc1\n\t" \
  "v_mfma_f32_16x16x32_bf16 v[200:203], v[48:51], a[68:71], v[200:203]\n\t" \
  "buffer_store_dwordx4 v[20:23], v34, s[60:63], s11 offen sc1\n\t" \
  "v_mfma_f32_16x16x32_bf16 v[204:207], v[52:55], a[68:71], v[204:207]\n\t" \
  "ds_write_b64 v77, v[8:9] offset:2048\n\t" \
  "s_waitcnt vmcnt(6)\n\t" \
  "v_mfma_f32_16x16x32_bf16 v[208:211], v[48:51], a[72:75], v[208:211]\n\t" \
  "ds_write_b64 v78, v[10:11] offset:2048\n\t" \
  "v_mfma_f32_16x16x32_bf16 v[212:215], v[52:55], a[72:75], v[212:215]\n\t" \
  "ds_write_b64 v79, v[12:13] offset:2048\n\t" \
  "v_mfma_f32_16x16x32_bf16 v[216:219], v[48:51], a[76:79], v[216:219]\n\t" \
  "ds_write_b64 v80, v[14:15] offset:2048\n\t" \
  "v_mfma_f32_16x16x32_bf16 v[220:223], v[52:55], a[76:79], v[220:223]\n\t" \
  "ds_read_b128 v[84:87], v81 offset:2048\n\t" \
  "s_waitcnt vmcnt(4)\n\t" \
  "v_mfma_f32_16x16x32_bf16 v[224:227], v[48:51], a[80:83], v[224:227]\n\t" \
  "ds_read_b128 v[88:91], v81 offset:3072\n\t" \
  "v_cvt_pk_bf16_f32 v0, v160, v161\n\t" \
  "v_cvt_pk_bf16_f32 v1, v162, v163\n\t" \
  "v_cvt_pk_bf16_f32 v2, v164, v165\n\t" \
  "v_mfma_f32_16x16x32_bf16 v[228:231], v[52:55], a[80:83], v[228:231]\n\t" \
  "v_cvt_pk_bf16_f32 v3, v166, v167\n\t" \
  "v_cvt_pk_bf16_f32 v4, v168, v169\n\t" \
  "v_cvt_pk_bf16_f32 v5, v170, v171\n\t" \
  "v_mfma_f32_16x16x32_bf16 v[232:235], v[48:51], a[84:87], v[232:235]\n\t" \
  "v_cvt_pk_bf16_f32 v6, v172, v173\n\t" \
  "v_cvt_pk_bf16_f32 v7, v174, v175\n\t" \
  "s_waitcnt lgkmcnt(0)\n\t" \
  "s_mov_b32 s10, 262144\n\t" \
  "s_mov_b32 s11, 393216\n\t" \
  "buffer_store_dwordx4 v[84:87], v34, s[60:63], s10 offen sc1\n\t" \
  "v_mfma_f32_16x16x32_bf16 v[236:239], v[52:55], a[84:87], v[236:239]\n\t" \
  "buffer_store_dwordx4 v[88:91], v34, s[60:63], s11 offen sc1\n\t" \
  "s_waitcnt vmcnt(4)\n\t" \
  "v_mfma_f32_16x16x32_bf16 v[240:243], v[48:51], a[88:91], v[240:243]\n\t" \
  "ds_write_b64 v77, v[0:1] offset:4096\n\t" \
  "v_mfma_f32_16x16x32_bf16 v[244:247], v[52:55], a[88:91], v[244:247]\n\t" \
  "ds_write_b64 v78, v[2:3] offset:4096\n\t" \
  "v_mfma_f32_16x16x32_bf16 v[248:251], v[48:51], a[92:95], v[248:251]\n\t" \
  "ds_write_b64 v79, v[4:5] offset:4096\n\t" \
  "v_mfma_f32_16x16x32_bf16 v[252:255], v[52:55], a[92:95], v[252:255]\n\t" \
  "ds_write_b64 v80, v[6:7] offset:4096\n\t" \
  "ds_read_b128 v[64:67], v81 offset:4096\n\t" \
  "ds_read_b128 v[68:71], v81 offset:5120\n\t" \
  "v_cvt_pk_bf16_f32 v8, v176, v177\n\t" \
  "v_cvt_pk_bf16_f32 v9, v178, v179\n\t" \
  "v_cvt_pk_bf16_f32 v10, v180, v181\n\t" \
  "v_cvt_pk_bf16_f32 v11, v182, v183\n\t" \
  "v_cvt_pk_bf16_f32 v12, v184, v185\n\t" \
  "v_cvt_pk_bf16_f32 v13, v186, v187\n\t" \
  "v_cvt_pk_bf16_f32 v14, v188, v189\n\t" \
  "v_cvt_pk_bf16_f32 v15, v190, v191\n\t" \
  "ds_write_b64 v77, v[8:9] offset:6144\n\t" \
  "s_waitcnt lgkmcnt(1)\n\t" \
  "s_mov_b32 s10, 524288\n\t" \
  "s_mov_b32 s11, 655360\n\t" \
  "buffer_store_dwordx4 v[64:67], v34, s[60:63], s10 offen sc1\n\t" \
  "buffer_store_dwordx4 v[68:71], v34, s[60:63], s11 offen sc1\n\t" \
  "ds_write_b64 v78, v[10:11] offset:6144\n\t" \
  "ds_write_b64 v79, v[12:13] offset:6144\n\t" \
  "ds_write_b64 v80, v[14:15] offset:6144\n\t" \
  "ds_read_b128 v[16:19], v81 offset:6144\n\t" \
  "ds_read_b128 v[20:23], v81 offset:7168\n\t" \
  "v_cvt_pk_bf16_f32 v0, v192, v193\n\t" \
  "v_cvt_pk_bf16_f32 v1, v194, v195\n\t" \
  "v_cvt_pk_bf16_f32 v2, v196, v197\n\t" \
  "v_cvt_pk_bf16_f32 v3, v198, v199\n\t" \
  "v_cvt_pk_bf16_f32 v4, v200, v201\n\t" \
  "v_cvt_pk_bf16_f32 v5, v202, v203\n\t" \
  "v_cvt_pk_bf16_f32 v6, v204, v205\n\t" \
  "v_cvt_pk_bf16_f32 v7, v206, v207\n\t" \
  "ds_write_b64 v77, v[0:1] offset:0\n\t" \
  "s_waitcnt lgkmcnt(1)\n\t" \
  "s_mov_b32 s10, 786432\n\t" \
  "s_mov_b32 s11, 917504\n\t" \
  "buffer_store_dwordx4 v[16:19], v34, s[60:63], s10 offen sc1\n\t" \
  "buffer_store_dwordx4 v[20:23], v34, s[60:63], s11 offen sc1\n\t" \
  "ds_write_b64 v78, v[2:3] offset:0\n\t" \
  "ds_write_b64 v79, v[4:5] offset:0\n\t" \
  "ds_write_b64 v80, v[6:7] offset:0\n\t" \
  "ds_read_b128 v[84:87], v81 offset:0\n\t" \
  "ds_read_b128 v[88:91], v81 offset:1024\n\t" \
  "v_cvt_pk_bf16_f32 v8, v208, v209\n\t" \
  "v_cvt_pk_bf16_f32 v9, v210, v211\n\t" \
  "v_cvt_pk_bf16_f32 v10, v212, v213\n\t" \
  "v_cvt_pk_bf16_f32 v11, v214, v215\n\t" \
  "v_cvt_pk_bf16_f32 v12, v216, v217\n\t" \
  "v_cvt_pk_bf16_f32 v13, v218, v219\n\t" \
  "v_cvt_pk_bf16_f32 v14, v220, v221\n\t" \
  "v_cvt_pk_bf16_f32 v15, v222, v223\n\t" \
  "ds_write_b64 v77, v[8:9] offset:2048\n\t" \
  "s_waitcnt lgkmcnt(1)\n\t" \
  "s_mov_b32 s10, 1048576\n\t" \
  "s_mov_b32 s11, 1179648\n\t" \
  "buffer_store_dwordx4 v[84:87], v34, s[60:63], s10 offen sc1\n\t" \
  "buffer_store_dwordx4 v[88:91], v34, s[60:63], s11 offen sc1\n\t" \
  "ds_write_b64 v78, v[10:11] offset:2048\n\t" \
  "ds_write_b64 v79, v[12:13] offset:2048\n\t" \
  "ds_write_b64 v80, v[14:15] offset:2048\n\t" \
  "ds_read_b128 v[64:67], v81 offset:2048\n\t" \
  "ds_read_b128 v[68:71], v81 offset:3072\n\t" \
  "v_cvt_pk_bf16_f32 v0, v224, v225\n\t" \
  "v_cvt_pk_bf16_f32 v1, v226, v227\n\t" \
  "v_cvt_pk_bf16_f32 v2, v228, v229\n\t" \
  "v_cvt_pk_bf16_f32 v3, v230, v231\n\t" \
  "v_cvt_pk_bf16_f32 v4, v232, v233\n\t" \
  "v_cvt_pk_bf16_f32 v5, v234, v235\n\t" \
  "v_cvt_pk_bf16_f32 v6, v236, v237\n\t" \
  "v_cvt_pk_bf16_f32 v7, v238, v239\n\t" \
  "ds_write_b64 v77, v[0:1] offset:4096\n\t" \
  "s_waitcnt lgkmcnt(1)\n\t" \
  "s_mov_b32 s10, 1310720\n\t" \
  "s_mov_b32 s11, 1441792\n\t" \
  "buffer_store_dwordx4 v[64:67], v34, s[60:63], s10 offen sc1\n\t" \
  "buffer_store_dwordx4 v[68:71], v34, s[60:63], s11 offen sc1\n\t" \
  "ds_write_b64 v78, v[2:3] offset:4096\n\t" \
  "ds_write_b64 v79, v[4:5] offset:4096\n\t" \
  "ds_write_b64 v80, v[6:7] offset:4096\n\t" \
  "ds_read_b128 v[16:19], v81 offset:4096\n\t" \
  "ds_read_b128 v[20:23], v81 offset:5120\n\t" \
  "s_nop 7\n\t" \
  "v_cvt_pk_bf16_f32 v8, v240, v241\n\t" \
  "v_cvt_pk_bf16_f32 v9, v242, v243\n\t" \
  "v_cvt_pk_bf16_f32 v10, v244, v245\n\t" \
  "v_cvt_pk_bf16_f32 v11, v246, v247\n\t" \
  "v_cvt_pk_bf16_f32 v12, v248, v249\n\t" \
  "v_cvt_pk_bf16_f32 v13, v250, v251\n\t" \
  "v_cvt_pk_bf16_f32 v14, v252, v253\n\t" \
  "v_cvt_pk_bf16_f32 v15, v254, v255\n\t" \
  "s_waitcnt lgkmcnt(0)\n\t" \
  "s_mov_b32 s10, 1572864\n\t" \
  "s_mov_b32 s11, 1703936\n\t" \
  "buffer_store_dwordx4 v[16:19], v34, s[60:63], s10 offen sc1\n\t" \
  "buffer_store_dwordx4 v[20:23], v34, s[60:63], s11 offen sc1\n\t" \
  "ds_write_b64 v77, v[8:9] offset:6144\n\t" \
  "ds_write_b64 v78, v[10:11] offset:6144\n\t" \
  "ds_write_b64 v79, v[12:13] offset:6144\n\t" \
  "ds_write_b64 v80, v[14:15] offset:6144\n\t" \
  "ds_read_b128 v[84:87], v81 offset:6144\n\t" \
  "ds_read_b128 v[88:91], v81 offset:7168\n\t" \
  "s_waitcnt lgkmcnt(0)\n\t" \
  "s_mov_b32 s10, 1835008\n\t" \
  "s_mov_b32 s11, 1966080\n\t" \
  "buffer_store_dwordx4 v[84:87], v34, s[60:63], s10 offen sc1\n\t" \
  "buffer_store_dwordx4 v[88:91], v34, s[60:63], s11 offen sc1\n\t" \
  "s_branch Lend_%=\n\t" \
  "Lsp2_%=:\n\t" \
  "s_call_b64 s[12:13], Lspin_%=\n\t" \
  "s_branch Lspr2_%=\n\t" \
  "Lsp3_%=:\n\t" \
  "s_call_b64 s[12:13], Lspin_%=\n\t" \
  "s_branch Lspr3_%=\n\t" \
  "Lsp4_%=:\n\t" \
  "s_call_b64 s[12:13], Lspin_%=\n\t" \
  "s_branch Lspr4_%=\n\t" \
  "Lsp5_%=:\n\t" \
  "s_call_b64 s[12:13], Lspin_%=\n\t" \
  "s_branch Lspr5_%=\n\t" \
  "Lsp6_%=:\n\t" \
  "s_call_b64 s[12:13], Lspin_%=\n\t" \
  "s_branch Lspr6_%=\n\t" \
  "Lsp7_%=:\n\t" \
  "s_call_b64 s[12:13], Lspin_%=\n\t" \
  "s_branch Lspr7_%=\n\t" \
  "Lsp8_%=:\n\t" \
  "s_call_b64 s[12:13], Lspin_%=\n\t" \
  "s_branch Lspr8_%=\n\t" \
  "Lsp9_%=:\n\t" \
  "s_call_b64 s[12:13], Lspin_%=\n\t" \
  "s_branch Lspr9_%=\n\t" \
  "Lsp10_%=:\n\t" \
  "s_call_b64 s[12:13], Lspin_%=\n\t" \
  "s_branch Lspr10_%=\n\t" \
  "Lsp11_%=:\n\t" \
  "s_call_b64 s[12:13], Lspin_%=\n\t" \
  "s_branch Lspr11_%=\n\t" \
  "Lsp12_%=:\n\t" \
  "s_call_b64 s[12:13], Lspin_%=\n\t" \
  "s_branch Lspr12_%=\n\t" \
  "LspH0_%=:\n\t" \
  "s_call_b64 s[12:13], Lspin_%=\n\t" \
  "s_branch LsprH0_%=\n\t" \
  "LspT01_%=:\n\t" \
  "s_call_b64 s[12:13], Lspin_%=\n\t" \
  "s_branch LsprT01_%=\n\t" \
  "Lnan01_%=:\n\t" \
  "v_mov_b32 v48, 0x7fc07fc0\n\t" \
  "v_mov_b32 v52, 0x7fc07fc0\n\t" \
  "s_nop 1\n\t" \
  "s_branch Lnanr01_%=\n\t" \
  "Lspin_%=:\n\t" \
  "s_memrealtime s[14:15]\n\t" \
  "Lspl_%=:\n\t" \
  "buffer_load_dword v28, v27, s[48:51], s90 offen sc0 sc1\n\t" \
  "s_memrealtime s[16:17]\n\t" \
  "s_waitcnt vmcnt(0) lgkmcnt(0)\n\t" \
  "v_cmp_gt_u32 vcc, s91, v28\n\t" \
  "s_cbranch_vccz Lspo_%=\n\t" \
  "s_sub_u32 s16, s16, s14\n\t" \
  "s_sleep 1\n\t" \
  "s_cmp_lt_u32 s16, 2000\n\t" \
  "s_cbranch_scc1 Lspl_%=\n\t" \
  "buffer_load_dword v45, v40, s[48:51], 0 offen sc1\n\t" \
  "s_waitcnt vmcnt(0)\n\t" \
  "v_cmp_eq_u32 vcc, s92, v45\n\t" \
  "s_cbranch_vccnz Lspo_%=\n\t" \
  "s_cmp_gt_u32 s16, 30000000\n\t" \
  "s_cbranch_scc0 Lspl_%=\n\t" \
  "Lspt_%=:\n\t" \
  "buffer_store_dword v43, v41, s[48:51], 0 offen sc1\n\t" \
  "s_waitcnt vmcnt(0)\n\t" \
  "Lspo_%=:\n\t" \
  "s_setpc_b64 s[12:13]\n\t" \
  "Lswu_%=:\n\t" \
  "s_mov_b32 s73, s77\n\t" \
  "s_mov_b64 s[8:9], s[78:79]\n\t" \
  "s_mov_b32 s87, 48\n\t" \
  "s_branch Lswur_%=\n\t" \
  "Lswx_%=:\n\t" \
  "s_cmp_ge_u32 s72, 2\n\t" \
  "s_cbranch_scc1 Lswxr_%=\n\t" \
  "s_mov_b32 s40, s44\n\t" \
  "s_mov_b32 s41, s45\n\t" \
  "s_mov_b32 s42, s46\n\t" \
  "s_mov_b32 s43, s47\n\t" \
  "v_add_u32 v0, s24, v26\n\t" \
  "v_add_u32 v1, s24, v26\n\t" \
  "v_add_u32 v2, s24, v26\n\t" \
  "v_add_u32 v3, s24, v26\n\t" \
  "v_add_u32 v4, s25, v26\n\t" \
  "v_add_u32 v5, s25, v26\n\t" \
  "v_add_u32 v6, s25, v26\n\t" \
  "v_add_u32 v7, s25, v26\n\t" \
  "v_add_u32 v8, s26, v26\n\t" \
  "v_add_u32 v9, s26, v26\n\t" \
  "v_add_u32 v10, s26, v26\n\t" \
  "v_add_u32 v11, s26, v26\n\t" \
  "v_add_u32 v12, s27, v26\n\t" \
  "v_add_u32 v13, s27, v26\n\t" \
  "v_add_u32 v14, s27, v26\n\t" \
  "v_add_u32 v15, s27, v26\n\t" \
  "s_mov_b32 s91, s92\n\t" \
  "s_mov_b32 s74, 17\n\t" \
  "s_mov_b32 s75, s76\n\t" \
  "s_branch Lswxr_%=\n\t" \
  "Ldsp_%=:\n\t" \
  "s_mov_b32 s94, s91\n\t" \
  "s_mov_b32 s88, s90\n\t" \
  "v_mov_b32 v46, v27\n\t" \
  "v_mov_b32 v27, v39\n\t" \
  "s_lshl_b32 s91, s92, 4\n\t" \
  "s_mov_b32 s90, 0\n\t" \
  "s_call_b64 s[12:13], Lspin_%=\n\t" \
  "v_cmp_gt_u32 vcc, s91, v28\n\t" \
  "s_mov_b32 s91, s94\n\t" \
  "s_mov_b32 s90, s88\n\t" \
  "v_mov_b32 v27, v46\n\t" \
  "v_mov_b32 v94, v28\n\t" \
  "s_mov_b32 s95, s71\n\t" \
  "s_cbranch_vccnz Ldecr_%=\n\t" \
  "s_branch Ldec_%=\n\t" \
  "Lend_%=:\n\t" \
  ""
#define GEMM_CLOBBERS "v60", "v61", "v62", "v63", "v64", "v65", "v66", "v67", "v68", "v69", "v70", "v71", "v72", "v73", "v74", "v75", "v84", "v85", "v87", "v88", "v89", "v90", "v91", "v92", "v93", "v94", "v95", "v96", "v97", "v98", "v99", "v100", "v101", "v102", "v103", "v104", "v105", "v106", "v107", "v108", "v109", "v110", "v111", "v112", "v113", "v114", "v115", "v116", "v117", "v118", "v119", "v120", "v121", "v122", "v123", "v124", "v125", "v126", "v127", "v128", "v129", "v130", "v131", "v132", "v133", "v134", "v135", "v136", "v137", "v138", "v139", "v140", "v141", "v142", "v143", "v144", "v145", "v146", "v147", "v148", "v149", "v150", "v151", "v152", "v153", "v154", "v155", "v156", "v157", "v158", "v159", "v160", "v161", "v162", "v163", "v164", "v165", "v166", "v167", "v168", "v169", "v170", "v171", "v172", "v173", "v174", "v175", "v176", "v177", "v178", "v179", "v180", "v181", "v182", "v183", "v184", "v185", "v186", "v187", "v188", "v189", "v190", "v191", "v192", "v193", "v194", "v195", "v196", "v197", "v198", "v199", "v200", "v201", "v202", "v203", "v204", "v205", "v206", "v207", "v208", "v209", "v210", "v211", "v212", "v213", "v214", "v215", "v216", "v217", "v218", "v219", "v220", "v221", "v222", "v223", "v224", "v225", "v226", "v227", "v228", "v229", "v230", "v231", "v232", "v233", "v234", "v235", "v236", "v237", "v238", "v239", "v240", "v241", "v242", "v243", "v244", "v245", "v246", "v247", "v248", "v249", "v250", "v251", "v252", "v253", "v254", "v255", "a0", "a1", "a2", "a3", "a4", "a5", "a6", "a7", "a8", "a9", "a10", "a11", "a12", "a13", "a14", "a15", "a16", "a17", "a18", "a19", "a20", "a21", "a22", "a23", "a24", "a25", "a26", "a27", "a28", "a29", "a30", "a31", "a32", "a33", "a34", "a35", "a36", "a37", "a38", "a39", "a40", "a41", "a42", "a43", "a44", "a45", "a46", "a47", "a48", "a49", "a50", "a51", "a52", "a53", "a54", "a55", "a56", "a57", "a58", "a59", "a60", "a61", "a62", "a63", "a64", "a65", "a66", "a67", "a68", "a69", "a70", "a71", "a72", "a73", "a74", "a75", "a76", "a77", "a78", "a79", "a80", "a81", "a82", "a83", "a84", "a85", "a86", "a87", "a88", "a89", "a90", "a91", "a92", "a93", "a94", "a95", "a96", "a97", "a98", "a99", "a100", "a101", "a102", "a103", "a104", "a105", "a106", "a107", "a108", "a109", "a110", "a111", "a112", "a113", "a114", "a115", "a116", "a117", "a118", "a119", "a120", "a121", "a122", "a123", "a124", "a125", "a126", "a127", "a128", "a129", "a130", "a131", "a132", "a133", "a134", "a135", "a136", "a137", "a138", "a139", "a140", "a141", "a142", "a143", "a144", "a145", "a146", "a147", "a148", "a149", "a150", "a151", "a152", "a153", "a154", "a155", "a156", "a157", "a158", "a159", "a160", "a161", "a162", "a163", "a164", "a165", "a166", "a167", "a168", "a169", "a170", "a171", "a172", "a173", "a174", "a175", "a176", "a177", "a178", "a179", "a180", "a181", "a182", "a183", "a184", "a185", "a186", "a187", "a188", "a189", "a190", "a191", "a192", "a193", "a194", "a195", "a196", "a197", "a198", "a199", "a200", "a201", "a202", "a203", "a204", "a205", "a206", "a207", "a208", "a209", "a210", "a211", "a212", "a213", "a214", "a215", "a216", "a217", "a218", "a219", "a220", "a221", "a222", "a223", "a224", "a225", "a226", "a227", "a228", "a229", "a230", "a231", "a232", "a233", "a234", "a235", "a236", "a237", "a238", "a239", "a240", "a241", "a242", "a243", "a244", "a245", "a246", "a247", "a248", "a249", "a250", "a251", "a252", "a253", "a254", "a255", "s10", "s11", "s12", "s13", "s14", "s15", "s16", "s17", "s18", "s19", "s20", "s21", "s22", "s23", "s84", "s85", "s86", "s87", "s88", "s89", "s90", "s91", "s92", "s93", "s94", "s95", "vcc", "scc", "memory"
typedef unsigned u32x16 __attribute__((ext_vector_type(16)));
typedef unsigned u32x8 __attribute__((ext_vector_type(8)));
typedef unsigned u32x4 __attribute__((ext_vector_type(4)));
constexpr unsigned STAGE = 16u << 20, CTRL = 2 * STAGE, CTRL_BYTES = 0x80000u, FLAG_BYTES = 0x10000u;
constexpr unsigned C_TMO = 0x11000u, C_TICKET = 0x20000u, C_XFLAG = 0x40000u, C_XCC = 0x60000u, OOB = 0x80000000u;
__device__ __forceinline__ unsigned rfl(unsigned x) { return __builtin_amdgcn_readfirstlane(x); }
__device__ __forceinline__ unsigned lo32(const void* p) { return rfl((unsigned)(unsigned long long)p); }
__device__ __forceinline__ unsigned hi32(const void* p) { return rfl((unsigned)((unsigned long long)p >> 32)) & 0xffffu; }
struct Args { const char* x; const char* W; char* out; char* ws[8]; unsigned long long delta[2]; unsigned u8[2]; unsigned rank, pad; };
static_assert(offsetof(Args, x) == 0 && offsetof(Args, W) == 8 && offsetof(Args, rank) == 0x70, "offsets used by the first asm statement");

__global__ __launch_bounds__(256, 1) void a2a_gemm(const Args A) {
  u32x4 kp; unsigned kr;
  asm volatile("s_load_dwordx4 s[12:15], %2, 0x0\n\ts_load_dword s16, %2, 0x70" : "={s[12:15]}"(kp), "={s16}"(kr) : "s"(__builtin_amdgcn_kernarg_segment_ptr()));
  __builtin_amdgcn_sched_barrier(0);
  const unsigned tid = threadIdx.x, w = rfl(tid >> 6), b = blockIdx.x; unsigned l = tid & 63;
  const unsigned idx = b >> 3, h = b & 1, m = idx & 3, nt = ((b & 7) >> 1) * 8 + (idx >> 2), wr = w >> 1, wc = w & 1, pb = b ^ 1;
  unsigned rot; unsigned long long ltab; const unsigned rank = A.rank; u32x8 V0, V1; u32x16 VB, VC; u32x8 VE; u32x4 SD, SG; u32x8 SA; u32x16 SB, SC;
  { const unsigned xe = (128 * w + (l >> 3)) * 2048u + 4096u, x0 = xe + ((l & 7) ^ (l >> 4)) * 16u, x1 = xe + 8 * 2048u + ((l & 7) ^ (4 + (l >> 4))) * 16u;
    const unsigned we = (32 * (w & 1) + (l >> 5)) * 16384u + 4096u, c = l & 31, hi = (l >> 5) << 2;
    const unsigned w0 = we + (c ^ hi) * 16u, w1 = we + 32768u + (c ^ hi ^ 8) * 16u, w2 = we + (c ^ hi ^ 2) * 16u, w3 = we + 32768u + (c ^ hi ^ 10) * 16u;
    const bool isx = w < 2; const unsigned q4 = isx ? nt & 3 : m, st = isx ? 16384u : 32768u;
    const unsigned b0 = isx ? x0 : w0, b1 = isx ? x1 : w1, b2 = isx ? x0 : w2, b3 = isx ? x1 : w3;
    const unsigned e0 = (q4 & 1) ? b2 : b0, e1 = (q4 & 1) ? b3 : b1, o0 = (q4 & 1) ? b0 : b2, o1 = (q4 & 1) ? b1 : b3;
#define VAJ(j) (((((j) >> 2) & 1) ? (((j) & 1) ? o1 : o0) : (((j) & 1) ? e1 : e0)) + ((4 * ((((j) >> 2) + q4) & 3) + ((j) & 2)) * st - ((j) & 3) * 1024u))
#pragma unroll
    for (unsigned j = 0; j < 8; ++j) V0[j] = VAJ(j);
#pragma unroll
    for (unsigned gi = 0; gi < 4; ++gi) SG[gi] = rfl((isx ? w * 16384u : 32768u + w * 16384u) + ((gi + q4) & 3) * 4096u);
    rot = b & 7; const unsigned u0 = rfl(8 * h + rot), bs = rfl(isx ? m * 256u * 2048u : nt * 512u);
    SC[6] = rfl(isx ? 7u : 20u); SD[2] = rfl((isx ? (16u << 20) : (128u << 20)) + 4096u); SD[3] = 0x20000;
    asm volatile(GEMM_EARLY : "+v"(l) : "{v[0:7]}"(V0), "{s[12:15]}"(kp), "{s16}"(kr), "{s17}"(rfl(isx ? 1u : 0u)), "{s42}"(SD[2]), "{s43}"(SD[3]), "{s[24:27]}"(SG), "{s73}"(u0), "{s74}"(SC[6]), "{s75}"(bs)
                 : "s40", "s41", "s84", "s10", "s18", "s19", "scc", "memory");
    __builtin_amdgcn_sched_barrier(0);
#pragma unroll
    for (unsigned j = 0; j < 8; ++j) V1[j] = VAJ(8 + j);
    const char* dsrc = (isx ? A.x : A.W) - 4096;
    SD[0] = lo32(dsrc); SD[1] = hi32(dsrc); SC[2] = 0;
    SC[5] = rfl(rank * 16 + 8 * h + rot); SC[7] = rfl(isx ? (rank * 1024u + m * 256u) * 2048u - rank * 2048u : nt * 512u);
    ltab = 0x0101010101010101ull ^ (0xf8ull << (8 * (7 - rot)));
    ltab = ((unsigned long long)rfl((unsigned)(ltab >> 32)) << 32) | rfl((unsigned)ltab); }
  __builtin_amdgcn_sched_barrier(0);
  const unsigned dst = (rank + 1 + (idx & 7)) & 7, si = (b & 7) | ((b >> 6) << 3);
  char* ws = A.ws[rank & 7]; char* peer = A.ws[dst];
  const unsigned g = l >> 4, pi = (rank - dst - 1) & 7, rf = pi > 3;
  const bool snd = dst != rank;
  unsigned tk; u32x4 X0;
  { u32x4 SCt, SXt; SCt[0] = lo32(ws + CTRL); SCt[1] = hi32(ws + CTRL); SCt[2] = CTRL_BYTES; SCt[3] = 0x20000; SXt[0] = lo32(A.x); SXt[1] = hi32(A.x); SXt[2] = rfl(snd ? 16u << 20 : 0u); SXt[3] = 0x20000;
    VB[13] = (dst * 1024u + 32 * si + 8 * w + (l >> 3)) * 2048u + ((l & 7) ^ (4 * (w & 1) + g)) * 16u;
    const unsigned tko = l == 0 ? C_TICKET + (b * 4 + w) * 128u : OOB;
    asm volatile(GEMM_EARLYB : "={v86}"(tk), "={v[56:59]}"(X0) : "v"(tko), "s"(SCt), "v"(VB[13]), "s"(SXt), "s"(rfl(rf ? 15 * 128u : 0u)) : "memory"); }
  __builtin_amdgcn_sched_barrier(0);
  { const unsigned q = (l >> 2) & 3, p = l & 3, f2 = (q << 1) | (g & 1);
#pragma unroll
    for (unsigned nb = 0; nb < 8; ++nb) { const unsigned cb = wc * 8 + (nb < 4 ? 4 * (1 - h) + nb : 4 * h + nb - 4); VB[nb] = 65536u + (8 * g + q) * 512u + 32u * (cb ^ f2) + 8u * p; } }
  VB[8] = (wr * 128 + (l & 15)) * 128u + ((g ^ ((l & 15) >> 1)) * 16u); VB[9] = VB[8] ^ 64u;
  VB[10] = l * 16u + 4096u; VB[11] = w < 2 ? m * 128u + w * 64u + 4u * (l & 15) : OOB; VB[12] = 0;
  VB[14] = rank * (2u << 20) + si * 4096u + w * 1024u + l * 16u;
  VB[15] = l == 0 ? rank * 8192u + 16 * si + 4 * w : OOB;
  { const unsigned o = (m * 256 + wr * 128) * 16384u + (nt * 256 + wc * 128) * 2u, lin = o + (l >> 3) * 16384u + (l & 7) * 16u;
    VC[0] = lin + (1 - h) * 128u; VC[2] = lin + h * 128u; VC[1] = 0; }
  VC[3] = 0;
  const unsigned me = (b * 4 + w) * 128u, pa = (pb * 4 + w) * 128u;
  VC[4] = l == 0 ? C_XFLAG + me : OOB; VC[5] = C_XFLAG + pa; VC[6] = l == 0 ? C_XCC + me : OOB; VC[7] = C_XCC + pa;
  VC[8] = C_TMO; VC[9] = l == 0 ? C_TMO : OOB; VC[10] = 0; VC[11] = 0; VC[12] = 0; VC[13] = 0; VC[14] = 0; VC[15] = 0;
  { const unsigned n = l & 15, one = 0x3f80u << (16 * (n & 1)), v = g == (n >> 2) ? one : 0u;
    VE[0] = (n & 2) ? 0u : v; VE[1] = (n & 2) ? v : 0u; VE[2] = 0; VE[3] = 0; VE[4] = 0; VE[5] = 0; VE[6] = VE[0]; VE[7] = VE[1]; }
  u32x8 VT;
  { const unsigned tb = 0x20000u + w * 8192u, wa = tb + (l & 15) * 128u + (g & 1) * 8u, ch = (l >> 5) ^ (l & 7);
    VT[0] = 0; VT[1] = wa + ch * 16u; VT[2] = wa + (ch ^ 2) * 16u; VT[3] = wa + (ch ^ 4) * 16u; VT[4] = wa + (ch ^ 6) * 16u; VT[5] = tb + (l >> 3) * 128u + ((l >> 3) ^ (l & 7)) * 16u; VT[6] = 0; VT[7] = 0; }
  SA[0] = lo32(ws - 4096); SA[1] = hi32(ws - 4096); SA[2] = STAGE + 4096u; SA[3] = 0x20000;
  SA[4] = lo32(ws + CTRL); SA[5] = hi32(ws + CTRL); SA[6] = CTRL_BYTES; SA[7] = 0x20000;
  SB[0] = lo32(peer); SB[1] = hi32(peer); SB[2] = rfl(snd ? CTRL : 0u); SB[3] = 0x20000;
  SB[4] = lo32(A.x); SB[5] = hi32(A.x); SB[6] = rfl(snd ? 16u << 20 : 0u); SB[7] = 0x20000;
  SB[8] = lo32(A.out); SB[9] = hi32(A.out); SB[10] = 16u << 20; SB[11] = 0x20000;
  SB[12] = lo32(peer + CTRL); SB[13] = hi32(peer + CTRL); SB[14] = rfl(snd ? FLAG_BYTES : 0u); SB[15] = 0x20000;
  SC[0] = 0; SC[1] = 0;
  SC[3] = rfl(5 - h); SC[4] = w; SC[8] = rfl(m * 32768u); SC[9] = rfl(A.u8[h]); SC[10] = rfl((unsigned)A.delta[h]); SC[11] = rfl((unsigned)(A.delta[h] >> 32));
  SC[12] = rfl(rf ? 15 * 128u : 0u); SC[13] = rfl(rf ? 0u : 15 * 128u); SC[14] = rfl(rf ? -128u : 128u); SC[15] = 0;
  asm volatile(GEMM_ASM
    : "+{v[0:7]}"(V0), "+{v[8:15]}"(V1), "+{v[16:31]}"(VB), "+{v[32:47]}"(VC), "+{v[48:55]}"(VE), "+{v[76:83]}"(VT), "+{v86}"(tk), "+{v[56:59]}"(X0), "+{s[40:43]}"(SD), "+{s[44:51]}"(SA), "+{s[52:67]}"(SB), "+{s[68:83]}"(SC), "+{s[8:9]}"(ltab), "+{s[24:27]}"(SG)
    :: GEMM_CLOBBERS);
}
extern "C" void kernel_launch(int rank, int nranks, ncclComm_t comm, hipStream_t stream, void* const* d_in, const int* in_sizes, int n_in,
                              void* d_out, int out_size, const int* in_dtypes, int out_dtype, void* const* d_out_all, void* const* d_ws_all, size_t ws_bytes) {
  if (nranks != 8 || (unsigned)rank >= 8u || ws_bytes < (size_t)CTRL + CTRL_BYTES || out_size != 1024 * 8192 || n_in != 2 || in_sizes[0] != 8192 * 1024 || in_sizes[1] != 8192 * 8192) {
    fprintf(stderr, "a2a_gemm: unexpected problem shape\n"); return; }
  hipError_t e = hipFuncSetAttribute((const void*)a2a_gemm, hipFuncAttributeMaxDynamicSharedMemorySize, 163840);
  if (e != hipSuccess) { fprintf(stderr, "HIP error %s\n", hipGetErrorString(e)); return; }
  Args A; A.x = (const char*)d_in[0]; A.W = (const char*)d_in[1]; A.out = (char*)d_out; A.rank = (unsigned)rank; A.pad = 0;
  for (int p = 0; p < 8; ++p) A.ws[p] = (char*)d_ws_all[p];
  for (int h = 0; h < 2; ++h) {
    int u[8]; for (int n = 0; n < 8; ++n) { const int q = h ? 111 - n : n; u[n] = 16 * ((rank + 1 + q % 7) & 7) + q / 7; }
    unsigned long long d = 0; for (int n = 0; n < 7; ++n) d |= (unsigned long long)(unsigned char)(signed char)(u[n + 1] - u[n]) << (8 * n);
    A.delta[h] = d; A.u8[h] = (unsigned)(u[6] + (h ? 1 : -1)); }
  a2a_gemm<<<256, 256, 163840, stream>>>(A);
}
```
